```python
import numpy as np

kernel_name = "gpu_a2a_gemm_m8192_k8192_n8192_bf16_gelu"


ABSMAX_RTOL = 0.008

COMPUTE_DTYPE = "bfloat16"
EPILOGUE = "gelu"
IN_QUANT = None
OUT_QUANT = None
IN_DTYPES = {"x": "bfloat16", "w_mat": "bfloat16"}
WORLD_SIZE = 8

def _to_bf16(x):
    u = np.ascontiguousarray(x, dtype=np.float32).view(np.uint32)
    return ((u + 0x7FFF + ((u >> 16) & 1)) & 0xFFFF0000).view(np.float32)

def setup_inputs():

    x = _to_bf16(np.random.randn(8192, 8192).astype(np.float32))
    w_mat = _to_bf16((np.random.randn(8192, 8192) / np.sqrt(8192)).astype(np.float32))
    return {"x": x, "w_mat": w_mat}

def _epilogue(y):
    c = 0.7978845608028654
    return (0.5 * y * (1.0 + np.tanh(c * (y + 0.044715 * y ** 3)))).astype(np.float32)

def reference(x, w_mat):
    y = (x @ w_mat).astype(np.float32)
    return _epilogue(y)
```

```cpp
#include <hip/hip_runtime.h>
#include <rccl/rccl.h>
#include <cstdio>

#define HIP_CHECK(expr) do { hipError_t _e = (expr); if (_e != hipSuccess) { \
    fprintf(stderr, "HIP error %s at %s:%d\n", hipGetErrorString(_e), __FILE__, __LINE__); \
    return; } } while (0)

namespace a2a {
#define LAS __attribute__((address_space(3)))
typedef short bf16x8 __attribute__((ext_vector_type(8)));
typedef short s16x4 __attribute__((ext_vector_type(4)));
typedef float f32x4 __attribute__((ext_vector_type(4)));
typedef unsigned u32x4 __attribute__((ext_vector_type(4)));

constexpr int NR = 8;
constexpr int HTB = 16384;
constexpr int LDS_X = 8 * HTB;
constexpr int LDS_BYTES = 8 * HTB + 8 * 2048;
constexpr size_t WS_CTR = 0;
constexpr size_t WS_TMO = 32768;
constexpr size_t WS_FLAG = 65536;
constexpr size_t WS_XFLAG = 131072;
constexpr size_t WS_STAGE = 1ull << 20;
constexpr size_t WS_XBUF = 33ull << 20;
constexpr size_t WS_END = 65ull << 20;
constexpr unsigned long long DEADLINE = 50000000ull;

struct Params {
    const char* x;
    const char* w;
    char* out;
    char* ws[NR];
    int rank;
};

typedef float f32x2 __attribute__((ext_vector_type(2)));
typedef __bf16 bf16x2 __attribute__((ext_vector_type(2)));
__device__ __forceinline__ unsigned cvt_pk_bf16(float lo, float hi) {
    return __builtin_bit_cast(unsigned, __builtin_convertvector((f32x2){lo, hi}, bf16x2));
}
__device__ __forceinline__ float gelu_tanh(float y) {
    const float c2 = 2.0f * 0.7978845608028654f * 1.4426950408889634f;
    const float u = y * (1.0f + 0.044715f * y * y);
    const float e = __builtin_amdgcn_exp2f(-c2 * u);
    return y * __builtin_amdgcn_rcpf(1.0f + e);
}

__global__ __launch_bounds__(512, 2) void a2a_gemm_kernel(Params p) {
    extern __shared__ __attribute__((aligned(16))) unsigned char shm[];
    LAS unsigned char* lds = (LAS unsigned char*)shm;
    const int tid = threadIdx.x, wid = __builtin_amdgcn_readfirstlane(tid >> 6), lane = tid & 63, wr = wid >> 2, wc = wid & 3, fr = lane & 15, fq = lane >> 4;
    const int bid = blockIdx.x, xcd = bid & 7, idx = bid >> 3;
    const int kh = idx & 1, mt = (idx >> 1) & 3, nt = xcd * 4 + (idx >> 3);
    const int rank = p.rank;
    char* const myws = p.ws[rank];

    const unsigned ra = tid >> 3, ca = (tid & 7) ^ ((tid >> 4) & 7);
    const unsigned voffA = ra * 2048u + ca * 16u;
    const unsigned kb = tid >> 4, cb = tid & 15, fb = (kb & 3) | ((kb >> 1) & 4);
    const unsigned voffB = kb * 16384u + (((cb >> 1) ^ fb) * 32u) + (cb & 1) * 16u;
    const unsigned ldsw = (unsigned)wid * 1024u;
    const int aoff0 = (wr * 64 + fr) * 128 + ((fq ^ (fr >> 1)) * 16);
    const int aoff1 = aoff0 ^ 64;
    const int q4 = (lane >> 2) & 3, p4 = lane & 3, fl = q4 | ((fq & 1) << 2);
    const int boff0 = (8 * fq + q4) * 256 + (((wc * 2) ^ fl) * 32) + 8 * p4;
    const unsigned vb0 = (unsigned)(size_t)lds + 65536u + (unsigned)boff0, vb1 = (unsigned)(size_t)lds + 65536u + (unsigned)(boff0 ^ 32);
    const unsigned voffP = (unsigned)(lane >> 3) * 2048u + (unsigned)(lane & 7) * 16u;
    const unsigned voffF = (unsigned)lane * 4u;
    const unsigned vzero = 0u;
    LAS unsigned char* const ldx = lds + LDS_X + wid * 2048;
    const unsigned vxp = (unsigned)(size_t)ldx + (unsigned)lane * 16u, vxf = (unsigned)(size_t)ldx + 1024u + (unsigned)lane * 4u;

#define SA(b, h) (((b) * 2 + (h)) * HTB)
#define SB(b, h) ((4 + (b) * 2 + (h)) * HTB)
#define AUX_STAGED 17
#define STAGE_A(bufoff, rs, soff) do { \
        __builtin_amdgcn_raw_ptr_buffer_load_lds(rs, (LAS void*)(lds + (bufoff) + ldsw), 16, voffA, (soff), 0, AUX_STAGED); \
        __builtin_amdgcn_raw_ptr_buffer_load_lds(rs, (LAS void*)(lds + (bufoff) + ldsw + 8192), 16, voffA, (soff) + 64 * 2048, 0, AUX_STAGED); } while (0)
#define STAGE_B(bufoff, soff) do { \
        __builtin_amdgcn_raw_ptr_buffer_load_lds(rsW, (LAS void*)(lds + (bufoff) + ldsw), 16, voffB, (soff), 0, 0); \
        __builtin_amdgcn_raw_ptr_buffer_load_lds(rsW, (LAS void*)(lds + (bufoff) + ldsw + 8192), 16, voffB, (soff) + 32 * 16384, 0, 0); } while (0)
#define LDA(dst, b, h) do { _Pragma("unroll") for (int m = 0; m < 4; ++m) { \
        dst[m][0] = *(const LAS bf16x8*)(lds + SA(b, h) + aoff0 + m * 2048); \
        dst[m][1] = *(const LAS bf16x8*)(lds + SA(b, h) + aoff1 + m * 2048); } } while (0)
#define TR(d, base, off) asm volatile("ds_read_b64_tr_b16 %0, %1 offset:%2" : "=v"(d) : "v"(base), "i"(off) : "memory")
#define LDB(dst, b, h) do { _Pragma("unroll") for (int k = 0; k < 2; ++k) { s16x4 l0, h0, l1, h1; \
        TR(l0, vb0, SB(b, h) - 65536 + k * 8192); TR(h0, vb0, SB(b, h) - 65536 + k * 8192 + 1024); \
        TR(l1, vb1, SB(b, h) - 65536 + k * 8192); TR(h1, vb1, SB(b, h) - 65536 + k * 8192 + 1024); \
        dst[0][k] = __builtin_shufflevector(l0, h0, 0, 1, 2, 3, 4, 5, 6, 7); \
        dst[1][k] = __builtin_shufflevector(l1, h1, 0, 1, 2, 3, 4, 5, 6, 7); } } while (0)
#define MMA(ai, bj, At, Bt) do { __builtin_amdgcn_s_setprio(1); _Pragma("unroll") for (int k = 0; k < 2; ++k) _Pragma("unroll") for (int m = 0; m < 4; ++m) _Pragma("unroll") for (int n = 0; n < 2; ++n) \
        acc[ai][bj][m][n] = __builtin_amdgcn_mfma_f32_16x16x32_bf16(Bt[n][k], At[m][k], acc[ai][bj][m][n], 0, 0, 0); __builtin_amdgcn_s_setprio(0); } while (0)
#define WAIT_V(n) asm volatile("s_waitcnt vmcnt(" #n ")" ::: "memory")
#define WAIT_L(n) do { asm volatile("s_waitcnt lgkmcnt(" #n ")" ::: "memory"); __builtin_amdgcn_sched_barrier(0); } while (0)
#define BAR __builtin_amdgcn_s_barrier()
#define SCHED __builtin_amdgcn_sched_barrier(0)
#define POLL(q) __builtin_amdgcn_raw_ptr_buffer_load_lds(rsF, (LAS void*)(ldx + 1024 + (q) * 256), 4, voffF, sflag, 0, 17)

    f32x4 acc[2][2][4][2];
#pragma unroll
    for (int a = 0; a < 2; ++a)
#pragma unroll
        for (int b = 0; b < 2; ++b)
#pragma unroll
            for (int m = 0; m < 4; ++m)
#pragma unroll
                for (int n = 0; n < 2; ++n) acc[a][b][m][n] = (f32x4){0.f, 0.f, 0.f, 0.f};
    bf16x8 At[4][2], B0[2][2], B1[2][2];

    const unsigned arow0 = (unsigned)(mt * 256 + kh * 128) * 2048u, arow1 = (unsigned)(mt * 256 + (1 - kh) * 128) * 2048u;
    const __amdgpu_buffer_rsrc_t rsW = __builtin_amdgcn_make_buffer_rsrc((void*)p.w, 0, 128u << 20, 0x00020000);
    const __amdgpu_buffer_rsrc_t rsX = __builtin_amdgcn_make_buffer_rsrc((void*)p.x, 0, 16u << 20, 0x00020000);
    const __amdgpu_buffer_rsrc_t rsF = __builtin_amdgcn_make_buffer_rsrc((void*)(myws + WS_FLAG), 0, 32768u, 0x00020000);
    unsigned sflag = (unsigned)((mt * 8 + wid) * 2 + kh) * 256u;
    const char* const pL = p.x + (size_t)rank * (2u << 20);
    const char* const pS = myws + WS_STAGE;
#define RSA(t) __builtin_amdgcn_make_buffer_rsrc((void*)((t) < 8 ? pL : pS), 0, 32u << 20, 0x00020000)
    const unsigned m0push = (unsigned)(size_t)ldx, m0poll = m0push + 1024u;

    unsigned tabA, tabB;
    {
        const int u = lane - 8, j = u / 7, i0 = u - j * 7, i = kh ? 6 - i0 : i0;
        const int c = lane < 8 ? rank : (rank + 1 + i) & 7, kt = kh * 8 + (lane < 8 ? lane : j);
        tabA = (lane < 8 ? 0u : (unsigned)c * (2u << 20)) + (unsigned)kt * 128u;
        tabB = (unsigned)(c * 16 + kt) * (64u * 16384u) + (unsigned)nt * 512u;
    }
    const int wq = wid * 256 + bid;
    const int oslot = (wq * 5) >> 11;
    unsigned tabLo, tabHi, tabD, tabF, tabS;
    {
        const int sl = lane < 7 ? lane : 6;
        const int Q = sl * 2048 + wq, b = Q / 896, rem = Q - b * 896, rg = rem / 7, di = rem - rg * 7;
        const int d = (rank + 1 + di) & 7, ipos = 6 - di, kk = (b & 1) ^ (ipos > 3 ? 1 : 0), kt = kk * 8 + (b >> 1);
        unsigned long long base = 0;
#pragma unroll
        for (int i = 0; i < NR; ++i) base = d == i ? (unsigned long long)p.ws[i] : base;
        tabLo = (unsigned)base; tabHi = (unsigned)(base >> 32);
        tabS = (unsigned)(d * 1024 + rg * 8) * 2048u + (unsigned)kt * 128u;
        tabD = (unsigned)WS_STAGE + (unsigned)rank * (2u << 20) + (unsigned)(rg * 8) * 2048u + (unsigned)kt * 128u;
        tabF = (unsigned)WS_FLAG + (unsigned)((((rg >> 5) * 8 + (rg & 7)) * 2 + kk) * 256 + (8 + (b >> 1) * 7 + (kk ? 6 - ipos : ipos)) * 4 + ((rg >> 3) & 3));
    }
    u32x4 pdat = {0u, 0u, 0u, 0u};
    unsigned fl0 = 0u, fl1 = 0u, callv = 0u;
#define PUSH_RD0() asm volatile("ds_read_b128 %0, %1" : "=v"(pdat) : "v"(vxp) : "memory")

    {
        __builtin_amdgcn_raw_ptr_buffer_load_lds(rsX, (LAS void*)ldx, 16, voffP, __builtin_amdgcn_readlane(tabS, 0), 0, 0);
        const unsigned one = 1u, coff = (unsigned)(bid * 8 + wid) * 16u;
        asm volatile("global_atomic_add %0, %1, %2, %3 sc0" : "=v"(callv) : "v"(coff), "v"(one), "s"(myws + WS_CTR) : "memory");
        const __amdgpu_buffer_rsrc_t rsL = RSA(0);
        const unsigned a0 = (unsigned)kh * 1024u, b0 = (unsigned)(rank * 16 + kh * 8) * (64u * 16384u) + (unsigned)nt * 512u, b1 = b0 + 64u * 16384u;
        STAGE_B(SB(0, 0), b0);
        STAGE_A(SA(0, 0), rsL, a0 + arow0); POLL(2);
        STAGE_B(SB(0, 1), b0 + 256);
        STAGE_A(SA(0, 1), rsL, a0 + arow1); POLL(2);
        STAGE_B(SB(1, 0), b1);
        STAGE_A(SA(1, 0), rsL, a0 + 128 + arow0); POLL(2);
        STAGE_B(SB(1, 1), b1 + 256);
        if (wr == 1) BAR;
        asm volatile("s_waitcnt vmcnt(15)" : "+v"(callv) :: "memory");
        BAR; BAR;
        LDB(B0, 0, 0); PUSH_RD0(); WAIT_V(13); BAR; WAIT_L(0); BAR;
    }
    const unsigned call = ((unsigned)__builtin_amdgcn_readfirstlane(callv) >> 6) + 1u;
    const unsigned spar = (call & 1u) << 24;
    const unsigned fpar = (call & 1u) << 14;
    const unsigned fval = (call & 0x7fu) | 0x80u, fpat = fval * 0x01010101u;
    sflag += fpar;
    tabA += lane < 8 ? 0u : spar; tabD += spar; tabF += fpar;
    unsigned long long ready = 0xffull;
    unsigned* const tmo = (unsigned*)(myws + WS_TMO);

    int ps = -oslot;
    int pn = 0;
    int mode = 1;
#define PUSH_SLOT() do { \
        const int pi = pn < 6 ? pn : 6; unsigned vt, sm; \
        const unsigned long long pb = (unsigned long long)(unsigned)__builtin_amdgcn_readlane(tabLo, pi) | ((unsigned long long)(unsigned)__builtin_amdgcn_readlane(tabHi, pi) << 32); \
        const unsigned pD = __builtin_amdgcn_readlane(tabD, pi), pF = __builtin_amdgcn_readlane(tabF, pi), pSr = __builtin_amdgcn_readlane(tabS, pi); \
        const int act = __builtin_amdgcn_readfirstlane(mode == 0 ? 5 : (pn >= 7 ? 1 : ps));       \
        asm volatile( \
            "s_cmp_eq_i32 %[act], 5\n\t" \
            "s_cbranch_scc1 .Lpe%=\n\t" \
            "s_mov_b32 %[sm], m0\n\t" \
            "s_nop 1\n\t" \
            "s_cmp_eq_i32 %[act], 0\n\t" \
            "s_cbranch_scc1 .Lps%=\n\t" \
            "s_cmp_eq_i32 %[act], 2\n\t" \
            "s_cbranch_scc1 .Lpf%=\n\t" \
            "s_cmp_eq_i32 %[act], 3\n\t" \
            "s_cbranch_scc1 .Lpl%=\n\t" \
            "s_mov_b32 m0, %[m0t]\n\t" \
            "s_nop 0\n\t" \
            "buffer_load_dword %[voffF], %[rsF], %[sflag] offen sc0 sc1 lds\n\t" \
            "s_branch .Lpn%=\n" \
            ".Lps%=:\n\t" \
            "v_add_u32 %[vt], %[pD], %[voffP]\n\t" \
            "global_store_dwordx4 %[vt], %[pdat], %[pb] sc0 sc1\n\t" \
            "s_branch .Lpn%=\n" \
            ".Lpf%=:\n\t" \
            "v_mov_b32 %[vt], %[pF]\n\t" \
            "s_mov_b64 exec, 1\n\t" \
            "global_store_byte %[vt], %[fval], %[pb] sc0 sc1\n\t" \
            "s_mov_b64 exec, -1\n\t" \
            "s_branch .Lpn%=\n" \
            ".Lpl%=:\n\t" \
            "s_mov_b32 m0, %[m0p]\n\t" \
            "s_nop 0\n\t" \
            "buffer_load_dwordx4 %[voffP], %[rsX], %[pSr] offen lds\n" \
            ".Lpn%=:\n\t" \
            "s_mov_b32 m0, %[sm]\n\t" \
            "s_nop 0\n" \
            ".Lpe%=:" \
            : [vt] "=&v"(vt), [sm] "=&s"(sm) \
            : [act] "s"(act), [m0t] "s"(m0poll + 512u), [m0p] "s"(m0push), [voffF] "v"(voffF), [rsF] "s"(rsF), [sflag] "s"(sflag), \
              [pD] "s"(pD), [pF] "s"(pF), [pSr] "s"(pSr), [pb] "s"(pb), [voffP] "v"(voffP), [pdat] "v"(pdat), [fval] "v"(fval), [rsX] "s"(rsX) \
            : "memory", "scc"); \
        pn += act == 2 ? 1 : 0; ps = ps == 4 ? 0 : ps + 1; } while (0)
#define POLLC(q) do { unsigned sm; asm volatile( \
            "s_cmp_eq_u32 %[mode], 0\n\t" \
            "s_cbranch_scc1 .Lqe%=\n\t" \
            "s_mov_b32 %[sm], m0\n\t" \
            "s_mov_b32 m0, %[m0q]\n\t" \
            "s_nop 0\n\t" \
            "buffer_load_dword %[voffF], %[rsF], %[sflag] offen sc0 sc1 lds\n\t" \
            "s_mov_b32 m0, %[sm]\n\t" \
            "s_nop 0\n" \
            ".Lqe%=:" \
            : [sm] "=&s"(sm) : [mode] "s"(mode), [m0q] "s"(m0poll + (q) * 256u), [voffF] "v"(voffF), [rsF] "s"(rsF), [sflag] "s"(sflag) : "memory", "scc"); } while (0)
#define POLL_RDC(q, f) asm volatile("s_cmp_eq_u32 %[mode], 0\n\ts_cbranch_scc1 .Lre%=\n\tds_read_b32 %[d], %[a] offset:%[o]\n.Lre%=:" \
            : [d] "+v"(f) : [a] "v"(vxf), [o] "i"((q) * 256), [mode] "s"(mode) : "memory", "scc")
#define PUSH_RDC() asm volatile("s_cmp_eq_u32 %[mode], 0\n\ts_cbranch_scc1 .Lre%=\n\tds_read_b128 %[d], %[a]\n.Lre%=:" \
            : [d] "+v"(pdat) : [a] "v"(vxp), [mode] "s"(mode) : "memory", "scc")
#define WAIT_C() asm volatile("s_waitcnt vmcnt(13)\n\ts_cmp_lg_u32 %[mode], 0\n\ts_cbranch_scc1 .Lwe%=\n\ts_waitcnt vmcnt(10)\n.Lwe%=:" \
            :: [mode] "s"(mode) : "memory", "scc")
#define NEED(tt, q, f) do { unsigned v1, sm; \
        ready |= __builtin_amdgcn_ballot_w64(f == fpat) & live; \
        asm volatile( \
            "s_bitcmp1_b64 %[rdy], %[t]\n\t" \
            "s_cbranch_scc1 .Lne%=\n\t" \
            "s_memrealtime s[96:97]\n\t" \
            "s_mov_b32 %[sm], m0\n\t" \
            "s_mov_b32 m0, %[m0q]\n\t" \
            "s_waitcnt lgkmcnt(0)\n" \
            ".Lnl%=:\n\t" \
            "buffer_load_dword %[voffF], %[rsF], %[sflag] offen sc0 sc1 lds\n\t" \
            "global_load_dword %[v1], %[vz], %[tmo] sc0 sc1\n\t" \
            "s_waitcnt vmcnt(0)\n\t" \
            "ds_read_b32 %[v0], %[a] offset:%[o]\n\t" \
            "s_memrealtime s[98:99]\n\t" \
            "s_waitcnt lgkmcnt(0)\n\t" \
            "v_cmp_eq_u32 vcc, %[fpat], %[v0]\n\t" \
            "s_sub_u32 s98, s98, s96\n\t" \
            "s_nop 0\n\t" \
            "s_bitcmp1_b64 vcc, %[t]\n\t" \
            "s_cbranch_scc1 .Lnx%=\n\t" \
            "v_cmp_eq_u32 vcc, %[call], %[v1]\n\t" \
            "s_cmp_gt_u32 s98, %[dl]\n\t" \
            "s_cbranch_scc1 .Lnd%=\n\t" \
            "s_cbranch_vccnz .Lnd%=\n\t" \
            "s_sleep 1\n\t" \
            "s_branch .Lnl%=\n" \
            ".Lnd%=:\n\t" \
            "v_mov_b32 %[v1], %[call]\n\t" \
            "v_mov_b32 %[v0], %[fpat]\n\t" \
            "global_store_dword %[vz], %[v1], %[tmo] sc0 sc1\n\t" \
            "s_waitcnt vmcnt(0)\n" \
            ".Lnx%=:\n\t" \
            "s_mov_b32 m0, %[sm]\n\t" \
            "s_nop 0\n" \
            ".Lne%=:" \
            : [v0] "+v"(f), [v1] "=&v"(v1), [sm] "=&s"(sm) \
            : [rdy] "s"(ready), [t] "s"(tt), [m0q] "s"(m0poll + (q) * 256u), [voffF] "v"(voffF), [rsF] "s"(rsF), [sflag] "s"(sflag), [vz] "v"(vzero), [tmo] "s"(tmo), \
              [a] "v"(vxf), [o] "i"((q) * 256), [fpat] "s"(fpat), [call] "s"(call), [dl] "s"((unsigned)DEADLINE) \
            : "memory", "scc", "vcc", "s96", "s97", "s98", "s99"); \
        ready |= __builtin_amdgcn_ballot_w64(f == fpat) & live; } while (0)

    unsigned a1 = (unsigned)kh * 1024u + 128u;
    for (int t = 0; t < 64; t += 2) {
        mode = __builtin_amdgcn_readfirstlane((pn == 7 && ready == ~0ull) ? 0 : 1);
        const int t2 = t + 2 < 63 ? t + 2 : 63, t3 = t + 3 < 63 ? t + 3 : 63;
        const unsigned a2 = __builtin_amdgcn_readlane(tabA, t2), b2 = __builtin_amdgcn_readlane(tabB, t2);
        const unsigned a3 = __builtin_amdgcn_readlane(tabA, t3), b3 = __builtin_amdgcn_readlane(tabB, t3);
        const __amdgpu_buffer_rsrc_t r1 = RSA(t + 1), r2 = RSA(t + 2);
        const unsigned long long live = t > 0 ? ~0ull : 0ull;
        LDA(At, 0, 0); STAGE_A(SA(1, 1), r1, a1 + arow1); PUSH_SLOT();
        WAIT_C(); BAR; WAIT_L(0); MMA(0, 0, At, B0); BAR; SCHED;
        LDB(B1, 0, 1); POLL_RDC(0, fl0); STAGE_B(SB(0, 0), b2);
        WAIT_C(); BAR; WAIT_L(0); MMA(0, 1, At, B1); BAR; SCHED;
        LDA(At, 0, 1); NEED(t + 2, 0, fl0);
        STAGE_A(SA(0, 0), r2, a2 + arow0); POLLC(0);
        WAIT_C(); BAR; WAIT_L(0); MMA(1, 0, At, B0); BAR; SCHED;
        LDB(B0, 1, 0); PUSH_RDC(); STAGE_B(SB(0, 1), b2 + 256);
        WAIT_C(); BAR; MMA(1, 1, At, B1); SCHED; WAIT_L(0); BAR; SCHED;
        LDA(At, 1, 0); STAGE_A(SA(0, 1), r2, a2 + arow1); PUSH_SLOT();
        WAIT_C(); BAR; WAIT_L(0); MMA(0, 0, At, B0); BAR; SCHED;
        LDB(B1, 1, 1); POLL_RDC(1, fl1); STAGE_B(SB(1, 0), b3);
        WAIT_C(); BAR; WAIT_L(0); MMA(0, 1, At, B1); BAR; SCHED;
        LDA(At, 1, 1); NEED(t + 3, 1, fl1);
        STAGE_A(SA(1, 0), r2, a3 + arow0); POLLC(1);
        WAIT_C(); BAR; WAIT_L(0); MMA(1, 0, At, B0); BAR; SCHED;
        LDB(B0, 0, 0); PUSH_RDC(); STAGE_B(SB(1, 1), b3 + 256);
        WAIT_C(); BAR; MMA(1, 1, At, B1); SCHED; WAIT_L(0); BAR; SCHED;
        a1 = a3;
    }
    WAIT_V(0);

    {
        int tz = threadIdx.x; asm volatile("" : "+v"(tz));
        const int lane2 = tz & 63, fr2 = lane2 & 15, fq2 = lane2 >> 4;
        const int pbid = bid ^ 8;
        const __amdgpu_buffer_rsrc_t xr = __builtin_amdgcn_make_buffer_rsrc(myws + WS_XBUF, 0, 256u * 131072u, 0x00020000);
        const unsigned myoff = (unsigned)bid * 131072u + (unsigned)tz * 16u, poff = (unsigned)pbid * 131072u + (unsigned)tz * 16u;
#pragma unroll
        for (int bj = 0; bj < 2; ++bj)
#pragma unroll
            for (int m = 0; m < 4; ++m)
#pragma unroll
                for (int n = 0; n < 2; ++n)
                    __builtin_amdgcn_raw_buffer_store_b128(__builtin_bit_cast(u32x4, acc[1][bj][m][n]), xr, myoff + ((bj * 4 + m) * 2 + n) * 8192, 0, 16);
        WAIT_V(0);
        unsigned* xf = (unsigned*)(myws + WS_XFLAG);
        if (lane2 == 0) __hip_atomic_store(xf + (bid * 8 + wid) * 4, call, __ATOMIC_RELAXED, __HIP_MEMORY_SCOPE_AGENT);
        {
            const unsigned long long t0 = __builtin_amdgcn_s_memrealtime();
            while ((unsigned)__builtin_amdgcn_readfirstlane(__hip_atomic_load(xf + (pbid * 8 + wid) * 4, __ATOMIC_RELAXED, __HIP_MEMORY_SCOPE_AGENT)) < call) {
                if ((unsigned)__builtin_amdgcn_readfirstlane(__hip_atomic_load(tmo, __ATOMIC_RELAXED, __HIP_MEMORY_SCOPE_AGENT)) == call ||
                    __builtin_amdgcn_s_memrealtime() - t0 > DEADLINE) {
                    __hip_atomic_store(tmo, call, __ATOMIC_RELAXED, __HIP_MEMORY_SCOPE_AGENT); break; }
                __builtin_amdgcn_s_sleep(1);
            }
            asm volatile("" ::: "memory");
        }
        const bool bad = (unsigned)__builtin_amdgcn_readfirstlane(__hip_atomic_load(tmo, __ATOMIC_RELAXED, __HIP_MEMORY_SCOPE_AGENT)) == call;
        f32x4 pv[2][4][2];
#pragma unroll
        for (int bj = 0; bj < 2; ++bj)
#pragma unroll
            for (int m = 0; m < 4; ++m)
#pragma unroll
                for (int n = 0; n < 2; ++n)
                    pv[bj][m][n] = __builtin_bit_cast(f32x4, __builtin_amdgcn_raw_buffer_load_b128(xr, poff + ((bj * 4 + m) * 2 + n) * 8192, 0, 16));
        char* orow = p.out + (size_t)(mt * 256 + kh * 128 + wr * 64 + fr2) * 16384 + (size_t)(nt * 256 + wc * 32 + (fq2 & 1) * 16 + (fq2 >> 1) * 8) * 2;
#pragma unroll
        for (int m = 0; m < 4; ++m)
#pragma unroll
            for (int bj = 0; bj < 2; ++bj) {
                const f32x4 v0 = acc[0][bj][m][0] + pv[bj][m][0], v1 = acc[0][bj][m][1] + pv[bj][m][1];
                const unsigned x0 = cvt_pk_bf16(gelu_tanh(v0[0]), gelu_tanh(v0[1])), x1 = cvt_pk_bf16(gelu_tanh(v0[2]), gelu_tanh(v0[3]));
                const unsigned y0 = cvt_pk_bf16(gelu_tanh(v1[0]), gelu_tanh(v1[1])), y1 = cvt_pk_bf16(gelu_tanh(v1[2]), gelu_tanh(v1[3]));
                const auto r0 = __builtin_amdgcn_permlane16_swap(x0, y0, false, false);
                const auto r1 = __builtin_amdgcn_permlane16_swap(x1, y1, false, false);
                u32x4 o; o.x = r0[0]; o.y = r1[0]; o.z = r0[1]; o.w = r1[1];
                if (bad) o = (u32x4){0x7fc07fc0u, 0x7fc07fc0u, 0x7fc07fc0u, 0x7fc07fc0u};
                *(u32x4*)(orow + (size_t)m * 16 * 16384 + bj * 256) = o;
            }
    }
    if (wr == 0) BAR;
}
}

extern "C" void kernel_launch(int rank, int nranks, ncclComm_t comm,
                              hipStream_t stream,
                              void* const* d_in, const int* in_sizes, int n_in,
                              void* d_out, int out_size,
                              const int* in_dtypes, int out_dtype,
                              void* const* d_out_all,
                              void* const* d_ws_all, size_t ws_bytes) {
    (void)comm; (void)in_sizes; (void)n_in; (void)out_size; (void)in_dtypes; (void)out_dtype; (void)d_out_all;
    if (nranks != a2a::NR || (unsigned)rank >= (unsigned)a2a::NR || ws_bytes < a2a::WS_END) {
        fprintf(stderr, "a2a_gemm: unexpected configuration (nranks %d, ws %zu)\n", nranks, ws_bytes);
        return;
    }
    HIP_CHECK(hipFuncSetAttribute((const void*)a2a::a2a_gemm_kernel, hipFuncAttributeMaxDynamicSharedMemorySize, a2a::LDS_BYTES));
    a2a::Params p;
    p.x = (const char*)d_in[0];
    p.w = (const char*)d_in[1];
    p.out = (char*)d_out;
    for (int i = 0; i < a2a::NR; ++i) p.ws[i] = (char*)d_ws_all[i];
    p.rank = rank;
    hipLaunchKernelGGL(a2a::a2a_gemm_kernel, dim3(256), dim3(512), a2a::LDS_BYTES, stream, p);
    HIP_CHECK(hipGetLastError());
}
```

```cpp
#include <hip/hip_runtime.h>
#include <rccl/rccl.h>
#include <cstdio>

#define HIP_CHECK(expr) do { hipError_t _e = (expr); if (_e != hipSuccess) { \
    fprintf(stderr, "HIP error %s at %s:%d\n", hipGetErrorString(_e), __FILE__, __LINE__); \
    return; } } while (0)

namespace a2a {
#define LAS __attribute__((address_space(3)))
typedef short bf16x8 __attribute__((ext_vector_type(8)));
typedef short s16x4 __attribute__((ext_vector_type(4)));
typedef float f32x4 __attribute__((ext_vector_type(4)));
typedef unsigned u32x4 __attribute__((ext_vector_type(4)));

constexpr int NR = 8;
constexpr int HTB = 16384;
constexpr int LDS_X = 8 * HTB;
constexpr int LDS_BYTES = 8 * HTB + 8 * 2048;
constexpr size_t WS_CTR = 0;
constexpr size_t WS_TMO = 32768;
constexpr size_t WS_FLAG = 65536;
constexpr size_t WS_XFLAG = 131072;
constexpr size_t WS_STAGE = 1ull << 20;
constexpr size_t WS_XBUF = 33ull << 20;
constexpr size_t WS_END = 65ull << 20;
constexpr unsigned long long DEADLINE = 50000000ull;

struct Params {
    const char* x;
    const char* w;
    char* out;
    char* ws[NR];
    int rank;
};

typedef float f32x2 __attribute__((ext_vector_type(2)));
typedef __bf16 bf16x2 __attribute__((ext_vector_type(2)));
__device__ __forceinline__ unsigned cvt_pk_bf16(float lo, float hi) {
    return __builtin_bit_cast(unsigned, __builtin_convertvector((f32x2){lo, hi}, bf16x2));
}
__device__ __forceinline__ float gelu_tanh(float y) {
    const float c2 = 2.0f * 0.7978845608028654f * 1.4426950408889634f;
    const float u = y * (1.0f + 0.044715f * y * y);
    const float e = __builtin_amdgcn_exp2f(-c2 * u);
    return y * __builtin_amdgcn_rcpf(1.0f + e);
}

#define ST(i)

__global__ __launch_bounds__(512, 2) void a2a_gemm_kernel(Params p) {
    extern __shared__ __attribute__((aligned(16))) unsigned char shm[];
    LAS unsigned char* lds = (LAS unsigned char*)shm;
    const int tid = threadIdx.x, wid = __builtin_amdgcn_readfirstlane(tid >> 6), lane = tid & 63, wr = wid >> 2, wc = wid & 3, fr = lane & 15, fq = lane >> 4;
    const int bid = blockIdx.x, xcd = bid & 7, idx = bid >> 3;
    const int kh = idx & 1, mt = (idx >> 1) & 3, nt = xcd * 4 + (idx >> 3);
    const int rank = p.rank;
    char* const myws = p.ws[rank];
    ST(0);

    const unsigned ra = tid >> 3, ca = (tid & 7) ^ ((tid >> 4) & 7);
    const unsigned voffA = ra * 2048u + ca * 16u;
    const unsigned kb = tid >> 4, cb = tid & 15, fb = (kb & 3) | ((kb >> 1) & 4);
    const unsigned voffB = kb * 16384u + (((cb >> 1) ^ fb) * 32u) + (cb & 1) * 16u;
    const unsigned ldsw = (unsigned)wid * 1024u;
    const int aoff0 = (wr * 64 + fr) * 128 + ((fq ^ (fr >> 1)) * 16);
    const int aoff1 = aoff0 ^ 64;
    const int q4 = (lane >> 2) & 3, p4 = lane & 3, fl = q4 | ((fq & 1) << 2);
    const int boff0 = (8 * fq + q4) * 256 + (((wc * 2) ^ fl) * 32) + 8 * p4;
    const unsigned vb0 = (unsigned)(size_t)lds + 65536u + (unsigned)boff0, vb1 = (unsigned)(size_t)lds + 65536u + (unsigned)(boff0 ^ 32);
    const unsigned voffP = (unsigned)(lane >> 3) * 2048u + (unsigned)(lane & 7) * 16u;
    const unsigned voffF = (unsigned)lane * 4u;
    const unsigned vzero = 0u;
    LAS unsigned char* const ldx = lds + LDS_X + wid * 2048;
    const unsigned vxp = (unsigned)(size_t)ldx + (unsigned)lane * 16u, vxf = (unsigned)(size_t)ldx + 1024u + (unsigned)lane * 4u;

#define SA(b, h) (((b) * 2 + (h)) * HTB)
#define SB(b, h) ((4 + (b) * 2 + (h)) * HTB)
#define AUX_STAGED 17
#define STAGE_A(bufoff, rs, soff) do { \
        __builtin_amdgcn_raw_ptr_buffer_load_lds(rs, (LAS void*)(lds + (bufoff) + ldsw), 16, voffA, (soff), 0, AUX_STAGED); \
        __builtin_amdgcn_raw_ptr_buffer_load_lds(rs, (LAS void*)(lds + (bufoff) + ldsw + 8192), 16, voffA, (soff) + 64 * 2048, 0, AUX_STAGED); } while (0)
#define STAGE_B(bufoff, soff) do { \
        __builtin_amdgcn_raw_ptr_buffer_load_lds(rsW, (LAS void*)(lds + (bufoff) + ldsw), 16, voffB, (soff), 0, 0); \
        __builtin_amdgcn_raw_ptr_buffer_load_lds(rsW, (LAS void*)(lds + (bufoff) + ldsw + 8192), 16, voffB, (soff) + 32 * 16384, 0, 0); } while (0)
#define LDA(dst, b, h) do { _Pragma("unroll") for (int m = 0; m < 4; ++m) { \
        dst[m][0] = *(const LAS bf16x8*)(lds + SA(b, h) + aoff0 + m * 2048); \
        dst[m][1] = *(const LAS bf16x8*)(lds + SA(b, h) + aoff1 + m * 2048); } } while (0)
#define TR(d, base, off) asm volatile("ds_read_b64_tr_b16 %0, %1 offset:%2" : "=v"(d) : "v"(base), "i"(off) : "memory")
#define LDB(dst, b, h) do { _Pragma("unroll") for (int k = 0; k < 2; ++k) { s16x4 l0, h0, l1, h1; \
        TR(l0, vb0, SB(b, h) - 65536 + k * 8192); TR(h0, vb0, SB(b, h) - 65536 + k * 8192 + 1024); \
        TR(l1, vb1, SB(b, h) - 65536 + k * 8192); TR(h1, vb1, SB(b, h) - 65536 + k * 8192 + 1024); \
        dst[0][k] = __builtin_shufflevector(l0, h0, 0, 1, 2, 3, 4, 5, 6, 7); \
        dst[1][k] = __builtin_shufflevector(l1, h1, 0, 1, 2, 3, 4, 5, 6, 7); } } while (0)
#define MMA(ai, bj, At, Bt) do { __builtin_amdgcn_s_setprio(1); _Pragma("unroll") for (int k = 0; k < 2; ++k) _Pragma("unroll") for (int m = 0; m < 4; ++m) _Pragma("unroll") for (int n = 0; n < 2; ++n) \
        acc[ai][bj][m][n] = __builtin_amdgcn_mfma_f32_16x16x32_bf16(Bt[n][k], At[m][k], acc[ai][bj][m][n], 0, 0, 0); __builtin_amdgcn_s_setprio(0); } while (0)
#define WAIT_V(n) asm volatile("s_waitcnt vmcnt(" #n ")" ::: "memory")
#define WAIT_L(n) do { asm volatile("s_waitcnt lgkmcnt(" #n ")" ::: "memory"); __builtin_amdgcn_sched_barrier(0); } while (0)
#define BAR __builtin_amdgcn_s_barrier()
#define SCHED __builtin_amdgcn_sched_barrier(0)
#define POLLAUX 17
#define POLL(q) __builtin_amdgcn_raw_ptr_buffer_load_lds(rsF, (LAS void*)(ldx + 1024 + (q) * 256), 4, voffF, sflag, 0, POLLAUX)

    f32x4 acc[2][2][4][2];
#pragma unroll
    for (int a = 0; a < 2; ++a)
#pragma unroll
        for (int b = 0; b < 2; ++b)
#pragma unroll
            for (int m = 0; m < 4; ++m)
#pragma unroll
                for (int n = 0; n < 2; ++n) acc[a][b][m][n] = (f32x4){0.f, 0.f, 0.f, 0.f};
    bf16x8 At[4][2], B0[2][2], B1[2][2];

    const unsigned arow0 = (unsigned)(mt * 256 + kh * 128) * 2048u, arow1 = (unsigned)(mt * 256 + (1 - kh) * 128) * 2048u;
    const __amdgpu_buffer_rsrc_t rsW = __builtin_amdgcn_make_buffer_rsrc((void*)p.w, 0, 128u << 20, 0x00020000);
    const __amdgpu_buffer_rsrc_t rsX = __builtin_amdgcn_make_buffer_rsrc((void*)p.x, 0, 16u << 20, 0x00020000);
    const __amdgpu_buffer_rsrc_t rsF = __builtin_amdgcn_make_buffer_rsrc((void*)(myws + WS_FLAG), 0, 32768u, 0x00020000);
    unsigned sflag = (unsigned)((mt * 8 + wid) * 2 + kh) * 256u;
    const char* const pL = p.x + (size_t)rank * (2u << 20);
    const char* const pS = myws + WS_STAGE;
#define RSA(t) __builtin_amdgcn_make_buffer_rsrc((void*)((t) < 8 ? pL : pS), 0, 32u << 20, 0x00020000)
    const unsigned m0push = (unsigned)(size_t)ldx, m0poll = m0push + 1024u;

    unsigned tabA, tabB;
    {
        const int u = lane - 8, j = u / 7, i0 = u - j * 7, i = kh ? 6 - i0 : i0;
        const int c = lane < 8 ? rank : (rank + 1 + i) & 7, kt = kh * 8 + (lane < 8 ? lane : j);
        tabA = (lane < 8 ? 0u : (unsigned)c * (2u << 20)) + (unsigned)kt * 128u;
        tabB = (unsigned)(c * 16 + kt) * (64u * 16384u) + (unsigned)nt * 512u;
    }
    const int wq = wid * 256 + bid;
    const int oslot = (wq * 5) >> 11;
    unsigned tabLo, tabHi, tabD, tabF, tabS;
    {
        const int sl = lane < 7 ? lane : 6;
        const int Q = sl * 2048 + wq, b = Q / 896, rem = Q - b * 896, rg = rem / 7, di = rem - rg * 7;
        const int d = (rank + 1 + di) & 7, ipos = 6 - di, kk = (b & 1) ^ (ipos > 3 ? 1 : 0), kt = kk * 8 + (b >> 1);
        unsigned long long base = 0;
#pragma unroll
        for (int i = 0; i < NR; ++i) base = d == i ? (unsigned long long)p.ws[i] : base;
        tabLo = (unsigned)base; tabHi = (unsigned)(base >> 32);
        tabS = (unsigned)(d * 1024 + rg * 8) * 2048u + (unsigned)kt * 128u;
        tabD = (unsigned)WS_STAGE + (unsigned)rank * (2u << 20) + (unsigned)(rg * 8) * 2048u + (unsigned)kt * 128u;
        tabF = (unsigned)WS_FLAG + (unsigned)((((rg >> 5) * 8 + (rg & 7)) * 2 + kk) * 256 + (8 + (b >> 1) * 7 + (kk ? 6 - ipos : ipos)) * 4 + ((rg >> 3) & 3));
    }
    char* const ctr = myws + WS_CTR + (size_t)(bid * 8 + wid) * 16;
    unsigned fl0 = 0u, fl1 = 0u, callv = 0u;

    {
        __builtin_amdgcn_raw_ptr_buffer_load_lds(rsX, (LAS void*)ldx, 16, voffP, __builtin_amdgcn_readlane(tabS, 0), 0, 0);
        asm volatile("global_load_dword %0, %1, %2 sc0 sc1" : "=v"(callv) : "v"(vzero), "s"(ctr) : "memory");
        const __amdgpu_buffer_rsrc_t rsL = RSA(0);
        const unsigned a0 = (unsigned)kh * 1024u, b0 = (unsigned)(rank * 16 + kh * 8) * (64u * 16384u) + (unsigned)nt * 512u, b1 = b0 + 64u * 16384u;
        STAGE_B(SB(0, 0), b0);
        STAGE_A(SA(0, 0), rsL, a0 + arow0); POLL(2);
        STAGE_B(SB(0, 1), b0 + 256);
        STAGE_A(SA(0, 1), rsL, a0 + arow1);
        STAGE_B(SB(1, 0), b1);
        STAGE_A(SA(1, 0), rsL, a0 + 128 + arow0); POLL(2);
        STAGE_B(SB(1, 1), b1 + 256);
        if (wr == 1) BAR;
        asm volatile("s_waitcnt vmcnt(14)" : "+v"(callv) :: "memory");
        BAR; BAR;
        LDB(B0, 0, 0); WAIT_V(12); BAR; WAIT_L(0); BAR;
    }
    const unsigned call = (unsigned)__builtin_amdgcn_readfirstlane(callv) + 1u;
    const unsigned spar = (call & 1u) << 24;
    const unsigned fpar = (call & 1u) << 14;
    const unsigned fval = (call & 0x7fu) | 0x80u, fpat = fval * 0x01010101u;
    sflag += fpar;
    tabA += lane < 8 ? 0u : spar; tabD += spar; tabF += fpar;
    unsigned long long ready = 0xffull;
    unsigned* const tmo = (unsigned*)(myws + WS_TMO);

    unsigned cd = (unsigned)oslot, pst = 0u;
    unsigned pn = 0u;
    asm volatile(".subsection 2\n\t.fill 256, 4, 0xbf800000\n\t.subsection 0");
#define PUSH_SLOT() do { \
        asm volatile( \
            "s_sub_u32 %[cd], %[cd], 1\n\t" \
            "s_cbranch_scc1 .Lpr%=\n" \
            ".Lpb%=:\n\t" \
            ".subsection 1\n" \
            ".Lpr%=:\n\t" \
            "v_readlane_b32 s96, %[tLo], %[pn]\n\t" \
            "v_readlane_b32 s97, %[tHi], %[pn]\n\t" \
            "s_cmp_eq_u32 %[pst], 1\n\t" \
            "s_cbranch_scc1 .Lpf%=\n\t" \
            "s_cmp_eq_u32 %[pst], 2\n\t" \
            "s_cbranch_scc1 .Lpl%=\n\t" \
            "ds_read_b128 v[248:251], %[vxp]\n\t" \
            "v_readlane_b32 s98, %[tD], %[pn]\n\t" \
            "s_mov_b32 %[pst], 1\n\t" \
            "s_mov_b32 %[cd], 1\n\t" \
            "s_nop 0\n\t" \
            "v_add_u32 v252, s98, %[voffP]\n\t" \
            "s_waitcnt lgkmcnt(0)\n\t" \
            "global_store_dwordx4 v252, v[248:251], s[96:97] sc0 sc1\n\t" \
            "s_nop 1\n\t" \
            "s_branch .Lpb%=\n" \
            ".Lpf%=:\n\t" \
            "v_readlane_b32 s98, %[tF], %[pn]\n\t" \
            "s_add_u32 %[pn], %[pn], 1\n\t" \
            "s_mov_b32 %[pst], 2\n\t" \
            "s_cmp_eq_u32 %[pn], 7\n\t" \
            "s_cselect_b32 %[cd], 0x7fffffff, 0\n\t" \
            "v_mov_b32 v252, s98\n\t" \
            "s_mov_b64 exec, 1\n\t" \
            "global_store_byte v252, %[fval], s[96:97] sc0 sc1\n\t" \
            "s_mov_b64 exec, -1\n\t" \
            "s_branch .Lpb%=\n" \
            ".Lpl%=:\n\t" \
            "v_readlane_b32 s98, %[tS], %[pn]\n\t" \
            "s_mov_b32 s99, m0\n\t" \
            "s_mov_b32 m0, %[m0p]\n\t" \
            "s_mov_b32 %[pst], 0\n\t" \
            "s_mov_b32 %[cd], 1\n\t" \
            "s_nop 1\n\t" \
            "buffer_load_dwordx4 %[voffP], %[rsX], s98 offen lds\n\t" \
            "s_mov_b32 m0, s99\n\t" \
            "s_branch .Lpb%=\n\t" \
            ".subsection 0" \
            : [cd] "+s"(cd), [pst] "+s"(pst), [pn] "+s"(pn) \
            : [m0p] "s"(m0push), [tLo] "v"(tabLo), [tHi] "v"(tabHi), [tD] "v"(tabD), [tF] "v"(tabF), [tS] "v"(tabS), \
              [voffP] "v"(voffP), [vxp] "v"(vxp), [fval] "v"(fval), [rsX] "s"(rsX) \
            : "memory", "scc", "s96", "s97", "s98", "s99", "v248", "v249", "v250", "v251", "v252"); } while (0)
#define POLL_RD(q, f) asm volatile("ds_read_b32 %0, %1 offset:%2" : "=v"(f) : "v"(vxf), "i"((q) * 256) : "memory")
#define NEED(tt, q, f) do { \
        ready |= __builtin_amdgcn_ballot_w64(f == fpat) & live; \
        asm volatile( \
            "s_bitcmp1_b64 %[rdy], %[t]\n\t" \
            "s_cbranch_scc0 .Lnr%=\n" \
            ".Lnb%=:\n\t" \
            ".subsection 1\n" \
            ".Lnr%=:\n\t" \
            "s_memrealtime s[96:97]\n\t" \
            "s_waitcnt lgkmcnt(0)\n\t" \
            "s_mov_b32 s97, m0\n\t" \
            "s_mov_b32 m0, %[m0q]\n\t" \
            "s_nop 0\n" \
            ".Lnl%=:\n\t" \
            "buffer_load_dword %[voffF], %[rsF], %[sflag] offen sc0 sc1 lds\n\t" \
            "global_load_dword v249, %[vz], %[tmo] sc0 sc1\n\t" \
            "s_waitcnt vmcnt(0)\n\t" \
            "ds_read_b32 v248, %[a] offset:%[o]\n\t" \
            "s_memrealtime s[98:99]\n\t" \
            "s_waitcnt lgkmcnt(0)\n\t" \
            "v_cmp_eq_u32 vcc, %[fpat], v248\n\t" \
            "s_sub_u32 s98, s98, s96\n\t" \
            "s_nop 0\n\t" \
            "s_or_b64 %[rdy], %[rdy], vcc\n\t" \
            "s_bitcmp1_b64 %[rdy], %[t]\n\t" \
            "s_cbranch_scc1 .Lnx%=\n\t" \
            "v_cmp_eq_u32 vcc, %[call], v249\n\t" \
            "s_cmp_gt_u32 s98, %[dl]\n\t" \
            "s_cbranch_scc1 .Lnd%=\n\t" \
            "s_cbranch_vccnz .Lnd%=\n\t" \
            "s_sleep 1\n\t" \
            "s_branch .Lnl%=\n" \
            ".Lnd%=:\n\t" \
            "v_mov_b32 v249, %[call]\n\t" \
            "s_mov_b64 %[rdy], -1\n\t" \
            "global_store_dword %[vz], v249, %[tmo] sc0 sc1\n\t" \
            "s_waitcnt vmcnt(0)\n" \
            ".Lnx%=:\n\t" \
            "s_mov_b32 m0, s97\n\t" \
            "s_branch .Lnb%=\n\t" \
            ".subsection 0" \
            : [rdy] "+s"(ready) \
            : [t] "s"(tt), [m0q] "s"(m0poll + (q) * 256u), [voffF] "v"(voffF), [rsF] "s"(rsF), [sflag] "s"(sflag), [vz] "v"(vzero), [tmo] "s"(tmo), \
              [a] "v"(vxf), [o] "i"((q) * 256), [fpat] "s"(fpat), [call] "s"(call), [dl] "s"((unsigned)DEADLINE) \
            : "memory", "scc", "vcc", "s96", "s97", "s98", "s99", "v248", "v249"); } while (0)

    unsigned a1 = (unsigned)kh * 1024u + 128u;
    int t = 0;
    do {
        ST(1 + (t >> 1));
        const unsigned a2 = __builtin_amdgcn_readlane(tabA, t + 2), b2 = __builtin_amdgcn_readlane(tabB, t + 2);
        const unsigned a3 = __builtin_amdgcn_readlane(tabA, t + 3), b3 = __builtin_amdgcn_readlane(tabB, t + 3);
        const __amdgpu_buffer_rsrc_t r1 = RSA(t + 1), r2 = RSA(t + 2);
        const unsigned long long live = t > 0 ? ~0ull : 0ull;
        LDA(At, 0, 0); STAGE_A(SA(1, 1), r1, a1 + arow1); PUSH_SLOT();
        WAIT_V(11); BAR; WAIT_L(0); MMA(0, 0, At, B0); BAR; SCHED;
        LDB(B1, 0, 1); POLL_RD(0, fl0); STAGE_B(SB(0, 0), b2);
        WAIT_V(11); BAR; WAIT_L(0); MMA(0, 1, At, B1); BAR; SCHED;
        LDA(At, 0, 1); NEED(t + 2, 0, fl0);
        STAGE_A(SA(0, 0), r2, a2 + arow0); POLL(0);
        WAIT_V(12); BAR; WAIT_L(0); MMA(1, 0, At, B0); BAR; SCHED;
        LDB(B0, 1, 0); STAGE_B(SB(0, 1), b2 + 256);
        WAIT_V(12); BAR; MMA(1, 1, At, B1); SCHED; WAIT_L(0); BAR; SCHED;
        LDA(At, 1, 0); STAGE_A(SA(0, 1), r2, a2 + arow1); PUSH_SLOT();
        WAIT_V(11); BAR; WAIT_L(0); MMA(0, 0, At, B0); BAR; SCHED;
        LDB(B1, 1, 1); POLL_RD(1, fl1); STAGE_B(SB(1, 0), b3);
        WAIT_V(11); BAR; WAIT_L(0); MMA(0, 1, At, B1); BAR; SCHED;
        LDA(At, 1, 1); NEED(t + 3, 1, fl1);
        STAGE_A(SA(1, 0), r2, a3 + arow0); POLL(1);
        WAIT_V(12); BAR; WAIT_L(0); MMA(1, 0, At, B0); BAR; SCHED;
        LDB(B0, 0, 0); STAGE_B(SB(1, 1), b3 + 256);
        WAIT_V(12); BAR; MMA(1, 1, At, B1); SCHED; WAIT_L(0); BAR; SCHED;
        a1 = a3; t += 2;
    } while (t < 62 && !(pn == 7u && ready == ~0ull && t >= 8));
    {
        const __amdgpu_buffer_rsrc_t rS = RSA(8);
        for (; t < 62; t += 2) {
            ST(1 + (t >> 1));
            const unsigned a2 = __builtin_amdgcn_readlane(tabA, t + 2), b2 = __builtin_amdgcn_readlane(tabB, t + 2);
            const unsigned a3 = __builtin_amdgcn_readlane(tabA, t + 3), b3 = __builtin_amdgcn_readlane(tabB, t + 3);
            LDA(At, 0, 0); STAGE_A(SA(1, 1), rS, a1 + arow1);
            WAIT_V(10); BAR; WAIT_L(0); MMA(0, 0, At, B0); BAR; SCHED;
            LDB(B1, 0, 1); STAGE_B(SB(0, 0), b2);
            WAIT_V(10); BAR; WAIT_L(0); MMA(0, 1, At, B1); BAR; SCHED;
            LDA(At, 0, 1); STAGE_A(SA(0, 0), rS, a2 + arow0);
            WAIT_V(10); BAR; WAIT_L(0); MMA(1, 0, At, B0); BAR; SCHED;
            LDB(B0, 1, 0); STAGE_B(SB(0, 1), b2 + 256);
            WAIT_V(10); BAR; MMA(1, 1, At, B1); SCHED; WAIT_L(0); BAR; SCHED;
            LDA(At, 1, 0); STAGE_A(SA(0, 1), rS, a2 + arow1);
            WAIT_V(10); BAR; WAIT_L(0); MMA(0, 0, At, B0); BAR; SCHED;
            LDB(B1, 1, 1); STAGE_B(SB(1, 0), b3);
            WAIT_V(10); BAR; WAIT_L(0); MMA(0, 1, At, B1); BAR; SCHED;
            LDA(At, 1, 1); STAGE_A(SA(1, 0), rS, a3 + arow0);
            WAIT_V(10); BAR; WAIT_L(0); MMA(1, 0, At, B0); BAR; SCHED;
            LDB(B0, 0, 0); STAGE_B(SB(1, 1), b3 + 256);
            WAIT_V(10); BAR; MMA(1, 1, At, B1); SCHED; WAIT_L(0); BAR; SCHED;
            a1 = a3;
        }
    }
    {
        ST(32);
        const __amdgpu_buffer_rsrc_t r1 = RSA(8);
        LDA(At, 0, 0); STAGE_A(SA(1, 1), r1, a1 + arow1);
        WAIT_V(10); BAR; WAIT_L(0); MMA(0, 0, At, B0); BAR; SCHED;
        LDB(B1, 0, 1);
        WAIT_V(8); BAR; WAIT_L(0); MMA(0, 1, At, B1); BAR; SCHED;
        LDA(At, 0, 1);
        WAIT_V(6); BAR; WAIT_L(0); MMA(1, 0, At, B0); BAR; SCHED;
        LDB(B0, 1, 0);
        WAIT_V(4); BAR; MMA(1, 1, At, B1); SCHED; WAIT_L(0); BAR; SCHED;
        LDA(At, 1, 0);
        WAIT_V(2); BAR; WAIT_L(0); MMA(0, 0, At, B0); BAR; SCHED;
        LDB(B1, 1, 1);
        WAIT_V(0); BAR; WAIT_L(0); MMA(0, 1, At, B1); BAR; SCHED;
        LDA(At, 1, 1);
        BAR; WAIT_L(0); MMA(1, 0, At, B0); BAR; SCHED;
        BAR; MMA(1, 1, At, B1); SCHED;
        if (wr == 0) BAR;
    }
    ST(33);

    {
        int tz = threadIdx.x; asm volatile("" : "+v"(tz));
        const int lane2 = tz & 63, fr2 = lane2 & 15, fq2 = lane2 >> 4;
        const int pbid = bid ^ 8;
        const __amdgpu_buffer_rsrc_t xr = __builtin_amdgcn_make_buffer_rsrc(myws + WS_XBUF, 0, 256u * 131072u, 0x00020000);
        const unsigned myoff = (unsigned)bid * 131072u + (unsigned)tz * 16u, poff = (unsigned)pbid * 131072u + (unsigned)tz * 16u;
#pragma unroll
        for (int bj = 0; bj < 2; ++bj)
#pragma unroll
            for (int m = 0; m < 4; ++m)
#pragma unroll
                for (int n = 0; n < 2; ++n)
                    __builtin_amdgcn_raw_buffer_store_b128(__builtin_bit_cast(u32x4, acc[1][bj][m][n]), xr, myoff + ((bj * 4 + m) * 2 + n) * 8192, 0, 16);
        WAIT_V(0);
        unsigned* xf = (unsigned*)(myws + WS_XFLAG);
        if (lane2 == 0) {
            __hip_atomic_store(xf + (bid * 8 + wid) * 4, call, __ATOMIC_RELAXED, __HIP_MEMORY_SCOPE_AGENT);
            __hip_atomic_store((unsigned*)ctr, call, __ATOMIC_RELAXED, __HIP_MEMORY_SCOPE_AGENT);
        }
        {
            const unsigned long long t0 = __builtin_amdgcn_s_memrealtime();
            while ((unsigned)__builtin_amdgcn_readfirstlane(__hip_atomic_load(xf + (pbid * 8 + wid) * 4, __ATOMIC_RELAXED, __HIP_MEMORY_SCOPE_AGENT)) < call) {
                if ((unsigned)__builtin_amdgcn_readfirstlane(__hip_atomic_load(tmo, __ATOMIC_RELAXED, __HIP_MEMORY_SCOPE_AGENT)) == call ||
                    __builtin_amdgcn_s_memrealtime() - t0 > DEADLINE) {
                    __hip_atomic_store(tmo, call, __ATOMIC_RELAXED, __HIP_MEMORY_SCOPE_AGENT); break; }
                __builtin_amdgcn_s_sleep(1);
            }
            asm volatile("" ::: "memory");
        }
        ST(34);
        const bool bad = (unsigned)__builtin_amdgcn_readfirstlane(__hip_atomic_load(tmo, __ATOMIC_RELAXED, __HIP_MEMORY_SCOPE_AGENT)) == call;
        f32x4 pv[2][4][2];
#pragma unroll
        for (int bj = 0; bj < 2; ++bj)
#pragma unroll
            for (int m = 0; m < 4; ++m)
#pragma unroll
                for (int n = 0; n < 2; ++n)
                    pv[bj][m][n] = __builtin_bit_cast(f32x4, __builtin_amdgcn_raw_buffer_load_b128(xr, poff + ((bj * 4 + m) * 2 + n) * 8192, 0, 16));
        char* orow = p.out + (size_t)(mt * 256 + kh * 128 + wr * 64 + fr2) * 16384 + (size_t)(nt * 256 + wc * 32 + (fq2 & 1) * 16 + (fq2 >> 1) * 8) * 2;
#pragma unroll
        for (int m = 0; m < 4; ++m)
#pragma unroll
            for (int bj = 0; bj < 2; ++bj) {
                const f32x4 v0 = acc[0][bj][m][0] + pv[bj][m][0], v1 = acc[0][bj][m][1] + pv[bj][m][1];
                const unsigned x0 = cvt_pk_bf16(gelu_tanh(v0[0]), gelu_tanh(v0[1])), x1 = cvt_pk_bf16(gelu_tanh(v0[2]), gelu_tanh(v0[3]));
                const unsigned y0 = cvt_pk_bf16(gelu_tanh(v1[0]), gelu_tanh(v1[1])), y1 = cvt_pk_bf16(gelu_tanh(v1[2]), gelu_tanh(v1[3]));
                const auto r0 = __builtin_amdgcn_permlane16_swap(x0, y0, false, false);
                const auto r1 = __builtin_amdgcn_permlane16_swap(x1, y1, false, false);
                u32x4 o; o.x = r0[0]; o.y = r1[0]; o.z = r0[1]; o.w = r1[1];
                if (bad) o = (u32x4){0x7fc07fc0u, 0x7fc07fc0u, 0x7fc07fc0u, 0x7fc07fc0u};
                *(u32x4*)(orow + (size_t)m * 16 * 16384 + bj * 256) = o;
            }
    }
}
}

extern "C" void kernel_launch(int rank, int nranks, ncclComm_t comm,
                              hipStream_t stream,
                              void* const* d_in, const int* in_sizes, int n_in,
                              void* d_out, int out_size,
                              const int* in_dtypes, int out_dtype,
                              void* const* d_out_all,
                              void* const* d_ws_all, size_t ws_bytes) {
    (void)comm; (void)in_sizes; (void)n_in; (void)out_size; (void)in_dtypes; (void)out_dtype; (void)d_out_all;
    if (nranks != a2a::NR || (unsigned)rank >= (unsigned)a2a::NR || ws_bytes < a2a::WS_END) {
        fprintf(stderr, "a2a_gemm: unexpected configuration (nranks %d, ws %zu)\n", nranks, ws_bytes);
        return;
    }
    HIP_CHECK(hipFuncSetAttribute((const void*)a2a::a2a_gemm_kernel, hipFuncAttributeMaxDynamicSharedMemorySize, a2a::LDS_BYTES));
    a2a::Params p;
    p.x = (const char*)d_in[0];
    p.w = (const char*)d_in[1];
    p.out = (char*)d_out;
    for (int i = 0; i < a2a::NR; ++i) p.ws[i] = (char*)d_ws_all[i];
    p.rank = rank;
    hipLaunchKernelGGL(a2a::a2a_gemm_kernel, dim3(256), dim3(512), a2a::LDS_BYTES, stream, p);
    HIP_CHECK(hipGetLastError());
}
```

```cpp
#include <hip/hip_runtime.h>
#include <rccl/rccl.h>
#include <cstdio>

#define HIP_CHECK(expr) do { hipError_t _e = (expr); if (_e != hipSuccess) { \
    fprintf(stderr, "HIP error %s at %s:%d\n", hipGetErrorString(_e), __FILE__, __LINE__); \
    return; } } while (0)

namespace a2a {
#define LAS __attribute__((address_space(3)))
typedef short bf16x8 __attribute__((ext_vector_type(8)));
typedef short s16x4 __attribute__((ext_vector_type(4)));
typedef float f32x4 __attribute__((ext_vector_type(4)));
typedef unsigned u32x4 __attribute__((ext_vector_type(4)));

constexpr int NR = 8;
constexpr int HTB = 16384;
constexpr int LDS_X = 8 * HTB;
constexpr int LDS_BYTES = 8 * HTB + 8 * 2048;
constexpr size_t WS_CTR = 0;
constexpr size_t WS_TMO = 32768;
constexpr size_t WS_FLAG = 65536;
constexpr size_t WS_XFLAG = 131072;
constexpr size_t WS_STAGE = 1ull << 20;
constexpr size_t WS_XBUF = 33ull << 20;
constexpr size_t WS_END = 65ull << 20;
constexpr unsigned long long DEADLINE = 50000000ull;

struct Params {
    const char* x;
    const char* w;
    char* out;
    char* ws[NR];
    int rank;
};

typedef float f32x2 __attribute__((ext_vector_type(2)));
typedef __bf16 bf16x2 __attribute__((ext_vector_type(2)));
__device__ __forceinline__ unsigned cvt_pk_bf16(float lo, float hi) {
    return __builtin_bit_cast(unsigned, __builtin_convertvector((f32x2){lo, hi}, bf16x2));
}
__device__ __forceinline__ float gelu_tanh(float y) {
    const float c2 = 2.0f * 0.7978845608028654f * 1.4426950408889634f;
    const float u = y * (1.0f + 0.044715f * y * y);
    const float e = __builtin_amdgcn_exp2f(-c2 * u);
    return y * __builtin_amdgcn_rcpf(1.0f + e);
}

#define ST(i)

__global__ __launch_bounds__(512, 2) void a2a_gemm_kernel(Params p) {
    extern __shared__ __attribute__((aligned(16))) unsigned char shm[];
    LAS unsigned char* lds = (LAS unsigned char*)shm;
    const int tid = threadIdx.x, wid = __builtin_amdgcn_readfirstlane(tid >> 6), lane = tid & 63, wr = wid >> 2, wc = wid & 3, fr = lane & 15, fq = lane >> 4;
    const int bid = blockIdx.x, xcd = bid & 7, idx = bid >> 3;
    const int kh = idx & 1, mt = (idx >> 1) & 3, nt = xcd * 4 + (idx >> 3);
    const int rank = p.rank;
    char* const myws = p.ws[rank];
    ST(0);

    const unsigned ra = tid >> 3, ca = (tid & 7) ^ ((tid >> 4) & 7);
    const unsigned voffA = ra * 2048u + ca * 16u;
    const unsigned kb = tid >> 4, cb = tid & 15, fb = (kb & 3) | ((kb >> 1) & 4);
    const unsigned voffB = kb * 16384u + (((cb >> 1) ^ fb) * 32u) + (cb & 1) * 16u;
    const unsigned ldsw = (unsigned)wid * 1024u;
    const int aoff0 = (wr * 64 + fr) * 128 + ((fq ^ (fr >> 1)) * 16);
    const int aoff1 = aoff0 ^ 64;
    const int q4 = (lane >> 2) & 3, p4 = lane & 3, fl = q4 | ((fq & 1) << 2);
    const int boff0 = (8 * fq + q4) * 256 + (((wc * 2) ^ fl) * 32) + 8 * p4;
    const unsigned vb0 = (unsigned)(size_t)lds + 65536u + (unsigned)boff0, vb1 = (unsigned)(size_t)lds + 65536u + (unsigned)(boff0 ^ 32);
    const unsigned voffP = (unsigned)(lane >> 3) * 2048u + (unsigned)(lane & 7) * 16u;
    const unsigned voffF = (unsigned)lane * 4u;
    const unsigned vzero = 0u;
    LAS unsigned char* const ldx = lds + LDS_X + wid * 2048;
    const unsigned vxf = (unsigned)(size_t)ldx + 1024u + (unsigned)lane * 4u;

#define SA(b, h) (((b) * 2 + (h)) * HTB)
#define SB(b, h) ((4 + (b) * 2 + (h)) * HTB)
#define AUX_STAGED 17
#define STAGE_A(bufoff, rs, soff) do { \
        __builtin_amdgcn_raw_ptr_buffer_load_lds(rs, (LAS void*)(lds + (bufoff) + ldsw), 16, voffA, (soff), 0, AUX_STAGED); \
        __builtin_amdgcn_raw_ptr_buffer_load_lds(rs, (LAS void*)(lds + (bufoff) + ldsw + 8192), 16, voffA, (soff) + 64 * 2048, 0, AUX_STAGED); } while (0)
#define AUX_W 0
#define STAGE_B(bufoff, soff) do { \
        __builtin_amdgcn_raw_ptr_buffer_load_lds(rsW, (LAS void*)(lds + (bufoff) + ldsw), 16, voffB, (soff), 0, AUX_W); \
        __builtin_amdgcn_raw_ptr_buffer_load_lds(rsW, (LAS void*)(lds + (bufoff) + ldsw + 8192), 16, voffB, (soff) + 32 * 16384, 0, AUX_W); } while (0)
#define LDA(dst, b, h) do { _Pragma("unroll") for (int m = 0; m < 4; ++m) { \
        dst[m][0] = *(const LAS bf16x8*)(lds + SA(b, h) + aoff0 + m * 2048); \
        dst[m][1] = *(const LAS bf16x8*)(lds + SA(b, h) + aoff1 + m * 2048); } } while (0)
#define TR(d, base, off) asm volatile("ds_read_b64_tr_b16 %0, %1 offset:%2" : "=v"(d) : "v"(base), "i"(off) : "memory")
#define LDB(dst, b, h) do { _Pragma("unroll") for (int k = 0; k < 2; ++k) { s16x4 l0, h0, l1, h1; \
        TR(l0, vb0, SB(b, h) - 65536 + k * 8192); TR(h0, vb0, SB(b, h) - 65536 + k * 8192 + 1024); \
        TR(l1, vb1, SB(b, h) - 65536 + k * 8192); TR(h1, vb1, SB(b, h) - 65536 + k * 8192 + 1024); \
        dst[0][k] = __builtin_shufflevector(l0, h0, 0, 1, 2, 3, 4, 5, 6, 7); \
        dst[1][k] = __builtin_shufflevector(l1, h1, 0, 1, 2, 3, 4, 5, 6, 7); } } while (0)
#define MMA(ai, bj, At, Bt) do { __builtin_amdgcn_s_setprio(1); _Pragma("unroll") for (int k = 0; k < 2; ++k) _Pragma("unroll") for (int m = 0; m < 4; ++m) _Pragma("unroll") for (int n = 0; n < 2; ++n) \
        acc[ai][bj][m][n] = __builtin_amdgcn_mfma_f32_16x16x32_bf16(Bt[n][k], At[m][k], acc[ai][bj][m][n], 0, 0, 0); __builtin_amdgcn_s_setprio(0); } while (0)
#define WAIT_V(n) asm volatile("s_waitcnt vmcnt(" #n ")" ::: "memory")
#define WAIT_L(n) do { asm volatile("s_waitcnt lgkmcnt(" #n ")" ::: "memory"); __builtin_amdgcn_sched_barrier(0); } while (0)
#define BAR __builtin_amdgcn_s_barrier()
#define SCHED __builtin_amdgcn_sched_barrier(0)
#define POLLAUX 17
#define POLL(q) __builtin_amdgcn_raw_ptr_buffer_load_lds(rsF, (LAS void*)(ldx + 1024 + (q) * 256), 4, voffF, sflag, 0, POLLAUX)

    f32x4 acc[2][2][4][2];
#pragma unroll
    for (int a = 0; a < 2; ++a)
#pragma unroll
        for (int b = 0; b < 2; ++b)
#pragma unroll
            for (int m = 0; m < 4; ++m)
#pragma unroll
                for (int n = 0; n < 2; ++n) acc[a][b][m][n] = (f32x4){0.f, 0.f, 0.f, 0.f};
    bf16x8 At[4][2], B0[2][2], B1[2][2];

    const unsigned arow0 = (unsigned)(mt * 256 + (1 - kh) * 128) * 2048u, arow1 = (unsigned)(mt * 256 + kh * 128) * 2048u;
    const __amdgpu_buffer_rsrc_t rsW = __builtin_amdgcn_make_buffer_rsrc((void*)p.w, 0, 128u << 20, 0x00020000);
    const __amdgpu_buffer_rsrc_t rsX = __builtin_amdgcn_make_buffer_rsrc((void*)p.x, 0, 16u << 20, 0x00020000);
    const __amdgpu_buffer_rsrc_t rsF = __builtin_amdgcn_make_buffer_rsrc((void*)(myws + WS_FLAG), 0, 32768u, 0x00020000);
    unsigned sflag = (unsigned)((mt * 8 + wid) * 2 + kh) * 256u;
    const char* const pL = p.x + (size_t)rank * (2u << 20);
    const char* const pS = myws + WS_STAGE;
#define RSA(t) __builtin_amdgcn_make_buffer_rsrc((void*)((t) < 8 ? pL : pS), 0, 32u << 20, 0x00020000)
    const unsigned m0poll = (unsigned)(size_t)ldx + 1024u;

    unsigned tabA, tabB;
    {
        const int u = lane - 8, j = u / 7, i0 = u - j * 7, i = kh ? 6 - i0 : i0;
        const int c = lane < 8 ? rank : (rank + 1 + i) & 7, kt = kh * 8 + (lane < 8 ? lane : j);
        tabA = (lane < 8 ? 0u : (unsigned)c * (2u << 20)) + (unsigned)kt * 128u;
        tabB = (unsigned)(c * 16 + kt) * (64u * 16384u) + (unsigned)nt * 512u;
    }
    const int cls = bid % 3;
    const int wq = (cls == 0 ? 0 : cls == 1 ? 688 : 1368) + (bid / 3) * 8 + wid;
    unsigned tabLo, tabHi, tabD, tabF, tabS;
    {
        const int sl = lane < 7 ? lane : 6;
        const int Q = sl * 2048 + wq, b = Q / 896, rem = Q - b * 896, rg = rem / 7, di = rem - rg * 7;
        const int d = (rank + 1 + di) & 7, ipos = 6 - di, kk = (b & 1) ^ (ipos > 3 ? 1 : 0), kt = kk * 8 + (b >> 1);
        unsigned long long base = 0;
#pragma unroll
        for (int i = 0; i < NR; ++i) base = d == i ? (unsigned long long)p.ws[i] : base;
        tabLo = (unsigned)base; tabHi = (unsigned)(base >> 32);
        tabS = (unsigned)(d * 1024 + rg * 8) * 2048u + (unsigned)kt * 128u;
        tabD = (unsigned)WS_STAGE + (unsigned)rank * (2u << 20) + (unsigned)(rg * 8) * 2048u + (unsigned)kt * 128u;
        tabF = (unsigned)WS_FLAG + (unsigned)((((rg >> 5) * 8 + (rg & 7)) * 2 + kk) * 256 + (8 + (b >> 1) * 7 + (kk ? 6 - ipos : ipos)) * 4 + ((rg >> 3) & 3));
    }
    char* const ctr = myws + WS_CTR + (size_t)(bid * 8 + wid) * 16;
    unsigned fl0 = 0u, callv = 0u;

    {
        __builtin_amdgcn_raw_ptr_buffer_load_lds(rsX, (LAS void*)ldx, 16, voffP, __builtin_amdgcn_readlane(tabS, 0), 0, 0);
        asm volatile("global_load_dword %0, %1, %2 sc0 sc1" : "=v"(callv) : "v"(vzero), "s"(ctr) : "memory");
        const __amdgpu_buffer_rsrc_t rsL = RSA(0);
        const unsigned a0 = (unsigned)kh * 1024u, b0 = (unsigned)(rank * 16 + kh * 8) * (64u * 16384u) + (unsigned)nt * 512u, b1 = b0 + 64u * 16384u;
        STAGE_B(SB(0, 0), b0);
        STAGE_A(SA(0, 0), rsL, a0 + arow0);
        STAGE_B(SB(0, 1), b0 + 256);
        STAGE_A(SA(0, 1), rsL, a0 + arow1);
        STAGE_B(SB(1, 0), b1);
        STAGE_A(SA(1, 0), rsL, a0 + 128 + arow0);
        STAGE_B(SB(1, 1), b1 + 256);
        asm volatile("ds_write_b32 %0, %1" :: "v"(vxf), "v"(vzero) : "memory");
        if (wr == 1) BAR;
        asm volatile("s_waitcnt vmcnt(12)" : "+v"(callv) :: "memory");
        BAR; BAR;
        LDB(B0, 0, 0); WAIT_V(10); BAR; WAIT_L(0); BAR;
    }
    const unsigned call = (unsigned)__builtin_amdgcn_readfirstlane(callv) + 1u;
    const unsigned spar = (call & 1u) << 24;
    const unsigned fpar = (call & 1u) << 14;
    const unsigned fval = (call & 0x7fu) | 0x80u, fpat = fval * 0x01010101u;
    sflag += fpar;
    tabA += lane < 8 ? 0u : spar; tabD += spar; tabF += fpar;
    unsigned long long ready = 0xffull;
    unsigned* const tmo = (unsigned*)(myws + WS_TMO);

    asm volatile(".subsection 2\n\t.fill 256, 4, 0xbf800000\n\t.subsection 0");
#define POLL_RD(f) asm volatile("ds_read_b32 %0, %1" : "=v"(f) : "v"(vxf) : "memory")
#define NEED(tt, f) do { \
        ready |= __builtin_amdgcn_ballot_w64(f == fpat); \
        asm volatile( \
            "s_lshr_b64 s[96:97], %[rdy], %[t]\n\t" \
            "s_andn2_b32 s96, 3, s96\n\t" \
            "s_cbranch_scc1 .Lnr%=\n" \
            ".Lnb%=:\n\t" \
            ".subsection 1\n" \
            ".Lnr%=:\n\t" \
            "s_memrealtime s[96:97]\n\t" \
            "s_waitcnt lgkmcnt(0)\n\t" \
            "s_mov_b32 s97, m0\n\t" \
            "s_mov_b32 m0, %[m0q]\n\t" \
            "s_nop 0\n" \
            ".Lnl%=:\n\t" \
            "buffer_load_dword %[voffF], %[rsF], %[sflag] offen sc0 sc1 lds\n\t" \
            "global_load_dword v249, %[vz], %[tmo] sc0 sc1\n\t" \
            "s_waitcnt vmcnt(0)\n\t" \
            "ds_read_b32 v248, %[a]\n\t" \
            "s_memrealtime s[98:99]\n\t" \
            "s_waitcnt lgkmcnt(0)\n\t" \
            "v_cmp_eq_u32 vcc, %[fpat], v248\n\t" \
            "s_sub_u32 s98, s98, s96\n\t" \
            "s_nop 0\n\t" \
            "s_or_b64 %[rdy], %[rdy], vcc\n\t" \
            "s_lshr_b64 vcc, %[rdy], %[t]\n\t" \
            "s_andn2_b32 s99, 3, vcc_lo\n\t" \
            "s_cbranch_scc0 .Lnx%=\n\t" \
            "v_cmp_eq_u32 vcc, %[call], v249\n\t" \
            "s_cmp_gt_u32 s98, %[dl]\n\t" \
            "s_cbranch_scc1 .Lnd%=\n\t" \
            "s_cbranch_vccnz .Lnd%=\n\t" \
            "s_sleep 1\n\t" \
            "s_branch .Lnl%=\n" \
            ".Lnd%=:\n\t" \
            "v_mov_b32 v249, %[call]\n\t" \
            "s_mov_b64 %[rdy], -1\n\t" \
            "global_store_dword %[vz], v249, %[tmo] sc0 sc1\n\t" \
            "s_waitcnt vmcnt(0)\n" \
            ".Lnx%=:\n\t" \
            "s_mov_b32 m0, s97\n\t" \
            "s_branch .Lnb%=\n\t" \
            ".subsection 0" \
            : [rdy] "+s"(ready) \
            : [t] "s"(tt), [m0q] "s"(m0poll), [voffF] "v"(voffF), [rsF] "s"(rsF), [sflag] "s"(sflag), [vz] "v"(vzero), [tmo] "s"(tmo), \
              [a] "v"(vxf), [fpat] "s"(fpat), [call] "s"(call), [dl] "s"((unsigned)DEADLINE) \
            : "memory", "scc", "vcc", "s96", "s97", "s98", "s99", "v248", "v249"); } while (0)

#define TRIP(W1, W2, W3, W4, W5, W6, W7, W8, X1, X2, X6, X8) do { \
        ST(1 + (t >> 1)); \
        const unsigned a2 = __builtin_amdgcn_readlane(tabA, t + 2), b2 = __builtin_amdgcn_readlane(tabB, t + 2); \
        const unsigned a3 = __builtin_amdgcn_readlane(tabA, t + 3), b3 = __builtin_amdgcn_readlane(tabB, t + 3); \
        const __amdgpu_buffer_rsrc_t r1 = RSA(t + 1), r2 = RSA(t + 2); \
        X1; LDA(At, 0, 0); STAGE_A(SA(1, 1), r1, a1 + arow1); \
        WAIT_V(W1); BAR; WAIT_L(0); MMA(0, 0, At, B0); BAR; SCHED; \
        LDB(B1, 0, 1); POLL_RD(fl0); X2; STAGE_B(SB(0, 0), b2); \
        WAIT_V(W2); BAR; WAIT_L(0); MMA(0, 1, At, B1); BAR; SCHED; \
        LDA(At, 0, 1); NEED(t + 2, fl0); STAGE_A(SA(0, 0), r2, a2 + arow0); \
        WAIT_V(W3); BAR; WAIT_L(0); MMA(1, 0, At, B0); BAR; SCHED; \
        LDB(B0, 1, 0); POLL(0); STAGE_B(SB(0, 1), b2 + 256); \
        WAIT_V(W4); BAR; MMA(1, 1, At, B1); SCHED; WAIT_L(0); BAR; SCHED; \
        LDA(At, 1, 0); STAGE_A(SA(0, 1), r2, a2 + arow1); \
        WAIT_V(W5); BAR; WAIT_L(0); MMA(0, 0, At, B0); BAR; SCHED; \
        LDB(B1, 1, 1); X6; STAGE_B(SB(1, 0), b3); \
        WAIT_V(W6); BAR; WAIT_L(0); MMA(0, 1, At, B1); BAR; SCHED; \
        LDA(At, 1, 1); STAGE_A(SA(1, 0), r2, a3 + arow0); \
        WAIT_V(W7); BAR; WAIT_L(0); MMA(1, 0, At, B0); BAR; SCHED; \
        LDB(B0, 0, 0); X8; STAGE_B(SB(1, 1), b3 + 256); \
        WAIT_V(W8); BAR; MMA(1, 1, At, B1); SCHED; WAIT_L(0); BAR; SCHED; \
        a1 = a3; t += 2; } while (0)

    unsigned a1 = (unsigned)kh * 1024u + 128u;
    int t = 0;
    {
        const unsigned voffL = lane ? 0x80000000u : 0u;
        int pn = 0; const int lim = 2 * cls;
        while (t < lim) TRIP(10, 10, 10, 11, 11, 11, 11, 11, , , , );
        for (; pn < 7; ++pn) {
            {
                const unsigned long long pb = (unsigned long long)(unsigned)__builtin_amdgcn_readlane(tabLo, pn) |
                                              ((unsigned long long)(unsigned)__builtin_amdgcn_readlane(tabHi, pn) << 32);
                const __amdgpu_buffer_rsrc_t rp = __builtin_amdgcn_make_buffer_rsrc((void*)pb, 0, (unsigned)WS_XBUF, 0x00020000);
                const unsigned sD = __builtin_amdgcn_readlane(tabD, pn), sF = __builtin_amdgcn_readlane(tabF, pn);
                const unsigned sS = __builtin_amdgcn_readlane(tabS, pn + 1);
                u32x4 pd;
                TRIP(10, 11, 11, 12, 12, 13, 12, 13,
                     pd = *(const LAS u32x4*)(ldx + lane * 16),
                     __builtin_amdgcn_raw_buffer_store_b128(pd, rp, voffP, sD, 17),
                     __builtin_amdgcn_raw_ptr_buffer_load_lds(rsX, (LAS void*)ldx, 16, voffP, sS, 0, 0),
                     __builtin_amdgcn_raw_buffer_store_b8((unsigned char)fval, rp, voffL, sF, 17));
            }
            TRIP(12, 12, 11, 12, 11, 11, 11, 11, , , , );
            TRIP(10, 10, 10, 11, 11, 11, 11, 11, , , , );
        }
        while (t < 62 && ready != ~0ull) TRIP(10, 10, 10, 11, 11, 11, 11, 11, , , , );
    }
    {
        const __amdgpu_buffer_rsrc_t rS = RSA(8);
        for (; t < 62; t += 2) {
            ST(1 + (t >> 1));
            const unsigned a2 = __builtin_amdgcn_readlane(tabA, t + 2), b2 = __builtin_amdgcn_readlane(tabB, t + 2);
            const unsigned a3 = __builtin_amdgcn_readlane(tabA, t + 3), b3 = __builtin_amdgcn_readlane(tabB, t + 3);
            LDA(At, 0, 0); STAGE_A(SA(1, 1), rS, a1 + arow1);
            WAIT_V(10); BAR; WAIT_L(0); MMA(0, 0, At, B0); BAR; SCHED;
            LDB(B1, 0, 1); STAGE_B(SB(0, 0), b2);
            WAIT_V(10); BAR; WAIT_L(0); MMA(0, 1, At, B1); BAR; SCHED;
            LDA(At, 0, 1); STAGE_A(SA(0, 0), rS, a2 + arow0);
            WAIT_V(10); BAR; WAIT_L(0); MMA(1, 0, At, B0); BAR; SCHED;
            LDB(B0, 1, 0); STAGE_B(SB(0, 1), b2 + 256);
            WAIT_V(10); BAR; MMA(1, 1, At, B1); SCHED; WAIT_L(0); BAR; SCHED;
            LDA(At, 1, 0); STAGE_A(SA(0, 1), rS, a2 + arow1);
            WAIT_V(10); BAR; WAIT_L(0); MMA(0, 0, At, B0); BAR; SCHED;
            LDB(B1, 1, 1); STAGE_B(SB(1, 0), b3);
            WAIT_V(10); BAR; WAIT_L(0); MMA(0, 1, At, B1); BAR; SCHED;
            LDA(At, 1, 1); STAGE_A(SA(1, 0), rS, a3 + arow0);
            WAIT_V(10); BAR; WAIT_L(0); MMA(1, 0, At, B0); BAR; SCHED;
            LDB(B0, 0, 0); STAGE_B(SB(1, 1), b3 + 256);
            WAIT_V(10); BAR; MMA(1, 1, At, B1); SCHED; WAIT_L(0); BAR; SCHED;
            a1 = a3;
        }
    }
    const int pbid = bid ^ 8;
    const __amdgpu_buffer_rsrc_t xr = __builtin_amdgcn_make_buffer_rsrc(myws + WS_XBUF, 0, 256u * 65536u, 0x00020000);
    const unsigned voffX = (unsigned)tid * 16u;
#define SEND(bj) do { _Pragma("unroll") for (int m = 0; m < 4; ++m) { u32x4 o_; \
        o_.x = cvt_pk_bf16(acc[0][bj][m][0][0], acc[0][bj][m][0][1]); o_.y = cvt_pk_bf16(acc[0][bj][m][0][2], acc[0][bj][m][0][3]); \
        o_.z = cvt_pk_bf16(acc[0][bj][m][1][0], acc[0][bj][m][1][1]); o_.w = cvt_pk_bf16(acc[0][bj][m][1][2], acc[0][bj][m][1][3]); \
        __builtin_amdgcn_raw_buffer_store_b128(o_, xr, voffX, (unsigned)bid * 65536u + ((bj) * 4 + m) * 8192, 16); } } while (0)
    {
        ST(32);
        const __amdgpu_buffer_rsrc_t r1 = RSA(8);
        LDA(At, 0, 0); STAGE_A(SA(1, 1), r1, a1 + arow1);
        WAIT_V(10); BAR; WAIT_L(0); MMA(0, 0, At, B0); BAR; SCHED;
        LDB(B1, 0, 1);
        WAIT_V(4); BAR; WAIT_L(0); MMA(0, 1, At, B1); BAR; SCHED;
        LDB(B0, 1, 0); LDA(At, 1, 0);
        WAIT_V(2); BAR; WAIT_L(0); MMA(0, 0, At, B0); BAR; SCHED;
        LDB(B1, 1, 1); LDB(B0, 0, 0); SEND(0);
        BAR; WAIT_L(0); MMA(0, 1, At, B1); BAR; SCHED;
        LDA(At, 0, 1); SEND(1);
        BAR; WAIT_L(0); MMA(1, 0, At, B0); BAR; SCHED;
        LDB(B1, 0, 1); LDB(B0, 1, 0);
        WAIT_V(8); BAR; WAIT_L(0); MMA(1, 1, At, B1); BAR; SCHED;
        LDA(At, 1, 1);
        BAR; WAIT_L(0); MMA(1, 0, At, B0); BAR; SCHED;
        LDB(B1, 1, 1);
        BAR; WAIT_L(0); MMA(1, 1, At, B1); SCHED;
        if (wr == 0) BAR;
    }
    ST(33);

    {
        int tz = threadIdx.x; asm volatile("" : "+v"(tz));
        const int lane2 = tz & 63, fr2 = lane2 & 15, fq2 = lane2 >> 4;
        WAIT_V(0);
        unsigned* xf = (unsigned*)(myws + WS_XFLAG);
        if (lane2 == 0) {
            __hip_atomic_store(xf + (bid * 8 + wid) * 4, call, __ATOMIC_RELAXED, __HIP_MEMORY_SCOPE_AGENT);
            __hip_atomic_store((unsigned*)ctr, call, __ATOMIC_RELAXED, __HIP_MEMORY_SCOPE_AGENT);
        }
        {
            const unsigned long long t0 = __builtin_amdgcn_s_memrealtime();
            while ((unsigned)__builtin_amdgcn_readfirstlane(__hip_atomic_load(xf + (pbid * 8 + wid) * 4, __ATOMIC_RELAXED, __HIP_MEMORY_SCOPE_AGENT)) < call) {
                if ((unsigned)__builtin_amdgcn_readfirstlane(__hip_atomic_load(tmo, __ATOMIC_RELAXED, __HIP_MEMORY_SCOPE_AGENT)) == call ||
                    __builtin_amdgcn_s_memrealtime() - t0 > DEADLINE) {
                    __hip_atomic_store(tmo, call, __ATOMIC_RELAXED, __HIP_MEMORY_SCOPE_AGENT); break; }
                __builtin_amdgcn_s_sleep(1);
            }
            asm volatile("" ::: "memory");
        }
        ST(34);
        const bool bad = (unsigned)__builtin_amdgcn_readfirstlane(__hip_atomic_load(tmo, __ATOMIC_RELAXED, __HIP_MEMORY_SCOPE_AGENT)) == call;
        u32x4 pv[2][4];
#pragma unroll
        for (int m = 0; m < 4; ++m)
#pragma unroll
            for (int bj = 0; bj < 2; ++bj)
                pv[bj][m] = __builtin_amdgcn_raw_buffer_load_b128(xr, (unsigned)tz * 16u, (unsigned)pbid * 65536u + (bj * 4 + m) * 8192, 16);
        char* orow = p.out + (size_t)(mt * 256 + kh * 128 + wr * 64 + fr2) * 16384 + (size_t)(nt * 256 + wc * 32 + (fq2 & 1) * 16 + (fq2 >> 1) * 8) * 2;
#define LO(u) __builtin_bit_cast(float, (u) << 16)
#define HI(u) __builtin_bit_cast(float, (u) & 0xffff0000u)
#pragma unroll
        for (int m = 0; m < 4; ++m)
#pragma unroll
            for (int bj = 0; bj < 2; ++bj) {
                const f32x4 a0 = acc[1][bj][m][0], a1_ = acc[1][bj][m][1]; const u32x4 q = pv[bj][m];
                const unsigned x0 = cvt_pk_bf16(gelu_tanh(a0[0] + LO(q.x)), gelu_tanh(a0[1] + HI(q.x))), x1 = cvt_pk_bf16(gelu_tanh(a0[2] + LO(q.y)), gelu_tanh(a0[3] + HI(q.y)));
                const unsigned y0 = cvt_pk_bf16(gelu_tanh(a1_[0] + LO(q.z)), gelu_tanh(a1_[1] + HI(q.z))), y1 = cvt_pk_bf16(gelu_tanh(a1_[2] + LO(q.w)), gelu_tanh(a1_[3] + HI(q.w)));
                const auto r0 = __builtin_amdgcn_permlane16_swap(x0, y0, false, false);
                const auto r1 = __builtin_amdgcn_permlane16_swap(x1, y1, false, false);
                u32x4 o; o.x = r0[0]; o.y = r1[0]; o.z = r0[1]; o.w = r1[1];
                if (bad) o = (u32x4){0x7fc07fc0u, 0x7fc07fc0u, 0x7fc07fc0u, 0x7fc07fc0u};
                *(u32x4*)(orow + (size_t)m * 16 * 16384 + bj * 256) = o;
            }
    }
}
}

extern "C" void kernel_launch(int rank, int nranks, ncclComm_t comm,
                              hipStream_t stream,
                              void* const* d_in, const int* in_sizes, int n_in,
                              void* d_out, int out_size,
                              const int* in_dtypes, int out_dtype,
                              void* const* d_out_all,
                              void* const* d_ws_all, size_t ws_bytes) {
    (void)comm; (void)in_sizes; (void)n_in; (void)out_size; (void)in_dtypes; (void)out_dtype; (void)d_out_all;
    if (nranks != a2a::NR || (unsigned)rank >= (unsigned)a2a::NR || ws_bytes < a2a::WS_END) {
        fprintf(stderr, "a2a_gemm: unexpected configuration (nranks %d, ws %zu)\n", nranks, ws_bytes);
        return;
    }
    HIP_CHECK(hipFuncSetAttribute((const void*)a2a::a2a_gemm_kernel, hipFuncAttributeMaxDynamicSharedMemorySize, a2a::LDS_BYTES));
    a2a::Params p;
    p.x = (const char*)d_in[0];
    p.w = (const char*)d_in[1];
    p.out = (char*)d_out;
    for (int i = 0; i < a2a::NR; ++i) p.ws[i] = (char*)d_ws_all[i];
    p.rank = rank;
    hipLaunchKernelGGL(a2a::a2a_gemm_kernel, dim3(256), dim3(512), a2a::LDS_BYTES, stream, p);
    HIP_CHECK(hipGetLastError());
}
```

```cpp
#include <hip/hip_runtime.h>
#include <rccl/rccl.h>
#include <cstdio>

#define HIP_CHECK(expr) do { hipError_t _e = (expr); if (_e != hipSuccess) { \
    fprintf(stderr, "HIP error %s at %s:%d\n", hipGetErrorString(_e), __FILE__, __LINE__); \
    return; } } while (0)

namespace a2a {
#define LAS __attribute__((address_space(3)))
typedef short bf16x8 __attribute__((ext_vector_type(8)));
typedef short s16x4 __attribute__((ext_vector_type(4)));
typedef float f32x4 __attribute__((ext_vector_type(4)));
typedef unsigned u32x4 __attribute__((ext_vector_type(4)));

constexpr int NR = 8;
constexpr int HTB = 16384;
constexpr int LDS_X = 8 * HTB;
constexpr int LDS_BYTES = 8 * HTB + 8 * 2048;
constexpr size_t WS_CTR = 0;
constexpr size_t WS_TMO = 32768;
constexpr size_t WS_FLAG = 65536;
constexpr size_t WS_STAGE = 2ull << 20;
constexpr size_t WS_XBUF = 34ull << 20;
constexpr size_t WS_END = 50ull << 20;
constexpr unsigned long long DEADLINE = 50000000ull;

struct Params {
    const char* x;
    const char* w;
    char* out;
    char* myws;
    int rank;
    char* ws[NR];
};

typedef float f32x2 __attribute__((ext_vector_type(2)));
typedef __bf16 bf16x2 __attribute__((ext_vector_type(2)));
__device__ __forceinline__ unsigned cvt_pk_bf16(float lo, float hi) {
    return __builtin_bit_cast(unsigned, __builtin_convertvector((f32x2){lo, hi}, bf16x2));
}
constexpr float GELU_C1 = -2.0f * 0.7978845608028654f * 1.4426950408889634f, GELU_C3 = GELU_C1 * 0.044715f;
__device__ __forceinline__ unsigned gelu2(f32x2 y, f32x2 c1, f32x2 c3) {
    const f32x2 z = y * __builtin_elementwise_fma(y * y, c3, c1);
    const f32x2 d = (f32x2){__builtin_amdgcn_exp2f(z[0]), __builtin_amdgcn_exp2f(z[1])} + 1.0f;
    const f32x2 g = y * (f32x2){__builtin_amdgcn_rcpf(d[0]), __builtin_amdgcn_rcpf(d[1])};
    return cvt_pk_bf16(g[0], g[1]);
}

#define ST(i)
#define SW(i)
#define SR(i)

__global__ __launch_bounds__(512, 2) void a2a_gemm_kernel(Params p) {
    extern __shared__ __attribute__((aligned(16))) unsigned char shm[];
    LAS unsigned char* lds = (LAS unsigned char*)shm;
    const int tid = threadIdx.x, wid = __builtin_amdgcn_readfirstlane(tid >> 6), lane = tid & 63, wr = wid >> 2, wc = wid & 3, fr = lane & 15, fq = lane >> 4;
    const int bid = blockIdx.x, xcd = bid & 7, idx = bid >> 3;
    const int kh = xcd & 1, mt = idx & 3, nt = (xcd >> 1) * 8 + (idx >> 2);
    const int rank = p.rank;
    char* const myws = p.myws;
    ST(0);

    const unsigned arow0 = (unsigned)(mt * 256 + (1 - kh) * 128) * 2048u, arow1 = (unsigned)(mt * 256 + kh * 128) * 2048u;
    const unsigned ra = (unsigned)(wid * 16 + (lane >> 3));
    const unsigned vA0 = ra * 2048u + (unsigned)((lane & 7) ^ (lane >> 4)) * 16u;
    const unsigned vA1 = (ra + 8u) * 2048u + (unsigned)((lane & 7) ^ (4 + (lane >> 4))) * 16u - 1024u;
    const unsigned voffA[2][2] = {{vA0 + arow0, vA1 + arow0}, {vA0 + arow1, vA1 + arow1}};
    const unsigned kb = (unsigned)(wid * 8 + (lane >> 4)), cb = lane & 15, fb = (unsigned)((lane >> 4) | ((wid & 1) << 2));
    const unsigned voffB0 = kb * 16384u + (((cb >> 1) ^ fb) * 32u) + (cb & 1) * 16u, voffB1 = voffB0 + 4u * 16384u - 1024u;
    const unsigned ldsw = (unsigned)wid * 2048u;
    const int aoff0 = (wr * 64 + fr) * 128 + ((fq ^ (fr >> 1)) * 16);
    const int aoff1 = aoff0 ^ 64;
    const int q4 = (lane >> 2) & 3, p4 = lane & 3, fl = q4 | ((fq & 1) << 2);
    const int boff0 = (8 * fq + q4) * 256 + (((wc * 2) ^ fl) * 32) + 8 * p4;
    const unsigned vb0 = (unsigned)(size_t)lds + 65536u + (unsigned)boff0, vb1 = (unsigned)(size_t)lds + 65536u + (unsigned)(boff0 ^ 32);
    const unsigned voffP = (unsigned)(lane >> 3) * 2048u + (unsigned)(lane & 7) * 16u;
    const unsigned voffF = (unsigned)lane * 4u;
    const unsigned vzero = 0u;
    LAS unsigned char* const ldx = lds + LDS_X + wid * 2048;
    const unsigned vxf = (unsigned)(size_t)ldx + 1024u + (unsigned)lane * 4u;

#define SA(b, h) (((b) * 2 + (h)) * HTB)
#define SB(b, h) ((4 + (b) * 2 + (h)) * HTB)
#define AUX_STAGED 17
#define STAGE_AX(slot, h, rs, soff) do { \
        __builtin_amdgcn_raw_ptr_buffer_load_lds(rs, (LAS void*)(lds + (slot) + ldsw), 16, voffA[h][0], (soff), 0, AUX_STAGED); \
        __builtin_amdgcn_raw_ptr_buffer_load_lds(rs, (LAS void*)(lds + (slot) + ldsw), 16, voffA[h][1], (soff), 1024, AUX_STAGED); } while (0)
#define STAGE_A(b, h, rs, soff) STAGE_AX(SA(b, h), h, rs, soff)
#define AUX_W 0
#define STAGE_BX(slot, h, soff) do { \
        __builtin_amdgcn_raw_ptr_buffer_load_lds(rsW, (LAS void*)(lds + (slot) - (h) * 256 + ldsw), 16, voffB0, (soff), (h) * 256, AUX_W); \
        __builtin_amdgcn_raw_ptr_buffer_load_lds(rsW, (LAS void*)(lds + (slot) - (h) * 256 + ldsw), 16, voffB1, (soff), (h) * 256 + 1024, AUX_W); } while (0)
#define STAGE_B(b, h, soff) STAGE_BX(SB(b, h), h, soff)
#define LDA(dst, b, h) do { _Pragma("unroll") for (int m = 0; m < 4; ++m) { \
        dst[m][0] = *(const LAS bf16x8*)(lds + SA(b, h) + aoff0 + m * 2048); \
        dst[m][1] = *(const LAS bf16x8*)(lds + SA(b, h) + aoff1 + m * 2048); } } while (0)
#undef LDA
#define RDA(d, base, off) asm volatile("ds_read_b128 %0, %1 offset:%2" : "=v"(d) : "v"(base), "i"(off) : "memory")
#define LDA(dst, b, h) do { _Pragma("unroll") for (int m = 0; m < 4; ++m) { RDA(dst[m][0], va0, SA(b, h) + m * 2048); } \
        _Pragma("unroll") for (int m = 0; m < 4; ++m) { RDA(dst[m][1], va1, SA(b, h) + m * 2048); } } while (0)
    const unsigned va0 = (unsigned)(size_t)lds + (unsigned)aoff0, va1 = (unsigned)(size_t)lds + (unsigned)aoff1;
#define TR(d, base, off) asm volatile("ds_read_b64_tr_b16 %0, %1 offset:%2" : "=v"(d) : "v"(base), "i"(off) : "memory")
#define LDBX(dst, v0, v1, off) do { _Pragma("unroll") for (int k = 0; k < 2; ++k) { s16x4 l0, h0, l1, h1; \
        TR(l0, v0, (off) + k * 8192); TR(h0, v0, (off) + k * 8192 + 1024); \
        TR(l1, v1, (off) + k * 8192); TR(h1, v1, (off) + k * 8192 + 1024); \
        dst[0][k] = __builtin_shufflevector(l0, h0, 0, 1, 2, 3, 4, 5, 6, 7); \
        dst[1][k] = __builtin_shufflevector(l1, h1, 0, 1, 2, 3, 4, 5, 6, 7); } } while (0)
#define LDB(dst, b, h) LDBX(dst, vb0, vb1, SB(b, h) - 65536)
#define MF_(c, a, b) "v_mfma_f32_16x16x32_bf16 %" #c ", %" #b ", %" #a ", %" #c "\n\t"
#define MSEQ MF_(0, 8, 16) MF_(0, 9, 17) MF_(2, 10, 16) MF_(2, 11, 17) MF_(4, 12, 16) MF_(4, 13, 17) MF_(6, 14, 16) MF_(6, 15, 17) \
             MF_(1, 8, 18) MF_(1, 9, 19) MF_(3, 10, 18) MF_(3, 11, 19) MF_(5, 12, 18) MF_(5, 13, 19) MF_(7, 14, 18) MF_(7, 15, 19)
#define MMAX(ai, bj, At, Bt, post) asm volatile("s_setprio 1\n\t" MSEQ "s_setprio 0" post \
        : "+v"(acc[ai][bj][0][0]), "+v"(acc[ai][bj][0][1]), "+v"(acc[ai][bj][1][0]), "+v"(acc[ai][bj][1][1]), \
          "+v"(acc[ai][bj][2][0]), "+v"(acc[ai][bj][2][1]), "+v"(acc[ai][bj][3][0]), "+v"(acc[ai][bj][3][1]) \
        : "v"(At[0][0]), "v"(At[0][1]), "v"(At[1][0]), "v"(At[1][1]), "v"(At[2][0]), "v"(At[2][1]), "v"(At[3][0]), "v"(At[3][1]), \
          "v"(Bt[0][0]), "v"(Bt[0][1]), "v"(Bt[1][0]), "v"(Bt[1][1]))
#define MMA(ai, bj, At, Bt) MMAX(ai, bj, At, Bt, "")
#define MMAT(ai, bj, At, Bt) MMAX(ai, bj, At, Bt, "\n\ts_nop 7\n\ts_nop 7")
#define WAIT_V(n) asm volatile("s_waitcnt vmcnt(" #n ")" ::: "memory")
#define WAIT_L(n) do { asm volatile("s_waitcnt lgkmcnt(" #n ")" ::: "memory"); __builtin_amdgcn_sched_barrier(0); } while (0)
#define BAR __builtin_amdgcn_s_barrier()
#define SCHED __builtin_amdgcn_sched_barrier(0)
#define POLLAUX 17
#define POLL(q) __builtin_amdgcn_raw_ptr_buffer_load_lds(rsF, (LAS void*)(ldx + 1024 + (q) * 256), 4, voffF, sflag, 0, POLLAUX)

    f32x4 acc[2][2][4][2];
#pragma unroll
    for (int a = 0; a < 2; ++a)
#pragma unroll
        for (int b = 0; b < 2; ++b)
#pragma unroll
            for (int m = 0; m < 4; ++m)
#pragma unroll
                for (int n = 0; n < 2; ++n) acc[a][b][m][n] = (f32x4){0.f, 0.f, 0.f, 0.f};
    bf16x8 At[4][2], B0[2][2], B1[2][2];

    const __amdgpu_buffer_rsrc_t rsW = __builtin_amdgcn_make_buffer_rsrc((void*)p.w, 0, 128u << 20, 0x00020000);
    const __amdgpu_buffer_rsrc_t rsX = __builtin_amdgcn_make_buffer_rsrc((void*)p.x, 0, 16u << 20, 0x00020000);
    const __amdgpu_buffer_rsrc_t rsF = __builtin_amdgcn_make_buffer_rsrc((void*)(myws + WS_FLAG), 0, 32768u, 0x00020000);
    unsigned sflag = (unsigned)((mt * 8 + wid) * 2 + kh) * 256u;
    const char* const pL = p.x + (size_t)rank * (2u << 20);
#define RSA(t) __builtin_amdgcn_make_buffer_rsrc((void*)((t) < 8 ? pL : pS), 0, 16u << 20, 0x00020000)
    const unsigned m0poll = (unsigned)(size_t)ldx + 1024u;

    const int cls = bid % 3;
    const int wq = (cls == 0 ? 0 : cls == 1 ? 688 : 1368) + (bid / 3) * 8 + wid;
    char* const ctr = myws + WS_CTR + (size_t)(bid * 8 + wid) * 16;
    unsigned fl0 = 0u, callv = 0u;
    unsigned tabA, tabB, tabLo, tabHi, tabD, tabF, tabS, voffL, vxp_;

    {
        asm volatile("global_load_dword %0, %1, %2 sc0 sc1" : "=v"(callv) : "v"(vzero), "s"(ctr) : "memory");
        const __amdgpu_buffer_rsrc_t rsL = __builtin_amdgcn_make_buffer_rsrc((void*)pL, 0, 16u << 20, 0x00020000);
        const unsigned a0 = (unsigned)kh * 1024u, b0 = (unsigned)(rank * 16 + kh * 8) * (64u * 16384u) + (unsigned)nt * 512u, b1 = b0 + 64u * 16384u;
        STAGE_B(0, 0, b0);
        STAGE_A(0, 0, rsL, a0);
        {
            const int Q = wq, b = Q / 896, rem = Q - b * 896, rg = rem / 7, di = rem - rg * 7;
            const int d = (rank + 1 + di) & 7, kk = (b ^ ((unsigned)(di - 3) >> 31)) & 1, kt = kk * 8 + (b >> 1);
            __builtin_amdgcn_raw_ptr_buffer_load_lds(rsX, (LAS void*)ldx, 16, voffP, (unsigned)(d * 1024 + rg * 8) * 2048u + (unsigned)kt * 128u, 0, 0);
        }
        STAGE_B(0, 1, b0);
        STAGE_A(0, 1, rsL, a0);
        STAGE_B(1, 0, b1);
        STAGE_A(1, 0, rsL, a0 + 128);
        STAGE_B(1, 1, b1);
        int lz = lane; asm volatile("" : "+v"(lz));
        {
            const int u = lz - 8, j = u / 7, i0 = u - j * 7, i = kh ? 6 - i0 : i0;
            const bool loc = lz >= 2 && lz < 8;
            const int c = loc ? rank : (rank + (lz < 2 ? 2 - lz : 1 + i)) & 7, kt = lz < 2 ? 15 : kh * 8 + (lz < 8 ? lz : j);
            tabA = (loc ? 0u : (unsigned)c * (2u << 20)) + (unsigned)kt * 128u;
            tabB = (unsigned)(c * 16 + kt) * (64u * 16384u) + (unsigned)nt * 512u;
        }
        {
            const int sl = lz < 7 ? lz : 6;
            const int Q = sl * 2048 + wq, b = Q / 896, rem = Q - b * 896, rg = rem / 7, di = rem - rg * 7;
            const int d = (rank + 1 + di) & 7, ipos = 6 - di, kk = (b & 1) ^ (ipos > 3 ? 1 : 0), kt = kk * 8 + (b >> 1);
            unsigned long long base = 0;
#pragma unroll
            for (int i = 0; i < NR; ++i) base = d == i ? (unsigned long long)p.ws[i] : base;
            base = lz >= 8 ? (unsigned long long)p.out : base;
            tabLo = (unsigned)base; tabHi = (unsigned)(base >> 32);
            tabS = (unsigned)(d * 1024 + rg * 8) * 2048u + (unsigned)kt * 128u;
            tabD = (unsigned)WS_STAGE + (unsigned)rank * (2u << 20) + (unsigned)(rg * 8) * 2048u + (unsigned)kt * 128u;
            const int tt = 8 + (b >> 1) * 7 + (kk ? 6 - ipos : ipos), half = (rg >> 4) & 1;
            const bool shared = kk && tt >= 62;
            tabF = (unsigned)WS_FLAG + (unsigned)((((rg >> 5) * 8 + ((rg >> 1) & 7)) * 2 + (shared ? half : kk)) * 256 + (shared ? 1 : tt) * 4 + (shared ? tt & 1 : half) * 2 + (rg & 1));
        }
        voffL = lz ? 0x80000000u : 0u;
        vxp_ = (unsigned)(size_t)ldx + (unsigned)lz * 16u;
        asm volatile("ds_write_b32 %0, %1\n\tds_write_b32 %0, %1 offset:512" :: "v"(vxf), "v"(vzero) : "memory");
        asm volatile("" : "+v"(tabLo), "+v"(tabHi), "+v"(tabD), "+v"(tabF), "+v"(tabS), "+v"(tabA), "+v"(tabB), "+v"(voffL), "+v"(vxp_));
#pragma unroll
        for (int a = 0; a < 2; ++a)
#pragma unroll
            for (int b = 0; b < 2; ++b)
                asm volatile("" : "+v"(acc[a][b][0][0]), "+v"(acc[a][b][0][1]), "+v"(acc[a][b][1][0]), "+v"(acc[a][b][1][1]),
                                  "+v"(acc[a][b][2][0]), "+v"(acc[a][b][2][1]), "+v"(acc[a][b][3][0]), "+v"(acc[a][b][3][1]));
        if (wr == 1) BAR;
                asm volatile("s_waitcnt vmcnt(13)" : "+v"(callv) :: "memory");
        BAR; BAR;
        LDB(B0, 0, 0); WAIT_V(10); BAR; WAIT_L(0); BAR;
    }
    const unsigned call = (unsigned)__builtin_amdgcn_readfirstlane(callv) + 1u;
    const unsigned spar = (call & 1u) << 24;
    const unsigned fpar = (call & 1u) << 14;
    const unsigned fval = (call & 0x7fu) | 0x80u, fpat = fval * 0x01010101u;
    sflag += fpar;
    const char* const pS = myws + WS_STAGE + spar;
    unsigned long long ready = kh ? 0xc0000000000000fdull : 0xfdull;
    const unsigned xbad = (unsigned)(size_t)lds + LDS_X + 1536u;
    unsigned* const tmo = (unsigned*)(myws + WS_TMO);

    asm volatile(".subsection 2\n\t.fill 256, 4, 0xbf800000\n\t.subsection 0");
#define POLL_RD(f) asm volatile("ds_read_b32 %0, %1" : "=v"(f) : "v"(vxf) : "memory")
#define NEED(tt, f) NEEDM(3ull << (tt), f)
#define NEEDM(msk, f) do { \
        asm volatile("s_waitcnt lgkmcnt(8)" : "+v"(f) :: "memory"); \
        ready |= __builtin_amdgcn_ballot_w64(f == fpat); \
        asm volatile( \
            "s_andn2_b64 s[96:97], %[m], %[rdy]\n\t" \
            "s_cbranch_scc1 .Lnr%=\n" \
            ".Lnb%=:\n\t" \
            ".subsection 1\n" \
            ".Lnr%=:\n\t" \
            "s_memrealtime s[96:97]\n\t" \
            "s_waitcnt lgkmcnt(0)\n\t" \
            "s_mov_b32 s97, m0\n\t" \
            "s_mov_b32 m0, %[m0q]\n\t" \
            "s_nop 0\n" \
            ".Lnl%=:\n\t" \
            "buffer_load_dword %[voffF], %[rsF], %[sflag] offen sc0 sc1 lds\n\t" \
            "global_load_dword v249, %[vz], %[tmo] sc0 sc1\n\t" \
            "s_waitcnt vmcnt(0)\n\t" \
            "ds_read_b32 v248, %[a]\n\t" \
            "s_memrealtime s[98:99]\n\t" \
            "s_waitcnt lgkmcnt(0)\n\t" \
            "v_cmp_eq_u32 vcc, %[fpat], v248\n\t" \
            "s_sub_u32 s98, s98, s96\n\t" \
            "s_nop 0\n\t" \
            "s_or_b64 %[rdy], %[rdy], vcc\n\t" \
            "s_andn2_b64 vcc, %[m], %[rdy]\n\t" \
            "s_cbranch_scc0 .Lnx%=\n\t" \
            "v_cmp_eq_u32 vcc, %[call], v249\n\t" \
            "s_cmp_gt_u32 s98, %[dl]\n\t" \
            "s_cbranch_scc1 .Lnd%=\n\t" \
            "s_cbranch_vccnz .Lnd%=\n\t" \
            "s_sleep 1\n\t" \
            "s_branch .Lnl%=\n" \
            ".Lnd%=:\n\t" \
            "v_mov_b32 v249, %[call]\n\t" \
            "s_mov_b64 %[rdy], -1\n\t" \
            "global_store_dword %[vz], v249, %[tmo] sc0 sc1\n\t" \
            "v_mbcnt_lo_u32_b32 v248, -1, 0\n\t" \
            "v_and_b32 v248, 7, v248\n\t" \
            "v_lshlrev_b32 v248, 11, v248\n\t" \
            "v_add_u32 v248, %[xb], v248\n\t" \
            "ds_write_b32 v248, v249\n\t" \
            "s_waitcnt vmcnt(0) lgkmcnt(0)\n" \
            ".Lnx%=:\n\t" \
            "s_mov_b32 m0, s97\n\t" \
            "s_branch .Lnb%=\n\t" \
            ".subsection 0" \
            : [rdy] "+s"(ready) \
            : [m] "s"((unsigned long long)(msk)), [xb] "s"(xbad), [m0q] "s"(m0poll), [voffF] "v"(voffF), [rsF] "s"(rsF), [sflag] "s"(sflag), [vz] "v"(vzero), [tmo] "s"(tmo), \
              [a] "v"(vxf), [fpat] "s"(fpat), [call] "s"(call), [dl] "s"((unsigned)DEADLINE) \
            : "memory", "scc", "vcc", "s96", "s97", "s98", "s99", "v248", "v249"); } while (0)

#define LDA2(a, b, c) LDA(a, b, c)
#define TRIPX(W1, W2, W3, W4, W5, W6, W7, W8, X1, X2, X6, X8, PRD, ND, PL, RSN) do { \
        ST(1 + (t >> 1)); \
        X1; LDA(At, 0, 0); STAGE_A(1, 1, r1, a1); \
        WAIT_V(W1); BAR; WAIT_L(0); MMA(0, 0, At, B0); BAR; SCHED; \
        PRD; LDB(B1, 0, 1); X2; STAGE_B(0, 0, b2); ND; \
        WAIT_V(W2); BAR; WAIT_L(0); MMA(0, 1, At, B1); BAR; SCHED; \
        LDA2(At, 0, 1); STAGE_A(0, 0, r2, a2); \
        WAIT_V(W3); BAR; WAIT_L(0); MMA(1, 0, At, B0); BAR; SCHED; \
        LDB(B0, 1, 0); PL; STAGE_B(0, 1, b2); \
        a1 = __builtin_amdgcn_readlane(tabA, t + 3); unsigned b3 = __builtin_amdgcn_readlane(tabB, t + 3); \
        asm volatile("" : "+s"(a1), "+s"(b3)); \
        WAIT_V(W4); BAR; MMA(1, 1, At, B1); SCHED; WAIT_L(0); BAR; SCHED; \
        LDA(At, 1, 0); STAGE_A(0, 1, r2, a2); \
        WAIT_V(W5); BAR; WAIT_L(0); MMA(0, 0, At, B0); BAR; SCHED; \
        LDB(B1, 1, 1); X6; STAGE_B(1, 0, b3); \
        WAIT_V(W6); BAR; WAIT_L(0); MMA(0, 1, At, B1); BAR; SCHED; \
        LDA2(At, 1, 1); STAGE_A(1, 0, r2, a1); \
        WAIT_V(W7); BAR; WAIT_L(0); MMA(1, 0, At, B0); BAR; SCHED; \
        LDB(B0, 0, 0); X8; STAGE_B(1, 1, b3); \
        t += 2; RSN; \
        a2 = __builtin_amdgcn_readlane(tabA, t + 2); b2 = __builtin_amdgcn_readlane(tabB, t + 2); \
        asm volatile("" : "+s"(a2), "+s"(b2), "+s"(t)); \
        WAIT_V(W8); BAR; MMA(1, 1, At, B1); SCHED; WAIT_L(0); BAR; SCHED; } while (0)
#define RS_NEXT do { r1 = r2; r2 = RSA(t + 2); asm volatile("" : "+s"(r1), "+s"(r2)); } while (0)
#define TRIP(W1, W2, W3, W4, W5, W6, W7, W8, X1, X2, X6, X8) \
        TRIPX(W1, W2, W3, W4, W5, W6, W7, W8, X1, X2, X6, X8, POLL_RD(fl0), NEED(t + 2, fl0), POLL(0), RS_NEXT)

    unsigned a1 = (unsigned)kh * 1024u + 128u, a2 = a1 + 128u, b2 = (unsigned)(rank * 16 + kh * 8 + 2) * (64u * 16384u) + (unsigned)nt * 512u;
    __amdgpu_buffer_rsrc_t r1 = RSA(0), r2 = r1;
    int t = 0;
    const int tl = 62 - 2 * kh;
    {
        int pn = 0; const int lim = 2 * cls;
        while (t < lim) TRIP(10, 10, 10, 11, 11, 11, 11, 11, , , , );
        for (; pn < 7; ++pn) {
            {
                const unsigned long long pb = (unsigned long long)(unsigned)__builtin_amdgcn_readlane(tabLo, pn) |
                                              ((unsigned long long)(unsigned)__builtin_amdgcn_readlane(tabHi, pn) << 32);
                const __amdgpu_buffer_rsrc_t rp = __builtin_amdgcn_make_buffer_rsrc((void*)pb, 0, (unsigned)WS_XBUF, 0x00020000);
                const unsigned sD = __builtin_amdgcn_readlane(tabD, pn) + spar, sF = __builtin_amdgcn_readlane(tabF, pn) + fpar;
                const unsigned sS = __builtin_amdgcn_readlane(tabS, pn + 1);
                u32x4 pd;
                TRIP(10, 11, 11, 12, 12, 13, 12, 13,
                     asm volatile("ds_read_b128 %0, %1" : "=v"(pd) : "v"(vxp_) : "memory"),
                     __builtin_amdgcn_raw_buffer_store_b128(pd, rp, voffP, sD, 17),
                     __builtin_amdgcn_raw_ptr_buffer_load_lds(rsX, (LAS void*)ldx, 16, voffP, sS, 0, 0),
                     __builtin_amdgcn_raw_buffer_store_b8((unsigned char)fval, rp, voffL, sF, 17));
            }
            TRIP(12, 12, 11, 12, 11, 11, 11, 11, , , , );
            TRIP(10, 10, 10, 11, 11, 11, 11, 11, , , , );
        }
        while (t < tl && ready != ~0ull) TRIP(10, 10, 10, 11, 11, 11, 11, 11, , , , );
    }
    while (t < tl) TRIPX(10, 10, 10, 10, 10, 10, 10, 10, , , , , , , , );
    {
        int tz = threadIdx.x; asm volatile("" : "+v"(tz));
        const unsigned ln = (unsigned)tz & 63u;
        const __amdgpu_buffer_rsrc_t xr = __builtin_amdgcn_make_buffer_rsrc(myws + WS_XBUF, 0, 256u * 65536u, 0x00020000);
        unsigned voffX = (unsigned)(bid * 8 + wid) * 8192u + ln * 16u;
        unsigned badv, badm, tagm = 0xfffe0000u;
        const unsigned tag = call & 1u, tagr = 0x10000u - (tag << 16);
#define RL(tab, i) __builtin_amdgcn_readlane(tab, i)
#define SEND(bj, m) do { u32x4 o_; \
        const float t_ = __builtin_bit_cast(float, ((__builtin_bit_cast(unsigned, acc[0][bj][m][0][0]) + tagr) & tagm) | tago); \
        o_.x = cvt_pk_bf16(t_, acc[0][bj][m][0][1]); o_.y = cvt_pk_bf16(acc[0][bj][m][0][2], acc[0][bj][m][0][3]); \
        o_.z = cvt_pk_bf16(acc[0][bj][m][1][0], acc[0][bj][m][1][1]); o_.w = cvt_pk_bf16(acc[0][bj][m][1][2], acc[0][bj][m][1][3]); \
        __builtin_amdgcn_raw_buffer_store_b128(o_, xr, voffX + ((bj) * 4 + (m)) * 1024, 0, 16); } while (0)
#define PH_A(W, ai, bj, Ax, Bx) WAIT_V(W); BAR; WAIT_L(0); MMAT(ai, bj, Ax, Bx); BAR; SCHED
#define PH_B(W, ai, bj, Ax, Bx) WAIT_V(W); BAR; WAIT_L(8); MMAT(ai, bj, Ax, Bx); SCHED; WAIT_L(0); BAR; SCHED
#define PH_C(W, ai, bj, Ax, Bx) WAIT_V(W); BAR; MMAT(ai, bj, Ax, Bx); SCHED; WAIT_L(0); BAR; SCHED
        u32x4 pv[2][4];
#define PVLD(bj) do { _Pragma("unroll") for (int m = 0; m < 4; ++m) \
        asm volatile("buffer_load_dwordx4 %0, %1, %2, %3 offen offset:%4 sc1" : "=v"(pv[bj][m]) : "v"(voffY), "s"(xr), "s"((bj) * 4096u), "i"(m * 1024) : "memory"); } while (0)
#define PVWAIT(bj, n) asm volatile("s_waitcnt vmcnt(" #n ")" : "+v"(pv[bj][0]), "+v"(pv[bj][1]), "+v"(pv[bj][2]), "+v"(pv[bj][3]) :: "memory")
        if (kh) { ST(31); } ST(32); SW(1); SR(62);
        LDA(At, 0, 0); STAGE_A(1, 1, r1, a1);
        PH_A(10, 0, 0, At, B0); SW(2);
        POLL_RD(fl0); LDB(B1, 0, 1); STAGE_B(0, 0, b2); NEEDM(~0ull, fl0);
        PH_A(10, 0, 1, At, B1); SW(3);
        LDA(At, 0, 1); asm volatile("ds_read_b32 %0, %1 offset:512" : "=v"(badv) : "v"(vxf) : "memory");
        STAGE_AX(SA(0, 0), 1, r2, a2);
        PH_A(10, 1, 0, At, B0); SW(4);
        LDB(B0, 1, 0); STAGE_B(0, 1, b2);
        asm volatile("" : "+v"(badv), "+v"(tagm));
        badm = __builtin_amdgcn_readfirstlane(badv) ? 0x7fc07fc0u : 0u;
        unsigned tago = (tag << 16) | (badm & 0xffff0000u);
        a1 = RL(tabA, t + 3); unsigned b3 = RL(tabB, t + 3);
        asm volatile("" : "+s"(a1), "+s"(b3), "+s"(badm), "+s"(tago));
        PH_C(10, 1, 1, At, B1); SW(5);
        LDA(At, 1, 0); STAGE_AX(SA(0, 1), 1, r2, a1);
        PH_A(10, 0, 0, At, B0); SW(6);
        LDB(B1, 1, 1); STAGE_B(1, 0, b3); SCHED; SEND(0, 0); SEND(0, 1);
        PH_A(12, 0, 1, At, B1); SW(7);
        LDA(At, 1, 1); SEND(0, 2); SEND(0, 3);
        PH_A(12, 1, 0, At, B0); SW(8);
        LDB(B0, 0, 0); STAGE_B(1, 1, b3); SCHED; SEND(1, 0); SEND(1, 1);
        PH_C(14, 1, 1, At, B1); SW(9);
        ST(33);
        LDA(At, 0, 0); SEND(1, 2); SEND(1, 3);
        PH_A(10, 1, 0, At, B0); SW(10);
        LDB(B1, 0, 1); LDB(B0, 1, 0);
        const unsigned voffY = voffX ^ 65536u;
        const unsigned one = ((ln >> 4) == ((ln >> 2) & 3u)) ? ((ln & 1u) ? 0x3f800000u : 0x00003f80u) : 0u;
        unsigned sd0 = (ln & 2u) ? 0u : one, sd1 = (ln & 2u) ? one : 0u;
        asm volatile("" : "+v"(sd0), "+v"(sd1));
        BAR; WAIT_L(8); MMAT(1, 1, At, B1); SCHED; WAIT_L(0); BAR; SCHED; SW(11);
        LDA(At, 0, 1); PVLD(0);
        char* orow = (char*)((size_t)(unsigned)RL(tabLo, 8) | ((size_t)(unsigned)RL(tabHi, 8) << 32)) +
                     (size_t)((unsigned)(mt * 256 + kh * 128 + wr * 64) + (ln & 15u)) * 16384u + (size_t)((unsigned)(nt * 256 + wc * 32) + ((ln >> 4) & 1u) * 16u + (ln >> 5) * 8u) * 2u;
        asm volatile("" : "+v"(orow));
        WAIT_V(8); BAR; WAIT_L(0); MMAT(1, 0, At, B0); SCHED;
        if (wr == 0) BAR; SW(12);
#define PMISS(bj) __builtin_amdgcn_ballot_w64((((pv[bj][0].x ^ tag) | (pv[bj][1].x ^ tag) | (pv[bj][2].x ^ tag) | (pv[bj][3].x ^ tag)) & 1u) != 0u)
#define NOW(t_) asm volatile("s_memrealtime %0\n\ts_waitcnt lgkmcnt(0)" : "=s"(t_) :: "memory")
#define ADDP(bj) do { \
        if (__builtin_expect(PMISS(bj) != 0ull, 0)) { \
            unsigned long long t0_, t1_; NOW(t0_); \
            for (;;) { unsigned tv_; PVLD(bj); \
                asm volatile("global_load_dword %0, %1, %2 sc0 sc1\n\ts_waitcnt vmcnt(0)" : "=v"(tv_) : "v"(vzero), "s"(tmo) : "memory"); \
                PVWAIT(bj, 0); \
                if (PMISS(bj) == 0ull) break; \
                NOW(t1_); \
                if ((unsigned)__builtin_amdgcn_readfirstlane(tv_) == call || t1_ - t0_ > DEADLINE) { \
                    asm volatile("global_store_dword %0, %1, %2 sc0 sc1\n\ts_waitcnt vmcnt(0)" :: "v"(vzero), "v"(call), "s"(tmo) : "memory"); \
                    badm = 0x7fc07fc0u; break; } \
                __builtin_amdgcn_s_sleep(1); } } \
        const bf16x8 s0_ = __builtin_bit_cast(bf16x8, (u32x4){sd0, sd1, badm, 0u}), s1_ = __builtin_bit_cast(bf16x8, (u32x4){badm, 0u, sd0, sd1}); \
        _Pragma("unroll") for (int m = 0; m < 4; ++m) { const bf16x8 q_ = __builtin_bit_cast(bf16x8, pv[bj][m]); \
            acc[1][bj][m][0] = __builtin_amdgcn_mfma_f32_16x16x32_bf16(s0_, q_, acc[1][bj][m][0], 0, 0, 0); \
            acc[1][bj][m][1] = __builtin_amdgcn_mfma_f32_16x16x32_bf16(s1_, q_, acc[1][bj][m][1], 0, 0, 0); } } while (0)
        f32x2 gc1 = {GELU_C1, GELU_C1}, gc3 = {GELU_C3, GELU_C3};
        asm volatile("" : "+s"(gc1), "+s"(gc3));
#define OSTORE(a, off, o) (*(__attribute__((address_space(1))) u32x4*)((a) + (off)) = (o))
#define G(bj, m) do { const f32x4 a0 = acc[1][bj][m][0], a1_ = acc[1][bj][m][1]; \
        const unsigned x0 = gelu2((f32x2){a0[0], a0[1]}, gc1, gc3), x1 = gelu2((f32x2){a0[2], a0[3]}, gc1, gc3); \
        const unsigned y0 = gelu2((f32x2){a1_[0], a1_[1]}, gc1, gc3), y1 = gelu2((f32x2){a1_[2], a1_[3]}, gc1, gc3); \
        const auto r0 = __builtin_amdgcn_permlane16_swap(x0, y0, false, false); \
        const auto r1_ = __builtin_amdgcn_permlane16_swap(x1, y1, false, false); \
        u32x4 o; o.x = r0[0]; o.y = r1_[0]; o.z = r0[1]; o.w = r1_[1]; \
        OSTORE(orow + (size_t)(m) * 16 * 16384, (bj) * 256, o); } while (0)
        LDB(B1, 1, 1); WAIT_L(0);
        MMAT(1, 1, At, B1); SCHED;
        PVLD(1);
        ST(34); SW(13); SR(61);
        PVWAIT(0, 4); SW(14); ADDP(0); SCHED; SW(15);
        G(0, 0); G(0, 1); G(0, 2); G(0, 3); SCHED; SW(16);
        PVWAIT(1, 4); SW(17); ADDP(1); SCHED; SW(18);
        G(1, 0); G(1, 1); G(1, 2); G(1, 3); SW(19);
        if (ln == 0u) __hip_atomic_store((unsigned*)ctr, call, __ATOMIC_RELAXED, __HIP_MEMORY_SCOPE_AGENT);
    }
}
}

extern "C" void kernel_launch(int rank, int nranks, ncclComm_t comm,
                              hipStream_t stream,
                              void* const* d_in, const int* in_sizes, int n_in,
                              void* d_out, int out_size,
                              const int* in_dtypes, int out_dtype,
                              void* const* d_out_all,
                              void* const* d_ws_all, size_t ws_bytes) {
    (void)comm; (void)in_sizes; (void)n_in; (void)out_size; (void)in_dtypes; (void)out_dtype; (void)d_out_all;
    if (nranks != a2a::NR || (unsigned)rank >= (unsigned)a2a::NR || ws_bytes < a2a::WS_END) {
        fprintf(stderr, "a2a_gemm: unexpected configuration (nranks %d, ws %zu)\n", nranks, ws_bytes);
        return;
    }
    HIP_CHECK(hipFuncSetAttribute((const void*)a2a::a2a_gemm_kernel, hipFuncAttributeMaxDynamicSharedMemorySize, a2a::LDS_BYTES));
    a2a::Params p;
    p.x = (const char*)d_in[0];
    p.w = (const char*)d_in[1];
    p.out = (char*)d_out;
    for (int i = 0; i < a2a::NR; ++i) p.ws[i] = (char*)d_ws_all[i];
    p.myws = (char*)d_ws_all[rank];
    p.rank = rank;
    hipLaunchKernelGGL(a2a::a2a_gemm_kernel, dim3(256), dim3(512), a2a::LDS_BYTES, stream, p);
    HIP_CHECK(hipGetLastError());
}
```

```cpp
#include <hip/hip_runtime.h>
#include <rccl/rccl.h>
#include <cstdio>

#define HIP_CHECK(expr) do { hipError_t _e = (expr); if (_e != hipSuccess) { \
    fprintf(stderr, "HIP error %s at %s:%d\n", hipGetErrorString(_e), __FILE__, __LINE__); \
    return; } } while (0)

namespace a2a {
#define LAS __attribute__((address_space(3)))
typedef short bf16x8 __attribute__((ext_vector_type(8)));
typedef short s16x4 __attribute__((ext_vector_type(4)));
typedef float f32x4 __attribute__((ext_vector_type(4)));
typedef unsigned u32x4 __attribute__((ext_vector_type(4)));

constexpr int NR = 8;
constexpr int HTB = 16384;
constexpr int LDS_X = 8 * HTB;
constexpr int LDS_BYTES = 8 * HTB + 8 * 2048;
constexpr size_t WS_CTR = 0;
constexpr size_t WS_TMO = 32768;
constexpr size_t WS_FLAG = 65536;
constexpr size_t WS_STAGE = 2ull << 20;
constexpr size_t WS_XBUF = 34ull << 20;
constexpr size_t WS_END = 50ull << 20;
constexpr unsigned long long DEADLINE = 50000000ull;
constexpr float TILE_CYC = 2300.f;

struct Params {
    const char* x;
    const char* w;
    char* out;
    char* myws;
    int rank;
    char* ws[NR];
};

typedef float f32x2 __attribute__((ext_vector_type(2)));
typedef __bf16 bf16x2 __attribute__((ext_vector_type(2)));
__device__ __forceinline__ unsigned cvt_pk_bf16(float lo, float hi) {
    return __builtin_bit_cast(unsigned, __builtin_convertvector((f32x2){lo, hi}, bf16x2));
}
constexpr float GELU_C1 = -2.0f * 0.7978845608028654f * 1.4426950408889634f, GELU_C3 = GELU_C1 * 0.044715f;
__device__ __forceinline__ unsigned gelu2(f32x2 y, f32x2 c1, f32x2 c3) {
    const f32x2 z = y * __builtin_elementwise_fma(y * y, c3, c1);
    const f32x2 d = (f32x2){__builtin_amdgcn_exp2f(z[0]), __builtin_amdgcn_exp2f(z[1])} + 1.0f;
    const f32x2 g = y * (f32x2){__builtin_amdgcn_rcpf(d[0]), __builtin_amdgcn_rcpf(d[1])};
    return cvt_pk_bf16(g[0], g[1]);
}

#define ST(i)
#define SW(i)
#define SR(i)

__global__ __launch_bounds__(512, 2) void a2a_gemm_kernel(Params p) {
    extern __shared__ __attribute__((aligned(16))) unsigned char shm[];
    LAS unsigned char* lds = (LAS unsigned char*)shm;
    const int tid = threadIdx.x, wid = __builtin_amdgcn_readfirstlane(tid >> 6), lane = tid & 63, wr = wid >> 2, wc = wid & 3, fr = lane & 15, fq = lane >> 4;
    const int bid = blockIdx.x, xcd = bid & 7, idx = bid >> 3;
    const int kh = xcd & 1, mt = idx & 3, nt = (xcd < 4 ? xcd : 7 - xcd) * 8 + (idx >> 2);
    const int rank = p.rank;
    char* const myws = p.myws;
    ST(0);

    const unsigned arow0 = (unsigned)(mt * 256 + (1 - kh) * 128) * 2048u, arow1 = (unsigned)(mt * 256 + kh * 128) * 2048u;
    const unsigned ra = (unsigned)(wid * 16 + (lane >> 3));
    const unsigned vA0 = ra * 2048u + (unsigned)((lane & 7) ^ (lane >> 4)) * 16u;
    const unsigned vA1 = (ra + 8u) * 2048u + (unsigned)((lane & 7) ^ (4 + (lane >> 4))) * 16u - 1024u;
    const unsigned voffA[2][2] = {{vA0 + arow0, vA1 + arow0}, {vA0 + arow1, vA1 + arow1}};
    const unsigned kb = (unsigned)(wid * 8 + (lane >> 4)), cb = lane & 15, fb = (unsigned)((lane >> 4) | ((wid & 1) << 2));
    const unsigned voffB0 = kb * 16384u + (((cb >> 1) ^ fb) * 32u) + (cb & 1) * 16u, voffB1 = voffB0 + 4u * 16384u - 1024u;
    const unsigned ldsw = (unsigned)wid * 2048u;
    const int aoff0 = (wr * 64 + fr) * 128 + ((fq ^ (fr >> 1)) * 16);
    const int aoff1 = aoff0 ^ 64;
    const int q4 = (lane >> 2) & 3, p4 = lane & 3, fl = q4 | ((fq & 1) << 2);
    const int boff0 = (8 * fq + q4) * 256 + (((wc * 2) ^ fl) * 32) + 8 * p4;
    unsigned va[2][2], vb[2][2];
    va[0][0] = (unsigned)(size_t)lds + (unsigned)aoff0; va[0][1] = (unsigned)(size_t)lds + (unsigned)aoff1;
    vb[0][0] = (unsigned)(size_t)lds + (unsigned)boff0; vb[0][1] = (unsigned)(size_t)lds + (unsigned)(boff0 ^ 32);
    va[1][0] = va[0][0] + 65536u; va[1][1] = va[0][1] + 65536u; vb[1][0] = vb[0][0] + 65536u; vb[1][1] = vb[0][1] + 65536u;
    LAS unsigned char* lb[2] = {lds + ldsw, lds + 65536 + ldsw};
    const unsigned voffP = (unsigned)(lane >> 3) * 2048u + (unsigned)(lane & 7) * 16u;
    const unsigned sfl0 = (unsigned)(mt * 8 + wid) * 512u, spw = 16384u + (unsigned)(nt * 4 + mt) * 64u;
    const unsigned voffF = lane >= 8 ? (unsigned)(kh * 256 + lane * 4) : lane < 4 ? (unsigned)((1 - kh) * 256 + (63 - lane) * 4) : spw - sfl0 + (unsigned)(lane - 4) * 4u;
    const unsigned vzero = 0u;
    LAS unsigned char* const ldx = lds + LDS_X + wid * 2048;
    const unsigned vxf = (unsigned)(size_t)ldx + 1024u + (unsigned)lane * 4u;

#define AUX_STAGED 17
#define AUX_W 0
#define STAGE_AX(b, sl, h, rs, soff) do { \
        __builtin_amdgcn_raw_ptr_buffer_load_lds(rs, (LAS void*)(lb[b] + (sl) * HTB), 16, voffA[h][0], (soff), 0, AUX_STAGED); \
        __builtin_amdgcn_raw_ptr_buffer_load_lds(rs, (LAS void*)(lb[b] + (sl) * HTB), 16, voffA[h][1], (soff), 1024, AUX_STAGED); } while (0)
#define STAGE_A(b, h, rs, soff) STAGE_AX(b, h, h, rs, soff)
#define STAGE_BX(b, sl, h, soff) do { \
        __builtin_amdgcn_raw_ptr_buffer_load_lds(rsW, (LAS void*)(lb[b] + (sl) * HTB - (h) * 256), 16, voffB0, (soff), (h) * 256, AUX_W); \
        __builtin_amdgcn_raw_ptr_buffer_load_lds(rsW, (LAS void*)(lb[b] + (sl) * HTB - (h) * 256), 16, voffB1, (soff), (h) * 256 + 1024, AUX_W); } while (0)
#define STAGE_B(b, h, soff) STAGE_BX(b, 2 + (h), h, soff)
#define RDA(d, base, off) asm volatile("ds_read_b128 %0, %1 offset:%2" : "=v"(d) : "v"(base), "i"(off) : "memory")
#define LDA(dst, b, sl) do { _Pragma("unroll") for (int m = 0; m < 4; ++m) { RDA(dst[m][0], va[b][0], (sl) * HTB + m * 2048); } \
        _Pragma("unroll") for (int m = 0; m < 4; ++m) { RDA(dst[m][1], va[b][1], (sl) * HTB + m * 2048); } } while (0)
#define TR(d, base, off) asm volatile("ds_read_b64_tr_b16 %0, %1 offset:%2" : "=v"(d) : "v"(base), "i"(off) : "memory")
#define LDBX(dst, b, sl) do { _Pragma("unroll") for (int k = 0; k < 2; ++k) { s16x4 l0, h0, l1, h1; \
        TR(l0, vb[b][0], (sl) * HTB + k * 8192); TR(h0, vb[b][0], (sl) * HTB + k * 8192 + 1024); \
        TR(l1, vb[b][1], (sl) * HTB + k * 8192); TR(h1, vb[b][1], (sl) * HTB + k * 8192 + 1024); \
        dst[0][k] = __builtin_shufflevector(l0, h0, 0, 1, 2, 3, 4, 5, 6, 7); \
        dst[1][k] = __builtin_shufflevector(l1, h1, 0, 1, 2, 3, 4, 5, 6, 7); } } while (0)
#define LDB(dst, b, h) LDBX(dst, b, 2 + (h))
#define MF_(c, a, b) "v_mfma_f32_16x16x32_bf16 %" #c ", %" #b ", %" #a ", %" #c "\n\t"
#define MSEQ MF_(0, 8, 16) MF_(0, 9, 17) MF_(2, 10, 16) MF_(2, 11, 17) MF_(4, 12, 16) MF_(4, 13, 17) MF_(6, 14, 16) MF_(6, 15, 17) \
             MF_(1, 8, 18) MF_(1, 9, 19) MF_(3, 10, 18) MF_(3, 11, 19) MF_(5, 12, 18) MF_(5, 13, 19) MF_(7, 14, 18) MF_(7, 15, 19)
#define MMAX(ai, bj, At, Bt, post) asm volatile("s_setprio 1\n\t" MSEQ "s_setprio 0" post \
        : "+v"(acc[ai][bj][0][0]), "+v"(acc[ai][bj][0][1]), "+v"(acc[ai][bj][1][0]), "+v"(acc[ai][bj][1][1]), \
          "+v"(acc[ai][bj][2][0]), "+v"(acc[ai][bj][2][1]), "+v"(acc[ai][bj][3][0]), "+v"(acc[ai][bj][3][1]) \
        : "v"(At[0][0]), "v"(At[0][1]), "v"(At[1][0]), "v"(At[1][1]), "v"(At[2][0]), "v"(At[2][1]), "v"(At[3][0]), "v"(At[3][1]), \
          "v"(Bt[0][0]), "v"(Bt[0][1]), "v"(Bt[1][0]), "v"(Bt[1][1]))
#define MMA(ai, bj, At, Bt) MMAX(ai, bj, At, Bt, "")
#define MMAT(ai, bj, At, Bt) MMAX(ai, bj, At, Bt, "\n\ts_nop 7\n\ts_nop 7")
#define WAIT_V(n) asm volatile("s_waitcnt vmcnt(" #n ")" ::: "memory")
#define WAIT_L(n) do { asm volatile("s_waitcnt lgkmcnt(" #n ")" ::: "memory"); __builtin_amdgcn_sched_barrier(0); } while (0)
#define BAR __builtin_amdgcn_s_barrier()
#define SCHED __builtin_amdgcn_sched_barrier(0)
#define POLLAUX 17
#define POLL(q) __builtin_amdgcn_raw_ptr_buffer_load_lds(rsF, (LAS void*)(ldx + 1024 + (q) * 256), 4, voffF, sflag, 0, POLLAUX)

    f32x4 acc[2][2][4][2];
#pragma unroll
    for (int a = 0; a < 2; ++a)
#pragma unroll
        for (int b = 0; b < 2; ++b)
#pragma unroll
            for (int m = 0; m < 4; ++m)
#pragma unroll
                for (int n = 0; n < 2; ++n) acc[a][b][m][n] = (f32x4){0.f, 0.f, 0.f, 0.f};
    bf16x8 At[4][2], B0[2][2], B1[2][2];

    const __amdgpu_buffer_rsrc_t rsW = __builtin_amdgcn_make_buffer_rsrc((void*)p.w, 0, 128u << 20, 0x00020000);
    const __amdgpu_buffer_rsrc_t rsX = __builtin_amdgcn_make_buffer_rsrc((void*)p.x, 0, 16u << 20, 0x00020000);
    const __amdgpu_buffer_rsrc_t rsF = __builtin_amdgcn_make_buffer_rsrc((void*)(myws + WS_FLAG), 0, 65536u, 0x00020000);
    unsigned sflag = sfl0;
    const char* const pL = p.x + (size_t)rank * (2u << 20);
#define RSA(t) __builtin_amdgcn_make_buffer_rsrc((void*)((t) < 8 ? pL : pS), 0, 16u << 20, 0x00020000)
    const unsigned m0poll = (unsigned)(size_t)ldx + 1024u;

    const int cls = bid % 3;
    const int wq = (cls == 0 ? 0 : cls == 1 ? 688 : 1368) + (bid / 3) * 8 + wid;
    char* const ctr = myws + WS_CTR + (size_t)(bid * 8 + wid) * 16;
    unsigned fl0 = 0u, callv = 0u;
    unsigned long long rt1, ck1;
    unsigned tabA, tabB, tabLo, tabHi, tabD, tabF, tabS, voffL, vxp_;

    {
        asm volatile("global_load_dword %0, %1, %2 sc0 sc1" : "=v"(callv) : "v"(vzero), "s"(ctr) : "memory");
        const __amdgpu_buffer_rsrc_t rsL = __builtin_amdgcn_make_buffer_rsrc((void*)pL, 0, 16u << 20, 0x00020000);
        const unsigned a0 = (unsigned)kh * 1024u, b0 = (unsigned)(rank * 16 + kh * 8) * (64u * 16384u) + (unsigned)nt * 512u, b1 = b0 + 64u * 16384u;
        STAGE_B(0, 0, b0);
        STAGE_A(0, 0, rsL, a0);
        {
            const int Q = wq, b = Q / 896, rem = Q - b * 896, rg = rem / 7, di = rem - rg * 7;
            const int d = (rank + 1 + di) & 7, kk = (b ^ ((unsigned)(di - 3) >> 31)) & 1, kt = kk * 8 + (b >> 1);
            __builtin_amdgcn_raw_ptr_buffer_load_lds(rsX, (LAS void*)ldx, 16, voffP, (unsigned)(d * 1024 + rg * 8) * 2048u + (unsigned)kt * 128u, 0, 0);
        }
        STAGE_B(0, 1, b0);
        STAGE_A(0, 1, rsL, a0);
        STAGE_B(1, 0, b1);
        STAGE_A(1, 0, rsL, a0 + 128);
        STAGE_B(1, 1, b1);
        int lz = lane; asm volatile("" : "+v"(lz));
        {
            const int u = lz - 8, j = u / 7, i0 = u - j * 7, i = kh ? 6 - i0 : i0;
            const bool loc = lz < 8;
            const int c = loc ? rank : (rank + 1 + i) & 7, kt = kh * 8 + (loc ? lz : j);
            tabA = (loc ? 0u : (unsigned)c * (2u << 20)) + (unsigned)kt * 128u;
            tabB = (unsigned)(c * 16 + kt) * (64u * 16384u) + (unsigned)nt * 512u;
        }
        {
            const int sl = lz < 7 ? lz : 6;
            const int Q = sl * 2048 + wq, b = Q / 896, rem = Q - b * 896, rg = rem / 7, di = rem - rg * 7;
            const int d = (rank + 1 + di) & 7, ipos = 6 - di, kk = (b & 1) ^ (ipos > 3 ? 1 : 0), kt = kk * 8 + (b >> 1);
            unsigned long long base = 0;
#pragma unroll
            for (int i = 0; i < NR; ++i) base = d == i ? (unsigned long long)p.ws[i] : base;
            base = lz >= 8 ? (unsigned long long)p.out : base;
            tabLo = (unsigned)base; tabHi = (unsigned)(base >> 32);
            tabS = (unsigned)(d * 1024 + rg * 8) * 2048u + (unsigned)kt * 128u;
            tabD = (unsigned)WS_STAGE + (unsigned)rank * (2u << 20) + (unsigned)(rg * 8) * 2048u + (unsigned)kt * 128u;
            const int tt = 8 + (b >> 1) * 7 + (kk ? 6 - ipos : ipos), half = (rg >> 4) & 1;
            tabF = (unsigned)WS_FLAG + (unsigned)((((rg >> 5) * 8 + ((rg >> 1) & 7)) * 2 + kk) * 256 + tt * 4 + half * 2 + (rg & 1));
        }
        voffL = lz ? 0x80000000u : 0u;
        vxp_ = (unsigned)(size_t)ldx + (unsigned)lz * 16u;
        asm volatile("ds_write_b32 %0, %1\n\tds_write_b32 %0, %1 offset:512" :: "v"(vxf), "v"(vzero) : "memory");
        asm volatile("" : "+v"(tabLo), "+v"(tabHi), "+v"(tabD), "+v"(tabF), "+v"(tabS), "+v"(tabA), "+v"(tabB), "+v"(voffL), "+v"(vxp_));
#pragma unroll
        for (int a = 0; a < 2; ++a)
#pragma unroll
            for (int b = 0; b < 2; ++b)
                asm volatile("" : "+v"(acc[a][b][0][0]), "+v"(acc[a][b][0][1]), "+v"(acc[a][b][1][0]), "+v"(acc[a][b][1][1]),
                                  "+v"(acc[a][b][2][0]), "+v"(acc[a][b][2][1]), "+v"(acc[a][b][3][0]), "+v"(acc[a][b][3][1]));
        asm volatile("s_memrealtime %0\n\ts_memtime %1\n\ts_waitcnt lgkmcnt(0)" : "=s"(rt1), "=s"(ck1) :: "memory");
        if (wr == 1) BAR;
        asm volatile("s_waitcnt vmcnt(13)" : "+v"(callv) :: "memory");
        BAR; BAR;
        LDB(B0, 0, 0); WAIT_V(10); BAR; WAIT_L(0); BAR;
    }
    const unsigned call = (unsigned)__builtin_amdgcn_readfirstlane(callv) + 1u;
    const unsigned spar = (call & 1u) << 24;
    const unsigned fpar = (call & 1u) << 15;
    const unsigned fval = (call & 0x7fu) | 0x80u, fpat = fval * 0x01010101u;
    sflag += fpar;
    const char* const pS = myws + WS_STAGE + spar;
    unsigned long long ready = 0xffull;
    const unsigned xbad = (unsigned)(size_t)lds + LDS_X + 1536u;
    unsigned* const tmo = (unsigned*)(myws + WS_TMO);

    asm volatile(".subsection 2\n\t.fill 256, 4, 0xbf800000\n\t.subsection 0");
#define POLL_RD(f) asm volatile("ds_read_b32 %0, %1" : "=v"(f) : "v"(vxf) : "memory")
#define NEED(tt, f) NEEDM(3ull << ((tt) & 63), f, 8)
#define NEEDM(msk, f, cnt) do { \
        asm volatile("s_waitcnt lgkmcnt(" #cnt ")" : "+v"(f) :: "memory"); \
        ready |= __builtin_amdgcn_ballot_w64(f == fpat); \
        asm volatile( \
            "s_andn2_b64 s[96:97], %[m], %[rdy]\n\t" \
            "s_cbranch_scc1 .Lnr%=\n" \
            ".Lnb%=:\n\t" \
            ".subsection 1\n" \
            ".Lnr%=:\n\t" \
            "s_memrealtime s[96:97]\n\t" \
            "s_waitcnt lgkmcnt(0)\n\t" \
            "s_mov_b32 s97, m0\n\t" \
            "s_mov_b32 m0, %[m0q]\n\t" \
            "s_nop 0\n" \
            ".Lnl%=:\n\t" \
            "buffer_load_dword %[voffF], %[rsF], %[sflag] offen sc0 sc1 lds\n\t" \
            "global_load_dword v249, %[vz], %[tmo] sc0 sc1\n\t" \
            "s_waitcnt vmcnt(0)\n\t" \
            "ds_read_b32 v248, %[a]\n\t" \
            "s_memrealtime s[98:99]\n\t" \
            "s_waitcnt lgkmcnt(0)\n\t" \
            "v_cmp_eq_u32 vcc, %[fpat], v248\n\t" \
            "s_sub_u32 s98, s98, s96\n\t" \
            "s_nop 0\n\t" \
            "s_or_b64 %[rdy], %[rdy], vcc\n\t" \
            "s_andn2_b64 vcc, %[m], %[rdy]\n\t" \
            "s_cbranch_scc0 .Lnx%=\n\t" \
            "v_cmp_eq_u32 vcc, %[call], v249\n\t" \
            "s_cmp_gt_u32 s98, %[dl]\n\t" \
            "s_cbranch_scc1 .Lnd%=\n\t" \
            "s_cbranch_vccnz .Lnd%=\n\t" \
            "s_sleep 1\n\t" \
            "s_branch .Lnl%=\n" \
            ".Lnd%=:\n\t" \
            "v_mov_b32 v249, %[call]\n\t" \
            "s_mov_b64 %[rdy], -1\n\t" \
            "global_store_dword %[vz], v249, %[tmo] sc0 sc1\n\t" \
            "v_mbcnt_lo_u32_b32 v248, -1, 0\n\t" \
            "v_and_b32 v248, 7, v248\n\t" \
            "v_lshlrev_b32 v248, 11, v248\n\t" \
            "v_add_u32 v248, %[xb], v248\n\t" \
            "ds_write_b32 v248, v249\n\t" \
            "s_waitcnt vmcnt(0) lgkmcnt(0)\n" \
            ".Lnx%=:\n\t" \
            "s_mov_b32 m0, s97\n\t" \
            "s_branch .Lnb%=\n\t" \
            ".subsection 0" \
            : [rdy] "+s"(ready) \
            : [m] "s"((unsigned long long)(msk)), [xb] "s"(xbad), [m0q] "s"(m0poll), [voffF] "v"(voffF), [rsF] "s"(rsF), [sflag] "s"(sflag), [vz] "v"(vzero), [tmo] "s"(tmo), \
              [a] "v"(vxf), [fpat] "s"(fpat), [call] "s"(call), [dl] "s"((unsigned)DEADLINE) \
            : "memory", "scc", "vcc", "s96", "s97", "s98", "s99", "v248", "v249"); } while (0)

#define LDA2(a, b, c) LDA(a, b, c)
#define RL(tab, i) __builtin_amdgcn_readlane(tab, i)
#define HALF1(W1, W2, W3, W4, X1, X2, PRD, ND, PL) do { \
        ST(1 + (t >> 1)); \
        X1; LDA(At, 0, 0); STAGE_A(1, 1, r1, a1); \
        WAIT_V(W1); BAR; WAIT_L(0); MMA(0, 0, At, B0); BAR; SCHED; \
        PRD; LDB(B1, 0, 1); X2; STAGE_B(0, 0, b2); ND; \
        WAIT_V(W2); BAR; WAIT_L(0); MMA(0, 1, At, B1); BAR; SCHED; \
        LDA2(At, 0, 1); STAGE_A(0, 0, r2, a2); \
        WAIT_V(W3); BAR; WAIT_L(0); MMA(1, 0, At, B0); BAR; SCHED; \
        LDB(B0, 1, 0); PL; STAGE_B(0, 1, b2); \
        a1 = RL(tabA, t + 3); b3 = RL(tabB, t + 3); \
        asm volatile("" : "+s"(a1), "+s"(b3)); \
        WAIT_V(W4); BAR; MMA(1, 1, At, B1); SCHED; WAIT_L(0); BAR; SCHED; } while (0)
#define HALF2(W5, W6, W7, W8, X6, X8, RSN) do { \
        LDA(At, 1, 0); STAGE_A(0, 1, r2, a2); \
        WAIT_V(W5); BAR; WAIT_L(0); MMA(0, 0, At, B0); BAR; SCHED; \
        LDB(B1, 1, 1); X6; STAGE_B(1, 0, b3); \
        WAIT_V(W6); BAR; WAIT_L(0); MMA(0, 1, At, B1); BAR; SCHED; \
        LDA2(At, 1, 1); STAGE_A(1, 0, r2, a1); \
        WAIT_V(W7); BAR; WAIT_L(0); MMA(1, 0, At, B0); BAR; SCHED; \
        LDB(B0, 0, 0); X8; STAGE_B(1, 1, b3); \
        t += 2; RSN; \
        a2 = RL(tabA, t + 2); b2 = RL(tabB, t + 2); \
        asm volatile("" : "+s"(a2), "+s"(b2), "+s"(t)); \
        WAIT_V(W8); BAR; MMA(1, 1, At, B1); SCHED; WAIT_L(0); BAR; SCHED; } while (0)
#define RS_NEXT do { r1 = r2; r2 = RSA(t + 2); asm volatile("" : "+s"(r1), "+s"(r2)); } while (0)
#define TRIP(W1, W2, W3, W4, W5, W6, W7, W8, X1, X2, X6, X8) \
        do { HALF1(W1, W2, W3, W4, X1, X2, POLL_RD(fl0), NEED(t + 2, fl0), POLL(0)); HALF2(W5, W6, W7, W8, X6, X8, RS_NEXT); } while (0)

    unsigned a1 = (unsigned)kh * 1024u + 128u, a2 = a1 + 128u, b2 = (unsigned)(rank * 16 + kh * 8 + 2) * (64u * 16384u) + (unsigned)nt * 512u, b3;
    __amdgpu_buffer_rsrc_t r1 = RSA(0), r2 = r1;
    int t = 0;
    int tl = 58;
#define NOW(t_) asm volatile("s_memrealtime %0\n\ts_waitcnt lgkmcnt(0)" : "=s"(t_) :: "memory")
    {
        int pn = 0; const int lim = 2 * cls;
        while (t < lim) TRIP(10, 10, 10, 11, 11, 11, 11, 11, , , , );
        for (; pn < 7; ++pn) {
            {
                const unsigned long long pb = (unsigned long long)(unsigned)__builtin_amdgcn_readlane(tabLo, pn) |
                                              ((unsigned long long)(unsigned)__builtin_amdgcn_readlane(tabHi, pn) << 32);
                const __amdgpu_buffer_rsrc_t rp = __builtin_amdgcn_make_buffer_rsrc((void*)pb, 0, (unsigned)WS_XBUF, 0x00020000);
                const unsigned sD = __builtin_amdgcn_readlane(tabD, pn) + spar, sF = __builtin_amdgcn_readlane(tabF, pn) + fpar;
                const unsigned sS = __builtin_amdgcn_readlane(tabS, pn + 1);
                u32x4 pd;
                TRIP(10, 11, 11, 12, 12, 13, 12, 13,
                     asm volatile("ds_read_b128 %0, %1" : "=v"(pd) : "v"(vxp_) : "memory"),
                     __builtin_amdgcn_raw_buffer_store_b128(pd, rp, voffP, sD, 17),
                     __builtin_amdgcn_raw_ptr_buffer_load_lds(rsX, (LAS void*)ldx, 16, voffP, sS, 0, 0),
                     __builtin_amdgcn_raw_buffer_store_b8((unsigned char)fval, rp, voffL, sF, 17));
            }
            TRIP(12, 12, 11, 12, 11, 11, 11, 11, , , , );
            TRIP(10, 10, 10, 11, 11, 11, 11, 11, , , , );
        }
        {
            const int lz = (int)__builtin_amdgcn_mbcnt_hi(~0u, __builtin_amdgcn_mbcnt_lo(~0u, 0u));
            const unsigned c = (unsigned)(rank + 1 + (kh ? 6 - lz : lz)) & 7u, kt = (unsigned)(15 - kh * 8);
            tabA = lz < 4 ? c * (2u << 20) + kt * 128u : tabA;
            tabB = lz < 4 ? (c * 16u + kt) * (64u * 16384u) + (unsigned)nt * 512u : tabB;
            asm volatile("" : "+v"(tabA), "+v"(tabB));
            ready &= ~0x5full;
            if (wid == 0) {
                unsigned long long rt2, ck2;
                asm volatile("s_memrealtime %0\n\ts_memtime %1\n\ts_waitcnt lgkmcnt(0)" : "=s"(rt2), "=s"(ck2) :: "memory");
                const float tau = TILE_CYC * (float)((unsigned)rt2 - (unsigned)rt1) * __builtin_amdgcn_rcpf((float)((unsigned)ck2 - (unsigned)ck1));
                const unsigned fin = (unsigned)rt2 + (unsigned)(tau * (float)(63 - t));
                const unsigned ti = (unsigned)(tau + 0.5f);
                typedef unsigned u32x2 __attribute__((ext_vector_type(2)));
                __builtin_amdgcn_raw_buffer_store_b64((u32x2){fpat, (fin << 16) | ((ti < 255u ? ti : 255u) << 8) | fval}, rsF, voffL, fpar + spw + (unsigned)kh * 8u, 17);
            }
        }
        unsigned long long dm = ~0x50ull;
#pragma nounroll
        for (int st = 0; st < 2; ++st) {
            while (t + 1 < tl && (ready | dm) != ~0ull) TRIP(10, 10, 10, 11, 11, 11, 11, 11, , , , );
            if (st) break;
            POLL_RD(fl0); asm volatile("s_waitcnt lgkmcnt(0)" : "+v"(fl0) :: "memory");
            unsigned wa = RL(fl0, 5), wb = RL(fl0, 7);
            if (__builtin_expect(((wa ^ fval) | (wb ^ fval)) & 0xffu, 0)) {
                unsigned long long t0_, t1_; NOW(t0_);
                for (;;) { unsigned tv_; u32x4 q_;
                    asm volatile("buffer_load_dwordx4 %0, %2, %3, %4 offen sc0 sc1\n\tglobal_load_dword %1, %2, %5 sc0 sc1\n\ts_waitcnt vmcnt(0)"
                                 : "=&v"(q_), "=&v"(tv_) : "v"(vzero), "s"(rsF), "s"(fpar + spw), "s"(tmo) : "memory");
                    wa = (unsigned)__builtin_amdgcn_readfirstlane(q_.y); wb = (unsigned)__builtin_amdgcn_readfirstlane(q_.w);
                    if ((((wa ^ fval) | (wb ^ fval)) & 0xffu) == 0u) break;
                    NOW(t1_);
                    if ((unsigned)__builtin_amdgcn_readfirstlane(tv_) == call || t1_ - t0_ > DEADLINE) {
                        const unsigned xa = xbad + (__builtin_amdgcn_mbcnt_lo(~0u, 0u) & 7u) * 2048u;
                        asm volatile("global_store_dword %0, %1, %2 sc0 sc1\n\tds_write_b32 %3, %1\n\ts_waitcnt vmcnt(0) lgkmcnt(0)" :: "v"(vzero), "v"(call), "s"(tmo), "v"(xa) : "memory");
                        wa = wb = 0u; ready = ~0ull; break; }
                    __builtin_amdgcn_s_sleep(1); }
            }
            const float d = (float)((int)(wb - (wa & 0xffff0000u)) >> 16), ts = (float)(((wa >> 8) & 0xffu) + ((wb >> 8) & 0xffu) + 1u);
            const int shv = (int)__builtin_rintf(__builtin_amdgcn_fmed3f(d * __builtin_amdgcn_rcpf(ts), -3.f, 3.f));
            int sh; asm volatile("s_nop 1\n\tv_readfirstlane_b32 %0, %1\n\ts_nop 3" : "=s"(sh) : "v"(shv));
            tl = 61 + (kh ? -sh : sh); dm = 0ull;
        }
        if (ready != ~0ull) { POLL_RD(fl0); NEEDM(~0ull, fl0, 0); }
    }
    unsigned par = 0u;
    while (t + 1 < tl) { HALF1(10, 10, 10, 10, , , , , ); HALF2(10, 10, 10, 10, , , ); }
    if (t < tl) {
        HALF1(10, 10, 10, 10, , , , , );
        const unsigned a_ = a1; a1 = a2; a2 = a_; b2 = b3; t += 1; par = 65536u;
    }
    {
        asm volatile("" : "+s"(par));
        va[1][0] = va[0][0] + (65536u - par); va[1][1] = va[0][1] + (65536u - par); vb[1][0] = vb[0][0] + (65536u - par); vb[1][1] = vb[0][1] + (65536u - par);
        va[0][0] += par; va[0][1] += par; vb[0][0] += par; vb[0][1] += par;
        lb[0] = lds + ldsw + par; lb[1] = lds + ldsw + (65536u - par);
        int tz = threadIdx.x; asm volatile("" : "+v"(tz));
        const unsigned ln = (unsigned)tz & 63u;
        const __amdgpu_buffer_rsrc_t xr = __builtin_amdgcn_make_buffer_rsrc(myws + WS_XBUF, 0, 256u * 65536u, 0x00020000);
        unsigned voffX = (unsigned)(bid * 8 + wid) * 8192u + ln * 16u;
        unsigned badv, badm, tagm = 0xfffe0000u;
        const unsigned tag = call & 1u, tagr = 0x10000u - (tag << 16);
#define SEND(bj, m) do { u32x4 o_; \
        const float t_ = __builtin_bit_cast(float, ((__builtin_bit_cast(unsigned, acc[0][bj][m][0][0]) + tagr) & tagm) | tago); \
        o_.x = cvt_pk_bf16(t_, acc[0][bj][m][0][1]); o_.y = cvt_pk_bf16(acc[0][bj][m][0][2], acc[0][bj][m][0][3]); \
        o_.z = cvt_pk_bf16(acc[0][bj][m][1][0], acc[0][bj][m][1][1]); o_.w = cvt_pk_bf16(acc[0][bj][m][1][2], acc[0][bj][m][1][3]); \
        __builtin_amdgcn_raw_buffer_store_b128(o_, xr, voffX + ((bj) * 4 + (m)) * 1024, 0, 16); } while (0)
#define PH_A(ai, bj, Ax, Bx) BAR; WAIT_L(0); MMAT(ai, bj, Ax, Bx); BAR; SCHED
#define PH_B(ai, bj, Ax, Bx) BAR; WAIT_L(8); MMAT(ai, bj, Ax, Bx); SCHED; WAIT_L(0); BAR; SCHED
#define PH_C(ai, bj, Ax, Bx) BAR; MMAT(ai, bj, Ax, Bx); SCHED; WAIT_L(0); BAR; SCHED
        u32x4 pv[2][4];
        bf16x8 A2[4][2];
#define PVLD(bj) do { _Pragma("unroll") for (int m = 0; m < 4; ++m) \
        asm volatile("buffer_load_dwordx4 %0, %1, %2, %3 offen offset:%4 sc1" : "=v"(pv[bj][m]) : "v"(voffY), "s"(xr), "s"((bj) * 4096u), "i"(m * 1024) : "memory"); } while (0)
#define PVWAIT(bj, n) asm volatile("s_waitcnt vmcnt(" #n ")" : "+v"(pv[bj][0]), "+v"(pv[bj][1]), "+v"(pv[bj][2]), "+v"(pv[bj][3]) :: "memory")
        SW(1); SR(62);
        LDA(At, 0, 0); STAGE_A(1, 1, r2, a1);
        WAIT_V(6); PH_A(0, 0, At, B0); SW(2);
        LDB(B1, 0, 1); LDB(B0, 1, 0);
        a1 = RL(tabA, t + 3); b3 = RL(tabB, t + 3);
        asm volatile("" : "+s"(a1), "+s"(b3));
        WAIT_V(4); PH_B(0, 1, At, B1); SW(3);
        LDA(At, 1, 0); asm volatile("ds_read_b32 %0, %1 offset:512" : "=v"(badv) : "v"(vxf) : "memory");
        STAGE_BX(0, 0, 0, b2);
        WAIT_V(4); PH_A(0, 0, At, B0); SW(4);
        LDB(B1, 1, 1); STAGE_AX(1, 2, 1, r2, a2); SCHED;
        asm volatile("" : "+v"(badv), "+v"(tagm));
        badm = __builtin_amdgcn_readfirstlane(badv) ? 0x7fc07fc0u : 0u;
        unsigned tago = (tag << 16) | (badm & 0xffff0000u);
        asm volatile("" : "+s"(badm), "+s"(tago));
        SEND(0, 0); SEND(0, 1);
        WAIT_V(6); PH_A(0, 1, At, B1); SW(5);
        LDA(At, 1, 1); STAGE_BX(1, 0, 0, b3); SCHED; SEND(0, 2); SEND(0, 3);
        PH_A(1, 0, At, B0); SW(6);
        LDB(B0, 0, 0); STAGE_AX(1, 3, 1, r2, a1); SCHED; SEND(1, 0);
        PH_C(1, 1, At, B1); SW(7);
        LDA(At, 0, 1); STAGE_BX(1, 1, 1, b2); SCHED; SEND(1, 1);
        WAIT_V(14); PH_A(1, 0, At, B0); SW(8);
        LDB(B1, 0, 1); LDBX(B0, 0, 0); STAGE_BX(0, 2, 1, b3); SCHED; SEND(1, 2);
        WAIT_V(11); PH_B(1, 1, At, B1); SW(9);
        LDA(At, 1, 2); LDBX(B1, 1, 0); SEND(1, 3);
        const unsigned voffY = voffX ^ (7u * 65536u);
        PVLD(0);
        WAIT_V(12); PH_B(1, 0, At, B0); SW(10);
        LDA(A2, 1, 3);
        const unsigned one = ((ln >> 4) == ((ln >> 2) & 3u)) ? ((ln & 1u) ? 0x3f800000u : 0x00003f80u) : 0u;
        unsigned sd0 = (ln & 2u) ? 0u : one, sd1 = (ln & 2u) ? one : 0u;
        char* orow = (char*)((size_t)(unsigned)RL(tabLo, 8) | ((size_t)(unsigned)RL(tabHi, 8) << 32)) +
                     (size_t)((unsigned)(mt * 256 + kh * 128 + wr * 64) + (ln & 15u)) * 16384u + (size_t)((unsigned)(nt * 256 + wc * 32) + ((ln >> 4) & 1u) * 16u + (ln >> 5) * 8u) * 2u;
        asm volatile("" : "+v"(sd0), "+v"(sd1), "+v"(orow));
        WAIT_V(6); BAR; WAIT_L(0); MMAT(1, 0, A2, B1); SCHED;
        if (wr == 0) BAR;
        SW(11);
#define PMISS(bj) __builtin_amdgcn_ballot_w64((((pv[bj][0].x ^ tag) | (pv[bj][1].x ^ tag) | (pv[bj][2].x ^ tag) | (pv[bj][3].x ^ tag)) & 1u) != 0u)
#define RCNT(bj)
#define ADDP(bj) do { \
        if (__builtin_expect(PMISS(bj) != 0ull, 0)) { \
            unsigned long long t0_, t1_; NOW(t0_); \
            for (;;) { unsigned tv_; PVLD(bj); RCNT(bj); \
                asm volatile("global_load_dword %0, %1, %2 sc0 sc1\n\ts_waitcnt vmcnt(0)" : "=v"(tv_) : "v"(vzero), "s"(tmo) : "memory"); \
                PVWAIT(bj, 0); \
                if (PMISS(bj) == 0ull) break; \
                NOW(t1_); \
                if ((unsigned)__builtin_amdgcn_readfirstlane(tv_) == call || t1_ - t0_ > DEADLINE) { \
                    asm volatile("global_store_dword %0, %1, %2 sc0 sc1\n\ts_waitcnt vmcnt(0)" :: "v"(vzero), "v"(call), "s"(tmo) : "memory"); \
                    badm = 0x7fc07fc0u; break; } \
                __builtin_amdgcn_s_sleep(1); } } \
        const bf16x8 s0_ = __builtin_bit_cast(bf16x8, (u32x4){sd0, sd1, badm, 0u}), s1_ = __builtin_bit_cast(bf16x8, (u32x4){badm, 0u, sd0, sd1}); \
        _Pragma("unroll") for (int m = 0; m < 4; ++m) { const bf16x8 q_ = __builtin_bit_cast(bf16x8, pv[bj][m]); \
            acc[1][bj][m][0] = __builtin_amdgcn_mfma_f32_16x16x32_bf16(s0_, q_, acc[1][bj][m][0], 0, 0, 0); \
            acc[1][bj][m][1] = __builtin_amdgcn_mfma_f32_16x16x32_bf16(s1_, q_, acc[1][bj][m][1], 0, 0, 0); } } while (0)
        f32x2 gc1 = {GELU_C1, GELU_C1}, gc3 = {GELU_C3, GELU_C3};
        asm volatile("" : "+s"(gc1), "+s"(gc3));
#define OSTORE(a, o) (*(__attribute__((address_space(1))) u32x4*)(a) = (o))
#define G(bj, m) do { const f32x4 a0 = acc[1][bj][m][0], a1_ = acc[1][bj][m][1]; \
        const unsigned x0 = gelu2((f32x2){a0[0], a0[1]}, gc1, gc3), x1 = gelu2((f32x2){a0[2], a0[3]}, gc1, gc3); \
        const unsigned y0 = gelu2((f32x2){a1_[0], a1_[1]}, gc1, gc3), y1 = gelu2((f32x2){a1_[2], a1_[3]}, gc1, gc3); \
        const auto r0 = __builtin_amdgcn_permlane16_swap(x0, y0, false, false); \
        const auto r1_ = __builtin_amdgcn_permlane16_swap(x1, y1, false, false); \
        u32x4 o; o.x = r0[0]; o.y = r1_[0]; o.z = r0[1]; o.w = r1_[1]; \
        OSTORE(orow + (size_t)(m) * 16 * 16384 + (bj) * 256, o); } while (0)
        LDBX(B0, 1, 1); LDBX(B1, 0, 2);
#define X_EORD 1
        WAIT_L(8); MMAT(1, 1, At, B0); SCHED; SW(12);
        PVWAIT(0, 0); SW(13); ADDP(0); SCHED; PVLD(1); SW(14);
        G(0, 0); G(0, 1); SCHED; WAIT_L(0); MMAT(1, 1, A2, B1); SCHED; SW(15); SR(61);
        G(0, 2); G(0, 3); SCHED; SW(16);
        PVWAIT(1, 4);
        SW(17); ADDP(1); SCHED; SW(18);
        G(1, 0); G(1, 1); G(1, 2); G(1, 3); SW(19);
        if (ln == 0u) __hip_atomic_store((unsigned*)ctr, call, __ATOMIC_RELAXED, __HIP_MEMORY_SCOPE_AGENT);
    }
}
}

extern "C" void kernel_launch(int rank, int nranks, ncclComm_t comm,
                              hipStream_t stream,
                              void* const* d_in, const int* in_sizes, int n_in,
                              void* d_out, int out_size,
                              const int* in_dtypes, int out_dtype,
                              void* const* d_out_all,
                              void* const* d_ws_all, size_t ws_bytes) {
    (void)comm; (void)in_sizes; (void)n_in; (void)out_size; (void)in_dtypes; (void)out_dtype; (void)d_out_all;
    if (nranks != a2a::NR || (unsigned)rank >= (unsigned)a2a::NR || ws_bytes < a2a::WS_END) {
        fprintf(stderr, "a2a_gemm: unexpected configuration (nranks %d, ws %zu)\n", nranks, ws_bytes);
        return;
    }
    HIP_CHECK(hipFuncSetAttribute((const void*)a2a::a2a_gemm_kernel, hipFuncAttributeMaxDynamicSharedMemorySize, a2a::LDS_BYTES));
    a2a::Params p;
    p.x = (const char*)d_in[0];
    p.w = (const char*)d_in[1];
    p.out = (char*)d_out;
    for (int i = 0; i < a2a::NR; ++i) p.ws[i] = (char*)d_ws_all[i];
    p.myws = (char*)d_ws_all[rank];
    p.rank = rank;
    hipLaunchKernelGGL(a2a::a2a_gemm_kernel, dim3(256), dim3(512), a2a::LDS_BYTES, stream, p);
    HIP_CHECK(hipGetLastError());
}
```

```cpp
#include <hip/hip_runtime.h>
#include <rccl/rccl.h>
#include <cstdio>

#define HIP_CHECK(expr) do { hipError_t _e = (expr); if (_e != hipSuccess) { \
    fprintf(stderr, "HIP error %s at %s:%d\n", hipGetErrorString(_e), __FILE__, __LINE__); \
    return; } } while (0)

namespace a2a {
#define LAS __attribute__((address_space(3)))
typedef short bf16x8 __attribute__((ext_vector_type(8)));
typedef float f32x4 __attribute__((ext_vector_type(4)));
typedef unsigned u32x4 __attribute__((ext_vector_type(4)));
typedef float f32x2 __attribute__((ext_vector_type(2)));
typedef __bf16 bf16x2 __attribute__((ext_vector_type(2)));

constexpr int NR = 8;
constexpr int HTB = 16384;
constexpr int LDS_BYTES = 10 * HTB;
constexpr size_t WS_CTR = 0;
constexpr size_t WS_FLAG = 65536;
constexpr size_t WS_STAGE = 2ull << 20;
constexpr size_t WS_XBUF = 34ull << 20;
constexpr size_t WS_END = 50ull << 20;
constexpr unsigned long long DEADLINE = 50000000ull;
#define PUSH_T 24
constexpr int PUSH_TRIPS = PUSH_T;

#define FIRST(Q) ((Q) * PUSH_TRIPS < 14336)
constexpr int PF = 0;
struct Params {
    char* out;
    char* ws[NR];
};

__device__ __forceinline__ unsigned cvt_pk_bf16(float lo, float hi) {
    return __builtin_bit_cast(unsigned, __builtin_convertvector((f32x2){lo, hi}, bf16x2));
}
constexpr float GELU_C1 = -2.0f * 0.7978845608028654f * 1.4426950408889634f, GELU_C3 = GELU_C1 * 0.044715f;

#define ST(i)
#define SW(i)
#define SR(i)
#define RCNT(bj)
#define BAR __builtin_amdgcn_s_barrier()
#define SCHED __builtin_amdgcn_sched_barrier(0)
#define RL(tab, i) __builtin_amdgcn_readlane(tab, i)

__global__ __launch_bounds__(256, 1) void a2a_gemm_kernel(const char* px, const char* pw, char* const myws, const int rank, Params p) {
    extern __shared__ __attribute__((aligned(16))) unsigned char shm[];
    LAS unsigned char* lds = (LAS unsigned char*)shm;
    const int tid = threadIdx.x, wid = __builtin_amdgcn_readfirstlane(tid >> 6), lane = tid & 63, wr = wid >> 1, wc = wid & 1, fr = lane & 15, fq = lane >> 4;
    const int bid = blockIdx.x, xcd = bid & 7, idx = bid >> 3;
    const int kh = xcd & 1, mt = idx & 3, nt = (xcd >> 1) + (idx >> 2) * 4;
    ST(0);
#define SWP (mt & 1)
    const int swp = SWP;
    const unsigned arow0 = (unsigned)(mt * 256 + (1 - kh) * 128) * 2048u;
    unsigned dl = kh ? 128u * 2048u : 0u - 128u * 2048u;
    unsigned voffA[4], voffB[4];
#pragma unroll
    for (int i = 0; i < 4; ++i) {
        const unsigned r = (unsigned)(wid * 32 + i * 8 + (lane >> 3));
        voffA[i] = r * 2048u + (unsigned)((lane & 7) ^ ((i & 1) * 4 + (lane >> 4))) * 16u - (unsigned)i * 1024u;
        const unsigned kb = (unsigned)(wid * 16 + i * 4 + (lane >> 4)), cb = lane & 15, fb = (unsigned)((lane >> 4) | ((i >> 1) << 2));
        voffB[i] = kb * 16384u + (((cb >> 1) ^ fb) * 32u) + (cb & 1) * 16u - (unsigned)i * 1024u;
    }
    const unsigned ldsw = (unsigned)(size_t)lds + (unsigned)wid * 4096u;
    LAS unsigned char* const lb = lds + wid * 4096;
    const unsigned voffP = (unsigned)(lane >> 3) * 2048u + (unsigned)(lane & 7) * 16u;
    const unsigned vzero = 0u;

    const __amdgpu_buffer_rsrc_t rsW = __builtin_amdgcn_make_buffer_rsrc((void*)pw, 0, 128u << 20, 0x00020000);
    const __amdgpu_buffer_rsrc_t rsX = __builtin_amdgcn_make_buffer_rsrc((void*)px, 0, 16u << 20, 0x00020000);
    const __amdgpu_buffer_rsrc_t rsF = __builtin_amdgcn_make_buffer_rsrc((void*)(myws + WS_FLAG), 0, 65536u, 0x00020000);
    const char* const pL = px + (size_t)rank * (2u << 20);
    const __amdgpu_buffer_rsrc_t rsL = __builtin_amdgcn_make_buffer_rsrc((void*)pL, 0, 16u << 20, 0x00020000);
    char* const ctr = myws + WS_CTR + (size_t)(bid * 4 + wid) * 16;
#define SA1(b, h, soff, i) __builtin_amdgcn_raw_ptr_buffer_load_lds(rsL, (LAS void*)(lb + ((b) * 2 + (h)) * HTB), 16, voffA[i], (soff), (i) * 1024, 17)
#define SB1(b, h, soff, i) __builtin_amdgcn_raw_ptr_buffer_load_lds(rsW, (LAS void*)(lb + (4 + (b) * 2 + (h)) * HTB - (h) * 256), 16, voffB[i], (soff), (h) * 256 + (i) * 1024, 0)
#define STAGE_A(b, h, soff) do { SA1(b, h, soff, 0); SA1(b, h, soff, 1); SA1(b, h, soff, 2); SA1(b, h, soff, 3); } while (0)
#define STAGE_B(b, h, soff) do { SB1(b, h, soff, 0); SB1(b, h, soff, 1); SB1(b, h, soff, 2); SB1(b, h, soff, 3); } while (0)

    const int wq = (idx * 4 + wid) * 8 + xcd;
    unsigned callv = 0u;
    u32x4 pd;
    unsigned long long pf = 0ull;
    unsigned tabA, tabB, tabLo, tabHi, tabD, tabF, voffL, voffF, voffQ, voffR, pr = 0u;
    int khs = kh; asm volatile("" : "+s"(khs));
    unsigned va[2], vb[4];
    unsigned vseq = 0u, vchk = 0u, vsw, vsr;
    f32x4 acc[2][2][4][4];

    {
        asm volatile("global_load_dword %0, %1, %2 sc0 sc1" : "=v"(callv) : "v"(vzero), "s"(ctr) : "memory");
        const unsigned a0 = (unsigned)kh * 1024u + arow0, b0 = (unsigned)(rank * 16 + kh * 8) * (64u * 16384u) + (unsigned)nt * 512u, b1 = b0 + 64u * 16384u;
        STAGE_A(0, 0, a0);
        STAGE_B(0, 0, b0);
        {
            const int Q = wq, rg = Q / 7, di = Q - rg * 7;
            const int d = (rank + 1 + di) & 7, kk = di < 3 ? 1 : 0;
            const unsigned so = FIRST(Q) ? (unsigned)(d * 1024 + rg * 8) * 2048u + (unsigned)kk * 1024u : 0x80000000u;
            asm volatile("buffer_load_dwordx4 %0, %1, %2, %3 offen" : "=v"(pd) : "v"(voffP), "s"(rsX), "s"(so) : "memory");
        }
        STAGE_B(0, 1, b0);
        STAGE_A(0, 1, a0 + dl);
#define T1POS 2
#define TILE1A do { STAGE_A(1, 0, a0 + 128); STAGE_B(1, 1, b1); \
        asm volatile("buffer_load_dwordx2 %0, %1, %2, %3 offen" : "+v"(pf) : "v"(vzero), "s"(rsF), "s"(0x80000000u) : "memory"); } while (0)
#define TILE1B do { STAGE_B(1, 0, b1); STAGE_A(1, 1, a0 + 128 + dl); } while (0)
        if (T1POS < 2) TILE1A;
        if (T1POS < 1) TILE1B;
        ST(40);
        voffR = (unsigned)((((idx & 7) * 4 + wid) * 2 + (lane >> 5)) * 16384 + ((xcd >> 1) + ((lane >> 2) & 7) * 4) * 512 + (lane & 3) * 128);
#pragma unroll
        for (int i = 2; i < PF; i += 2)
            asm volatile("buffer_load_dword %0, %1, %2, %3 offen" : "+v"(pr) : "v"(voffR), "s"(rsW), "s"((unsigned)(rank * 16 + kh * 8 + i + (idx >> 4)) * (64u * 16384u)) : "memory");
        int lz = lane; asm volatile("" : "+v"(lz));
        {
            const int fr_ = lz & 15, fq_ = lz >> 4, q4 = (lz >> 2) & 3, p4 = lz & 3, fl = q4 | ((fq_ & 1) << 2);
            const int aoff0 = (wr * 64 + fr_) * 128 + ((fq_ ^ (fr_ >> 1)) * 16);
            const int boff0 = (8 * fq_ + q4) * 256 + (((wc * 4) ^ fl) * 32) + 8 * p4;
#pragma unroll
            for (int k = 0; k < 2; ++k) va[k] = (unsigned)(size_t)lds + (unsigned)(aoff0 ^ (k * 64));
#pragma unroll
            for (int n = 0; n < 4; ++n) vb[n] = (unsigned)(size_t)lds + (unsigned)(boff0 ^ (n * 32)) + 65536u;
        }
        {
            const int lt = lz ^ swp;
            const int u = lt - 8, j = u / 7, i0 = u - j * 7, i = kh ? 6 - i0 : i0;
            const bool loc = lz >= 2 && lz < 8;
            const int c = loc ? rank : (rank + (lz < 2 ? 2 - lt : 1 + i)) & 7, kt = lz < 2 ? 15 : kh * 8 + (lz < 8 ? lt : j);
            tabA = loc ? (unsigned)kt * 128u + arow0 : (unsigned)c * (2u << 20) + (unsigned)kt * 131072u + (arow0 >> 4);
            tabB = (unsigned)(c * 16 + kt) * (64u * 16384u) + (unsigned)nt * 512u;
        }
        {
            const int u = ((lz + 1) >> 1) - 1;
            const int s = u < 0 ? 14 : (u * 14336 - wq * PUSH_TRIPS + 1024 * PUSH_TRIPS - 1) / (1024 * PUSH_TRIPS);
            const int Q = s * 1024 + wq, b = Q / 896, rem = Q - b * 896, rg = rem / 7, di = rem - rg * 7;
            const bool act = s < 14 && Q * PUSH_TRIPS < (u + 1) * 14336;
            const int d = (rank + 1 + di) & 7, ipos = 6 - di, kk = (b & 1) ^ (ipos > 3 ? 1 : 0), kt = kk * 8 + (b >> 1);
            unsigned long long base = 0;
#pragma unroll
            for (int i = 0; i < NR; ++i) base = d == i ? (unsigned long long)p.ws[i] : base;
            tabLo = (unsigned)base; tabHi = (unsigned)(base >> 32);
            const unsigned src = (unsigned)(d * 1024 + rg * 8) * 2048u + (unsigned)kt * 128u;
            const unsigned dst = (unsigned)WS_STAGE + (unsigned)rank * (2u << 20) + (unsigned)kt * 131072u + (unsigned)rg * 1024u;
            tabD = act ? ((lz & 1) ? src : dst) : 0x80000000u;
            const int tt = 8 + (b >> 1) * 7 + (kk ? 6 - ipos : ipos), half = (rg >> 4) & 1;
            const bool shared = kk && tt >= 62;
            tabF = act ? (unsigned)WS_FLAG + (unsigned)((((rg >> 5) * 4 + ((rg >> 2) & 3)) * 2 + (shared ? half : kk)) * 512 + (shared ? 1 : tt) * 8 + (shared ? tt & 1 : half) * 4 + (rg & 3)) : 0x80000000u;
        }
        voffL = lz ? 0x80000000u : 0u;
        voffF = lz == 2 ? 16384u : (unsigned)((mt * 4 + wid) * 2 + kh) * 512u + ((kh && lz >= 62) ? 8u : (unsigned)lz * 8u);
        voffQ = (unsigned)lz * 16u;
        asm volatile("" : "+v"(tabLo), "+v"(tabHi), "+v"(tabD), "+v"(tabF), "+v"(tabA), "+v"(tabB), "+v"(voffL), "+v"(voffF), "+v"(voffQ));
        asm volatile("" : "+v"(va[0]), "+v"(va[1]), "+v"(vb[0]), "+v"(vb[1]), "+v"(vb[2]), "+v"(vb[3]));
        SCHED; if (T1POS == 2) TILE1A; SCHED;
#pragma unroll
        for (int a = 0; a < 2; ++a)
#pragma unroll
            for (int b = 0; b < 2; ++b)
#pragma unroll
                for (int m = 0; m < 4; ++m)
#pragma unroll
                    for (int n = 0; n < 4; ++n) if (a || !b) { acc[a][b][m][n] = (f32x4){0.f, 0.f, 0.f, 0.f}; asm volatile("" : "+v"(acc[a][b][m][n])); }
        asm volatile(".set .Lzi, 0\n\t.rept 64\n\tv_accvgpr_write_b32 a[.Lzi], 0\n\t.set .Lzi, .Lzi + 1\n\t.endr" ::: "a255");
        SCHED; if (T1POS == 3) TILE1A; if (T1POS) TILE1B; SCHED;
        ST(41);
        asm volatile("s_waitcnt vmcnt(25)" : "+v"(callv), "+v"(pd) :: "memory");
        ST(42);
        BAR;
        ST(43);
#pragma unroll
        for (int j = 0; j < 8; ++j) asm volatile("ds_read_b128 a[%c1:%c2], %0 offset:%c3" :: "v"(va[j & 1]), "i"(64 + j * 4), "i"(64 + j * 4 + 3), "i"((j >> 1) * 2048) : "memory");
#pragma unroll
        for (int j = 0; j < 16; ++j) asm volatile("ds_read_b64_tr_b16 a[%c1:%c2], %0 offset:%c3" :: "v"(vb[j >> 2]), "i"(128 + j * 2), "i"(128 + j * 2 + 1), "i"(((j >> 1) & 1) * 8192 + (j & 1) * 1024) : "memory");
    }
    const unsigned call = (unsigned)__builtin_amdgcn_readfirstlane(callv) + 1u;
    const unsigned spar = (call & 1u) << 24;
    const unsigned fpar = (call & 1u) << 15;
    const unsigned fval = (call & 0x7fu) | 0x80u, fpat = fval * 0x01010101u;
    const unsigned long long fpat2 = (unsigned long long)fpat * 0x100000001ull;
    const unsigned sflag = fpar, stmo = fpar + 16384u;
    const unsigned long long pLu = (unsigned long long)pL, pSu = (unsigned long long)(myws + WS_STAGE + spar);
    unsigned long long rdy = 0xf9ull;
    unsigned vfv = fpat;
    tabD += (lane & 1) ? 0u : spar; tabF += fpar;
    unsigned a2 = (unsigned)kh * 1024u + arow0 + 256u + (unsigned)swp * 128u, b2 = (unsigned)(rank * 16 + kh * 8 + 2 + swp) * (64u * 16384u) + (unsigned)nt * 512u, a3 = 0u, b3 = 0u;
    u32x4 rsa = {(unsigned)pLu, (unsigned)(pLu >> 32), 16u << 20, 0x00020000u};
    u32x4 rp = {(unsigned)RL(tabLo, 2), (unsigned)RL(tabHi, 2), (unsigned)WS_XBUF, 0x00020000u};
    unsigned sD = RL(tabD, 2), sF = RL(tabF, 2), sS = 0u, tmp = 0u, spo = 0u, sR = 0u, tp = (unsigned)(PF + (idx >> 4));
    unsigned long long msk = 0ull;
    int t = 0;
#define TPF 4
    const int tend = 60 - 2 * khs;
    const int tli = (tend + 2 + (idx >> 3)) & 63, tl2 = tend - TPF - 2 * mt;
    const unsigned dl2 = khs ? 16384u : 0u - 16384u, voffS = (unsigned)wid * 4096u + (voffQ & 0x380u);
    int tl = 6;
    asm volatile("" : "+v"(vfv), "+v"(tabD), "+v"(tabF), "+s"(sD), "+s"(sF), "+{s[92:95]}"(rp), "+{s[88:91]}"(rsa));
    asm volatile(".subsection 2\n\t.fill 256, 4, 0xbf800000\n\t.subsection 0");
    asm volatile("s_waitcnt vmcnt(17) lgkmcnt(0)\n\ts_barrier" ::: "memory");
    SCHED;
    {
        do {
            ST(1 + (t >> 1));
            asm volatile(
                "v_mfma_f32_16x16x32_bf16 %[c0000], a[128:131], a[64:67], %[c0000]\n\t"
                "ds_read_b64_tr_b16 a[160:161], %[vb0] offset:16384\n\t"
                "v_mfma_f32_16x16x32_bf16 %[c0000], a[132:135], a[68:71], %[c0000]\n\t"
                "ds_read_b64_tr_b16 a[162:163], %[vb0] offset:17408\n\t"
                "buffer_store_dwordx4 %[pd], %[voffQ], s[92:95], %[sD] offen sc0 sc1\n\t"
                "v_mfma_f32_16x16x32_bf16 %[c0010], a[128:131], a[72:75], %[c0010]\n\t"
                "ds_read_b64_tr_b16 a[164:165], %[vb0] offset:24576\n\t"
                "v_mfma_f32_16x16x32_bf16 %[c0010], a[132:135], a[76:79], %[c0010]\n\t"
                "ds_read_b64_tr_b16 a[166:167], %[vb0] offset:25600\n\t"
                "s_cmp_eq_u32 %[t], %[tl]\n\t"
                "v_mfma_f32_16x16x32_bf16 %[c0020], a[128:131], a[80:83], %[c0020]\n\t"
                "ds_read_b64_tr_b16 a[168:169], %[vb1] offset:16384\n\t"
                "s_cbranch_scc1 .Lsr%=\n\t"
                ".Lsb%=:\n\t"
                ".subsection 1\n\t"
                ".Lsr%=:\n\t"
                "v_readlane_b32 s98, %[tabB], %[tli]\n\t"
                "v_and_or_b32 %[oa0], %[oa0], %[mk], %[voffS]\n\t"
                "v_and_or_b32 %[oa1], %[oa1], %[mk], %[voffS]\n\t"
                "v_and_or_b32 %[oa2], %[oa2], %[mk], %[voffS]\n\t"
                "v_and_or_b32 %[oa3], %[oa3], %[mk], %[voffS]\n\t"
                "s_mov_b64 s[88:89], %[ps]\n\t"
                "s_mov_b32 %[dl], %[dl2]\n\t"
                "s_and_b32 s98, s98, 0xfff00000\n\t"
                "s_cmp_eq_u32 %[t], 6\n\t"
                "s_cselect_b32 s98, 0x80000000, s98\n\t"
                "s_mov_b32 %[tl], %[tl2]\n\t"
                "buffer_load_dword a255, %[voffR], %[rsw], s98 offen\n\t"
                "s_branch .Lsb%=\n\t"
                ".subsection 0\n\t"
                "v_mfma_f32_16x16x32_bf16 %[c0020], a[132:135], a[84:87], %[c0020]\n\t"
                "ds_read_b64_tr_b16 a[170:171], %[vb1] offset:17408\n\t"
                "v_mfma_f32_16x16x32_bf16 %[c0030], a[128:131], a[88:91], %[c0030]\n\t"
                "ds_read_b64_tr_b16 a[172:173], %[vb1] offset:24576\n\t"
                "v_mfma_f32_16x16x32_bf16 %[c0030], a[132:135], a[92:95], %[c0030]\n\t"
                "ds_read_b64_tr_b16 a[174:175], %[vb1] offset:25600\n\t"
                "v_mfma_f32_16x16x32_bf16 %[c0001], a[136:139], a[64:67], %[c0001]\n\t"
                "ds_read_b64_tr_b16 a[176:177], %[vb2] offset:16384\n\t"
                "v_mfma_f32_16x16x32_bf16 %[c0001], a[140:143], a[68:71], %[c0001]\n\t"
                "ds_read_b64_tr_b16 a[178:179], %[vb2] offset:17408\n\t"
                "v_mfma_f32_16x16x32_bf16 %[c0011], a[136:139], a[72:75], %[c0011]\n\t"
                "ds_read_b64_tr_b16 a[180:181], %[vb2] offset:24576\n\t"
                "v_mfma_f32_16x16x32_bf16 %[c0011], a[140:143], a[76:79], %[c0011]\n\t"
                "ds_read_b64_tr_b16 a[182:183], %[vb2] offset:25600\n\t"
                "v_mfma_f32_16x16x32_bf16 %[c0021], a[136:139], a[80:83], %[c0021]\n\t"
                "ds_read_b64_tr_b16 a[184:185], %[vb3] offset:16384\n\t"
                "v_mfma_f32_16x16x32_bf16 %[c0021], a[140:143], a[84:87], %[c0021]\n\t"
                "ds_read_b64_tr_b16 a[186:187], %[vb3] offset:17408\n\t"
                "v_mfma_f32_16x16x32_bf16 %[c0031], a[136:139], a[88:91], %[c0031]\n\t"
                "ds_read_b64_tr_b16 a[188:189], %[vb3] offset:24576\n\t"
                "v_mfma_f32_16x16x32_bf16 %[c0031], a[140:143], a[92:95], %[c0031]\n\t"
                "ds_read_b64_tr_b16 a[190:191], %[vb3] offset:25600\n\t"
                "s_add_u32 m0, %[ldsw], 0x0\n\t"
                "v_mfma_f32_16x16x32_bf16 %[c0002], a[144:147], a[64:67], %[c0002]\n\t"
                "buffer_load_dwordx4 %[oa0], s[88:91], %[a2] offen offset:0 sc0 sc1 lds\n\t"
                "v_mfma_f32_16x16x32_bf16 %[c0002], a[148:151], a[68:71], %[c0002]\n\t"
                "v_mfma_f32_16x16x32_bf16 %[c0012], a[144:147], a[72:75], %[c0012]\n\t"
                "v_mfma_f32_16x16x32_bf16 %[c0012], a[148:151], a[76:79], %[c0012]\n\t"
                "v_mfma_f32_16x16x32_bf16 %[c0022], a[144:147], a[80:83], %[c0022]\n\t"
                "buffer_load_dwordx4 %[oa1], s[88:91], %[a2] offen offset:1024 sc0 sc1 lds\n\t"
                "v_mfma_f32_16x16x32_bf16 %[c0022], a[148:151], a[84:87], %[c0022]\n\t"
                "v_mfma_f32_16x16x32_bf16 %[c0032], a[144:147], a[88:91], %[c0032]\n\t"
                "v_mfma_f32_16x16x32_bf16 %[c0032], a[148:151], a[92:95], %[c0032]\n\t"
                "v_mfma_f32_16x16x32_bf16 %[c0003], a[152:155], a[64:67], %[c0003]\n\t"
                "buffer_load_dwordx4 %[oa2], s[88:91], %[a2] offen offset:2048 sc0 sc1 lds\n\t"
                "v_mfma_f32_16x16x32_bf16 %[c0003], a[156:159], a[68:71], %[c0003]\n\t"
                "v_mfma_f32_16x16x32_bf16 %[c0013], a[152:155], a[72:75], %[c0013]\n\t"
                "v_mfma_f32_16x16x32_bf16 %[c0013], a[156:159], a[76:79], %[c0013]\n\t"
                "s_waitcnt lgkmcnt(0)\n\t"
                "v_mfma_f32_16x16x32_bf16 %[c0023], a[152:155], a[80:83], %[c0023]\n\t"
                "v_mfma_f32_16x16x32_bf16 %[c0023], a[156:159], a[84:87], %[c0023]\n\t"
                "buffer_load_dwordx4 %[oa3], s[88:91], %[a2] offen offset:3072 sc0 sc1 lds\n\t"
                "v_mfma_f32_16x16x32_bf16 %[c0033], a[152:155], a[88:91], %[c0033]\n\t"
                "v_mfma_f32_16x16x32_bf16 %[c0033], a[156:159], a[92:95], %[c0033]\n\t"
                : [c0000] "+v"(acc[0][0][0][0]), [pd] "+v"(pd), [sD] "+s"(sD), [c0010] "+v"(acc[0][0][1][0]), [t] "+s"(t), [tl] "+s"(tl), [c0020] "+v"(acc[0][0][2][0]), [oa0] "+v"(voffA[0]), [oa1] "+v"(voffA[1]), [oa2] "+v"(voffA[2]), [oa3] "+v"(voffA[3]), [dl] "+s"(dl), [c0030] "+v"(acc[0][0][3][0]), [c0001] "+v"(acc[0][0][0][1]), [c0011] "+v"(acc[0][0][1][1]), [c0021] "+v"(acc[0][0][2][1]), [c0031] "+v"(acc[0][0][3][1]), [c0002] "+v"(acc[0][0][0][2]), [a2] "+s"(a2), [c0012] "+v"(acc[0][0][1][2]), [c0022] "+v"(acc[0][0][2][2]), [c0032] "+v"(acc[0][0][3][2]), [c0003] "+v"(acc[0][0][0][3]), [c0013] "+v"(acc[0][0][1][3]), [c0023] "+v"(acc[0][0][2][3]), [c0033] "+v"(acc[0][0][3][3]), [rp] "+{s[92:95]}"(rp), [rsa] "+{s[88:91]}"(rsa)
                : [vb0] "v"(vb[0]), [voffQ] "v"(voffQ), [vb1] "v"(vb[1]), [tabB] "v"(tabB), [tli] "s"(tli), [mk] "s"(0x70u), [voffS] "v"(voffS), [ps] "s"(pSu), [dl2] "s"(dl2), [tl2] "s"(tl2), [voffR] "v"(voffR), [rsw] "s"(rsW), [vb2] "v"(vb[2]), [vb3] "v"(vb[3]), [ldsw] "s"(ldsw)
                : "memory", "scc", "s98");
            asm volatile(
                "v_mfma_f32_16x16x32_bf16 a[0:3], a[160:163], a[64:67], a[0:3]\n\t"
                "ds_read_b128 a[96:99], %[va0] offset:16384\n\t"
                "v_mfma_f32_16x16x32_bf16 a[0:3], a[164:167], a[68:71], a[0:3]\n\t"
                "ds_read_b128 a[100:103], %[va1] offset:16384\n\t"
                "v_mfma_f32_16x16x32_bf16 a[16:19], a[160:163], a[72:75], a[16:19]\n\t"
                "ds_read_b128 a[104:107], %[va0] offset:18432\n\t"
                "v_mfma_f32_16x16x32_bf16 a[16:19], a[164:167], a[76:79], a[16:19]\n\t"
                "ds_read_b128 a[108:111], %[va1] offset:18432\n\t"
                "v_mfma_f32_16x16x32_bf16 a[32:35], a[160:163], a[80:83], a[32:35]\n\t"
                "ds_read_b128 a[112:115], %[va0] offset:20480\n\t"
                "v_mfma_f32_16x16x32_bf16 a[32:35], a[164:167], a[84:87], a[32:35]\n\t"
                "ds_read_b128 a[116:119], %[va1] offset:20480\n\t"
                "v_mfma_f32_16x16x32_bf16 a[48:51], a[160:163], a[88:91], a[48:51]\n\t"
                "ds_read_b128 a[120:123], %[va0] offset:22528\n\t"
                "v_mfma_f32_16x16x32_bf16 a[48:51], a[164:167], a[92:95], a[48:51]\n\t"
                "ds_read_b128 a[124:127], %[va1] offset:22528\n\t"
                "s_add_u32 m0, %[ldsw], 0x10000\n\t"
                "v_mfma_f32_16x16x32_bf16 a[4:7], a[168:171], a[64:67], a[4:7]\n\t"
                "buffer_load_dwordx4 %[ob0], %[rsw], %[b2] offen offset:0  lds\n\t"
                "v_mfma_f32_16x16x32_bf16 a[4:7], a[172:175], a[68:71], a[4:7]\n\t"
                "s_add_u32 %[a2], %[a2], %[dl]\n\t"
                "v_mfma_f32_16x16x32_bf16 a[20:23], a[168:171], a[72:75], a[20:23]\n\t"
                "v_mfma_f32_16x16x32_bf16 a[20:23], a[172:175], a[76:79], a[20:23]\n\t"
                "s_add_u32 %[tmp], %[t], 3\n\t"
                "v_mfma_f32_16x16x32_bf16 a[36:39], a[168:171], a[80:83], a[36:39]\n\t"
                "v_mfma_f32_16x16x32_bf16 a[36:39], a[172:175], a[84:87], a[36:39]\n\t"
                "v_readlane_b32 %[a3], %[tabA], %[tmp]\n\t"
                "v_mfma_f32_16x16x32_bf16 a[52:55], a[168:171], a[88:91], a[52:55]\n\t"
                "buffer_load_dwordx4 %[ob1], %[rsw], %[b2] offen offset:1024  lds\n\t"
                "v_mfma_f32_16x16x32_bf16 a[52:55], a[172:175], a[92:95], a[52:55]\n\t"
                "v_mfma_f32_16x16x32_bf16 a[8:11], a[176:179], a[64:67], a[8:11]\n\t"
                "v_mfma_f32_16x16x32_bf16 a[8:11], a[180:183], a[68:71], a[8:11]\n\t"
                "v_readlane_b32 %[b3], %[tabB], %[tmp]\n\t"
                "v_mfma_f32_16x16x32_bf16 a[24:27], a[176:179], a[72:75], a[24:27]\n\t"
                "v_mfma_f32_16x16x32_bf16 a[24:27], a[180:183], a[76:79], a[24:27]\n\t"
                "v_readlane_b32 %[sS], %[tabD], %[tmp]\n\t"
                "v_mfma_f32_16x16x32_bf16 a[40:43], a[176:179], a[80:83], a[40:43]\n\t"
                "buffer_load_dwordx4 %[ob2], %[rsw], %[b2] offen offset:2048  lds\n\t"
                "v_mfma_f32_16x16x32_bf16 a[40:43], a[180:183], a[84:87], a[40:43]\n\t"
                "v_mfma_f32_16x16x32_bf16 a[56:59], a[176:179], a[88:91], a[56:59]\n\t"
                "v_mfma_f32_16x16x32_bf16 a[56:59], a[180:183], a[92:95], a[56:59]\n\t"
                "v_mfma_f32_16x16x32_bf16 a[12:15], a[184:187], a[64:67], a[12:15]\n\t"
                "v_mfma_f32_16x16x32_bf16 a[12:15], a[188:191], a[68:71], a[12:15]\n\t"
                "buffer_load_dwordx4 %[ob3], %[rsw], %[b2] offen offset:3072  lds\n\t"
                "v_mfma_f32_16x16x32_bf16 a[28:31], a[184:187], a[72:75], a[28:31]\n\t"
                "v_mfma_f32_16x16x32_bf16 a[28:31], a[188:191], a[76:79], a[28:31]\n\t"
                "s_waitcnt vmcnt(18) lgkmcnt(0)\n\t"
                "s_barrier\n\t"
                "v_mfma_f32_16x16x32_bf16 a[44:47], a[184:187], a[80:83], a[44:47]\n\t"
                "v_mfma_f32_16x16x32_bf16 a[44:47], a[188:191], a[84:87], a[44:47]\n\t"
                "v_mfma_f32_16x16x32_bf16 a[60:63], a[184:187], a[88:91], a[60:63]\n\t"
                "v_mfma_f32_16x16x32_bf16 a[60:63], a[188:191], a[92:95], a[60:63]\n\t"
                : [b2] "+s"(b2), [a2] "+s"(a2), [dl] "+s"(dl), [tmp] "+s"(tmp), [t] "+s"(t), [a3] "+s"(a3), [b3] "+s"(b3), [sS] "+s"(sS)
                : [va0] "v"(va[0]), [va1] "v"(va[1]), [ldsw] "s"(ldsw), [ob0] "v"(voffB[0]), [rsw] "s"(rsW), [tabA] "v"(tabA), [ob1] "v"(voffB[1]), [tabB] "v"(tabB), [tabD] "v"(tabD), [ob2] "v"(voffB[2]), [ob3] "v"(voffB[3])
                : "memory", "scc");
            asm volatile(
                "v_mfma_f32_16x16x32_bf16 %[c1100], a[160:163], a[96:99], %[c1100]\n\t"
                "ds_read_b128 a[64:67], %[va0] offset:32768\n\t"
                "v_mfma_f32_16x16x32_bf16 %[c1100], a[164:167], a[100:103], %[c1100]\n\t"
                "ds_read_b128 a[68:71], %[va1] offset:32768\n\t"
                "v_mfma_f32_16x16x32_bf16 %[c1110], a[160:163], a[104:107], %[c1110]\n\t"
                "ds_read_b128 a[72:75], %[va0] offset:34816\n\t"
                "buffer_load_dwordx4 %[pd], %[voffP], %[rsx], %[sS] offen\n\t"
                "v_mfma_f32_16x16x32_bf16 %[c1110], a[164:167], a[108:111], %[c1110]\n\t"
                "ds_read_b128 a[76:79], %[va1] offset:34816\n\t"
                "v_mfma_f32_16x16x32_bf16 %[c1120], a[160:163], a[112:115], %[c1120]\n\t"
                "ds_read_b128 a[80:83], %[va0] offset:36864\n\t"
                "v_mfma_f32_16x16x32_bf16 %[c1120], a[164:167], a[116:119], %[c1120]\n\t"
                "ds_read_b128 a[84:87], %[va1] offset:36864\n\t"
                "v_mfma_f32_16x16x32_bf16 %[c1130], a[160:163], a[120:123], %[c1130]\n\t"
                "ds_read_b128 a[88:91], %[va0] offset:38912\n\t"
                "v_mfma_f32_16x16x32_bf16 %[c1130], a[164:167], a[124:127], %[c1130]\n\t"
                "ds_read_b128 a[92:95], %[va1] offset:38912\n\t"
                "s_add_u32 m0, %[ldsw], 0x13f00\n\t"
                "v_mfma_f32_16x16x32_bf16 %[c1101], a[168:171], a[96:99], %[c1101]\n\t"
                "buffer_load_dwordx4 %[ob0], %[rsw], %[b2] offen offset:256  lds\n\t"
                "v_mfma_f32_16x16x32_bf16 %[c1101], a[172:175], a[100:103], %[c1101]\n\t"
                "v_mfma_f32_16x16x32_bf16 %[c1111], a[168:171], a[104:107], %[c1111]\n\t"
                "v_mfma_f32_16x16x32_bf16 %[c1111], a[172:175], a[108:111], %[c1111]\n\t"
                "v_mfma_f32_16x16x32_bf16 %[c1121], a[168:171], a[112:115], %[c1121]\n\t"
                "v_mfma_f32_16x16x32_bf16 %[c1121], a[172:175], a[116:119], %[c1121]\n\t"
                "v_mfma_f32_16x16x32_bf16 %[c1131], a[168:171], a[120:123], %[c1131]\n\t"
                "buffer_load_dwordx4 %[ob1], %[rsw], %[b2] offen offset:1280  lds\n\t"
                "v_mfma_f32_16x16x32_bf16 %[c1131], a[172:175], a[124:127], %[c1131]\n\t"
                "v_mfma_f32_16x16x32_bf16 %[c1102], a[176:179], a[96:99], %[c1102]\n\t"
                "v_mfma_f32_16x16x32_bf16 %[c1102], a[180:183], a[100:103], %[c1102]\n\t"
                "v_mfma_f32_16x16x32_bf16 %[c1112], a[176:179], a[104:107], %[c1112]\n\t"
                "v_mfma_f32_16x16x32_bf16 %[c1112], a[180:183], a[108:111], %[c1112]\n\t"
                "v_mfma_f32_16x16x32_bf16 %[c1122], a[176:179], a[112:115], %[c1122]\n\t"
                "buffer_load_dwordx4 %[ob2], %[rsw], %[b2] offen offset:2304  lds\n\t"
                "v_mfma_f32_16x16x32_bf16 %[c1122], a[180:183], a[116:119], %[c1122]\n\t"
                "v_mfma_f32_16x16x32_bf16 %[c1132], a[176:179], a[120:123], %[c1132]\n\t"
                "v_mfma_f32_16x16x32_bf16 %[c1132], a[180:183], a[124:127], %[c1132]\n\t"
                "v_mfma_f32_16x16x32_bf16 %[c1103], a[184:187], a[96:99], %[c1103]\n\t"
                "v_mfma_f32_16x16x32_bf16 %[c1103], a[188:191], a[100:103], %[c1103]\n\t"
                "buffer_load_dwordx4 %[ob3], %[rsw], %[b2] offen offset:3328  lds\n\t"
                "v_mfma_f32_16x16x32_bf16 %[c1113], a[184:187], a[104:107], %[c1113]\n\t"
                "v_mfma_f32_16x16x32_bf16 %[c1113], a[188:191], a[108:111], %[c1113]\n\t"
                "s_waitcnt lgkmcnt(0)\n\t"
                "v_mfma_f32_16x16x32_bf16 %[c1123], a[184:187], a[112:115], %[c1123]\n\t"
                "v_mfma_f32_16x16x32_bf16 %[c1123], a[188:191], a[116:119], %[c1123]\n\t"
                "v_mfma_f32_16x16x32_bf16 %[c1133], a[184:187], a[120:123], %[c1133]\n\t"
                "v_mfma_f32_16x16x32_bf16 %[c1133], a[188:191], a[124:127], %[c1133]\n\t"
                : [c1100] "+v"(acc[1][1][0][0]), [c1110] "+v"(acc[1][1][1][0]), [pd] "+v"(pd), [sS] "+s"(sS), [c1120] "+v"(acc[1][1][2][0]), [c1130] "+v"(acc[1][1][3][0]), [c1101] "+v"(acc[1][1][0][1]), [b2] "+s"(b2), [c1111] "+v"(acc[1][1][1][1]), [c1121] "+v"(acc[1][1][2][1]), [c1131] "+v"(acc[1][1][3][1]), [c1102] "+v"(acc[1][1][0][2]), [c1112] "+v"(acc[1][1][1][2]), [c1122] "+v"(acc[1][1][2][2]), [c1132] "+v"(acc[1][1][3][2]), [c1103] "+v"(acc[1][1][0][3]), [c1113] "+v"(acc[1][1][1][3]), [c1123] "+v"(acc[1][1][2][3]), [c1133] "+v"(acc[1][1][3][3])
                : [va0] "v"(va[0]), [va1] "v"(va[1]), [voffP] "v"(voffP), [rsx] "s"(rsX), [ldsw] "s"(ldsw), [ob0] "v"(voffB[0]), [rsw] "s"(rsW), [ob1] "v"(voffB[1]), [ob2] "v"(voffB[2]), [ob3] "v"(voffB[3])
                : "memory", "scc");
            asm volatile(
                "v_mfma_f32_16x16x32_bf16 %[c1000], a[128:131], a[96:99], %[c1000]\n\t"
                "ds_read_b64_tr_b16 a[160:161], %[vb0] offset:49152\n\t"
                "v_mfma_f32_16x16x32_bf16 %[c1000], a[132:135], a[100:103], %[c1000]\n\t"
                "ds_read_b64_tr_b16 a[162:163], %[vb0] offset:50176\n\t"
                "v_mfma_f32_16x16x32_bf16 %[c1010], a[128:131], a[104:107], %[c1010]\n\t"
                "ds_read_b64_tr_b16 a[164:165], %[vb0] offset:57344\n\t"
                "v_mfma_f32_16x16x32_bf16 %[c1010], a[132:135], a[108:111], %[c1010]\n\t"
                "ds_read_b64_tr_b16 a[166:167], %[vb0] offset:58368\n\t"
                "v_mfma_f32_16x16x32_bf16 %[c1020], a[128:131], a[112:115], %[c1020]\n\t"
                "ds_read_b64_tr_b16 a[168:169], %[vb1] offset:49152\n\t"
                "v_mfma_f32_16x16x32_bf16 %[c1020], a[132:135], a[116:119], %[c1020]\n\t"
                "ds_read_b64_tr_b16 a[170:171], %[vb1] offset:50176\n\t"
                "v_mfma_f32_16x16x32_bf16 %[c1030], a[128:131], a[120:123], %[c1030]\n\t"
                "ds_read_b64_tr_b16 a[172:173], %[vb1] offset:57344\n\t"
                "v_mfma_f32_16x16x32_bf16 %[c1030], a[132:135], a[124:127], %[c1030]\n\t"
                "ds_read_b64_tr_b16 a[174:175], %[vb1] offset:58368\n\t"
                "v_mfma_f32_16x16x32_bf16 %[c1001], a[136:139], a[96:99], %[c1001]\n\t"
                "ds_read_b64_tr_b16 a[176:177], %[vb2] offset:49152\n\t"
                "v_mfma_f32_16x16x32_bf16 %[c1001], a[140:143], a[100:103], %[c1001]\n\t"
                "ds_read_b64_tr_b16 a[178:179], %[vb2] offset:50176\n\t"
                "v_mfma_f32_16x16x32_bf16 %[c1011], a[136:139], a[104:107], %[c1011]\n\t"
                "ds_read_b64_tr_b16 a[180:181], %[vb2] offset:57344\n\t"
                "v_mfma_f32_16x16x32_bf16 %[c1011], a[140:143], a[108:111], %[c1011]\n\t"
                "ds_read_b64_tr_b16 a[182:183], %[vb2] offset:58368\n\t"
                "v_mfma_f32_16x16x32_bf16 %[c1021], a[136:139], a[112:115], %[c1021]\n\t"
                "ds_read_b64_tr_b16 a[184:185], %[vb3] offset:49152\n\t"
                "v_mfma_f32_16x16x32_bf16 %[c1021], a[140:143], a[116:119], %[c1021]\n\t"
                "ds_read_b64_tr_b16 a[186:187], %[vb3] offset:50176\n\t"
                "v_mfma_f32_16x16x32_bf16 %[c1031], a[136:139], a[120:123], %[c1031]\n\t"
                "ds_read_b64_tr_b16 a[188:189], %[vb3] offset:57344\n\t"
                "v_mfma_f32_16x16x32_bf16 %[c1031], a[140:143], a[124:127], %[c1031]\n\t"
                "ds_read_b64_tr_b16 a[190:191], %[vb3] offset:58368\n\t"
                "s_add_u32 m0, %[ldsw], 0x4000\n\t"
                "v_mfma_f32_16x16x32_bf16 %[c1002], a[144:147], a[96:99], %[c1002]\n\t"
                "buffer_load_dwordx4 %[oa0], s[88:91], %[a2] offen offset:0 sc0 sc1 lds\n\t"
                "v_mfma_f32_16x16x32_bf16 %[c1002], a[148:151], a[100:103], %[c1002]\n\t"
                "v_mfma_f32_16x16x32_bf16 %[c1012], a[144:147], a[104:107], %[c1012]\n\t"
                "v_mfma_f32_16x16x32_bf16 %[c1012], a[148:151], a[108:111], %[c1012]\n\t"
                "s_lshl_b64 %[msk], 6, %[tmp]\n\t"
                "v_mfma_f32_16x16x32_bf16 %[c1022], a[144:147], a[112:115], %[c1022]\n\t"
                "buffer_load_dwordx4 %[oa1], s[88:91], %[a2] offen offset:1024 sc0 sc1 lds\n\t"
                "v_mfma_f32_16x16x32_bf16 %[c1022], a[148:151], a[116:119], %[c1022]\n\t"
                "s_andn2_b64 s[98:99], %[msk], %[rdy]\n\t"
                "s_cselect_b32 %[spo], %[sflag], 0x80000000\n\t"
                "v_mfma_f32_16x16x32_bf16 %[c1032], a[144:147], a[120:123], %[c1032]\n\t"
                "v_mfma_f32_16x16x32_bf16 %[c1032], a[148:151], a[124:127], %[c1032]\n\t"
                "v_mfma_f32_16x16x32_bf16 %[c1003], a[152:155], a[96:99], %[c1003]\n\t"
                "buffer_load_dwordx4 %[oa2], s[88:91], %[a2] offen offset:2048 sc0 sc1 lds\n\t"
                "buffer_load_dwordx2 %[pf], %[voffF], %[rsf], %[spo] offen sc0 sc1\n\t"
                "v_mfma_f32_16x16x32_bf16 %[c1003], a[156:159], a[100:103], %[c1003]\n\t"
                "v_mfma_f32_16x16x32_bf16 %[c1013], a[152:155], a[104:107], %[c1013]\n\t"
                "v_mfma_f32_16x16x32_bf16 %[c1013], a[156:159], a[108:111], %[c1013]\n\t"
                "s_waitcnt vmcnt(18) lgkmcnt(0)\n\t"
                "s_barrier\n\t"
                "v_mfma_f32_16x16x32_bf16 %[c1023], a[152:155], a[112:115], %[c1023]\n\t"
                "v_mfma_f32_16x16x32_bf16 %[c1023], a[156:159], a[116:119], %[c1023]\n\t"
                "buffer_load_dwordx4 %[oa3], s[88:91], %[a2] offen offset:3072 sc0 sc1 lds\n\t"
                "v_mfma_f32_16x16x32_bf16 %[c1033], a[152:155], a[120:123], %[c1033]\n\t"
                "v_mfma_f32_16x16x32_bf16 %[c1033], a[156:159], a[124:127], %[c1033]\n\t"
                : [c1000] "+v"(acc[1][0][0][0]), [c1010] "+v"(acc[1][0][1][0]), [c1020] "+v"(acc[1][0][2][0]), [c1030] "+v"(acc[1][0][3][0]), [c1001] "+v"(acc[1][0][0][1]), [c1011] "+v"(acc[1][0][1][1]), [c1021] "+v"(acc[1][0][2][1]), [c1031] "+v"(acc[1][0][3][1]), [c1002] "+v"(acc[1][0][0][2]), [oa0] "+v"(voffA[0]), [a2] "+s"(a2), [c1012] "+v"(acc[1][0][1][2]), [msk] "+s"(msk), [tmp] "+s"(tmp), [c1022] "+v"(acc[1][0][2][2]), [oa1] "+v"(voffA[1]), [rdy] "+s"(rdy), [spo] "+s"(spo), [c1032] "+v"(acc[1][0][3][2]), [c1003] "+v"(acc[1][0][0][3]), [oa2] "+v"(voffA[2]), [pf] "+v"(pf), [c1013] "+v"(acc[1][0][1][3]), [c1023] "+v"(acc[1][0][2][3]), [oa3] "+v"(voffA[3]), [c1033] "+v"(acc[1][0][3][3]), [rsa] "+{s[88:91]}"(rsa)
                : [vb0] "v"(vb[0]), [vb1] "v"(vb[1]), [vb2] "v"(vb[2]), [vb3] "v"(vb[3]), [ldsw] "s"(ldsw), [sflag] "s"(sflag), [voffF] "v"(voffF), [rsf] "s"(rsF)
                : "memory", "scc", "s98", "s99");
            asm volatile(
                "v_mfma_f32_16x16x32_bf16 a[0:3], a[160:163], a[64:67], a[0:3]\n\t"
                "ds_read_b64_tr_b16 a[128:129], %[vb0] offset:32768\n\t"
                "v_mfma_f32_16x16x32_bf16 a[0:3], a[164:167], a[68:71], a[0:3]\n\t"
                "ds_read_b64_tr_b16 a[130:131], %[vb0] offset:33792\n\t"
                "v_mfma_f32_16x16x32_bf16 a[16:19], a[160:163], a[72:75], a[16:19]\n\t"
                "ds_read_b64_tr_b16 a[132:133], %[vb0] offset:40960\n\t"
                "v_mfma_f32_16x16x32_bf16 a[16:19], a[164:167], a[76:79], a[16:19]\n\t"
                "ds_read_b64_tr_b16 a[134:135], %[vb0] offset:41984\n\t"
                "v_mfma_f32_16x16x32_bf16 a[32:35], a[160:163], a[80:83], a[32:35]\n\t"
                "ds_read_b64_tr_b16 a[136:137], %[vb1] offset:32768\n\t"
                "v_mfma_f32_16x16x32_bf16 a[32:35], a[164:167], a[84:87], a[32:35]\n\t"
                "ds_read_b64_tr_b16 a[138:139], %[vb1] offset:33792\n\t"
                "v_mfma_f32_16x16x32_bf16 a[48:51], a[160:163], a[88:91], a[48:51]\n\t"
                "ds_read_b64_tr_b16 a[140:141], %[vb1] offset:40960\n\t"
                "v_mfma_f32_16x16x32_bf16 a[48:51], a[164:167], a[92:95], a[48:51]\n\t"
                "ds_read_b64_tr_b16 a[142:143], %[vb1] offset:41984\n\t"
                "v_mfma_f32_16x16x32_bf16 a[4:7], a[168:171], a[64:67], a[4:7]\n\t"
                "ds_read_b64_tr_b16 a[144:145], %[vb2] offset:32768\n\t"
                "v_mfma_f32_16x16x32_bf16 a[4:7], a[172:175], a[68:71], a[4:7]\n\t"
                "ds_read_b64_tr_b16 a[146:147], %[vb2] offset:33792\n\t"
                "v_mfma_f32_16x16x32_bf16 a[20:23], a[168:171], a[72:75], a[20:23]\n\t"
                "ds_read_b64_tr_b16 a[148:149], %[vb2] offset:40960\n\t"
                "v_mfma_f32_16x16x32_bf16 a[20:23], a[172:175], a[76:79], a[20:23]\n\t"
                "ds_read_b64_tr_b16 a[150:151], %[vb2] offset:41984\n\t"
                "v_mfma_f32_16x16x32_bf16 a[36:39], a[168:171], a[80:83], a[36:39]\n\t"
                "ds_read_b64_tr_b16 a[152:153], %[vb3] offset:32768\n\t"
                "v_mfma_f32_16x16x32_bf16 a[36:39], a[172:175], a[84:87], a[36:39]\n\t"
                "ds_read_b64_tr_b16 a[154:155], %[vb3] offset:33792\n\t"
                "v_mfma_f32_16x16x32_bf16 a[52:55], a[168:171], a[88:91], a[52:55]\n\t"
                "ds_read_b64_tr_b16 a[156:157], %[vb3] offset:40960\n\t"
                "v_mfma_f32_16x16x32_bf16 a[52:55], a[172:175], a[92:95], a[52:55]\n\t"
                "ds_read_b64_tr_b16 a[158:159], %[vb3] offset:41984\n\t"
                "s_add_u32 m0, %[ldsw], 0x8000\n\t"
                "v_mfma_f32_16x16x32_bf16 a[8:11], a[176:179], a[64:67], a[8:11]\n\t"
                "buffer_load_dwordx4 %[oa0], s[88:91], %[a3] offen offset:0 sc0 sc1 lds\n\t"
                "v_mfma_f32_16x16x32_bf16 a[8:11], a[180:183], a[68:71], a[8:11]\n\t"
                "v_mfma_f32_16x16x32_bf16 a[24:27], a[176:179], a[72:75], a[24:27]\n\t"
                "v_mfma_f32_16x16x32_bf16 a[24:27], a[180:183], a[76:79], a[24:27]\n\t"
                "v_mfma_f32_16x16x32_bf16 a[40:43], a[176:179], a[80:83], a[40:43]\n\t"
                "buffer_load_dwordx4 %[oa1], s[88:91], %[a3] offen offset:1024 sc0 sc1 lds\n\t"
                "v_mfma_f32_16x16x32_bf16 a[40:43], a[180:183], a[84:87], a[40:43]\n\t"
                "v_mfma_f32_16x16x32_bf16 a[56:59], a[176:179], a[88:91], a[56:59]\n\t"
                "v_mfma_f32_16x16x32_bf16 a[56:59], a[180:183], a[92:95], a[56:59]\n\t"
                "v_mfma_f32_16x16x32_bf16 a[12:15], a[184:187], a[64:67], a[12:15]\n\t"
                "buffer_load_dwordx4 %[oa2], s[88:91], %[a3] offen offset:2048 sc0 sc1 lds\n\t"
                "v_mfma_f32_16x16x32_bf16 a[12:15], a[188:191], a[68:71], a[12:15]\n\t"
                "v_mfma_f32_16x16x32_bf16 a[28:31], a[184:187], a[72:75], a[28:31]\n\t"
                "v_mfma_f32_16x16x32_bf16 a[28:31], a[188:191], a[76:79], a[28:31]\n\t"
                "s_waitcnt lgkmcnt(0)\n\t"
                "v_mfma_f32_16x16x32_bf16 a[44:47], a[184:187], a[80:83], a[44:47]\n\t"
                "v_mfma_f32_16x16x32_bf16 a[44:47], a[188:191], a[84:87], a[44:47]\n\t"
                "buffer_load_dwordx4 %[oa3], s[88:91], %[a3] offen offset:3072 sc0 sc1 lds\n\t"
                "v_mfma_f32_16x16x32_bf16 a[60:63], a[184:187], a[88:91], a[60:63]\n\t"
                "v_mfma_f32_16x16x32_bf16 a[60:63], a[188:191], a[92:95], a[60:63]\n\t"
                : [oa0] "+v"(voffA[0]), [a3] "+s"(a3), [oa1] "+v"(voffA[1]), [oa2] "+v"(voffA[2]), [oa3] "+v"(voffA[3]), [rsa] "+{s[88:91]}"(rsa)
                : [vb0] "v"(vb[0]), [vb1] "v"(vb[1]), [vb2] "v"(vb[2]), [vb3] "v"(vb[3]), [ldsw] "s"(ldsw)
                : "memory", "scc");
            asm volatile(
                "v_mfma_f32_16x16x32_bf16 %[c0000], a[128:131], a[64:67], %[c0000]\n\t"
                "ds_read_b128 a[96:99], %[va0] offset:49152\n\t"
                "v_mfma_f32_16x16x32_bf16 %[c0000], a[132:135], a[68:71], %[c0000]\n\t"
                "ds_read_b128 a[100:103], %[va1] offset:49152\n\t"
                "v_mfma_f32_16x16x32_bf16 %[c0010], a[128:131], a[72:75], %[c0010]\n\t"
                "ds_read_b128 a[104:107], %[va0] offset:51200\n\t"
                "v_mfma_f32_16x16x32_bf16 %[c0010], a[132:135], a[76:79], %[c0010]\n\t"
                "ds_read_b128 a[108:111], %[va1] offset:51200\n\t"
                "v_mfma_f32_16x16x32_bf16 %[c0020], a[128:131], a[80:83], %[c0020]\n\t"
                "ds_read_b128 a[112:115], %[va0] offset:53248\n\t"
                "v_mfma_f32_16x16x32_bf16 %[c0020], a[132:135], a[84:87], %[c0020]\n\t"
                "ds_read_b128 a[116:119], %[va1] offset:53248\n\t"
                "v_mfma_f32_16x16x32_bf16 %[c0030], a[128:131], a[88:91], %[c0030]\n\t"
                "ds_read_b128 a[120:123], %[va0] offset:55296\n\t"
                "v_mfma_f32_16x16x32_bf16 %[c0030], a[132:135], a[92:95], %[c0030]\n\t"
                "ds_read_b128 a[124:127], %[va1] offset:55296\n\t"
                "s_add_u32 m0, %[ldsw], 0x1bf00\n\t"
                "v_mfma_f32_16x16x32_bf16 %[c0001], a[136:139], a[64:67], %[c0001]\n\t"
                "buffer_load_dwordx4 %[ob0], %[rsw], %[b3] offen offset:256  lds\n\t"
                "v_mfma_f32_16x16x32_bf16 %[c0001], a[140:143], a[68:71], %[c0001]\n\t"
                "s_add_u32 %[a3], %[a3], %[dl]\n\t"
                "v_mfma_f32_16x16x32_bf16 %[c0011], a[136:139], a[72:75], %[c0011]\n\t"
                "v_mfma_f32_16x16x32_bf16 %[c0011], a[140:143], a[76:79], %[c0011]\n\t"
                "s_add_u32 %[tmp], %[t], 4\n\t"
                "v_mfma_f32_16x16x32_bf16 %[c0021], a[136:139], a[80:83], %[c0021]\n\t"
                "v_mfma_f32_16x16x32_bf16 %[c0021], a[140:143], a[84:87], %[c0021]\n\t"
                "v_readlane_b32 %[a2], %[tabA], %[tmp]\n\t"
                "v_mfma_f32_16x16x32_bf16 %[c0031], a[136:139], a[88:91], %[c0031]\n\t"
                "buffer_load_dwordx4 %[ob1], %[rsw], %[b3] offen offset:1280  lds\n\t"
                "v_mfma_f32_16x16x32_bf16 %[c0031], a[140:143], a[92:95], %[c0031]\n\t"
                "v_mfma_f32_16x16x32_bf16 %[c0002], a[144:147], a[64:67], %[c0002]\n\t"
                "v_mfma_f32_16x16x32_bf16 %[c0002], a[148:151], a[68:71], %[c0002]\n\t"
                "v_readlane_b32 %[b2], %[tabB], %[tmp]\n\t"
                "v_mfma_f32_16x16x32_bf16 %[c0012], a[144:147], a[72:75], %[c0012]\n\t"
                "v_mfma_f32_16x16x32_bf16 %[c0012], a[148:151], a[76:79], %[c0012]\n\t"
                "s_add_u32 %[t], %[t], 2\n\t"
                "v_mfma_f32_16x16x32_bf16 %[c0022], a[144:147], a[80:83], %[c0022]\n\t"
                "buffer_load_dwordx4 %[ob2], %[rsw], %[b3] offen offset:2304  lds\n\t"
                "v_mfma_f32_16x16x32_bf16 %[c0022], a[148:151], a[84:87], %[c0022]\n\t"
                "v_mfma_f32_16x16x32_bf16 %[c0032], a[144:147], a[88:91], %[c0032]\n\t"
                "v_mfma_f32_16x16x32_bf16 %[c0032], a[148:151], a[92:95], %[c0032]\n\t"
                "v_mfma_f32_16x16x32_bf16 %[c0003], a[152:155], a[64:67], %[c0003]\n\t"
                "v_mfma_f32_16x16x32_bf16 %[c0003], a[156:159], a[68:71], %[c0003]\n\t"
                "buffer_load_dwordx4 %[ob3], %[rsw], %[b3] offen offset:3328  lds\n\t"
                "v_mfma_f32_16x16x32_bf16 %[c0013], a[152:155], a[72:75], %[c0013]\n\t"
                "v_mfma_f32_16x16x32_bf16 %[c0013], a[156:159], a[76:79], %[c0013]\n\t"
                "s_waitcnt vmcnt(18) lgkmcnt(0)\n\t"
                "s_barrier\n\t"
                "v_mfma_f32_16x16x32_bf16 %[c0023], a[152:155], a[80:83], %[c0023]\n\t"
                "v_mfma_f32_16x16x32_bf16 %[c0023], a[156:159], a[84:87], %[c0023]\n\t"
                "buffer_store_byte %[vfv], %[voffL], s[92:95], %[sF] offen sc0 sc1\n\t"
                "v_mfma_f32_16x16x32_bf16 %[c0033], a[152:155], a[88:91], %[c0033]\n\t"
                "v_mfma_f32_16x16x32_bf16 %[c0033], a[156:159], a[92:95], %[c0033]\n\t"
                : [c0000] "+v"(acc[0][0][0][0]), [c0010] "+v"(acc[0][0][1][0]), [c0020] "+v"(acc[0][0][2][0]), [c0030] "+v"(acc[0][0][3][0]), [c0001] "+v"(acc[0][0][0][1]), [b3] "+s"(b3), [a3] "+s"(a3), [dl] "+s"(dl), [c0011] "+v"(acc[0][0][1][1]), [tmp] "+s"(tmp), [t] "+s"(t), [c0021] "+v"(acc[0][0][2][1]), [a2] "+s"(a2), [c0031] "+v"(acc[0][0][3][1]), [c0002] "+v"(acc[0][0][0][2]), [b2] "+s"(b2), [c0012] "+v"(acc[0][0][1][2]), [c0022] "+v"(acc[0][0][2][2]), [c0032] "+v"(acc[0][0][3][2]), [c0003] "+v"(acc[0][0][0][3]), [c0013] "+v"(acc[0][0][1][3]), [c0023] "+v"(acc[0][0][2][3]), [sF] "+s"(sF), [c0033] "+v"(acc[0][0][3][3]), [rp] "+{s[92:95]}"(rp)
                : [va0] "v"(va[0]), [va1] "v"(va[1]), [ldsw] "s"(ldsw), [ob0] "v"(voffB[0]), [rsw] "s"(rsW), [tabA] "v"(tabA), [ob1] "v"(voffB[1]), [tabB] "v"(tabB), [ob2] "v"(voffB[2]), [ob3] "v"(voffB[3]), [vfv] "v"(vfv), [voffL] "v"(voffL)
                : "memory", "scc");
            asm volatile(
                "v_mfma_f32_16x16x32_bf16 %[c1000], a[128:131], a[96:99], %[c1000]\n\t"
                "ds_read_b128 a[64:67], %[va0] offset:0\n\t"
                "v_mfma_f32_16x16x32_bf16 %[c1000], a[132:135], a[100:103], %[c1000]\n\t"
                "ds_read_b128 a[68:71], %[va1] offset:0\n\t"
                "v_mfma_f32_16x16x32_bf16 %[c1010], a[128:131], a[104:107], %[c1010]\n\t"
                "ds_read_b128 a[72:75], %[va0] offset:2048\n\t"
                "v_mfma_f32_16x16x32_bf16 %[c1010], a[132:135], a[108:111], %[c1010]\n\t"
                "ds_read_b128 a[76:79], %[va1] offset:2048\n\t"
                "v_mfma_f32_16x16x32_bf16 %[c1020], a[128:131], a[112:115], %[c1020]\n\t"
                "ds_read_b128 a[80:83], %[va0] offset:4096\n\t"
                "v_mfma_f32_16x16x32_bf16 %[c1020], a[132:135], a[116:119], %[c1020]\n\t"
                "ds_read_b128 a[84:87], %[va1] offset:4096\n\t"
                "v_mfma_f32_16x16x32_bf16 %[c1030], a[128:131], a[120:123], %[c1030]\n\t"
                "ds_read_b128 a[88:91], %[va0] offset:6144\n\t"
                "v_mfma_f32_16x16x32_bf16 %[c1030], a[132:135], a[124:127], %[c1030]\n\t"
                "ds_read_b128 a[92:95], %[va1] offset:6144\n\t"
                "s_add_u32 m0, %[ldsw], 0x18000\n\t"
                "v_mfma_f32_16x16x32_bf16 %[c1001], a[136:139], a[96:99], %[c1001]\n\t"
                "buffer_load_dwordx4 %[ob0], %[rsw], %[b3] offen offset:0  lds\n\t"
                "v_mfma_f32_16x16x32_bf16 %[c1001], a[140:143], a[100:103], %[c1001]\n\t"
                "v_readlane_b32 s92, %[tabLo], %[tmp]\n\t"
                "v_mfma_f32_16x16x32_bf16 %[c1011], a[136:139], a[104:107], %[c1011]\n\t"
                "v_mfma_f32_16x16x32_bf16 %[c1011], a[140:143], a[108:111], %[c1011]\n\t"
                "v_readlane_b32 s93, %[tabHi], %[tmp]\n\t"
                "v_mfma_f32_16x16x32_bf16 %[c1021], a[136:139], a[112:115], %[c1021]\n\t"
                "v_mfma_f32_16x16x32_bf16 %[c1021], a[140:143], a[116:119], %[c1021]\n\t"
                "v_readlane_b32 %[sD], %[tabD], %[tmp]\n\t"
                "v_mfma_f32_16x16x32_bf16 %[c1031], a[136:139], a[120:123], %[c1031]\n\t"
                "buffer_load_dwordx4 %[ob1], %[rsw], %[b3] offen offset:1024  lds\n\t"
                "v_mfma_f32_16x16x32_bf16 %[c1031], a[140:143], a[124:127], %[c1031]\n\t"
                "v_mfma_f32_16x16x32_bf16 %[c1002], a[144:147], a[96:99], %[c1002]\n\t"
                "v_mfma_f32_16x16x32_bf16 %[c1002], a[148:151], a[100:103], %[c1002]\n\t"
                "v_readlane_b32 %[sF], %[tabF], %[tmp]\n\t"
                "v_mfma_f32_16x16x32_bf16 %[c1012], a[144:147], a[104:107], %[c1012]\n\t"
                "v_mfma_f32_16x16x32_bf16 %[c1012], a[148:151], a[108:111], %[c1012]\n\t"
                "v_mfma_f32_16x16x32_bf16 %[c1022], a[144:147], a[112:115], %[c1022]\n\t"
                "buffer_load_dwordx4 %[ob2], %[rsw], %[b3] offen offset:2048  lds\n\t"
                "v_mfma_f32_16x16x32_bf16 %[c1022], a[148:151], a[116:119], %[c1022]\n\t"
                "v_mfma_f32_16x16x32_bf16 %[c1032], a[144:147], a[120:123], %[c1032]\n\t"
                "v_mfma_f32_16x16x32_bf16 %[c1032], a[148:151], a[124:127], %[c1032]\n\t"
                "v_mfma_f32_16x16x32_bf16 %[c1003], a[152:155], a[96:99], %[c1003]\n\t"
                "v_mfma_f32_16x16x32_bf16 %[c1003], a[156:159], a[100:103], %[c1003]\n\t"
                "buffer_load_dwordx4 %[ob3], %[rsw], %[b3] offen offset:3072  lds\n\t"
                "v_mfma_f32_16x16x32_bf16 %[c1013], a[152:155], a[104:107], %[c1013]\n\t"
                "v_mfma_f32_16x16x32_bf16 %[c1013], a[156:159], a[108:111], %[c1013]\n\t"
                "s_waitcnt lgkmcnt(0)\n\t"
                "v_mfma_f32_16x16x32_bf16 %[c1023], a[152:155], a[112:115], %[c1023]\n\t"
                "v_mfma_f32_16x16x32_bf16 %[c1023], a[156:159], a[116:119], %[c1023]\n\t"
                "v_mfma_f32_16x16x32_bf16 %[c1033], a[152:155], a[120:123], %[c1033]\n\t"
                "v_mfma_f32_16x16x32_bf16 %[c1033], a[156:159], a[124:127], %[c1033]\n\t"
                : [c1000] "+v"(acc[1][0][0][0]), [c1010] "+v"(acc[1][0][1][0]), [c1020] "+v"(acc[1][0][2][0]), [c1030] "+v"(acc[1][0][3][0]), [c1001] "+v"(acc[1][0][0][1]), [b3] "+s"(b3), [tmp] "+s"(tmp), [c1011] "+v"(acc[1][0][1][1]), [c1021] "+v"(acc[1][0][2][1]), [sD] "+s"(sD), [c1031] "+v"(acc[1][0][3][1]), [c1002] "+v"(acc[1][0][0][2]), [sF] "+s"(sF), [c1012] "+v"(acc[1][0][1][2]), [c1022] "+v"(acc[1][0][2][2]), [c1032] "+v"(acc[1][0][3][2]), [c1003] "+v"(acc[1][0][0][3]), [c1013] "+v"(acc[1][0][1][3]), [c1023] "+v"(acc[1][0][2][3]), [c1033] "+v"(acc[1][0][3][3]), [rp] "+{s[92:95]}"(rp)
                : [va0] "v"(va[0]), [va1] "v"(va[1]), [ldsw] "s"(ldsw), [ob0] "v"(voffB[0]), [rsw] "s"(rsW), [tabLo] "v"(tabLo), [tabHi] "v"(tabHi), [tabD] "v"(tabD), [ob1] "v"(voffB[1]), [tabF] "v"(tabF), [ob2] "v"(voffB[2]), [ob3] "v"(voffB[3])
                : "memory", "scc");
            asm volatile(
                "v_mfma_f32_16x16x32_bf16 %[c1100], a[160:163], a[96:99], %[c1100]\n\t"
                "ds_read_b64_tr_b16 a[128:129], %[vb0] offset:0\n\t"
                "v_mfma_f32_16x16x32_bf16 %[c1100], a[164:167], a[100:103], %[c1100]\n\t"
                "ds_read_b64_tr_b16 a[130:131], %[vb0] offset:1024\n\t"
                "v_mfma_f32_16x16x32_bf16 %[c1110], a[160:163], a[104:107], %[c1110]\n\t"
                "ds_read_b64_tr_b16 a[132:133], %[vb0] offset:8192\n\t"
                "v_mfma_f32_16x16x32_bf16 %[c1110], a[164:167], a[108:111], %[c1110]\n\t"
                "ds_read_b64_tr_b16 a[134:135], %[vb0] offset:9216\n\t"
                "v_mfma_f32_16x16x32_bf16 %[c1120], a[160:163], a[112:115], %[c1120]\n\t"
                "ds_read_b64_tr_b16 a[136:137], %[vb1] offset:0\n\t"
                "v_mfma_f32_16x16x32_bf16 %[c1120], a[164:167], a[116:119], %[c1120]\n\t"
                "ds_read_b64_tr_b16 a[138:139], %[vb1] offset:1024\n\t"
                "v_mfma_f32_16x16x32_bf16 %[c1130], a[160:163], a[120:123], %[c1130]\n\t"
                "ds_read_b64_tr_b16 a[140:141], %[vb1] offset:8192\n\t"
                "v_mfma_f32_16x16x32_bf16 %[c1130], a[164:167], a[124:127], %[c1130]\n\t"
                "ds_read_b64_tr_b16 a[142:143], %[vb1] offset:9216\n\t"
                "v_mfma_f32_16x16x32_bf16 %[c1101], a[168:171], a[96:99], %[c1101]\n\t"
                "ds_read_b64_tr_b16 a[144:145], %[vb2] offset:0\n\t"
                "v_mfma_f32_16x16x32_bf16 %[c1101], a[172:175], a[100:103], %[c1101]\n\t"
                "ds_read_b64_tr_b16 a[146:147], %[vb2] offset:1024\n\t"
                "v_mfma_f32_16x16x32_bf16 %[c1111], a[168:171], a[104:107], %[c1111]\n\t"
                "ds_read_b64_tr_b16 a[148:149], %[vb2] offset:8192\n\t"
                "v_mfma_f32_16x16x32_bf16 %[c1111], a[172:175], a[108:111], %[c1111]\n\t"
                "ds_read_b64_tr_b16 a[150:151], %[vb2] offset:9216\n\t"
                "v_mfma_f32_16x16x32_bf16 %[c1121], a[168:171], a[112:115], %[c1121]\n\t"
                "ds_read_b64_tr_b16 a[152:153], %[vb3] offset:0\n\t"
                "v_mfma_f32_16x16x32_bf16 %[c1121], a[172:175], a[116:119], %[c1121]\n\t"
                "ds_read_b64_tr_b16 a[154:155], %[vb3] offset:1024\n\t"
                "v_mfma_f32_16x16x32_bf16 %[c1131], a[168:171], a[120:123], %[c1131]\n\t"
                "ds_read_b64_tr_b16 a[156:157], %[vb3] offset:8192\n\t"
                "v_mfma_f32_16x16x32_bf16 %[c1131], a[172:175], a[124:127], %[c1131]\n\t"
                "ds_read_b64_tr_b16 a[158:159], %[vb3] offset:9216\n\t"
                "s_add_u32 m0, %[ldsw], 0xc000\n\t"
                "v_mfma_f32_16x16x32_bf16 %[c1102], a[176:179], a[96:99], %[c1102]\n\t"
                "buffer_load_dwordx4 %[oa0], s[88:91], %[a3] offen offset:0 sc0 sc1 lds\n\t"
                "v_mfma_f32_16x16x32_bf16 %[c1102], a[180:183], a[100:103], %[c1102]\n\t"
                "v_mfma_f32_16x16x32_bf16 %[c1112], a[176:179], a[104:107], %[c1112]\n\t"
                "v_mfma_f32_16x16x32_bf16 %[c1112], a[180:183], a[108:111], %[c1112]\n\t"
                "v_mfma_f32_16x16x32_bf16 %[c1122], a[176:179], a[112:115], %[c1122]\n\t"
                "buffer_load_dwordx4 %[oa1], s[88:91], %[a3] offen offset:1024 sc0 sc1 lds\n\t"
                "v_mfma_f32_16x16x32_bf16 %[c1122], a[180:183], a[116:119], %[c1122]\n\t"
                "v_mfma_f32_16x16x32_bf16 %[c1132], a[176:179], a[120:123], %[c1132]\n\t"
                "v_mfma_f32_16x16x32_bf16 %[c1132], a[180:183], a[124:127], %[c1132]\n\t"
                "v_mfma_f32_16x16x32_bf16 %[c1103], a[184:187], a[96:99], %[c1103]\n\t"
                "buffer_load_dwordx4 %[oa2], s[88:91], %[a3] offen offset:2048 sc0 sc1 lds\n\t"
                "v_mfma_f32_16x16x32_bf16 %[c1103], a[188:191], a[100:103], %[c1103]\n\t"
                "v_mfma_f32_16x16x32_bf16 %[c1113], a[184:187], a[104:107], %[c1113]\n\t"
                "v_mfma_f32_16x16x32_bf16 %[c1113], a[188:191], a[108:111], %[c1113]\n\t"
                "s_waitcnt vmcnt(16) lgkmcnt(0)\n\t"
                "s_barrier\n\t"
                "v_mfma_f32_16x16x32_bf16 %[c1123], a[184:187], a[112:115], %[c1123]\n\t"
                "v_cmp_eq_u64 vcc, %[fpat2], %[pf]\n\t"
                "v_mfma_f32_16x16x32_bf16 %[c1123], a[188:191], a[116:119], %[c1123]\n\t"
                "buffer_load_dwordx4 %[oa3], s[88:91], %[a3] offen offset:3072 sc0 sc1 lds\n\t"
                "s_or_b64 %[rdy], %[rdy], vcc\n\t"
                "v_mfma_f32_16x16x32_bf16 %[c1133], a[184:187], a[120:123], %[c1133]\n\t"
                "s_andn2_b64 s[98:99], %[msk], %[rdy]\n\t"
                "s_cbranch_scc1 .Lnr%=\n\t"
                ".Lnb%=:\n\t"
                ".subsection 1\n\t"
                ".Lnr%=:\n\t"
                "s_memrealtime s[98:99]\n\t"
                ".Lnl%=:\n\t"
                "buffer_load_dwordx2 %[pf], %[voffF], %[rsf], %[sflag] offen sc0 sc1\n\t"
                "s_waitcnt vmcnt(0)\n\t"
                "s_memrealtime s[86:87]\n\t"
                "v_cmp_eq_u64 vcc, %[fpat2], %[pf]\n\t"
                "s_waitcnt lgkmcnt(0)\n\t"
                "s_sub_u32 s86, s86, s98\n\t"
                "s_or_b64 %[rdy], %[rdy], vcc\n\t"
                "s_andn2_b64 vcc, %[msk], %[rdy]\n\t"
                "s_cbranch_scc0 .Lnb%=\n\t"
                "s_bitcmp1_b64 %[rdy], 2\n\t"
                "s_cbranch_scc1 .Lnd%=\n\t"
                "s_cmp_gt_u32 s86, %[ddl]\n\t"
                "s_cbranch_scc1 .Lnd%=\n\t"
                "s_sleep 1\n\t"
                "s_branch .Lnl%=\n\t"
                ".Lnd%=:\n\t"
                "s_mov_b64 %[rdy], -1\n\t"
                "buffer_store_dword %[vfv], %[voffL], %[rsf], %[stmo] offen sc0 sc1\n\t"
                "buffer_store_dword %[vfv], %[voffL], %[rsf], %[stmo] offen offset:4 sc0 sc1\n\t"
                "s_waitcnt vmcnt(0)\n\t"
                "s_branch .Lnb%=\n\t"
                ".subsection 0\n\t"
                "v_mfma_f32_16x16x32_bf16 %[c1133], a[188:191], a[124:127], %[c1133]\n\t"
                : [c1100] "+v"(acc[1][1][0][0]), [c1110] "+v"(acc[1][1][1][0]), [c1120] "+v"(acc[1][1][2][0]), [c1130] "+v"(acc[1][1][3][0]), [c1101] "+v"(acc[1][1][0][1]), [c1111] "+v"(acc[1][1][1][1]), [c1121] "+v"(acc[1][1][2][1]), [c1131] "+v"(acc[1][1][3][1]), [c1102] "+v"(acc[1][1][0][2]), [oa0] "+v"(voffA[0]), [a3] "+s"(a3), [c1112] "+v"(acc[1][1][1][2]), [c1122] "+v"(acc[1][1][2][2]), [oa1] "+v"(voffA[1]), [c1132] "+v"(acc[1][1][3][2]), [c1103] "+v"(acc[1][1][0][3]), [oa2] "+v"(voffA[2]), [c1113] "+v"(acc[1][1][1][3]), [c1123] "+v"(acc[1][1][2][3]), [pf] "+v"(pf), [oa3] "+v"(voffA[3]), [rdy] "+s"(rdy), [c1133] "+v"(acc[1][1][3][3]), [msk] "+s"(msk), [rsa] "+{s[88:91]}"(rsa)
                : [vb0] "v"(vb[0]), [vb1] "v"(vb[1]), [vb2] "v"(vb[2]), [vb3] "v"(vb[3]), [ldsw] "s"(ldsw), [fpat2] "s"(fpat2), [voffF] "v"(voffF), [rsf] "s"(rsF), [sflag] "s"(sflag), [ddl] "s"((unsigned)DEADLINE), [vfv] "v"(vfv), [voffL] "v"(voffL), [stmo] "s"(stmo)
                : "memory", "scc", "vcc", "s98", "s99", "s86", "s87");
        } while (t < tend);
    }
    asm volatile("" :: "v"(pf), "v"(pd));
    {
        int tz = threadIdx.x; asm volatile("" : "+v"(tz));
        const unsigned ln = (unsigned)tz & 63u;
        unsigned a4 = 0u, b4 = 0u, a5 = 0u, b5 = 0u;
        unsigned vax[2], vbx[4], vbl[4];
#pragma unroll
        for (int k = 0; k < 2; ++k) vax[k] = va[k] + 131072u;
#pragma unroll
        for (int n = 0; n < 4; ++n) vbx[n] = vb[n] + 65536u;
        const __amdgpu_buffer_rsrc_t xr = __builtin_amdgcn_make_buffer_rsrc(myws + WS_XBUF, 0, 256u * 65536u, 0x00020000);
        const __amdgpu_buffer_rsrc_t ro = __builtin_amdgcn_make_buffer_rsrc(p.out, 0, 16u << 20, 0x00020000);
        const unsigned voffX = (unsigned)(bid * 4 + wid) * 16384u + ln * 16u, voffY = voffX ^ 65536u;
#define XG 1024
#define OAUX 0
        unsigned voffO, sd0, sd1;
#define SETUP0 do { voffO = ((unsigned)(mt * 256 + kh * 128 + wr * 64) + (ln & 15u)) * 16384u + ((unsigned)(nt * 256 + wc * 64) + ((ln >> 4) & 1u) * 16u + (ln >> 5) * 8u) * 2u; \
        asm volatile("" : "+v"(voffO)); } while (0)
#define SETUP1 do { _Pragma("unroll") for (int n = 0; n < 4; ++n) vbl[n] = vb[n] - 65536u; asm volatile("" : "+v"(vbl[0]), "+v"(vbl[1]), "+v"(vbl[2]), "+v"(vbl[3])); } while (0)
        unsigned badm, tago, tagm = 0xfffe0000u;
        const unsigned tag = call & 1u, tagr = 0x10000u - (tag << 16);
#define SETUP2 do { const unsigned one = ((ln >> 4) == ((ln >> 2) & 3u)) ? ((ln & 1u) ? 0x3f800000u : 0x00003f80u) : 0u; \
        sd0 = (ln & 2u) ? 0u : one; sd1 = (ln & 2u) ? one : 0u; asm volatile("" : "+v"(sd0), "+v"(sd1)); } while (0)
        f32x2 gc1 = {GELU_C1, GELU_C1}, gc3 = {GELU_C3, GELU_C3};
        asm volatile("" : "+s"(gc1), "+s"(gc3), "+v"(tagm));
        u32x4 pv[2][8], so_;
        f32x4 s0_, s1_;
        f32x2 ez[2][4];
#define TAGV(v) __builtin_bit_cast(float, ((__builtin_bit_cast(unsigned, v) + tagr) & tagm) | tago)
#define SENDST(g) __builtin_amdgcn_raw_buffer_store_b128(so_, xr, voffX + (g) * XG, 0, 16)
#define ARD(d, i) asm volatile("v_accvgpr_read_b32 %0, a[%c1]" : "=v"(d) : "i"(i))
#define PVLD1(bj, g) asm volatile("buffer_load_dwordx4 %0, %1, %2, %3 offen offset:%4 sc1" : "=v"(pv[bj][g]) : "v"(voffY), "s"(xr), "s"(XG == 1024 ? ((bj) * 8 + (g)) / 4 * 4096u : ((bj) * 8 + (g)) * (unsigned)XG), "i"(XG == 1024 ? ((g) & 3) * 1024 : 0) : "memory")
#define PVLD(bj) do { PVLD1(bj, 0); PVLD1(bj, 1); PVLD1(bj, 2); PVLD1(bj, 3); PVLD1(bj, 4); PVLD1(bj, 5); PVLD1(bj, 6); PVLD1(bj, 7); } while (0)
#define PVPIN(bj) asm volatile("" : "+v"(pv[bj][0]), "+v"(pv[bj][1]), "+v"(pv[bj][2]), "+v"(pv[bj][3]), "+v"(pv[bj][4]), "+v"(pv[bj][5]), "+v"(pv[bj][6]), "+v"(pv[bj][7]) :: "memory")
#define PMISS(bj) __builtin_amdgcn_ballot_w64(((((pv[bj][0].x & pv[bj][1].x & pv[bj][2].x & pv[bj][3].x & pv[bj][4].x & pv[bj][5].x & pv[bj][6].x & pv[bj][7].x) ^ tag) | \
                                                ((pv[bj][0].x | pv[bj][1].x | pv[bj][2].x | pv[bj][3].x | pv[bj][4].x | pv[bj][5].x | pv[bj][6].x | pv[bj][7].x) ^ tag)) & 1u) != 0u)
#define NOW(t_) asm volatile("s_memrealtime %0\n\ts_waitcnt lgkmcnt(0)" : "=s"(t_) :: "memory")
#define ADDP(bj) do { PVPIN(bj); \
        if (__builtin_expect(PMISS(bj) != 0ull, 0)) { \
            unsigned long long t0_, t1_; NOW(t0_); \
            for (;;) { unsigned tv_; PVLD(bj); RCNT(bj); \
                asm volatile("buffer_load_dword %0, %1, %2, %3 offen sc0 sc1\n\ts_waitcnt vmcnt(0)" : "=v"(tv_) : "v"(vzero), "s"(rsF), "s"(stmo) : "memory"); \
                PVPIN(bj); \
                if (PMISS(bj) == 0ull) break; \
                NOW(t1_); \
                if ((unsigned)__builtin_amdgcn_readfirstlane(tv_) == fpat || t1_ - t0_ > DEADLINE) { \
                    asm volatile("buffer_store_dword %0, %1, %2, %3 offen sc0 sc1\n\tbuffer_store_dword %0, %1, %2, %3 offen offset:4 sc0 sc1\n\ts_waitcnt vmcnt(0)" :: "v"(vfv), "v"(voffL), "s"(rsF), "s"(stmo) : "memory"); \
                    badm = 0x7fc07fc0u; break; } \
                __builtin_amdgcn_s_sleep(1); } } \
        const u32x4 e0_ = {sd0, sd1, badm, 0u}, e1_ = {badm, 0u, sd0, sd1}; \
        asm volatile("s_nop 1\n\tv_mfma_f32_16x16x32_bf16 %0, %24, %16, %0\n\tv_mfma_f32_16x16x32_bf16 %1, %25, %16, %1\n\tv_mfma_f32_16x16x32_bf16 %2, %24, %17, %2\n\tv_mfma_f32_16x16x32_bf16 %3, %25, %17, %3\n\tv_mfma_f32_16x16x32_bf16 %4, %24, %18, %4\n\tv_mfma_f32_16x16x32_bf16 %5, %25, %18, %5\n\tv_mfma_f32_16x16x32_bf16 %6, %24, %19, %6\n\tv_mfma_f32_16x16x32_bf16 %7, %25, %19, %7\n\tv_mfma_f32_16x16x32_bf16 %8, %24, %20, %8\n\tv_mfma_f32_16x16x32_bf16 %9, %25, %20, %9\n\tv_mfma_f32_16x16x32_bf16 %10, %24, %21, %10\n\tv_mfma_f32_16x16x32_bf16 %11, %25, %21, %11\n\tv_mfma_f32_16x16x32_bf16 %12, %24, %22, %12\n\tv_mfma_f32_16x16x32_bf16 %13, %25, %22, %13\n\tv_mfma_f32_16x16x32_bf16 %14, %24, %23, %14\n\tv_mfma_f32_16x16x32_bf16 %15, %25, %23, %15\n\ts_nop 7" \
            : "+v"(acc[1][bj][0][0]), "+v"(acc[1][bj][0][1]), "+v"(acc[1][bj][0][2]), "+v"(acc[1][bj][0][3]), "+v"(acc[1][bj][1][0]), "+v"(acc[1][bj][1][1]), "+v"(acc[1][bj][1][2]), "+v"(acc[1][bj][1][3]), "+v"(acc[1][bj][2][0]), "+v"(acc[1][bj][2][1]), "+v"(acc[1][bj][2][2]), "+v"(acc[1][bj][2][3]), "+v"(acc[1][bj][3][0]), "+v"(acc[1][bj][3][1]), "+v"(acc[1][bj][3][2]), "+v"(acc[1][bj][3][3]) \
            : "v"(pv[bj][0]), "v"(pv[bj][1]), "v"(pv[bj][2]), "v"(pv[bj][3]), "v"(pv[bj][4]), "v"(pv[bj][5]), "v"(pv[bj][6]), "v"(pv[bj][7]), "v"(e0_), "v"(e1_)); } while (0)
#define YV(bj, u, i) ((f32x2){acc[1][bj][(u) >> 1][((u) & 1) * 2 + ((i) >> 1)][((i) & 1) * 2], acc[1][bj][(u) >> 1][((u) & 1) * 2 + ((i) >> 1)][((i) & 1) * 2 + 1]})
#define E1(bj, u) do { _Pragma("unroll") for (int i = 0; i < 4; ++i) { const f32x2 y = YV(bj, u, i); ez[(u) & 1][i] = y * __builtin_elementwise_fma(y * y, gc3, gc1); } } while (0)
#define E2(u, j) ez[(u) & 1][(j) >> 1][(j) & 1] = __builtin_amdgcn_exp2f(ez[(u) & 1][(j) >> 1][(j) & 1])
#define E3(u) do { _Pragma("unroll") for (int i = 0; i < 4; ++i) ez[(u) & 1][i] += 1.0f; } while (0)
#define E4(u, j) ez[(u) & 1][(j) >> 1][(j) & 1] = __builtin_amdgcn_rcpf(ez[(u) & 1][(j) >> 1][(j) & 1])
#define E5(bj, u) do { unsigned c_[4]; \
        _Pragma("unroll") for (int i = 0; i < 4; ++i) { const f32x2 g = YV(bj, u, i) * ez[(u) & 1][i]; c_[i] = cvt_pk_bf16(g[0], g[1]); } \
        const auto r0 = __builtin_amdgcn_permlane16_swap(c_[0], c_[2], false, false); \
        const auto r1 = __builtin_amdgcn_permlane16_swap(c_[1], c_[3], false, false); \
        u32x4 o; o.x = r0[0]; o.y = r1[0]; o.z = r0[1]; o.w = r1[1]; \
        __builtin_amdgcn_raw_buffer_store_b128(o, ro, voffO + ((u) >> 1) * (16u * 16384u) + (bj) * 256 + ((u) & 1) * 64, 0, OAUX); } while (0)
        ST(1 + (t >> 1));
        asm volatile(
            "s_add_u32 %[tmp], %[t], 3\n\t"
            "v_mfma_f32_16x16x32_bf16 %[c0000], a[128:131], a[64:67], %[c0000]\n\t"
            "ds_read_b64_tr_b16 a[160:161], %[vb0] offset:16384\n\t"
            "v_mfma_f32_16x16x32_bf16 %[c0000], a[132:135], a[68:71], %[c0000]\n\t"
            "ds_read_b64_tr_b16 a[162:163], %[vb0] offset:17408\n\t"
            "v_readlane_b32 %[a3], %[tabA], %[tmp]\n\t"
            "v_mfma_f32_16x16x32_bf16 %[c0010], a[128:131], a[72:75], %[c0010]\n\t"
            "ds_read_b64_tr_b16 a[164:165], %[vb0] offset:24576\n\t"
            "v_mfma_f32_16x16x32_bf16 %[c0010], a[132:135], a[76:79], %[c0010]\n\t"
            "ds_read_b64_tr_b16 a[166:167], %[vb0] offset:25600\n\t"
            "v_readlane_b32 %[b3], %[tabB], %[tmp]\n\t"
            "v_mfma_f32_16x16x32_bf16 %[c0020], a[128:131], a[80:83], %[c0020]\n\t"
            "ds_read_b64_tr_b16 a[168:169], %[vb1] offset:16384\n\t"
            "v_mfma_f32_16x16x32_bf16 %[c0020], a[132:135], a[84:87], %[c0020]\n\t"
            "ds_read_b64_tr_b16 a[170:171], %[vb1] offset:17408\n\t"
            "s_add_u32 %[tmp], %[t], 5\n\t"
            "v_mfma_f32_16x16x32_bf16 %[c0030], a[128:131], a[88:91], %[c0030]\n\t"
            "ds_read_b64_tr_b16 a[172:173], %[vb1] offset:24576\n\t"
            "v_mfma_f32_16x16x32_bf16 %[c0030], a[132:135], a[92:95], %[c0030]\n\t"
            "ds_read_b64_tr_b16 a[174:175], %[vb1] offset:25600\n\t"
            "v_readlane_b32 %[a5], %[tabA], %[tmp]\n\t"
            "v_mfma_f32_16x16x32_bf16 %[c0001], a[136:139], a[64:67], %[c0001]\n\t"
            "ds_read_b64_tr_b16 a[176:177], %[vb2] offset:16384\n\t"
            "v_mfma_f32_16x16x32_bf16 %[c0001], a[140:143], a[68:71], %[c0001]\n\t"
            "ds_read_b64_tr_b16 a[178:179], %[vb2] offset:17408\n\t"
            "v_readlane_b32 %[b5], %[tabB], %[tmp]\n\t"
            "v_mfma_f32_16x16x32_bf16 %[c0011], a[136:139], a[72:75], %[c0011]\n\t"
            "ds_read_b64_tr_b16 a[180:181], %[vb2] offset:24576\n\t"
            "v_mfma_f32_16x16x32_bf16 %[c0011], a[140:143], a[76:79], %[c0011]\n\t"
            "ds_read_b64_tr_b16 a[182:183], %[vb2] offset:25600\n\t"
            "s_add_u32 %[tmp], %[t], 4\n\t"
            "v_mfma_f32_16x16x32_bf16 %[c0021], a[136:139], a[80:83], %[c0021]\n\t"
            "ds_read_b64_tr_b16 a[184:185], %[vb3] offset:16384\n\t"
            "v_mfma_f32_16x16x32_bf16 %[c0021], a[140:143], a[84:87], %[c0021]\n\t"
            "ds_read_b64_tr_b16 a[186:187], %[vb3] offset:17408\n\t"
            "v_readlane_b32 %[a4], %[tabA], %[tmp]\n\t"
            "v_mfma_f32_16x16x32_bf16 %[c0031], a[136:139], a[88:91], %[c0031]\n\t"
            "ds_read_b64_tr_b16 a[188:189], %[vb3] offset:24576\n\t"
            "v_mfma_f32_16x16x32_bf16 %[c0031], a[140:143], a[92:95], %[c0031]\n\t"
            "ds_read_b64_tr_b16 a[190:191], %[vb3] offset:25600\n\t"
            "s_add_u32 m0, %[ldsw], 0x0\n\t"
            "v_mfma_f32_16x16x32_bf16 %[c0002], a[144:147], a[64:67], %[c0002]\n\t"
            "buffer_load_dwordx4 %[oa0], s[88:91], %[a2] offen offset:0 sc0 sc1 lds\n\t"
            "v_mfma_f32_16x16x32_bf16 %[c0002], a[148:151], a[68:71], %[c0002]\n\t"
            "v_readlane_b32 %[b4], %[tabB], %[tmp]\n\t"
            "v_mfma_f32_16x16x32_bf16 %[c0012], a[144:147], a[72:75], %[c0012]\n\t"
            "buffer_load_dwordx4 %[oa1], s[88:91], %[a2] offen offset:1024 sc0 sc1 lds\n\t"
            "v_mfma_f32_16x16x32_bf16 %[c0012], a[148:151], a[76:79], %[c0012]\n\t"
            "v_mfma_f32_16x16x32_bf16 %[c0022], a[144:147], a[80:83], %[c0022]\n\t"
            "buffer_load_dwordx4 %[oa2], s[88:91], %[a2] offen offset:2048 sc0 sc1 lds\n\t"
            "v_mfma_f32_16x16x32_bf16 %[c0022], a[148:151], a[84:87], %[c0022]\n\t"
            "v_mfma_f32_16x16x32_bf16 %[c0032], a[144:147], a[88:91], %[c0032]\n\t"
            "buffer_load_dwordx4 %[oa3], s[88:91], %[a2] offen offset:3072 sc0 sc1 lds\n\t"
            "v_mfma_f32_16x16x32_bf16 %[c0032], a[148:151], a[92:95], %[c0032]\n\t"
            "v_mfma_f32_16x16x32_bf16 %[c0003], a[152:155], a[64:67], %[c0003]\n\t"
            "v_mfma_f32_16x16x32_bf16 %[c0003], a[156:159], a[68:71], %[c0003]\n\t"
            "v_mfma_f32_16x16x32_bf16 %[c0013], a[152:155], a[72:75], %[c0013]\n\t"
            "v_mfma_f32_16x16x32_bf16 %[c0013], a[156:159], a[76:79], %[c0013]\n\t"
            "s_waitcnt lgkmcnt(0)\n\t"
            "v_mfma_f32_16x16x32_bf16 %[c0023], a[152:155], a[80:83], %[c0023]\n\t"
            "v_mfma_f32_16x16x32_bf16 %[c0023], a[156:159], a[84:87], %[c0023]\n\t"
            "v_mfma_f32_16x16x32_bf16 %[c0033], a[152:155], a[88:91], %[c0033]\n\t"
            "v_mfma_f32_16x16x32_bf16 %[c0033], a[156:159], a[92:95], %[c0033]\n\t"
            : [tmp] "+s"(tmp), [t] "+s"(t), [c0000] "+v"(acc[0][0][0][0]), [a3] "+s"(a3), [c0010] "+v"(acc[0][0][1][0]), [b3] "+s"(b3), [c0020] "+v"(acc[0][0][2][0]), [c0030] "+v"(acc[0][0][3][0]), [a5] "+s"(a5), [c0001] "+v"(acc[0][0][0][1]), [b5] "+s"(b5), [c0011] "+v"(acc[0][0][1][1]), [c0021] "+v"(acc[0][0][2][1]), [a4] "+s"(a4), [c0031] "+v"(acc[0][0][3][1]), [c0002] "+v"(acc[0][0][0][2]), [oa0] "+v"(voffA[0]), [a2] "+s"(a2), [b4] "+s"(b4), [c0012] "+v"(acc[0][0][1][2]), [oa1] "+v"(voffA[1]), [c0022] "+v"(acc[0][0][2][2]), [oa2] "+v"(voffA[2]), [c0032] "+v"(acc[0][0][3][2]), [oa3] "+v"(voffA[3]), [c0003] "+v"(acc[0][0][0][3]), [c0013] "+v"(acc[0][0][1][3]), [c0023] "+v"(acc[0][0][2][3]), [c0033] "+v"(acc[0][0][3][3]), [rsa] "+{s[88:91]}"(rsa)
            : [vb0] "v"(vb[0]), [tabA] "v"(tabA), [tabB] "v"(tabB), [vb1] "v"(vb[1]), [vb2] "v"(vb[2]), [vb3] "v"(vb[3]), [ldsw] "s"(ldsw)
            : "memory", "scc");
        asm volatile(
            "v_mfma_f32_16x16x32_bf16 a[0:3], a[160:163], a[64:67], a[0:3]\n\t"
            "ds_read_b128 a[96:99], %[va0] offset:16384\n\t"
            "s_add_u32 m0, %[ldsw], 0x20000\n\t"
            "v_mfma_f32_16x16x32_bf16 a[0:3], a[164:167], a[68:71], a[0:3]\n\t"
            "ds_read_b128 a[100:103], %[va1] offset:16384\n\t"
            "buffer_load_dwordx4 %[oa0], s[88:91], %[a3] offen offset:0 sc0 sc1 lds\n\t"
            "v_mfma_f32_16x16x32_bf16 a[16:19], a[160:163], a[72:75], a[16:19]\n\t"
            "ds_read_b128 a[104:107], %[va0] offset:18432\n\t"
            "v_mfma_f32_16x16x32_bf16 a[16:19], a[164:167], a[76:79], a[16:19]\n\t"
            "ds_read_b128 a[108:111], %[va1] offset:18432\n\t"
            "buffer_load_dwordx4 %[oa1], s[88:91], %[a3] offen offset:1024 sc0 sc1 lds\n\t"
            "v_mfma_f32_16x16x32_bf16 a[32:35], a[160:163], a[80:83], a[32:35]\n\t"
            "ds_read_b128 a[112:115], %[va0] offset:20480\n\t"
            "v_mfma_f32_16x16x32_bf16 a[32:35], a[164:167], a[84:87], a[32:35]\n\t"
            "ds_read_b128 a[116:119], %[va1] offset:20480\n\t"
            "buffer_load_dwordx4 %[oa2], s[88:91], %[a3] offen offset:2048 sc0 sc1 lds\n\t"
            "v_mfma_f32_16x16x32_bf16 a[48:51], a[160:163], a[88:91], a[48:51]\n\t"
            "ds_read_b128 a[120:123], %[va0] offset:22528\n\t"
            "v_mfma_f32_16x16x32_bf16 a[48:51], a[164:167], a[92:95], a[48:51]\n\t"
            "ds_read_b128 a[124:127], %[va1] offset:22528\n\t"
            "buffer_load_dwordx4 %[oa3], s[88:91], %[a3] offen offset:3072 sc0 sc1 lds\n\t"
            "v_mfma_f32_16x16x32_bf16 a[4:7], a[168:171], a[64:67], a[4:7]\n\t"
            "v_mfma_f32_16x16x32_bf16 a[4:7], a[172:175], a[68:71], a[4:7]\n\t"
            "s_add_u32 %[a2], %[a2], %[dl]\n\t"
            "v_mfma_f32_16x16x32_bf16 a[20:23], a[168:171], a[72:75], a[20:23]\n\t"
            "v_mfma_f32_16x16x32_bf16 a[20:23], a[172:175], a[76:79], a[20:23]\n\t"
            "s_add_u32 %[a3], %[a3], %[dl]\n\t"
            "v_mfma_f32_16x16x32_bf16 a[36:39], a[168:171], a[80:83], a[36:39]\n\t"
            "s_add_u32 %[a4], %[a4], %[dl]\n\t"
            "v_mfma_f32_16x16x32_bf16 a[36:39], a[172:175], a[84:87], a[36:39]\n\t"
            "s_add_u32 %[a5], %[a5], %[dl]\n\t"
            "v_mfma_f32_16x16x32_bf16 a[52:55], a[168:171], a[88:91], a[52:55]\n\t"
            "v_mfma_f32_16x16x32_bf16 a[52:55], a[172:175], a[92:95], a[52:55]\n\t"
            "s_add_u32 m0, %[ldsw], 0x10000\n\t"
            "v_mfma_f32_16x16x32_bf16 a[8:11], a[176:179], a[64:67], a[8:11]\n\t"
            "buffer_load_dwordx4 %[ob0], %[rsw], %[b2] offen offset:0  lds\n\t"
            "v_mfma_f32_16x16x32_bf16 a[8:11], a[180:183], a[68:71], a[8:11]\n\t"
            "v_mfma_f32_16x16x32_bf16 a[24:27], a[176:179], a[72:75], a[24:27]\n\t"
            "buffer_load_dwordx4 %[ob1], %[rsw], %[b2] offen offset:1024  lds\n\t"
            "v_mfma_f32_16x16x32_bf16 a[24:27], a[180:183], a[76:79], a[24:27]\n\t"
            "v_mfma_f32_16x16x32_bf16 a[40:43], a[176:179], a[80:83], a[40:43]\n\t"
            "buffer_load_dwordx4 %[ob2], %[rsw], %[b2] offen offset:2048  lds\n\t"
            "v_mfma_f32_16x16x32_bf16 a[40:43], a[180:183], a[84:87], a[40:43]\n\t"
            "v_mfma_f32_16x16x32_bf16 a[56:59], a[176:179], a[88:91], a[56:59]\n\t"
            "buffer_load_dwordx4 %[ob3], %[rsw], %[b2] offen offset:3072  lds\n\t"
            "v_mfma_f32_16x16x32_bf16 a[56:59], a[180:183], a[92:95], a[56:59]\n\t"
            "v_mfma_f32_16x16x32_bf16 a[12:15], a[184:187], a[64:67], a[12:15]\n\t"
            "buffer_load_dwordx2 %[pf], %[voffF], %[rsf], %[sflag] offen sc0 sc1\n\t"
            "v_mfma_f32_16x16x32_bf16 a[12:15], a[188:191], a[68:71], a[12:15]\n\t"
            "v_mfma_f32_16x16x32_bf16 a[28:31], a[184:187], a[72:75], a[28:31]\n\t"
            "v_mfma_f32_16x16x32_bf16 a[28:31], a[188:191], a[76:79], a[28:31]\n\t"
            "s_waitcnt vmcnt(22) lgkmcnt(0)\n\t"
            "s_barrier\n\t"
            "v_mfma_f32_16x16x32_bf16 a[44:47], a[184:187], a[80:83], a[44:47]\n\t"
            : [oa0] "+v"(voffA[0]), [a3] "+s"(a3), [oa1] "+v"(voffA[1]), [oa2] "+v"(voffA[2]), [oa3] "+v"(voffA[3]), [a2] "+s"(a2), [dl] "+s"(dl), [a4] "+s"(a4), [a5] "+s"(a5), [b2] "+s"(b2), [pf] "+v"(pf), [rsa] "+{s[88:91]}"(rsa)
            : [va0] "v"(va[0]), [ldsw] "s"(ldsw), [va1] "v"(va[1]), [ob0] "v"(voffB[0]), [rsw] "s"(rsW), [ob1] "v"(voffB[1]), [ob2] "v"(voffB[2]), [ob3] "v"(voffB[3]), [voffF] "v"(voffF), [rsf] "s"(rsF), [sflag] "s"(sflag)
            : "memory", "scc");
        SCHED; asm volatile("" :: "v"(pr)); SCHED;
        asm volatile(
            "v_mfma_f32_16x16x32_bf16 a[44:47], a[188:191], a[84:87], a[44:47]\n\t"
            "v_mfma_f32_16x16x32_bf16 a[60:63], a[184:187], a[88:91], a[60:63]\n\t"
            "v_mfma_f32_16x16x32_bf16 a[60:63], a[188:191], a[92:95], a[60:63]\n\t"
            :
            :
            : "memory", "scc");
        asm volatile(
            "v_mfma_f32_16x16x32_bf16 %[c1100], a[160:163], a[96:99], %[c1100]\n\t"
            "ds_read_b128 a[64:67], %[va0] offset:32768\n\t"
            "s_add_u32 m0, %[ldsw], 0x24000\n\t"
            "v_mfma_f32_16x16x32_bf16 %[c1100], a[164:167], a[100:103], %[c1100]\n\t"
            "ds_read_b128 a[68:71], %[va1] offset:32768\n\t"
            "buffer_load_dwordx4 %[ob0], %[rsw], %[b3] offen offset:0  lds\n\t"
            "v_mfma_f32_16x16x32_bf16 %[c1110], a[160:163], a[104:107], %[c1110]\n\t"
            "ds_read_b128 a[72:75], %[va0] offset:34816\n\t"
            "v_mfma_f32_16x16x32_bf16 %[c1110], a[164:167], a[108:111], %[c1110]\n\t"
            "ds_read_b128 a[76:79], %[va1] offset:34816\n\t"
            "buffer_load_dwordx4 %[ob1], %[rsw], %[b3] offen offset:1024  lds\n\t"
            "v_mfma_f32_16x16x32_bf16 %[c1120], a[160:163], a[112:115], %[c1120]\n\t"
            "ds_read_b128 a[80:83], %[va0] offset:36864\n\t"
            "v_mfma_f32_16x16x32_bf16 %[c1120], a[164:167], a[116:119], %[c1120]\n\t"
            "ds_read_b128 a[84:87], %[va1] offset:36864\n\t"
            "buffer_load_dwordx4 %[ob2], %[rsw], %[b3] offen offset:2048  lds\n\t"
            "v_mfma_f32_16x16x32_bf16 %[c1130], a[160:163], a[120:123], %[c1130]\n\t"
            "ds_read_b128 a[88:91], %[va0] offset:38912\n\t"
            "v_mfma_f32_16x16x32_bf16 %[c1130], a[164:167], a[124:127], %[c1130]\n\t"
            "ds_read_b128 a[92:95], %[va1] offset:38912\n\t"
            "buffer_load_dwordx4 %[ob3], %[rsw], %[b3] offen offset:3072  lds\n\t"
            "v_mfma_f32_16x16x32_bf16 %[c1101], a[168:171], a[96:99], %[c1101]\n\t"
            "v_mfma_f32_16x16x32_bf16 %[c1101], a[172:175], a[100:103], %[c1101]\n\t"
            "v_mfma_f32_16x16x32_bf16 %[c1111], a[168:171], a[104:107], %[c1111]\n\t"
            "v_mfma_f32_16x16x32_bf16 %[c1111], a[172:175], a[108:111], %[c1111]\n\t"
            "v_mfma_f32_16x16x32_bf16 %[c1121], a[168:171], a[112:115], %[c1121]\n\t"
            "v_mfma_f32_16x16x32_bf16 %[c1121], a[172:175], a[116:119], %[c1121]\n\t"
            "v_mfma_f32_16x16x32_bf16 %[c1131], a[168:171], a[120:123], %[c1131]\n\t"
            "v_mfma_f32_16x16x32_bf16 %[c1131], a[172:175], a[124:127], %[c1131]\n\t"
            "s_add_u32 m0, %[ldsw], 0x13f00\n\t"
            "v_mfma_f32_16x16x32_bf16 %[c1102], a[176:179], a[96:99], %[c1102]\n\t"
            "buffer_load_dwordx4 %[ob0], %[rsw], %[b2] offen offset:256  lds\n\t"
            "v_mfma_f32_16x16x32_bf16 %[c1102], a[180:183], a[100:103], %[c1102]\n\t"
            "v_mfma_f32_16x16x32_bf16 %[c1112], a[176:179], a[104:107], %[c1112]\n\t"
            "buffer_load_dwordx4 %[ob1], %[rsw], %[b2] offen offset:1280  lds\n\t"
            "v_mfma_f32_16x16x32_bf16 %[c1112], a[180:183], a[108:111], %[c1112]\n\t"
            "v_mfma_f32_16x16x32_bf16 %[c1122], a[176:179], a[112:115], %[c1122]\n\t"
            "buffer_load_dwordx4 %[ob2], %[rsw], %[b2] offen offset:2304  lds\n\t"
            "v_mfma_f32_16x16x32_bf16 %[c1122], a[180:183], a[116:119], %[c1122]\n\t"
            "v_mfma_f32_16x16x32_bf16 %[c1132], a[176:179], a[120:123], %[c1132]\n\t"
            "buffer_load_dwordx4 %[ob3], %[rsw], %[b2] offen offset:3328  lds\n\t"
            "v_mfma_f32_16x16x32_bf16 %[c1132], a[180:183], a[124:127], %[c1132]\n\t"
            "v_mfma_f32_16x16x32_bf16 %[c1103], a[184:187], a[96:99], %[c1103]\n\t"
            "v_mfma_f32_16x16x32_bf16 %[c1103], a[188:191], a[100:103], %[c1103]\n\t"
            "v_mfma_f32_16x16x32_bf16 %[c1113], a[184:187], a[104:107], %[c1113]\n\t"
            "v_mfma_f32_16x16x32_bf16 %[c1113], a[188:191], a[108:111], %[c1113]\n\t"
            "s_waitcnt lgkmcnt(0)\n\t"
            "v_mfma_f32_16x16x32_bf16 %[c1123], a[184:187], a[112:115], %[c1123]\n\t"
            "v_mfma_f32_16x16x32_bf16 %[c1123], a[188:191], a[116:119], %[c1123]\n\t"
            "v_mfma_f32_16x16x32_bf16 %[c1133], a[184:187], a[120:123], %[c1133]\n\t"
            "v_mfma_f32_16x16x32_bf16 %[c1133], a[188:191], a[124:127], %[c1133]\n\t"
            : [c1100] "+v"(acc[1][1][0][0]), [b3] "+s"(b3), [c1110] "+v"(acc[1][1][1][0]), [c1120] "+v"(acc[1][1][2][0]), [c1130] "+v"(acc[1][1][3][0]), [c1101] "+v"(acc[1][1][0][1]), [c1111] "+v"(acc[1][1][1][1]), [c1121] "+v"(acc[1][1][2][1]), [c1131] "+v"(acc[1][1][3][1]), [c1102] "+v"(acc[1][1][0][2]), [b2] "+s"(b2), [c1112] "+v"(acc[1][1][1][2]), [c1122] "+v"(acc[1][1][2][2]), [c1132] "+v"(acc[1][1][3][2]), [c1103] "+v"(acc[1][1][0][3]), [c1113] "+v"(acc[1][1][1][3]), [c1123] "+v"(acc[1][1][2][3]), [c1133] "+v"(acc[1][1][3][3])
            : [va0] "v"(va[0]), [ldsw] "s"(ldsw), [va1] "v"(va[1]), [ob0] "v"(voffB[0]), [rsw] "s"(rsW), [ob1] "v"(voffB[1]), [ob2] "v"(voffB[2]), [ob3] "v"(voffB[3])
            : "memory", "scc");
        asm volatile(
            "v_mfma_f32_16x16x32_bf16 %[c1000], a[128:131], a[96:99], %[c1000]\n\t"
            "ds_read_b64_tr_b16 a[160:161], %[vb0] offset:49152\n\t"
            "v_mfma_f32_16x16x32_bf16 %[c1000], a[132:135], a[100:103], %[c1000]\n\t"
            "ds_read_b64_tr_b16 a[162:163], %[vb0] offset:50176\n\t"
            "v_mfma_f32_16x16x32_bf16 %[c1010], a[128:131], a[104:107], %[c1010]\n\t"
            "ds_read_b64_tr_b16 a[164:165], %[vb0] offset:57344\n\t"
            "v_mfma_f32_16x16x32_bf16 %[c1010], a[132:135], a[108:111], %[c1010]\n\t"
            "ds_read_b64_tr_b16 a[166:167], %[vb0] offset:58368\n\t"
            "v_mfma_f32_16x16x32_bf16 %[c1020], a[128:131], a[112:115], %[c1020]\n\t"
            "ds_read_b64_tr_b16 a[168:169], %[vb1] offset:49152\n\t"
            "v_mfma_f32_16x16x32_bf16 %[c1020], a[132:135], a[116:119], %[c1020]\n\t"
            "ds_read_b64_tr_b16 a[170:171], %[vb1] offset:50176\n\t"
            "v_mfma_f32_16x16x32_bf16 %[c1030], a[128:131], a[120:123], %[c1030]\n\t"
            "ds_read_b64_tr_b16 a[172:173], %[vb1] offset:57344\n\t"
            "v_mfma_f32_16x16x32_bf16 %[c1030], a[132:135], a[124:127], %[c1030]\n\t"
            "ds_read_b64_tr_b16 a[174:175], %[vb1] offset:58368\n\t"
            "v_mfma_f32_16x16x32_bf16 %[c1001], a[136:139], a[96:99], %[c1001]\n\t"
            "ds_read_b64_tr_b16 a[176:177], %[vb2] offset:49152\n\t"
            "v_mfma_f32_16x16x32_bf16 %[c1001], a[140:143], a[100:103], %[c1001]\n\t"
            "ds_read_b64_tr_b16 a[178:179], %[vb2] offset:50176\n\t"
            "v_mfma_f32_16x16x32_bf16 %[c1011], a[136:139], a[104:107], %[c1011]\n\t"
            "ds_read_b64_tr_b16 a[180:181], %[vb2] offset:57344\n\t"
            "v_mfma_f32_16x16x32_bf16 %[c1011], a[140:143], a[108:111], %[c1011]\n\t"
            "ds_read_b64_tr_b16 a[182:183], %[vb2] offset:58368\n\t"
            "v_mfma_f32_16x16x32_bf16 %[c1021], a[136:139], a[112:115], %[c1021]\n\t"
            "ds_read_b64_tr_b16 a[184:185], %[vb3] offset:49152\n\t"
            "v_mfma_f32_16x16x32_bf16 %[c1021], a[140:143], a[116:119], %[c1021]\n\t"
            "ds_read_b64_tr_b16 a[186:187], %[vb3] offset:50176\n\t"
            "v_mfma_f32_16x16x32_bf16 %[c1031], a[136:139], a[120:123], %[c1031]\n\t"
            "ds_read_b64_tr_b16 a[188:189], %[vb3] offset:57344\n\t"
            "v_mfma_f32_16x16x32_bf16 %[c1031], a[140:143], a[124:127], %[c1031]\n\t"
            "ds_read_b64_tr_b16 a[190:191], %[vb3] offset:58368\n\t"
            "s_add_u32 m0, %[ldsw], 0x4000\n\t"
            "v_mfma_f32_16x16x32_bf16 %[c1002], a[144:147], a[96:99], %[c1002]\n\t"
            "buffer_load_dwordx4 %[oa0], s[88:91], %[a2] offen offset:0 sc0 sc1 lds\n\t"
            "v_mfma_f32_16x16x32_bf16 %[c1002], a[148:151], a[100:103], %[c1002]\n\t"
            "v_mfma_f32_16x16x32_bf16 %[c1012], a[144:147], a[104:107], %[c1012]\n\t"
            "buffer_load_dwordx4 %[oa1], s[88:91], %[a2] offen offset:1024 sc0 sc1 lds\n\t"
            "v_mfma_f32_16x16x32_bf16 %[c1012], a[148:151], a[108:111], %[c1012]\n\t"
            "v_mfma_f32_16x16x32_bf16 %[c1022], a[144:147], a[112:115], %[c1022]\n\t"
            "buffer_load_dwordx4 %[oa2], s[88:91], %[a2] offen offset:2048 sc0 sc1 lds\n\t"
            "v_mfma_f32_16x16x32_bf16 %[c1022], a[148:151], a[116:119], %[c1022]\n\t"
            "v_mfma_f32_16x16x32_bf16 %[c1032], a[144:147], a[120:123], %[c1032]\n\t"
            "buffer_load_dwordx4 %[oa3], s[88:91], %[a2] offen offset:3072 sc0 sc1 lds\n\t"
            "v_mfma_f32_16x16x32_bf16 %[c1032], a[148:151], a[124:127], %[c1032]\n\t"
            "v_mfma_f32_16x16x32_bf16 %[c1003], a[152:155], a[96:99], %[c1003]\n\t"
            "v_mfma_f32_16x16x32_bf16 %[c1003], a[156:159], a[100:103], %[c1003]\n\t"
            "v_mfma_f32_16x16x32_bf16 %[c1013], a[152:155], a[104:107], %[c1013]\n\t"
            "v_mfma_f32_16x16x32_bf16 %[c1013], a[156:159], a[108:111], %[c1013]\n\t"
            "s_waitcnt vmcnt(25) lgkmcnt(0)\n\t"
            "s_barrier\n\t"
            "v_mfma_f32_16x16x32_bf16 %[c1023], a[152:155], a[112:115], %[c1023]\n\t"
            "v_mfma_f32_16x16x32_bf16 %[c1023], a[156:159], a[116:119], %[c1023]\n\t"
            "v_mfma_f32_16x16x32_bf16 %[c1033], a[152:155], a[120:123], %[c1033]\n\t"
            "v_mfma_f32_16x16x32_bf16 %[c1033], a[156:159], a[124:127], %[c1033]\n\t"
            : [c1000] "+v"(acc[1][0][0][0]), [c1010] "+v"(acc[1][0][1][0]), [c1020] "+v"(acc[1][0][2][0]), [c1030] "+v"(acc[1][0][3][0]), [c1001] "+v"(acc[1][0][0][1]), [c1011] "+v"(acc[1][0][1][1]), [c1021] "+v"(acc[1][0][2][1]), [c1031] "+v"(acc[1][0][3][1]), [c1002] "+v"(acc[1][0][0][2]), [oa0] "+v"(voffA[0]), [a2] "+s"(a2), [c1012] "+v"(acc[1][0][1][2]), [oa1] "+v"(voffA[1]), [c1022] "+v"(acc[1][0][2][2]), [oa2] "+v"(voffA[2]), [c1032] "+v"(acc[1][0][3][2]), [oa3] "+v"(voffA[3]), [c1003] "+v"(acc[1][0][0][3]), [c1013] "+v"(acc[1][0][1][3]), [c1023] "+v"(acc[1][0][2][3]), [c1033] "+v"(acc[1][0][3][3]), [rsa] "+{s[88:91]}"(rsa)
            : [vb0] "v"(vb[0]), [vb1] "v"(vb[1]), [vb2] "v"(vb[2]), [vb3] "v"(vb[3]), [ldsw] "s"(ldsw)
            : "memory", "scc");
        asm volatile(
            "v_mfma_f32_16x16x32_bf16 a[0:3], a[160:163], a[64:67], a[0:3]\n\t"
            "ds_read_b64_tr_b16 a[128:129], %[vb0] offset:32768\n\t"
            "v_mfma_f32_16x16x32_bf16 a[0:3], a[164:167], a[68:71], a[0:3]\n\t"
            "ds_read_b64_tr_b16 a[130:131], %[vb0] offset:33792\n\t"
            "v_mfma_f32_16x16x32_bf16 a[16:19], a[160:163], a[72:75], a[16:19]\n\t"
            "ds_read_b64_tr_b16 a[132:133], %[vb0] offset:40960\n\t"
            "v_mfma_f32_16x16x32_bf16 a[16:19], a[164:167], a[76:79], a[16:19]\n\t"
            "ds_read_b64_tr_b16 a[134:135], %[vb0] offset:41984\n\t"
            "v_mfma_f32_16x16x32_bf16 a[32:35], a[160:163], a[80:83], a[32:35]\n\t"
            "ds_read_b64_tr_b16 a[136:137], %[vb1] offset:32768\n\t"
            "v_mfma_f32_16x16x32_bf16 a[32:35], a[164:167], a[84:87], a[32:35]\n\t"
            "ds_read_b64_tr_b16 a[138:139], %[vb1] offset:33792\n\t"
            "v_mfma_f32_16x16x32_bf16 a[48:51], a[160:163], a[88:91], a[48:51]\n\t"
            "ds_read_b64_tr_b16 a[140:141], %[vb1] offset:40960\n\t"
            "v_mfma_f32_16x16x32_bf16 a[48:51], a[164:167], a[92:95], a[48:51]\n\t"
            "ds_read_b64_tr_b16 a[142:143], %[vb1] offset:41984\n\t"
            "v_mfma_f32_16x16x32_bf16 a[4:7], a[168:171], a[64:67], a[4:7]\n\t"
            "ds_read_b64_tr_b16 a[144:145], %[vb2] offset:32768\n\t"
            "v_mfma_f32_16x16x32_bf16 a[4:7], a[172:175], a[68:71], a[4:7]\n\t"
            "ds_read_b64_tr_b16 a[146:147], %[vb2] offset:33792\n\t"
            "v_mfma_f32_16x16x32_bf16 a[20:23], a[168:171], a[72:75], a[20:23]\n\t"
            "ds_read_b64_tr_b16 a[148:149], %[vb2] offset:40960\n\t"
            "v_mfma_f32_16x16x32_bf16 a[20:23], a[172:175], a[76:79], a[20:23]\n\t"
            "ds_read_b64_tr_b16 a[150:151], %[vb2] offset:41984\n\t"
            "v_mfma_f32_16x16x32_bf16 a[36:39], a[168:171], a[80:83], a[36:39]\n\t"
            "ds_read_b64_tr_b16 a[152:153], %[vb3] offset:32768\n\t"
            "v_mfma_f32_16x16x32_bf16 a[36:39], a[172:175], a[84:87], a[36:39]\n\t"
            "ds_read_b64_tr_b16 a[154:155], %[vb3] offset:33792\n\t"
            "v_mfma_f32_16x16x32_bf16 a[52:55], a[168:171], a[88:91], a[52:55]\n\t"
            "ds_read_b64_tr_b16 a[156:157], %[vb3] offset:40960\n\t"
            "v_mfma_f32_16x16x32_bf16 a[52:55], a[172:175], a[92:95], a[52:55]\n\t"
            "ds_read_b64_tr_b16 a[158:159], %[vb3] offset:41984\n\t"
            "s_add_u32 m0, %[ldsw], 0x8000\n\t"
            "v_mfma_f32_16x16x32_bf16 a[8:11], a[176:179], a[64:67], a[8:11]\n\t"
            "buffer_load_dwordx4 %[oa0], s[88:91], %[a3] offen offset:0 sc0 sc1 lds\n\t"
            "v_mfma_f32_16x16x32_bf16 a[8:11], a[180:183], a[68:71], a[8:11]\n\t"
            "v_mfma_f32_16x16x32_bf16 a[24:27], a[176:179], a[72:75], a[24:27]\n\t"
            "buffer_load_dwordx4 %[oa1], s[88:91], %[a3] offen offset:1024 sc0 sc1 lds\n\t"
            "v_mfma_f32_16x16x32_bf16 a[24:27], a[180:183], a[76:79], a[24:27]\n\t"
            "v_mfma_f32_16x16x32_bf16 a[40:43], a[176:179], a[80:83], a[40:43]\n\t"
            "buffer_load_dwordx4 %[oa2], s[88:91], %[a3] offen offset:2048 sc0 sc1 lds\n\t"
            "v_mfma_f32_16x16x32_bf16 a[40:43], a[180:183], a[84:87], a[40:43]\n\t"
            "v_mfma_f32_16x16x32_bf16 a[56:59], a[176:179], a[88:91], a[56:59]\n\t"
            "buffer_load_dwordx4 %[oa3], s[88:91], %[a3] offen offset:3072 sc0 sc1 lds\n\t"
            "v_mfma_f32_16x16x32_bf16 a[56:59], a[180:183], a[92:95], a[56:59]\n\t"
            "v_mfma_f32_16x16x32_bf16 a[12:15], a[184:187], a[64:67], a[12:15]\n\t"
            "v_mfma_f32_16x16x32_bf16 a[12:15], a[188:191], a[68:71], a[12:15]\n\t"
            "v_mfma_f32_16x16x32_bf16 a[28:31], a[184:187], a[72:75], a[28:31]\n\t"
            "v_mfma_f32_16x16x32_bf16 a[28:31], a[188:191], a[76:79], a[28:31]\n\t"
            "s_waitcnt lgkmcnt(0)\n\t"
            "v_mfma_f32_16x16x32_bf16 a[44:47], a[184:187], a[80:83], a[44:47]\n\t"
            "v_mfma_f32_16x16x32_bf16 a[44:47], a[188:191], a[84:87], a[44:47]\n\t"
            "v_mfma_f32_16x16x32_bf16 a[60:63], a[184:187], a[88:91], a[60:63]\n\t"
            "v_mfma_f32_16x16x32_bf16 a[60:63], a[188:191], a[92:95], a[60:63]\n\t"
            : [oa0] "+v"(voffA[0]), [a3] "+s"(a3), [oa1] "+v"(voffA[1]), [oa2] "+v"(voffA[2]), [oa3] "+v"(voffA[3]), [rsa] "+{s[88:91]}"(rsa)
            : [vb0] "v"(vb[0]), [vb1] "v"(vb[1]), [vb2] "v"(vb[2]), [vb3] "v"(vb[3]), [ldsw] "s"(ldsw)
            : "memory", "scc");
        asm volatile(
            "v_mfma_f32_16x16x32_bf16 %[c0000], a[128:131], a[64:67], %[c0000]\n\t"
            "ds_read_b128 a[96:99], %[va0] offset:49152\n\t"
            "v_mfma_f32_16x16x32_bf16 %[c0000], a[132:135], a[68:71], %[c0000]\n\t"
            "ds_read_b128 a[100:103], %[va1] offset:49152\n\t"
            "v_mfma_f32_16x16x32_bf16 %[c0010], a[128:131], a[72:75], %[c0010]\n\t"
            "ds_read_b128 a[104:107], %[va0] offset:51200\n\t"
            "v_mfma_f32_16x16x32_bf16 %[c0010], a[132:135], a[76:79], %[c0010]\n\t"
            "ds_read_b128 a[108:111], %[va1] offset:51200\n\t"
            "v_mfma_f32_16x16x32_bf16 %[c0020], a[128:131], a[80:83], %[c0020]\n\t"
            "ds_read_b128 a[112:115], %[va0] offset:53248\n\t"
            "v_mfma_f32_16x16x32_bf16 %[c0020], a[132:135], a[84:87], %[c0020]\n\t"
            "ds_read_b128 a[116:119], %[va1] offset:53248\n\t"
            "v_mfma_f32_16x16x32_bf16 %[c0030], a[128:131], a[88:91], %[c0030]\n\t"
            "ds_read_b128 a[120:123], %[va0] offset:55296\n\t"
            "v_mfma_f32_16x16x32_bf16 %[c0030], a[132:135], a[92:95], %[c0030]\n\t"
            "ds_read_b128 a[124:127], %[va1] offset:55296\n\t"
            "v_mfma_f32_16x16x32_bf16 %[c0001], a[136:139], a[64:67], %[c0001]\n\t"
            "v_mfma_f32_16x16x32_bf16 %[c0001], a[140:143], a[68:71], %[c0001]\n\t"
            "v_mfma_f32_16x16x32_bf16 %[c0011], a[136:139], a[72:75], %[c0011]\n\t"
            "v_mfma_f32_16x16x32_bf16 %[c0011], a[140:143], a[76:79], %[c0011]\n\t"
            "v_mfma_f32_16x16x32_bf16 %[c0021], a[136:139], a[80:83], %[c0021]\n\t"
            "v_mfma_f32_16x16x32_bf16 %[c0021], a[140:143], a[84:87], %[c0021]\n\t"
            "v_mfma_f32_16x16x32_bf16 %[c0031], a[136:139], a[88:91], %[c0031]\n\t"
            "v_mfma_f32_16x16x32_bf16 %[c0031], a[140:143], a[92:95], %[c0031]\n\t"
            "s_add_u32 m0, %[ldsw], 0x1bf00\n\t"
            "v_mfma_f32_16x16x32_bf16 %[c0002], a[144:147], a[64:67], %[c0002]\n\t"
            "buffer_load_dwordx4 %[ob0], %[rsw], %[b3] offen offset:256  lds\n\t"
            "v_mfma_f32_16x16x32_bf16 %[c0002], a[148:151], a[68:71], %[c0002]\n\t"
            "v_mfma_f32_16x16x32_bf16 %[c0012], a[144:147], a[72:75], %[c0012]\n\t"
            "buffer_load_dwordx4 %[ob1], %[rsw], %[b3] offen offset:1280  lds\n\t"
            "v_mfma_f32_16x16x32_bf16 %[c0012], a[148:151], a[76:79], %[c0012]\n\t"
            "v_mfma_f32_16x16x32_bf16 %[c0022], a[144:147], a[80:83], %[c0022]\n\t"
            "buffer_load_dwordx4 %[ob2], %[rsw], %[b3] offen offset:2304  lds\n\t"
            "v_mfma_f32_16x16x32_bf16 %[c0022], a[148:151], a[84:87], %[c0022]\n\t"
            "v_mfma_f32_16x16x32_bf16 %[c0032], a[144:147], a[88:91], %[c0032]\n\t"
            "buffer_load_dwordx4 %[ob3], %[rsw], %[b3] offen offset:3328  lds\n\t"
            "v_mfma_f32_16x16x32_bf16 %[c0032], a[148:151], a[92:95], %[c0032]\n\t"
            "v_mfma_f32_16x16x32_bf16 %[c0003], a[152:155], a[64:67], %[c0003]\n\t"
            "v_mfma_f32_16x16x32_bf16 %[c0003], a[156:159], a[68:71], %[c0003]\n\t"
            "v_mfma_f32_16x16x32_bf16 %[c0013], a[152:155], a[72:75], %[c0013]\n\t"
            "v_mfma_f32_16x16x32_bf16 %[c0013], a[156:159], a[76:79], %[c0013]\n\t"
            "s_waitcnt vmcnt(20) lgkmcnt(0)\n\t"
            "s_barrier\n\t"
            "v_mfma_f32_16x16x32_bf16 %[c0023], a[152:155], a[80:83], %[c0023]\n\t"
            "v_mfma_f32_16x16x32_bf16 %[c0023], a[156:159], a[84:87], %[c0023]\n\t"
            "v_mfma_f32_16x16x32_bf16 %[c0033], a[152:155], a[88:91], %[c0033]\n\t"
            "v_mfma_f32_16x16x32_bf16 %[c0033], a[156:159], a[92:95], %[c0033]\n\t"
            : [c0000] "+v"(acc[0][0][0][0]), [c0010] "+v"(acc[0][0][1][0]), [c0020] "+v"(acc[0][0][2][0]), [c0030] "+v"(acc[0][0][3][0]), [c0001] "+v"(acc[0][0][0][1]), [c0011] "+v"(acc[0][0][1][1]), [c0021] "+v"(acc[0][0][2][1]), [c0031] "+v"(acc[0][0][3][1]), [c0002] "+v"(acc[0][0][0][2]), [b3] "+s"(b3), [c0012] "+v"(acc[0][0][1][2]), [c0022] "+v"(acc[0][0][2][2]), [c0032] "+v"(acc[0][0][3][2]), [c0003] "+v"(acc[0][0][0][3]), [c0013] "+v"(acc[0][0][1][3]), [c0023] "+v"(acc[0][0][2][3]), [c0033] "+v"(acc[0][0][3][3])
            : [va0] "v"(va[0]), [va1] "v"(va[1]), [ldsw] "s"(ldsw), [ob0] "v"(voffB[0]), [rsw] "s"(rsW), [ob1] "v"(voffB[1]), [ob2] "v"(voffB[2]), [ob3] "v"(voffB[3])
            : "memory", "scc");
        asm volatile(
            "v_mfma_f32_16x16x32_bf16 %[c1000], a[128:131], a[96:99], %[c1000]\n\t"
            "ds_read_b128 a[64:67], %[va0] offset:0\n\t"
            "v_mfma_f32_16x16x32_bf16 %[c1000], a[132:135], a[100:103], %[c1000]\n\t"
            "ds_read_b128 a[68:71], %[va1] offset:0\n\t"
            "v_cmp_eq_u64 vcc, %[fpat2], %[pf]\n\t"
            "s_lshl_b64 %[msk], 3, %[tmp]\n\t"
            "v_mfma_f32_16x16x32_bf16 %[c1010], a[128:131], a[104:107], %[c1010]\n\t"
            "ds_read_b128 a[72:75], %[va0] offset:2048\n\t"
            "s_or_b64 %[rdy], %[rdy], vcc\n\t"
            "v_mfma_f32_16x16x32_bf16 %[c1010], a[132:135], a[108:111], %[c1010]\n\t"
            "ds_read_b128 a[76:79], %[va1] offset:2048\n\t"
            "s_andn2_b64 s[98:99], %[msk], %[rdy]\n\t"
            "s_cbranch_scc1 .Lnr%=\n\t"
            ".Lnb%=:\n\t"
            ".subsection 1\n\t"
            ".Lnr%=:\n\t"
            "s_memrealtime s[98:99]\n\t"
            ".Lnl%=:\n\t"
            "buffer_load_dwordx2 %[pf], %[voffF], %[rsf], %[sflag] offen sc0 sc1\n\t"
            "s_waitcnt vmcnt(0)\n\t"
            "s_memrealtime s[86:87]\n\t"
            "v_cmp_eq_u64 vcc, %[fpat2], %[pf]\n\t"
            "s_waitcnt lgkmcnt(0)\n\t"
            "s_sub_u32 s86, s86, s98\n\t"
            "s_or_b64 %[rdy], %[rdy], vcc\n\t"
            "s_andn2_b64 vcc, %[msk], %[rdy]\n\t"
            "s_cbranch_scc0 .Lnb%=\n\t"
            "s_bitcmp1_b64 %[rdy], 2\n\t"
            "s_cbranch_scc1 .Lnd%=\n\t"
            "s_cmp_gt_u32 s86, %[ddl]\n\t"
            "s_cbranch_scc1 .Lnd%=\n\t"
            "s_sleep 1\n\t"
            "s_branch .Lnl%=\n\t"
            ".Lnd%=:\n\t"
            "s_mov_b64 %[rdy], -1\n\t"
            "buffer_store_dword %[vfv], %[voffL], %[rsf], %[stmo] offen sc0 sc1\n\t"
            "buffer_store_dword %[vfv], %[voffL], %[rsf], %[stmo] offen offset:4 sc0 sc1\n\t"
            "s_waitcnt vmcnt(0)\n\t"
            "s_branch .Lnb%=\n\t"
            ".subsection 0\n\t"
            "v_mfma_f32_16x16x32_bf16 %[c1020], a[128:131], a[112:115], %[c1020]\n\t"
            "ds_read_b128 a[80:83], %[va0] offset:4096\n\t"
            "v_mfma_f32_16x16x32_bf16 %[c1020], a[132:135], a[116:119], %[c1020]\n\t"
            "ds_read_b128 a[84:87], %[va1] offset:4096\n\t"
            "v_mfma_f32_16x16x32_bf16 %[c1030], a[128:131], a[120:123], %[c1030]\n\t"
            "ds_read_b128 a[88:91], %[va0] offset:6144\n\t"
            "v_mfma_f32_16x16x32_bf16 %[c1030], a[132:135], a[124:127], %[c1030]\n\t"
            "ds_read_b128 a[92:95], %[va1] offset:6144\n\t"
            "v_mfma_f32_16x16x32_bf16 %[c1001], a[136:139], a[96:99], %[c1001]\n\t"
            "v_mfma_f32_16x16x32_bf16 %[c1001], a[140:143], a[100:103], %[c1001]\n\t"
            "v_mfma_f32_16x16x32_bf16 %[c1011], a[136:139], a[104:107], %[c1011]\n\t"
            "v_mfma_f32_16x16x32_bf16 %[c1011], a[140:143], a[108:111], %[c1011]\n\t"
            "v_mfma_f32_16x16x32_bf16 %[c1021], a[136:139], a[112:115], %[c1021]\n\t"
            "v_mfma_f32_16x16x32_bf16 %[c1021], a[140:143], a[116:119], %[c1021]\n\t"
            "v_mfma_f32_16x16x32_bf16 %[c1031], a[136:139], a[120:123], %[c1031]\n\t"
            "v_mfma_f32_16x16x32_bf16 %[c1031], a[140:143], a[124:127], %[c1031]\n\t"
            "s_add_u32 m0, %[ldsw], 0x18000\n\t"
            "v_mfma_f32_16x16x32_bf16 %[c1002], a[144:147], a[96:99], %[c1002]\n\t"
            "buffer_load_dwordx4 %[ob0], %[rsw], %[b4] offen offset:0  lds\n\t"
            "v_mfma_f32_16x16x32_bf16 %[c1002], a[148:151], a[100:103], %[c1002]\n\t"
            "v_mfma_f32_16x16x32_bf16 %[c1012], a[144:147], a[104:107], %[c1012]\n\t"
            "buffer_load_dwordx4 %[ob1], %[rsw], %[b4] offen offset:1024  lds\n\t"
            "v_mfma_f32_16x16x32_bf16 %[c1012], a[148:151], a[108:111], %[c1012]\n\t"
            "v_mfma_f32_16x16x32_bf16 %[c1022], a[144:147], a[112:115], %[c1022]\n\t"
            "buffer_load_dwordx4 %[ob2], %[rsw], %[b4] offen offset:2048  lds\n\t"
            "v_mfma_f32_16x16x32_bf16 %[c1022], a[148:151], a[116:119], %[c1022]\n\t"
            "v_mfma_f32_16x16x32_bf16 %[c1032], a[144:147], a[120:123], %[c1032]\n\t"
            "buffer_load_dwordx4 %[ob3], %[rsw], %[b4] offen offset:3072  lds\n\t"
            "v_mfma_f32_16x16x32_bf16 %[c1032], a[148:151], a[124:127], %[c1032]\n\t"
            "v_mfma_f32_16x16x32_bf16 %[c1003], a[152:155], a[96:99], %[c1003]\n\t"
            "v_mfma_f32_16x16x32_bf16 %[c1003], a[156:159], a[100:103], %[c1003]\n\t"
            "v_mfma_f32_16x16x32_bf16 %[c1013], a[152:155], a[104:107], %[c1013]\n\t"
            "v_mfma_f32_16x16x32_bf16 %[c1013], a[156:159], a[108:111], %[c1013]\n\t"
            "s_waitcnt lgkmcnt(0)\n\t"
            "v_mfma_f32_16x16x32_bf16 %[c1023], a[152:155], a[112:115], %[c1023]\n\t"
            "v_mfma_f32_16x16x32_bf16 %[c1023], a[156:159], a[116:119], %[c1023]\n\t"
            "v_mfma_f32_16x16x32_bf16 %[c1033], a[152:155], a[120:123], %[c1033]\n\t"
            "v_mfma_f32_16x16x32_bf16 %[c1033], a[156:159], a[124:127], %[c1033]\n\t"
            : [c1000] "+v"(acc[1][0][0][0]), [pf] "+v"(pf), [msk] "+s"(msk), [tmp] "+s"(tmp), [c1010] "+v"(acc[1][0][1][0]), [rdy] "+s"(rdy), [c1020] "+v"(acc[1][0][2][0]), [c1030] "+v"(acc[1][0][3][0]), [c1001] "+v"(acc[1][0][0][1]), [c1011] "+v"(acc[1][0][1][1]), [c1021] "+v"(acc[1][0][2][1]), [c1031] "+v"(acc[1][0][3][1]), [c1002] "+v"(acc[1][0][0][2]), [b4] "+s"(b4), [c1012] "+v"(acc[1][0][1][2]), [c1022] "+v"(acc[1][0][2][2]), [c1032] "+v"(acc[1][0][3][2]), [c1003] "+v"(acc[1][0][0][3]), [c1013] "+v"(acc[1][0][1][3]), [c1023] "+v"(acc[1][0][2][3]), [c1033] "+v"(acc[1][0][3][3])
            : [va0] "v"(va[0]), [va1] "v"(va[1]), [fpat2] "s"(fpat2), [voffF] "v"(voffF), [rsf] "s"(rsF), [sflag] "s"(sflag), [ddl] "s"((unsigned)DEADLINE), [vfv] "v"(vfv), [voffL] "v"(voffL), [stmo] "s"(stmo), [ldsw] "s"(ldsw), [ob0] "v"(voffB[0]), [rsw] "s"(rsW), [ob1] "v"(voffB[1]), [ob2] "v"(voffB[2]), [ob3] "v"(voffB[3])
            : "memory", "scc", "vcc", "s98", "s99", "s86", "s87");
        asm volatile(
            "v_mfma_f32_16x16x32_bf16 %[c1100], a[160:163], a[96:99], %[c1100]\n\t"
            "ds_read_b64_tr_b16 a[128:129], %[vb0] offset:0\n\t"
            "v_mfma_f32_16x16x32_bf16 %[c1100], a[164:167], a[100:103], %[c1100]\n\t"
            "ds_read_b64_tr_b16 a[130:131], %[vb0] offset:1024\n\t"
            "v_mfma_f32_16x16x32_bf16 %[c1110], a[160:163], a[104:107], %[c1110]\n\t"
            "ds_read_b64_tr_b16 a[132:133], %[vb0] offset:8192\n\t"
            "v_mfma_f32_16x16x32_bf16 %[c1110], a[164:167], a[108:111], %[c1110]\n\t"
            "ds_read_b64_tr_b16 a[134:135], %[vb0] offset:9216\n\t"
            "v_mfma_f32_16x16x32_bf16 %[c1120], a[160:163], a[112:115], %[c1120]\n\t"
            "ds_read_b64_tr_b16 a[136:137], %[vb1] offset:0\n\t"
            "v_mfma_f32_16x16x32_bf16 %[c1120], a[164:167], a[116:119], %[c1120]\n\t"
            "ds_read_b64_tr_b16 a[138:139], %[vb1] offset:1024\n\t"
            "v_mfma_f32_16x16x32_bf16 %[c1130], a[160:163], a[120:123], %[c1130]\n\t"
            "ds_read_b64_tr_b16 a[140:141], %[vb1] offset:8192\n\t"
            "v_mfma_f32_16x16x32_bf16 %[c1130], a[164:167], a[124:127], %[c1130]\n\t"
            "ds_read_b64_tr_b16 a[142:143], %[vb1] offset:9216\n\t"
            "v_mfma_f32_16x16x32_bf16 %[c1101], a[168:171], a[96:99], %[c1101]\n\t"
            "ds_read_b64_tr_b16 a[144:145], %[vb2] offset:0\n\t"
            "v_mfma_f32_16x16x32_bf16 %[c1101], a[172:175], a[100:103], %[c1101]\n\t"
            "ds_read_b64_tr_b16 a[146:147], %[vb2] offset:1024\n\t"
            "v_mfma_f32_16x16x32_bf16 %[c1111], a[168:171], a[104:107], %[c1111]\n\t"
            "ds_read_b64_tr_b16 a[148:149], %[vb2] offset:8192\n\t"
            "v_mfma_f32_16x16x32_bf16 %[c1111], a[172:175], a[108:111], %[c1111]\n\t"
            "ds_read_b64_tr_b16 a[150:151], %[vb2] offset:9216\n\t"
            "v_mfma_f32_16x16x32_bf16 %[c1121], a[168:171], a[112:115], %[c1121]\n\t"
            "ds_read_b64_tr_b16 a[152:153], %[vb3] offset:0\n\t"
            "v_mfma_f32_16x16x32_bf16 %[c1121], a[172:175], a[116:119], %[c1121]\n\t"
            "ds_read_b64_tr_b16 a[154:155], %[vb3] offset:1024\n\t"
            "v_mfma_f32_16x16x32_bf16 %[c1131], a[168:171], a[120:123], %[c1131]\n\t"
            "ds_read_b64_tr_b16 a[156:157], %[vb3] offset:8192\n\t"
            "v_mfma_f32_16x16x32_bf16 %[c1131], a[172:175], a[124:127], %[c1131]\n\t"
            "ds_read_b64_tr_b16 a[158:159], %[vb3] offset:9216\n\t"
            "s_add_u32 m0, %[ldsw], 0xc000\n\t"
            "v_mfma_f32_16x16x32_bf16 %[c1102], a[176:179], a[96:99], %[c1102]\n\t"
            "buffer_load_dwordx4 %[oa0], s[88:91], %[a4] offen offset:0 sc0 sc1 lds\n\t"
            "v_mfma_f32_16x16x32_bf16 %[c1102], a[180:183], a[100:103], %[c1102]\n\t"
            "v_mfma_f32_16x16x32_bf16 %[c1112], a[176:179], a[104:107], %[c1112]\n\t"
            "buffer_load_dwordx4 %[oa1], s[88:91], %[a4] offen offset:1024 sc0 sc1 lds\n\t"
            "v_mfma_f32_16x16x32_bf16 %[c1112], a[180:183], a[108:111], %[c1112]\n\t"
            "v_mfma_f32_16x16x32_bf16 %[c1122], a[176:179], a[112:115], %[c1122]\n\t"
            "buffer_load_dwordx4 %[oa2], s[88:91], %[a4] offen offset:2048 sc0 sc1 lds\n\t"
            "v_mfma_f32_16x16x32_bf16 %[c1122], a[180:183], a[116:119], %[c1122]\n\t"
            "v_mfma_f32_16x16x32_bf16 %[c1132], a[176:179], a[120:123], %[c1132]\n\t"
            "buffer_load_dwordx4 %[oa3], s[88:91], %[a4] offen offset:3072 sc0 sc1 lds\n\t"
            "v_mfma_f32_16x16x32_bf16 %[c1132], a[180:183], a[124:127], %[c1132]\n\t"
            "v_mfma_f32_16x16x32_bf16 %[c1103], a[184:187], a[96:99], %[c1103]\n\t"
            "v_mfma_f32_16x16x32_bf16 %[c1103], a[188:191], a[100:103], %[c1103]\n\t"
            "v_mfma_f32_16x16x32_bf16 %[c1113], a[184:187], a[104:107], %[c1113]\n\t"
            "v_mfma_f32_16x16x32_bf16 %[c1113], a[188:191], a[108:111], %[c1113]\n\t"
            "s_waitcnt vmcnt(16) lgkmcnt(0)\n\t"
            "s_barrier\n\t"
            "v_mfma_f32_16x16x32_bf16 %[c1123], a[184:187], a[112:115], %[c1123]\n\t"
            "v_mfma_f32_16x16x32_bf16 %[c1123], a[188:191], a[116:119], %[c1123]\n\t"
            "v_mfma_f32_16x16x32_bf16 %[c1133], a[184:187], a[120:123], %[c1133]\n\t"
            "v_mfma_f32_16x16x32_bf16 %[c1133], a[188:191], a[124:127], %[c1133]\n\t"
            : [c1100] "+v"(acc[1][1][0][0]), [c1110] "+v"(acc[1][1][1][0]), [c1120] "+v"(acc[1][1][2][0]), [c1130] "+v"(acc[1][1][3][0]), [c1101] "+v"(acc[1][1][0][1]), [c1111] "+v"(acc[1][1][1][1]), [c1121] "+v"(acc[1][1][2][1]), [c1131] "+v"(acc[1][1][3][1]), [c1102] "+v"(acc[1][1][0][2]), [oa0] "+v"(voffA[0]), [a4] "+s"(a4), [c1112] "+v"(acc[1][1][1][2]), [oa1] "+v"(voffA[1]), [c1122] "+v"(acc[1][1][2][2]), [oa2] "+v"(voffA[2]), [c1132] "+v"(acc[1][1][3][2]), [oa3] "+v"(voffA[3]), [c1103] "+v"(acc[1][1][0][3]), [c1113] "+v"(acc[1][1][1][3]), [c1123] "+v"(acc[1][1][2][3]), [c1133] "+v"(acc[1][1][3][3]), [rsa] "+{s[88:91]}"(rsa)
            : [vb0] "v"(vb[0]), [vb1] "v"(vb[1]), [vb2] "v"(vb[2]), [vb3] "v"(vb[3]), [ldsw] "s"(ldsw)
            : "memory", "scc");
        SCHED; SW(1); SCHED;
        SCHED; SR(62); ST(33); SCHED;
        asm volatile(
            "v_mfma_f32_16x16x32_bf16 %[c0000], a[128:131], a[64:67], %[c0000]\n\t"
            "ds_read_b64_tr_b16 a[160:161], %[vbx0] offset:16384\n\t"
            "v_mfma_f32_16x16x32_bf16 %[c0000], a[132:135], a[68:71], %[c0000]\n\t"
            "ds_read_b64_tr_b16 a[162:163], %[vbx0] offset:17408\n\t"
            "v_mfma_f32_16x16x32_bf16 %[c0010], a[128:131], a[72:75], %[c0010]\n\t"
            "ds_read_b64_tr_b16 a[164:165], %[vbx0] offset:24576\n\t"
            "v_mfma_f32_16x16x32_bf16 %[c0010], a[132:135], a[76:79], %[c0010]\n\t"
            "ds_read_b64_tr_b16 a[166:167], %[vbx0] offset:25600\n\t"
            "v_mfma_f32_16x16x32_bf16 %[c0020], a[128:131], a[80:83], %[c0020]\n\t"
            "ds_read_b64_tr_b16 a[168:169], %[vbx1] offset:16384\n\t"
            "v_mfma_f32_16x16x32_bf16 %[c0020], a[132:135], a[84:87], %[c0020]\n\t"
            "ds_read_b64_tr_b16 a[170:171], %[vbx1] offset:17408\n\t"
            "v_mfma_f32_16x16x32_bf16 %[c0030], a[128:131], a[88:91], %[c0030]\n\t"
            "ds_read_b64_tr_b16 a[172:173], %[vbx1] offset:24576\n\t"
            "v_mfma_f32_16x16x32_bf16 %[c0030], a[132:135], a[92:95], %[c0030]\n\t"
            "ds_read_b64_tr_b16 a[174:175], %[vbx1] offset:25600\n\t"
            "v_mfma_f32_16x16x32_bf16 %[c0001], a[136:139], a[64:67], %[c0001]\n\t"
            "ds_read_b64_tr_b16 a[176:177], %[vbx2] offset:16384\n\t"
            "v_mfma_f32_16x16x32_bf16 %[c0001], a[140:143], a[68:71], %[c0001]\n\t"
            "ds_read_b64_tr_b16 a[178:179], %[vbx2] offset:17408\n\t"
            "v_mfma_f32_16x16x32_bf16 %[c0011], a[136:139], a[72:75], %[c0011]\n\t"
            "ds_read_b64_tr_b16 a[180:181], %[vbx2] offset:24576\n\t"
            "v_mfma_f32_16x16x32_bf16 %[c0011], a[140:143], a[76:79], %[c0011]\n\t"
            "ds_read_b64_tr_b16 a[182:183], %[vbx2] offset:25600\n\t"
            "v_mfma_f32_16x16x32_bf16 %[c0021], a[136:139], a[80:83], %[c0021]\n\t"
            "ds_read_b64_tr_b16 a[184:185], %[vbx3] offset:16384\n\t"
            "v_mfma_f32_16x16x32_bf16 %[c0021], a[140:143], a[84:87], %[c0021]\n\t"
            "ds_read_b64_tr_b16 a[186:187], %[vbx3] offset:17408\n\t"
            "v_mfma_f32_16x16x32_bf16 %[c0031], a[136:139], a[88:91], %[c0031]\n\t"
            "ds_read_b64_tr_b16 a[188:189], %[vbx3] offset:24576\n\t"
            "v_mfma_f32_16x16x32_bf16 %[c0031], a[140:143], a[92:95], %[c0031]\n\t"
            "ds_read_b64_tr_b16 a[190:191], %[vbx3] offset:25600\n\t"
            "v_mfma_f32_16x16x32_bf16 %[c0002], a[144:147], a[64:67], %[c0002]\n\t"
            "ds_read_b128 a[96:99], %[vax0] offset:0\n\t"
            "v_mfma_f32_16x16x32_bf16 %[c0002], a[148:151], a[68:71], %[c0002]\n\t"
            "ds_read_b128 a[100:103], %[vax1] offset:0\n\t"
            "v_mfma_f32_16x16x32_bf16 %[c0012], a[144:147], a[72:75], %[c0012]\n\t"
            "ds_read_b128 a[104:107], %[vax0] offset:2048\n\t"
            "v_mfma_f32_16x16x32_bf16 %[c0012], a[148:151], a[76:79], %[c0012]\n\t"
            "ds_read_b128 a[108:111], %[vax1] offset:2048\n\t"
            "v_mfma_f32_16x16x32_bf16 %[c0022], a[144:147], a[80:83], %[c0022]\n\t"
            "ds_read_b128 a[112:115], %[vax0] offset:4096\n\t"
            "v_mfma_f32_16x16x32_bf16 %[c0022], a[148:151], a[84:87], %[c0022]\n\t"
            "ds_read_b128 a[116:119], %[vax1] offset:4096\n\t"
            "v_mfma_f32_16x16x32_bf16 %[c0032], a[144:147], a[88:91], %[c0032]\n\t"
            "ds_read_b128 a[120:123], %[vax0] offset:6144\n\t"
            "v_mfma_f32_16x16x32_bf16 %[c0032], a[148:151], a[92:95], %[c0032]\n\t"
            "ds_read_b128 a[124:127], %[vax1] offset:6144\n\t"
            "s_add_u32 m0, %[ldsw], 0x0\n\t"
            "v_mfma_f32_16x16x32_bf16 %[c0003], a[152:155], a[64:67], %[c0003]\n\t"
            "buffer_load_dwordx4 %[oa0], s[88:91], %[a5] offen offset:0 sc0 sc1 lds\n\t"
            "v_mfma_f32_16x16x32_bf16 %[c0003], a[156:159], a[68:71], %[c0003]\n\t"
            "v_mfma_f32_16x16x32_bf16 %[c0013], a[152:155], a[72:75], %[c0013]\n\t"
            "buffer_load_dwordx4 %[oa1], s[88:91], %[a5] offen offset:1024 sc0 sc1 lds\n\t"
            "v_mfma_f32_16x16x32_bf16 %[c0013], a[156:159], a[76:79], %[c0013]\n\t"
            "v_mfma_f32_16x16x32_bf16 %[c0023], a[152:155], a[80:83], %[c0023]\n\t"
            "buffer_load_dwordx4 %[oa2], s[88:91], %[a5] offen offset:2048 sc0 sc1 lds\n\t"
            "v_mfma_f32_16x16x32_bf16 %[c0023], a[156:159], a[84:87], %[c0023]\n\t"
            "s_waitcnt lgkmcnt(0)\n\t"
            "v_mfma_f32_16x16x32_bf16 %[c0033], a[152:155], a[88:91], %[c0033]\n\t"
            "buffer_load_dwordx4 %[oa3], s[88:91], %[a5] offen offset:3072 sc0 sc1 lds\n\t"
            "v_mfma_f32_16x16x32_bf16 %[c0033], a[156:159], a[92:95], %[c0033]\n\t"
            "buffer_load_dwordx2 %[pf], %[voffF], %[rsf], %[sflag] offen sc0 sc1\n\t"
            : [c0000] "+v"(acc[0][0][0][0]), [c0010] "+v"(acc[0][0][1][0]), [c0020] "+v"(acc[0][0][2][0]), [c0030] "+v"(acc[0][0][3][0]), [c0001] "+v"(acc[0][0][0][1]), [c0011] "+v"(acc[0][0][1][1]), [c0021] "+v"(acc[0][0][2][1]), [c0031] "+v"(acc[0][0][3][1]), [c0002] "+v"(acc[0][0][0][2]), [c0012] "+v"(acc[0][0][1][2]), [c0022] "+v"(acc[0][0][2][2]), [c0032] "+v"(acc[0][0][3][2]), [c0003] "+v"(acc[0][0][0][3]), [oa0] "+v"(voffA[0]), [a5] "+s"(a5), [c0013] "+v"(acc[0][0][1][3]), [oa1] "+v"(voffA[1]), [c0023] "+v"(acc[0][0][2][3]), [oa2] "+v"(voffA[2]), [c0033] "+v"(acc[0][0][3][3]), [oa3] "+v"(voffA[3]), [pf] "+v"(pf), [rsa] "+{s[88:91]}"(rsa)
            : [vbx0] "v"(vbx[0]), [vbx1] "v"(vbx[1]), [vbx2] "v"(vbx[2]), [vbx3] "v"(vbx[3]), [vax0] "v"(vax[0]), [vax1] "v"(vax[1]), [ldsw] "s"(ldsw), [voffF] "v"(voffF), [rsf] "s"(rsF), [sflag] "s"(sflag)
            : "memory", "scc");
        SCHED; SW(2); SCHED;
        asm volatile(
            "v_mfma_f32_16x16x32_bf16 %[c0000], a[160:163], a[96:99], %[c0000]\n\t"
            "ds_read_b64_tr_b16 a[128:129], %[vb0] offset:16384\n\t"
            "v_mfma_f32_16x16x32_bf16 %[c0000], a[164:167], a[100:103], %[c0000]\n\t"
            : [c0000] "+v"(acc[0][0][0][0])
            : [vb0] "v"(vb[0])
            : "memory", "scc");
        SCHED; badm = (rdy & 4ull) ? 0x7fc07fc0u : 0u; tago = (tag << 16) | (badm & 0xffff0000u); asm volatile("" : "+s"(badm), "+s"(tago)); SCHED;
        asm volatile(
            "ds_read_b64_tr_b16 a[130:131], %[vb0] offset:17408\n\t"
            "v_mfma_f32_16x16x32_bf16 %[c0010], a[160:163], a[104:107], %[c0010]\n\t"
            "ds_read_b64_tr_b16 a[132:133], %[vb0] offset:24576\n\t"
            "v_mfma_f32_16x16x32_bf16 %[c0010], a[164:167], a[108:111], %[c0010]\n\t"
            "ds_read_b64_tr_b16 a[134:135], %[vb0] offset:25600\n\t"
            "v_mfma_f32_16x16x32_bf16 %[c0020], a[160:163], a[112:115], %[c0020]\n\t"
            "ds_read_b64_tr_b16 a[136:137], %[vb1] offset:16384\n\t"
            "v_mfma_f32_16x16x32_bf16 %[c0020], a[164:167], a[116:119], %[c0020]\n\t"
            "ds_read_b64_tr_b16 a[138:139], %[vb1] offset:17408\n\t"
            "v_mfma_f32_16x16x32_bf16 %[c0030], a[160:163], a[120:123], %[c0030]\n\t"
            "ds_read_b64_tr_b16 a[140:141], %[vb1] offset:24576\n\t"
            "v_mfma_f32_16x16x32_bf16 %[c0030], a[164:167], a[124:127], %[c0030]\n\t"
            "ds_read_b64_tr_b16 a[142:143], %[vb1] offset:25600\n\t"
            "v_mfma_f32_16x16x32_bf16 %[c0001], a[168:171], a[96:99], %[c0001]\n\t"
            "ds_read_b64_tr_b16 a[144:145], %[vb2] offset:16384\n\t"
            "v_mfma_f32_16x16x32_bf16 %[c0001], a[172:175], a[100:103], %[c0001]\n\t"
            "ds_read_b64_tr_b16 a[146:147], %[vb2] offset:17408\n\t"
            "v_mfma_f32_16x16x32_bf16 %[c0011], a[168:171], a[104:107], %[c0011]\n\t"
            "ds_read_b64_tr_b16 a[148:149], %[vb2] offset:24576\n\t"
            "v_mfma_f32_16x16x32_bf16 %[c0011], a[172:175], a[108:111], %[c0011]\n\t"
            "ds_read_b64_tr_b16 a[150:151], %[vb2] offset:25600\n\t"
            "v_mfma_f32_16x16x32_bf16 %[c0021], a[168:171], a[112:115], %[c0021]\n\t"
            "ds_read_b64_tr_b16 a[152:153], %[vb3] offset:16384\n\t"
            "v_mfma_f32_16x16x32_bf16 %[c0021], a[172:175], a[116:119], %[c0021]\n\t"
            "ds_read_b64_tr_b16 a[154:155], %[vb3] offset:17408\n\t"
            "v_mfma_f32_16x16x32_bf16 %[c0031], a[168:171], a[120:123], %[c0031]\n\t"
            "ds_read_b64_tr_b16 a[156:157], %[vb3] offset:24576\n\t"
            "v_mfma_f32_16x16x32_bf16 %[c0031], a[172:175], a[124:127], %[c0031]\n\t"
            "ds_read_b64_tr_b16 a[158:159], %[vb3] offset:25600\n\t"
            "v_mfma_f32_16x16x32_bf16 %[c0002], a[176:179], a[96:99], %[c0002]\n\t"
            : [c0010] "+v"(acc[0][0][1][0]), [c0020] "+v"(acc[0][0][2][0]), [c0030] "+v"(acc[0][0][3][0]), [c0001] "+v"(acc[0][0][0][1]), [c0011] "+v"(acc[0][0][1][1]), [c0021] "+v"(acc[0][0][2][1]), [c0031] "+v"(acc[0][0][3][1]), [c0002] "+v"(acc[0][0][0][2])
            : [vb0] "v"(vb[0]), [vb1] "v"(vb[1]), [vb2] "v"(vb[2]), [vb3] "v"(vb[3])
            : "memory", "scc");
        SCHED; s0_[0] = TAGV(acc[0][0][0][0][0]); SCHED;
        asm volatile(
            "v_mfma_f32_16x16x32_bf16 %[c0002], a[180:183], a[100:103], %[c0002]\n\t"
            : [c0002] "+v"(acc[0][0][0][2])
            :
            : "memory", "scc");
        SCHED; so_.y = cvt_pk_bf16(acc[0][0][0][0][2], acc[0][0][0][0][3]); SCHED;
        asm volatile(
            "v_mfma_f32_16x16x32_bf16 %[c0012], a[176:179], a[104:107], %[c0012]\n\t"
            : [c0012] "+v"(acc[0][0][1][2])
            :
            : "memory", "scc");
        SCHED; so_.x = cvt_pk_bf16(s0_[0], acc[0][0][0][0][1]); SCHED;
        asm volatile(
            "v_mfma_f32_16x16x32_bf16 %[c0012], a[180:183], a[108:111], %[c0012]\n\t"
            : [c0012] "+v"(acc[0][0][1][2])
            :
            : "memory", "scc");
        SCHED; so_.z = cvt_pk_bf16(acc[0][0][0][1][0], acc[0][0][0][1][1]); SCHED;
        asm volatile(
            "v_mfma_f32_16x16x32_bf16 %[c0022], a[176:179], a[112:115], %[c0022]\n\t"
            : [c0022] "+v"(acc[0][0][2][2])
            :
            : "memory", "scc");
        SCHED; so_.w = cvt_pk_bf16(acc[0][0][0][1][2], acc[0][0][0][1][3]); SCHED;
        asm volatile(
            "v_mfma_f32_16x16x32_bf16 %[c0022], a[180:183], a[116:119], %[c0022]\n\t"
            : [c0022] "+v"(acc[0][0][2][2])
            :
            : "memory", "scc");
        SCHED; SENDST(0); SCHED;
        asm volatile(
            "v_mfma_f32_16x16x32_bf16 %[c0032], a[176:179], a[120:123], %[c0032]\n\t"
            "v_mfma_f32_16x16x32_bf16 %[c0032], a[180:183], a[124:127], %[c0032]\n\t"
            "v_mfma_f32_16x16x32_bf16 %[c0003], a[184:187], a[96:99], %[c0003]\n\t"
            : [c0032] "+v"(acc[0][0][3][2]), [c0003] "+v"(acc[0][0][0][3])
            :
            : "memory", "scc");
        SCHED; s0_[0] = TAGV(acc[0][0][1][0][0]); SCHED;
        asm volatile(
            "v_mfma_f32_16x16x32_bf16 %[c0003], a[188:191], a[100:103], %[c0003]\n\t"
            : [c0003] "+v"(acc[0][0][0][3])
            :
            : "memory", "scc");
        SCHED; so_.y = cvt_pk_bf16(acc[0][0][1][0][2], acc[0][0][1][0][3]); SCHED;
        asm volatile(
            "v_mfma_f32_16x16x32_bf16 %[c0013], a[184:187], a[104:107], %[c0013]\n\t"
            : [c0013] "+v"(acc[0][0][1][3])
            :
            : "memory", "scc");
        SCHED; so_.x = cvt_pk_bf16(s0_[0], acc[0][0][1][0][1]); SCHED;
        asm volatile(
            "v_mfma_f32_16x16x32_bf16 %[c0013], a[188:191], a[108:111], %[c0013]\n\t"
            "s_waitcnt vmcnt(14) lgkmcnt(0)\n\t"
            "s_barrier\n\t"
            "s_add_u32 m0, %[ldsw], 0x24000\n\t"
            : [c0013] "+v"(acc[0][0][1][3])
            : [ldsw] "s"(ldsw)
            : "memory", "scc");
        SCHED; so_.z = cvt_pk_bf16(acc[0][0][1][1][0], acc[0][0][1][1][1]); SCHED;
        asm volatile(
            "v_mfma_f32_16x16x32_bf16 %[c0023], a[184:187], a[112:115], %[c0023]\n\t"
            "buffer_load_dwordx4 %[ob0], %[rsw], %[b5] offen offset:0  lds\n\t"
            : [c0023] "+v"(acc[0][0][2][3]), [b5] "+s"(b5)
            : [ob0] "v"(voffB[0]), [rsw] "s"(rsW)
            : "memory", "scc");
        SCHED; so_.w = cvt_pk_bf16(acc[0][0][1][1][2], acc[0][0][1][1][3]); SCHED;
        asm volatile(
            "v_mfma_f32_16x16x32_bf16 %[c0023], a[188:191], a[116:119], %[c0023]\n\t"
            "buffer_load_dwordx4 %[ob1], %[rsw], %[b5] offen offset:1024  lds\n\t"
            : [c0023] "+v"(acc[0][0][2][3]), [b5] "+s"(b5)
            : [ob1] "v"(voffB[1]), [rsw] "s"(rsW)
            : "memory", "scc");
        SCHED; SENDST(2); SCHED;
        asm volatile(
            "v_mfma_f32_16x16x32_bf16 %[c0033], a[184:187], a[120:123], %[c0033]\n\t"
            "buffer_load_dwordx4 %[ob2], %[rsw], %[b5] offen offset:2048  lds\n\t"
            "v_mfma_f32_16x16x32_bf16 %[c0033], a[188:191], a[124:127], %[c0033]\n\t"
            "buffer_load_dwordx4 %[ob3], %[rsw], %[b5] offen offset:3072  lds\n\t"
            : [c0033] "+v"(acc[0][0][3][3]), [b5] "+s"(b5)
            : [ob2] "v"(voffB[2]), [rsw] "s"(rsW), [ob3] "v"(voffB[3])
            : "memory", "scc");
        SCHED; SW(3); SCHED;
        asm volatile(
            "v_mfma_f32_16x16x32_bf16 a[0:3], a[128:131], a[64:67], a[0:3]\n\t"
            "ds_read_b64_tr_b16 a[224:225], %[vb0] offset:49152\n\t"
            :
            : [vb0] "v"(vb[0])
            : "memory", "scc");
        SCHED; s0_[0] = TAGV(acc[0][0][2][0][0]); SCHED;
        asm volatile(
            "v_mfma_f32_16x16x32_bf16 a[0:3], a[132:135], a[68:71], a[0:3]\n\t"
            "ds_read_b64_tr_b16 a[226:227], %[vb0] offset:50176\n\t"
            :
            : [vb0] "v"(vb[0])
            : "memory", "scc");
        SCHED; so_.y = cvt_pk_bf16(acc[0][0][2][0][2], acc[0][0][2][0][3]); SCHED;
        asm volatile(
            "v_mfma_f32_16x16x32_bf16 a[16:19], a[128:131], a[72:75], a[16:19]\n\t"
            "ds_read_b64_tr_b16 a[228:229], %[vb0] offset:57344\n\t"
            :
            : [vb0] "v"(vb[0])
            : "memory", "scc");
        SCHED; so_.x = cvt_pk_bf16(s0_[0], acc[0][0][2][0][1]); SCHED;
        asm volatile(
            "v_mfma_f32_16x16x32_bf16 a[16:19], a[132:135], a[76:79], a[16:19]\n\t"
            "ds_read_b64_tr_b16 a[230:231], %[vb0] offset:58368\n\t"
            :
            : [vb0] "v"(vb[0])
            : "memory", "scc");
        SCHED; so_.z = cvt_pk_bf16(acc[0][0][2][1][0], acc[0][0][2][1][1]); SCHED;
        asm volatile(
            "v_mfma_f32_16x16x32_bf16 a[32:35], a[128:131], a[80:83], a[32:35]\n\t"
            "ds_read_b64_tr_b16 a[232:233], %[vb1] offset:49152\n\t"
            :
            : [vb1] "v"(vb[1])
            : "memory", "scc");
        SCHED; so_.w = cvt_pk_bf16(acc[0][0][2][1][2], acc[0][0][2][1][3]); SCHED;
        asm volatile(
            "v_mfma_f32_16x16x32_bf16 a[32:35], a[132:135], a[84:87], a[32:35]\n\t"
            "ds_read_b64_tr_b16 a[234:235], %[vb1] offset:50176\n\t"
            :
            : [vb1] "v"(vb[1])
            : "memory", "scc");
        SCHED; SENDST(4); SCHED;
        asm volatile(
            "v_mfma_f32_16x16x32_bf16 a[48:51], a[128:131], a[88:91], a[48:51]\n\t"
            "ds_read_b64_tr_b16 a[236:237], %[vb1] offset:57344\n\t"
            "v_mfma_f32_16x16x32_bf16 a[48:51], a[132:135], a[92:95], a[48:51]\n\t"
            "ds_read_b64_tr_b16 a[238:239], %[vb1] offset:58368\n\t"
            "v_mfma_f32_16x16x32_bf16 a[4:7], a[136:139], a[64:67], a[4:7]\n\t"
            "ds_read_b64_tr_b16 a[240:241], %[vb2] offset:49152\n\t"
            :
            : [vb1] "v"(vb[1]), [vb2] "v"(vb[2])
            : "memory", "scc");
        SCHED; s0_[0] = TAGV(acc[0][0][3][0][0]); SCHED;
        asm volatile(
            "v_mfma_f32_16x16x32_bf16 a[4:7], a[140:143], a[68:71], a[4:7]\n\t"
            "ds_read_b64_tr_b16 a[242:243], %[vb2] offset:50176\n\t"
            :
            : [vb2] "v"(vb[2])
            : "memory", "scc");
        SCHED; so_.y = cvt_pk_bf16(acc[0][0][3][0][2], acc[0][0][3][0][3]); SCHED;
        asm volatile(
            "v_mfma_f32_16x16x32_bf16 a[20:23], a[136:139], a[72:75], a[20:23]\n\t"
            "ds_read_b64_tr_b16 a[244:245], %[vb2] offset:57344\n\t"
            :
            : [vb2] "v"(vb[2])
            : "memory", "scc");
        SCHED; so_.x = cvt_pk_bf16(s0_[0], acc[0][0][3][0][1]); SCHED;
        asm volatile(
            "v_mfma_f32_16x16x32_bf16 a[20:23], a[140:143], a[76:79], a[20:23]\n\t"
            "ds_read_b64_tr_b16 a[246:247], %[vb2] offset:58368\n\t"
            :
            : [vb2] "v"(vb[2])
            : "memory", "scc");
        SCHED; so_.z = cvt_pk_bf16(acc[0][0][3][1][0], acc[0][0][3][1][1]); SCHED;
        asm volatile(
            "v_mfma_f32_16x16x32_bf16 a[36:39], a[136:139], a[80:83], a[36:39]\n\t"
            "ds_read_b64_tr_b16 a[248:249], %[vb3] offset:49152\n\t"
            :
            : [vb3] "v"(vb[3])
            : "memory", "scc");
        SCHED; so_.w = cvt_pk_bf16(acc[0][0][3][1][2], acc[0][0][3][1][3]); SCHED;
        asm volatile(
            "v_mfma_f32_16x16x32_bf16 a[36:39], a[140:143], a[84:87], a[36:39]\n\t"
            "ds_read_b64_tr_b16 a[250:251], %[vb3] offset:50176\n\t"
            :
            : [vb3] "v"(vb[3])
            : "memory", "scc");
        SCHED; SENDST(6); SCHED;
        asm volatile(
            "v_mfma_f32_16x16x32_bf16 a[52:55], a[136:139], a[88:91], a[52:55]\n\t"
            "ds_read_b64_tr_b16 a[252:253], %[vb3] offset:57344\n\t"
            "v_mfma_f32_16x16x32_bf16 a[52:55], a[140:143], a[92:95], a[52:55]\n\t"
            "ds_read_b64_tr_b16 a[254:255], %[vb3] offset:58368\n\t"
            "v_mfma_f32_16x16x32_bf16 a[8:11], a[144:147], a[64:67], a[8:11]\n\t"
            "ds_read_b128 a[192:195], %[va0] offset:32768\n\t"
            :
            : [vb3] "v"(vb[3]), [va0] "v"(va[0])
            : "memory", "scc");
        SCHED; s0_[0] = TAGV(acc[0][0][0][2][0]); SCHED;
        asm volatile(
            "v_mfma_f32_16x16x32_bf16 a[8:11], a[148:151], a[68:71], a[8:11]\n\t"
            "ds_read_b128 a[196:199], %[va1] offset:32768\n\t"
            :
            : [va1] "v"(va[1])
            : "memory", "scc");
        SCHED; so_.y = cvt_pk_bf16(acc[0][0][0][2][2], acc[0][0][0][2][3]); SCHED;
        asm volatile(
            "v_mfma_f32_16x16x32_bf16 a[24:27], a[144:147], a[72:75], a[24:27]\n\t"
            "ds_read_b128 a[200:203], %[va0] offset:34816\n\t"
            :
            : [va0] "v"(va[0])
            : "memory", "scc");
        SCHED; so_.x = cvt_pk_bf16(s0_[0], acc[0][0][0][2][1]); SCHED;
        asm volatile(
            "v_mfma_f32_16x16x32_bf16 a[24:27], a[148:151], a[76:79], a[24:27]\n\t"
            "ds_read_b128 a[204:207], %[va1] offset:34816\n\t"
            :
            : [va1] "v"(va[1])
            : "memory", "scc");
        SCHED; so_.z = cvt_pk_bf16(acc[0][0][0][3][0], acc[0][0][0][3][1]); SCHED;
        asm volatile(
            "v_mfma_f32_16x16x32_bf16 a[40:43], a[144:147], a[80:83], a[40:43]\n\t"
            "ds_read_b128 a[208:211], %[va0] offset:36864\n\t"
            :
            : [va0] "v"(va[0])
            : "memory", "scc");
        SCHED; so_.w = cvt_pk_bf16(acc[0][0][0][3][2], acc[0][0][0][3][3]); SCHED;
        asm volatile(
            "v_mfma_f32_16x16x32_bf16 a[40:43], a[148:151], a[84:87], a[40:43]\n\t"
            "ds_read_b128 a[212:215], %[va1] offset:36864\n\t"
            :
            : [va1] "v"(va[1])
            : "memory", "scc");
        SCHED; SENDST(1); SCHED;
        asm volatile(
            "v_mfma_f32_16x16x32_bf16 a[56:59], a[144:147], a[88:91], a[56:59]\n\t"
            "ds_read_b128 a[216:219], %[va0] offset:38912\n\t"
            "v_mfma_f32_16x16x32_bf16 a[56:59], a[148:151], a[92:95], a[56:59]\n\t"
            "ds_read_b128 a[220:223], %[va1] offset:38912\n\t"
            "s_add_u32 m0, %[ldsw], 0x1ff00\n\t"
            "v_mfma_f32_16x16x32_bf16 a[12:15], a[152:155], a[64:67], a[12:15]\n\t"
            "buffer_load_dwordx4 %[ob0], %[rsw], %[b4] offen offset:256  lds\n\t"
            : [b4] "+s"(b4)
            : [va0] "v"(va[0]), [va1] "v"(va[1]), [ldsw] "s"(ldsw), [ob0] "v"(voffB[0]), [rsw] "s"(rsW)
            : "memory", "scc");
        SCHED; s0_[0] = TAGV(acc[0][0][1][2][0]); SCHED;
        asm volatile(
            "v_mfma_f32_16x16x32_bf16 a[12:15], a[156:159], a[68:71], a[12:15]\n\t"
            :
            :
            : "memory", "scc");
        SCHED; so_.y = cvt_pk_bf16(acc[0][0][1][2][2], acc[0][0][1][2][3]); SCHED;
        asm volatile(
            "v_mfma_f32_16x16x32_bf16 a[28:31], a[152:155], a[72:75], a[28:31]\n\t"
            "buffer_load_dwordx4 %[ob1], %[rsw], %[b4] offen offset:1280  lds\n\t"
            : [b4] "+s"(b4)
            : [ob1] "v"(voffB[1]), [rsw] "s"(rsW)
            : "memory", "scc");
        SCHED; so_.x = cvt_pk_bf16(s0_[0], acc[0][0][1][2][1]); SCHED;
        asm volatile(
            "v_mfma_f32_16x16x32_bf16 a[28:31], a[156:159], a[76:79], a[28:31]\n\t"
            :
            :
            : "memory", "scc");
        SCHED; so_.z = cvt_pk_bf16(acc[0][0][1][3][0], acc[0][0][1][3][1]); SCHED;
        asm volatile(
            "v_mfma_f32_16x16x32_bf16 a[44:47], a[152:155], a[80:83], a[44:47]\n\t"
            "buffer_load_dwordx4 %[ob2], %[rsw], %[b4] offen offset:2304  lds\n\t"
            : [b4] "+s"(b4)
            : [ob2] "v"(voffB[2]), [rsw] "s"(rsW)
            : "memory", "scc");
        SCHED; so_.w = cvt_pk_bf16(acc[0][0][1][3][2], acc[0][0][1][3][3]); SCHED;
        asm volatile(
            "v_mfma_f32_16x16x32_bf16 a[44:47], a[156:159], a[84:87], a[44:47]\n\t"
            "s_waitcnt lgkmcnt(0)\n\t"
            :
            :
            : "memory", "scc");
        SCHED; SENDST(3); SCHED;
        asm volatile(
            "v_mfma_f32_16x16x32_bf16 a[60:63], a[152:155], a[88:91], a[60:63]\n\t"
            "buffer_load_dwordx4 %[ob3], %[rsw], %[b4] offen offset:3328  lds\n\t"
            "v_mfma_f32_16x16x32_bf16 a[60:63], a[156:159], a[92:95], a[60:63]\n\t"
            : [b4] "+s"(b4)
            : [ob3] "v"(voffB[3]), [rsw] "s"(rsW)
            : "memory", "scc");
        SCHED; SW(4); SCHED;
        asm volatile(
            "v_mfma_f32_16x16x32_bf16 a[0:3], a[224:227], a[96:99], a[0:3]\n\t"
            "ds_read_b128 a[64:67], %[va0] offset:16384\n\t"
            :
            : [va0] "v"(va[0])
            : "memory", "scc");
        SCHED; s0_[0] = TAGV(acc[0][0][2][2][0]); SCHED;
        asm volatile(
            "v_mfma_f32_16x16x32_bf16 a[0:3], a[228:231], a[100:103], a[0:3]\n\t"
            "ds_read_b128 a[68:71], %[va1] offset:16384\n\t"
            :
            : [va1] "v"(va[1])
            : "memory", "scc");
        SCHED; so_.y = cvt_pk_bf16(acc[0][0][2][2][2], acc[0][0][2][2][3]); SCHED;
        asm volatile(
            "v_mfma_f32_16x16x32_bf16 a[16:19], a[224:227], a[104:107], a[16:19]\n\t"
            "ds_read_b128 a[72:75], %[va0] offset:18432\n\t"
            :
            : [va0] "v"(va[0])
            : "memory", "scc");
        SCHED; so_.x = cvt_pk_bf16(s0_[0], acc[0][0][2][2][1]); SCHED;
        asm volatile(
            "v_mfma_f32_16x16x32_bf16 a[16:19], a[228:231], a[108:111], a[16:19]\n\t"
            "ds_read_b128 a[76:79], %[va1] offset:18432\n\t"
            :
            : [va1] "v"(va[1])
            : "memory", "scc");
        SCHED; so_.z = cvt_pk_bf16(acc[0][0][2][3][0], acc[0][0][2][3][1]); SCHED;
        asm volatile(
            "v_mfma_f32_16x16x32_bf16 a[32:35], a[224:227], a[112:115], a[32:35]\n\t"
            "ds_read_b128 a[80:83], %[va0] offset:20480\n\t"
            :
            : [va0] "v"(va[0])
            : "memory", "scc");
        SCHED; so_.w = cvt_pk_bf16(acc[0][0][2][3][2], acc[0][0][2][3][3]); SCHED;
        asm volatile(
            "v_mfma_f32_16x16x32_bf16 a[32:35], a[228:231], a[116:119], a[32:35]\n\t"
            "ds_read_b128 a[84:87], %[va1] offset:20480\n\t"
            :
            : [va1] "v"(va[1])
            : "memory", "scc");
        SCHED; SENDST(5); SCHED;
        asm volatile(
            "v_mfma_f32_16x16x32_bf16 a[48:51], a[224:227], a[120:123], a[48:51]\n\t"
            "ds_read_b128 a[88:91], %[va0] offset:22528\n\t"
            "v_mfma_f32_16x16x32_bf16 a[48:51], a[228:231], a[124:127], a[48:51]\n\t"
            "ds_read_b128 a[92:95], %[va1] offset:22528\n\t"
            "v_mfma_f32_16x16x32_bf16 a[4:7], a[232:235], a[96:99], a[4:7]\n\t"
            :
            : [va0] "v"(va[0]), [va1] "v"(va[1])
            : "memory", "scc");
        SCHED; s0_[0] = TAGV(acc[0][0][3][2][0]); SCHED;
        asm volatile(
            "v_mfma_f32_16x16x32_bf16 a[4:7], a[236:239], a[100:103], a[4:7]\n\t"
            :
            :
            : "memory", "scc");
        SCHED; so_.y = cvt_pk_bf16(acc[0][0][3][2][2], acc[0][0][3][2][3]); SCHED;
        asm volatile(
            "v_mfma_f32_16x16x32_bf16 a[20:23], a[232:235], a[104:107], a[20:23]\n\t"
            :
            :
            : "memory", "scc");
        SCHED; so_.x = cvt_pk_bf16(s0_[0], acc[0][0][3][2][1]); SCHED;
        asm volatile(
            "v_mfma_f32_16x16x32_bf16 a[20:23], a[236:239], a[108:111], a[20:23]\n\t"
            :
            :
            : "memory", "scc");
        SCHED; so_.z = cvt_pk_bf16(acc[0][0][3][3][0], acc[0][0][3][3][1]); SCHED;
        asm volatile(
            "v_mfma_f32_16x16x32_bf16 a[36:39], a[232:235], a[112:115], a[36:39]\n\t"
            :
            :
            : "memory", "scc");
        SCHED; so_.w = cvt_pk_bf16(acc[0][0][3][3][2], acc[0][0][3][3][3]); SCHED;
        asm volatile(
            "v_mfma_f32_16x16x32_bf16 a[36:39], a[236:239], a[116:119], a[36:39]\n\t"
            :
            :
            : "memory", "scc");
        SCHED; SENDST(7); SCHED;
        asm volatile(
            "v_mfma_f32_16x16x32_bf16 a[52:55], a[232:235], a[120:123], a[52:55]\n\t"
            "v_mfma_f32_16x16x32_bf16 a[52:55], a[236:239], a[124:127], a[52:55]\n\t"
            "v_mfma_f32_16x16x32_bf16 a[8:11], a[240:243], a[96:99], a[8:11]\n\t"
            "v_mfma_f32_16x16x32_bf16 a[8:11], a[244:247], a[100:103], a[8:11]\n\t"
            :
            :
            : "memory", "scc");
        SCHED; ARD(s0_[0], 0); ARD(s0_[1], 1); SCHED;
        asm volatile(
            "v_mfma_f32_16x16x32_bf16 a[24:27], a[240:243], a[104:107], a[24:27]\n\t"
            :
            :
            : "memory", "scc");
        SCHED; ARD(s0_[2], 2); ARD(s0_[3], 3); SCHED;
        asm volatile(
            "v_mfma_f32_16x16x32_bf16 a[24:27], a[244:247], a[108:111], a[24:27]\n\t"
            :
            :
            : "memory", "scc");
        SCHED; s0_[0] = TAGV(s0_[0]); SCHED;
        asm volatile(
            "v_mfma_f32_16x16x32_bf16 a[40:43], a[240:243], a[112:115], a[40:43]\n\t"
            :
            :
            : "memory", "scc");
        SCHED; ARD(s1_[0], 4); ARD(s1_[1], 5); SCHED;
        asm volatile(
            "v_mfma_f32_16x16x32_bf16 a[40:43], a[244:247], a[116:119], a[40:43]\n\t"
            :
            :
            : "memory", "scc");
        SCHED; ARD(s1_[2], 6); ARD(s1_[3], 7); SCHED;
        asm volatile(
            "v_mfma_f32_16x16x32_bf16 a[56:59], a[240:243], a[120:123], a[56:59]\n\t"
            :
            :
            : "memory", "scc");
        SCHED; so_.x = cvt_pk_bf16(s0_[0], s0_[1]); SCHED;
        asm volatile(
            "v_mfma_f32_16x16x32_bf16 a[56:59], a[244:247], a[124:127], a[56:59]\n\t"
            :
            :
            : "memory", "scc");
        SCHED; so_.y = cvt_pk_bf16(s0_[2], s0_[3]); SCHED;
        asm volatile(
            "v_mfma_f32_16x16x32_bf16 a[12:15], a[248:251], a[96:99], a[12:15]\n\t"
            :
            :
            : "memory", "scc");
        SCHED; so_.z = cvt_pk_bf16(s1_[0], s1_[1]); SCHED;
        asm volatile(
            "v_mfma_f32_16x16x32_bf16 a[12:15], a[252:255], a[100:103], a[12:15]\n\t"
            :
            :
            : "memory", "scc");
        SCHED; so_.w = cvt_pk_bf16(s1_[2], s1_[3]); SCHED;
        asm volatile(
            "v_mfma_f32_16x16x32_bf16 a[28:31], a[248:251], a[104:107], a[28:31]\n\t"
            :
            :
            : "memory", "scc");
        SCHED; SENDST(8); SCHED;
        asm volatile(
            "v_mfma_f32_16x16x32_bf16 a[28:31], a[252:255], a[108:111], a[28:31]\n\t"
            "s_waitcnt vmcnt(22) lgkmcnt(0)\n\t"
            "s_barrier\n\t"
            "s_add_u32 m0, %[ldsw], 0x7f00\n\t"
            "v_mfma_f32_16x16x32_bf16 a[44:47], a[248:251], a[112:115], a[44:47]\n\t"
            "buffer_load_dwordx4 %[ob0], %[rsw], %[b5] offen offset:256  lds\n\t"
            "v_mfma_f32_16x16x32_bf16 a[44:47], a[252:255], a[116:119], a[44:47]\n\t"
            "buffer_load_dwordx4 %[ob1], %[rsw], %[b5] offen offset:1280  lds\n\t"
            : [b5] "+s"(b5)
            : [ldsw] "s"(ldsw), [ob0] "v"(voffB[0]), [rsw] "s"(rsW), [ob1] "v"(voffB[1])
            : "memory", "scc");
        SCHED; ARD(s0_[0], 16); ARD(s0_[1], 17); SCHED;
        asm volatile(
            "v_mfma_f32_16x16x32_bf16 a[60:63], a[248:251], a[120:123], a[60:63]\n\t"
            "buffer_load_dwordx4 %[ob2], %[rsw], %[b5] offen offset:2304  lds\n\t"
            : [b5] "+s"(b5)
            : [ob2] "v"(voffB[2]), [rsw] "s"(rsW)
            : "memory", "scc");
        SCHED; ARD(s0_[2], 18); ARD(s0_[3], 19); SCHED;
        asm volatile(
            "v_mfma_f32_16x16x32_bf16 a[60:63], a[252:255], a[124:127], a[60:63]\n\t"
            "buffer_load_dwordx4 %[ob3], %[rsw], %[b5] offen offset:3328  lds\n\t"
            : [b5] "+s"(b5)
            : [ob3] "v"(voffB[3]), [rsw] "s"(rsW)
            : "memory", "scc");
        SCHED; s0_[0] = TAGV(s0_[0]); SCHED;
        SCHED; SW(5); SCHED;
        asm volatile(
            "v_mfma_f32_16x16x32_bf16 %[c1000], a[160:163], a[192:195], %[c1000]\n\t"
            "ds_read_b64_tr_b16 a[128:129], %[vb0] offset:0\n\t"
            : [c1000] "+v"(acc[1][0][0][0])
            : [vb0] "v"(vb[0])
            : "memory", "scc");
        SCHED; ARD(s1_[0], 20); ARD(s1_[1], 21); SCHED;
        asm volatile(
            "v_mfma_f32_16x16x32_bf16 %[c1000], a[164:167], a[196:199], %[c1000]\n\t"
            "ds_read_b64_tr_b16 a[130:131], %[vb0] offset:1024\n\t"
            : [c1000] "+v"(acc[1][0][0][0])
            : [vb0] "v"(vb[0])
            : "memory", "scc");
        SCHED; ARD(s1_[2], 22); ARD(s1_[3], 23); SCHED;
        asm volatile(
            "v_mfma_f32_16x16x32_bf16 %[c1010], a[160:163], a[200:203], %[c1010]\n\t"
            "ds_read_b64_tr_b16 a[132:133], %[vb0] offset:8192\n\t"
            : [c1010] "+v"(acc[1][0][1][0])
            : [vb0] "v"(vb[0])
            : "memory", "scc");
        SCHED; so_.x = cvt_pk_bf16(s0_[0], s0_[1]); SCHED;
        asm volatile(
            "v_mfma_f32_16x16x32_bf16 %[c1010], a[164:167], a[204:207], %[c1010]\n\t"
            "ds_read_b64_tr_b16 a[134:135], %[vb0] offset:9216\n\t"
            : [c1010] "+v"(acc[1][0][1][0])
            : [vb0] "v"(vb[0])
            : "memory", "scc");
        SCHED; so_.y = cvt_pk_bf16(s0_[2], s0_[3]); SCHED;
        asm volatile(
            "v_mfma_f32_16x16x32_bf16 %[c1020], a[160:163], a[208:211], %[c1020]\n\t"
            "ds_read_b64_tr_b16 a[136:137], %[vb1] offset:0\n\t"
            : [c1020] "+v"(acc[1][0][2][0])
            : [vb1] "v"(vb[1])
            : "memory", "scc");
        SCHED; so_.z = cvt_pk_bf16(s1_[0], s1_[1]); SCHED;
        asm volatile(
            "v_mfma_f32_16x16x32_bf16 %[c1020], a[164:167], a[212:215], %[c1020]\n\t"
            "ds_read_b64_tr_b16 a[138:139], %[vb1] offset:1024\n\t"
            : [c1020] "+v"(acc[1][0][2][0])
            : [vb1] "v"(vb[1])
            : "memory", "scc");
        SCHED; so_.w = cvt_pk_bf16(s1_[2], s1_[3]); SCHED;
        asm volatile(
            "v_mfma_f32_16x16x32_bf16 %[c1030], a[160:163], a[216:219], %[c1030]\n\t"
            "ds_read_b64_tr_b16 a[140:141], %[vb1] offset:8192\n\t"
            : [c1030] "+v"(acc[1][0][3][0])
            : [vb1] "v"(vb[1])
            : "memory", "scc");
        SCHED; SENDST(10); SCHED;
        asm volatile(
            "v_mfma_f32_16x16x32_bf16 %[c1030], a[164:167], a[220:223], %[c1030]\n\t"
            "ds_read_b64_tr_b16 a[142:143], %[vb1] offset:9216\n\t"
            "v_mfma_f32_16x16x32_bf16 %[c1001], a[168:171], a[192:195], %[c1001]\n\t"
            "ds_read_b64_tr_b16 a[144:145], %[vb2] offset:0\n\t"
            "v_mfma_f32_16x16x32_bf16 %[c1001], a[172:175], a[196:199], %[c1001]\n\t"
            "ds_read_b64_tr_b16 a[146:147], %[vb2] offset:1024\n\t"
            : [c1030] "+v"(acc[1][0][3][0]), [c1001] "+v"(acc[1][0][0][1])
            : [vb1] "v"(vb[1]), [vb2] "v"(vb[2])
            : "memory", "scc");
        SCHED; ARD(s0_[0], 32); ARD(s0_[1], 33); SCHED;
        asm volatile(
            "v_mfma_f32_16x16x32_bf16 %[c1011], a[168:171], a[200:203], %[c1011]\n\t"
            "ds_read_b64_tr_b16 a[148:149], %[vb2] offset:8192\n\t"
            : [c1011] "+v"(acc[1][0][1][1])
            : [vb2] "v"(vb[2])
            : "memory", "scc");
        SCHED; ARD(s0_[2], 34); ARD(s0_[3], 35); SCHED;
        asm volatile(
            "v_mfma_f32_16x16x32_bf16 %[c1011], a[172:175], a[204:207], %[c1011]\n\t"
            "ds_read_b64_tr_b16 a[150:151], %[vb2] offset:9216\n\t"
            : [c1011] "+v"(acc[1][0][1][1])
            : [vb2] "v"(vb[2])
            : "memory", "scc");
        SCHED; s0_[0] = TAGV(s0_[0]); SCHED;
        asm volatile(
            "v_mfma_f32_16x16x32_bf16 %[c1021], a[168:171], a[208:211], %[c1021]\n\t"
            "ds_read_b64_tr_b16 a[152:153], %[vb3] offset:0\n\t"
            : [c1021] "+v"(acc[1][0][2][1])
            : [vb3] "v"(vb[3])
            : "memory", "scc");
        SCHED; ARD(s1_[0], 36); ARD(s1_[1], 37); SCHED;
        asm volatile(
            "v_mfma_f32_16x16x32_bf16 %[c1021], a[172:175], a[212:215], %[c1021]\n\t"
            "ds_read_b64_tr_b16 a[154:155], %[vb3] offset:1024\n\t"
            : [c1021] "+v"(acc[1][0][2][1])
            : [vb3] "v"(vb[3])
            : "memory", "scc");
        SCHED; ARD(s1_[2], 38); ARD(s1_[3], 39); SCHED;
        asm volatile(
            "v_mfma_f32_16x16x32_bf16 %[c1031], a[168:171], a[216:219], %[c1031]\n\t"
            "ds_read_b64_tr_b16 a[156:157], %[vb3] offset:8192\n\t"
            : [c1031] "+v"(acc[1][0][3][1])
            : [vb3] "v"(vb[3])
            : "memory", "scc");
        SCHED; so_.x = cvt_pk_bf16(s0_[0], s0_[1]); SCHED;
        asm volatile(
            "v_mfma_f32_16x16x32_bf16 %[c1031], a[172:175], a[220:223], %[c1031]\n\t"
            "ds_read_b64_tr_b16 a[158:159], %[vb3] offset:9216\n\t"
            : [c1031] "+v"(acc[1][0][3][1])
            : [vb3] "v"(vb[3])
            : "memory", "scc");
        SCHED; so_.y = cvt_pk_bf16(s0_[2], s0_[3]); SCHED;
        asm volatile(
            "v_mfma_f32_16x16x32_bf16 %[c1002], a[176:179], a[192:195], %[c1002]\n\t"
            : [c1002] "+v"(acc[1][0][0][2])
            :
            : "memory", "scc");
        SCHED; so_.z = cvt_pk_bf16(s1_[0], s1_[1]); SCHED;
        asm volatile(
            "v_mfma_f32_16x16x32_bf16 %[c1002], a[180:183], a[196:199], %[c1002]\n\t"
            : [c1002] "+v"(acc[1][0][0][2])
            :
            : "memory", "scc");
        SCHED; so_.w = cvt_pk_bf16(s1_[2], s1_[3]); SCHED;
        asm volatile(
            "v_mfma_f32_16x16x32_bf16 %[c1012], a[176:179], a[200:203], %[c1012]\n\t"
            : [c1012] "+v"(acc[1][0][1][2])
            :
            : "memory", "scc");
        SCHED; SENDST(12); SCHED;
        asm volatile(
            "v_mfma_f32_16x16x32_bf16 %[c1012], a[180:183], a[204:207], %[c1012]\n\t"
            "v_mfma_f32_16x16x32_bf16 %[c1022], a[176:179], a[208:211], %[c1022]\n\t"
            "v_mfma_f32_16x16x32_bf16 %[c1022], a[180:183], a[212:215], %[c1022]\n\t"
            : [c1012] "+v"(acc[1][0][1][2]), [c1022] "+v"(acc[1][0][2][2])
            :
            : "memory", "scc");
        SCHED; ARD(s0_[0], 48); ARD(s0_[1], 49); SCHED;
        asm volatile(
            "v_mfma_f32_16x16x32_bf16 %[c1032], a[176:179], a[216:219], %[c1032]\n\t"
            : [c1032] "+v"(acc[1][0][3][2])
            :
            : "memory", "scc");
        SCHED; ARD(s0_[2], 50); ARD(s0_[3], 51); SCHED;
        asm volatile(
            "v_mfma_f32_16x16x32_bf16 %[c1032], a[180:183], a[220:223], %[c1032]\n\t"
            : [c1032] "+v"(acc[1][0][3][2])
            :
            : "memory", "scc");
        SCHED; s0_[0] = TAGV(s0_[0]); SCHED;
        asm volatile(
            "v_mfma_f32_16x16x32_bf16 %[c1003], a[184:187], a[192:195], %[c1003]\n\t"
            : [c1003] "+v"(acc[1][0][0][3])
            :
            : "memory", "scc");
        SCHED; ARD(s1_[0], 52); ARD(s1_[1], 53); SCHED;
        asm volatile(
            "v_mfma_f32_16x16x32_bf16 %[c1003], a[188:191], a[196:199], %[c1003]\n\t"
            : [c1003] "+v"(acc[1][0][0][3])
            :
            : "memory", "scc");
        SCHED; ARD(s1_[2], 54); ARD(s1_[3], 55); SCHED;
        asm volatile(
            "v_mfma_f32_16x16x32_bf16 %[c1013], a[184:187], a[200:203], %[c1013]\n\t"
            : [c1013] "+v"(acc[1][0][1][3])
            :
            : "memory", "scc");
        SCHED; so_.x = cvt_pk_bf16(s0_[0], s0_[1]); SCHED;
        asm volatile(
            "v_mfma_f32_16x16x32_bf16 %[c1013], a[188:191], a[204:207], %[c1013]\n\t"
            "s_waitcnt lgkmcnt(0)\n\t"
            : [c1013] "+v"(acc[1][0][1][3])
            :
            : "memory", "scc");
        SCHED; so_.y = cvt_pk_bf16(s0_[2], s0_[3]); SCHED;
        asm volatile(
            "v_mfma_f32_16x16x32_bf16 %[c1023], a[184:187], a[208:211], %[c1023]\n\t"
            : [c1023] "+v"(acc[1][0][2][3])
            :
            : "memory", "scc");
        SCHED; so_.z = cvt_pk_bf16(s1_[0], s1_[1]); SCHED;
        asm volatile(
            "v_mfma_f32_16x16x32_bf16 %[c1023], a[188:191], a[212:215], %[c1023]\n\t"
            : [c1023] "+v"(acc[1][0][2][3])
            :
            : "memory", "scc");
        SCHED; so_.w = cvt_pk_bf16(s1_[2], s1_[3]); SCHED;
        asm volatile(
            "v_mfma_f32_16x16x32_bf16 %[c1033], a[184:187], a[216:219], %[c1033]\n\t"
            : [c1033] "+v"(acc[1][0][3][3])
            :
            : "memory", "scc");
        SCHED; SENDST(14); SCHED;
        asm volatile(
            "v_mfma_f32_16x16x32_bf16 %[c1033], a[188:191], a[220:223], %[c1033]\n\t"
            : [c1033] "+v"(acc[1][0][3][3])
            :
            : "memory", "scc");
        SCHED; SW(6); SCHED;
        asm volatile(
            "v_mfma_f32_16x16x32_bf16 %[c1000], a[128:131], a[64:67], %[c1000]\n\t"
            "ds_read_b64_tr_b16 a[160:161], %[vb0] offset:32768\n\t"
            "v_mfma_f32_16x16x32_bf16 %[c1000], a[132:135], a[68:71], %[c1000]\n\t"
            "ds_read_b64_tr_b16 a[162:163], %[vb0] offset:33792\n\t"
            : [c1000] "+v"(acc[1][0][0][0])
            : [vb0] "v"(vb[0])
            : "memory", "scc");
        SCHED; ARD(s0_[0], 8); ARD(s0_[1], 9); SCHED;
        asm volatile(
            "v_mfma_f32_16x16x32_bf16 %[c1010], a[128:131], a[72:75], %[c1010]\n\t"
            "ds_read_b64_tr_b16 a[164:165], %[vb0] offset:40960\n\t"
            : [c1010] "+v"(acc[1][0][1][0])
            : [vb0] "v"(vb[0])
            : "memory", "scc");
        SCHED; ARD(s0_[2], 10); ARD(s0_[3], 11); SCHED;
        asm volatile(
            "v_mfma_f32_16x16x32_bf16 %[c1010], a[132:135], a[76:79], %[c1010]\n\t"
            "ds_read_b64_tr_b16 a[166:167], %[vb0] offset:41984\n\t"
            : [c1010] "+v"(acc[1][0][1][0])
            : [vb0] "v"(vb[0])
            : "memory", "scc");
        SCHED; s0_[0] = TAGV(s0_[0]); SCHED;
        asm volatile(
            "v_mfma_f32_16x16x32_bf16 %[c1020], a[128:131], a[80:83], %[c1020]\n\t"
            "ds_read_b64_tr_b16 a[168:169], %[vb1] offset:32768\n\t"
            : [c1020] "+v"(acc[1][0][2][0])
            : [vb1] "v"(vb[1])
            : "memory", "scc");
        SCHED; ARD(s1_[0], 12); ARD(s1_[1], 13); SCHED;
        asm volatile(
            "v_mfma_f32_16x16x32_bf16 %[c1020], a[132:135], a[84:87], %[c1020]\n\t"
            "ds_read_b64_tr_b16 a[170:171], %[vb1] offset:33792\n\t"
            : [c1020] "+v"(acc[1][0][2][0])
            : [vb1] "v"(vb[1])
            : "memory", "scc");
        SCHED; ARD(s1_[2], 14); ARD(s1_[3], 15); SCHED;
        asm volatile(
            "v_mfma_f32_16x16x32_bf16 %[c1030], a[128:131], a[88:91], %[c1030]\n\t"
            "ds_read_b64_tr_b16 a[172:173], %[vb1] offset:40960\n\t"
            : [c1030] "+v"(acc[1][0][3][0])
            : [vb1] "v"(vb[1])
            : "memory", "scc");
        SCHED; so_.x = cvt_pk_bf16(s0_[0], s0_[1]); SCHED;
        asm volatile(
            "v_mfma_f32_16x16x32_bf16 %[c1030], a[132:135], a[92:95], %[c1030]\n\t"
            "ds_read_b64_tr_b16 a[174:175], %[vb1] offset:41984\n\t"
            : [c1030] "+v"(acc[1][0][3][0])
            : [vb1] "v"(vb[1])
            : "memory", "scc");
        SCHED; so_.y = cvt_pk_bf16(s0_[2], s0_[3]); SCHED;
        asm volatile(
            "v_mfma_f32_16x16x32_bf16 %[c1001], a[136:139], a[64:67], %[c1001]\n\t"
            "ds_read_b64_tr_b16 a[176:177], %[vb2] offset:32768\n\t"
            : [c1001] "+v"(acc[1][0][0][1])
            : [vb2] "v"(vb[2])
            : "memory", "scc");
        SCHED; so_.z = cvt_pk_bf16(s1_[0], s1_[1]); SCHED;
        asm volatile(
            "v_mfma_f32_16x16x32_bf16 %[c1001], a[140:143], a[68:71], %[c1001]\n\t"
            "ds_read_b64_tr_b16 a[178:179], %[vb2] offset:33792\n\t"
            : [c1001] "+v"(acc[1][0][0][1])
            : [vb2] "v"(vb[2])
            : "memory", "scc");
        SCHED; so_.w = cvt_pk_bf16(s1_[2], s1_[3]); SCHED;
        asm volatile(
            "v_mfma_f32_16x16x32_bf16 %[c1011], a[136:139], a[72:75], %[c1011]\n\t"
            "ds_read_b64_tr_b16 a[180:181], %[vb2] offset:40960\n\t"
            : [c1011] "+v"(acc[1][0][1][1])
            : [vb2] "v"(vb[2])
            : "memory", "scc");
        SCHED; SENDST(9); SCHED;
        asm volatile(
            "v_mfma_f32_16x16x32_bf16 %[c1011], a[140:143], a[76:79], %[c1011]\n\t"
            "ds_read_b64_tr_b16 a[182:183], %[vb2] offset:41984\n\t"
            "v_mfma_f32_16x16x32_bf16 %[c1021], a[136:139], a[80:83], %[c1021]\n\t"
            "ds_read_b64_tr_b16 a[184:185], %[vb3] offset:32768\n\t"
            "v_mfma_f32_16x16x32_bf16 %[c1021], a[140:143], a[84:87], %[c1021]\n\t"
            "ds_read_b64_tr_b16 a[186:187], %[vb3] offset:33792\n\t"
            : [c1011] "+v"(acc[1][0][1][1]), [c1021] "+v"(acc[1][0][2][1])
            : [vb2] "v"(vb[2]), [vb3] "v"(vb[3])
            : "memory", "scc");
        SCHED; ARD(s0_[0], 24); ARD(s0_[1], 25); SCHED;
        asm volatile(
            "v_mfma_f32_16x16x32_bf16 %[c1031], a[136:139], a[88:91], %[c1031]\n\t"
            "ds_read_b64_tr_b16 a[188:189], %[vb3] offset:40960\n\t"
            : [c1031] "+v"(acc[1][0][3][1])
            : [vb3] "v"(vb[3])
            : "memory", "scc");
        SCHED; ARD(s0_[2], 26); ARD(s0_[3], 27); SCHED;
        asm volatile(
            "v_mfma_f32_16x16x32_bf16 %[c1031], a[140:143], a[92:95], %[c1031]\n\t"
            "ds_read_b64_tr_b16 a[190:191], %[vb3] offset:41984\n\t"
            : [c1031] "+v"(acc[1][0][3][1])
            : [vb3] "v"(vb[3])
            : "memory", "scc");
        SCHED; s0_[0] = TAGV(s0_[0]); SCHED;
        asm volatile(
            "v_mfma_f32_16x16x32_bf16 %[c1002], a[144:147], a[64:67], %[c1002]\n\t"
            "ds_read_b128 a[96:99], %[va0] offset:49152\n\t"
            : [c1002] "+v"(acc[1][0][0][2])
            : [va0] "v"(va[0])
            : "memory", "scc");
        SCHED; ARD(s1_[0], 28); ARD(s1_[1], 29); SCHED;
        asm volatile(
            "v_mfma_f32_16x16x32_bf16 %[c1002], a[148:151], a[68:71], %[c1002]\n\t"
            "ds_read_b128 a[100:103], %[va1] offset:49152\n\t"
            : [c1002] "+v"(acc[1][0][0][2])
            : [va1] "v"(va[1])
            : "memory", "scc");
        SCHED; ARD(s1_[2], 30); ARD(s1_[3], 31); SCHED;
        asm volatile(
            "v_mfma_f32_16x16x32_bf16 %[c1012], a[144:147], a[72:75], %[c1012]\n\t"
            "ds_read_b128 a[104:107], %[va0] offset:51200\n\t"
            : [c1012] "+v"(acc[1][0][1][2])
            : [va0] "v"(va[0])
            : "memory", "scc");
        SCHED; so_.x = cvt_pk_bf16(s0_[0], s0_[1]); SCHED;
        asm volatile(
            "v_mfma_f32_16x16x32_bf16 %[c1012], a[148:151], a[76:79], %[c1012]\n\t"
            "ds_read_b128 a[108:111], %[va1] offset:51200\n\t"
            : [c1012] "+v"(acc[1][0][1][2])
            : [va1] "v"(va[1])
            : "memory", "scc");
        SCHED; so_.y = cvt_pk_bf16(s0_[2], s0_[3]); SCHED;
        asm volatile(
            "v_mfma_f32_16x16x32_bf16 %[c1022], a[144:147], a[80:83], %[c1022]\n\t"
            "ds_read_b128 a[112:115], %[va0] offset:53248\n\t"
            : [c1022] "+v"(acc[1][0][2][2])
            : [va0] "v"(va[0])
            : "memory", "scc");
        SCHED; so_.z = cvt_pk_bf16(s1_[0], s1_[1]); SCHED;
        asm volatile(
            "v_mfma_f32_16x16x32_bf16 %[c1022], a[148:151], a[84:87], %[c1022]\n\t"
            "ds_read_b128 a[116:119], %[va1] offset:53248\n\t"
            : [c1022] "+v"(acc[1][0][2][2])
            : [va1] "v"(va[1])
            : "memory", "scc");
        SCHED; so_.w = cvt_pk_bf16(s1_[2], s1_[3]); SCHED;
        asm volatile(
            "v_mfma_f32_16x16x32_bf16 %[c1032], a[144:147], a[88:91], %[c1032]\n\t"
            "ds_read_b128 a[120:123], %[va0] offset:55296\n\t"
            : [c1032] "+v"(acc[1][0][3][2])
            : [va0] "v"(va[0])
            : "memory", "scc");
        SCHED; SENDST(11); SCHED;
        asm volatile(
            "v_mfma_f32_16x16x32_bf16 %[c1032], a[148:151], a[92:95], %[c1032]\n\t"
            "ds_read_b128 a[124:127], %[va1] offset:55296\n\t"
            "v_mfma_f32_16x16x32_bf16 %[c1003], a[152:155], a[64:67], %[c1003]\n\t"
            "v_mfma_f32_16x16x32_bf16 %[c1003], a[156:159], a[68:71], %[c1003]\n\t"
            : [c1032] "+v"(acc[1][0][3][2]), [c1003] "+v"(acc[1][0][0][3])
            : [va1] "v"(va[1])
            : "memory", "scc");
        SCHED; ARD(s0_[0], 40); ARD(s0_[1], 41); SCHED;
        asm volatile(
            "v_mfma_f32_16x16x32_bf16 %[c1013], a[152:155], a[72:75], %[c1013]\n\t"
            : [c1013] "+v"(acc[1][0][1][3])
            :
            : "memory", "scc");
        SCHED; ARD(s0_[2], 42); ARD(s0_[3], 43); SCHED;
        asm volatile(
            "v_mfma_f32_16x16x32_bf16 %[c1013], a[156:159], a[76:79], %[c1013]\n\t"
            : [c1013] "+v"(acc[1][0][1][3])
            :
            : "memory", "scc");
        SCHED; s0_[0] = TAGV(s0_[0]); SCHED;
        asm volatile(
            "v_mfma_f32_16x16x32_bf16 %[c1023], a[152:155], a[80:83], %[c1023]\n\t"
            "s_waitcnt vmcnt(20) lgkmcnt(0)\n\t"
            "s_barrier\n\t"
            : [c1023] "+v"(acc[1][0][2][3])
            :
            : "memory", "scc");
        SCHED; ARD(s1_[0], 44); ARD(s1_[1], 45); SCHED;
        asm volatile(
            "v_mfma_f32_16x16x32_bf16 %[c1023], a[156:159], a[84:87], %[c1023]\n\t"
            : [c1023] "+v"(acc[1][0][2][3])
            :
            : "memory", "scc");
        SCHED; ARD(s1_[2], 46); ARD(s1_[3], 47); SCHED;
        asm volatile(
            "v_mfma_f32_16x16x32_bf16 %[c1033], a[152:155], a[88:91], %[c1033]\n\t"
            : [c1033] "+v"(acc[1][0][3][3])
            :
            : "memory", "scc");
        SCHED; so_.x = cvt_pk_bf16(s0_[0], s0_[1]); SCHED;
        asm volatile(
            "v_mfma_f32_16x16x32_bf16 %[c1033], a[156:159], a[92:95], %[c1033]\n\t"
            : [c1033] "+v"(acc[1][0][3][3])
            :
            : "memory", "scc");
        SCHED; so_.y = cvt_pk_bf16(s0_[2], s0_[3]); SCHED;
        SCHED; SW(7); SCHED;
        asm volatile(
            "v_mfma_f32_16x16x32_bf16 %[c1000], a[160:163], a[96:99], %[c1000]\n\t"
            "ds_read_b64_tr_b16 a[128:129], %[vbx0] offset:16384\n\t"
            : [c1000] "+v"(acc[1][0][0][0])
            : [vbx0] "v"(vbx[0])
            : "memory", "scc");
        SCHED; so_.z = cvt_pk_bf16(s1_[0], s1_[1]); SCHED;
        asm volatile(
            "v_mfma_f32_16x16x32_bf16 %[c1000], a[164:167], a[100:103], %[c1000]\n\t"
            "ds_read_b64_tr_b16 a[130:131], %[vbx0] offset:17408\n\t"
            : [c1000] "+v"(acc[1][0][0][0])
            : [vbx0] "v"(vbx[0])
            : "memory", "scc");
        SCHED; so_.w = cvt_pk_bf16(s1_[2], s1_[3]); SCHED;
        asm volatile(
            "v_mfma_f32_16x16x32_bf16 %[c1010], a[160:163], a[104:107], %[c1010]\n\t"
            "ds_read_b64_tr_b16 a[132:133], %[vbx0] offset:24576\n\t"
            : [c1010] "+v"(acc[1][0][1][0])
            : [vbx0] "v"(vbx[0])
            : "memory", "scc");
        SCHED; SENDST(13); SCHED;
        asm volatile(
            "v_mfma_f32_16x16x32_bf16 %[c1010], a[164:167], a[108:111], %[c1010]\n\t"
            "ds_read_b64_tr_b16 a[134:135], %[vbx0] offset:25600\n\t"
            "v_mfma_f32_16x16x32_bf16 %[c1020], a[160:163], a[112:115], %[c1020]\n\t"
            "ds_read_b64_tr_b16 a[136:137], %[vbx1] offset:16384\n\t"
            "v_mfma_f32_16x16x32_bf16 %[c1020], a[164:167], a[116:119], %[c1020]\n\t"
            "ds_read_b64_tr_b16 a[138:139], %[vbx1] offset:17408\n\t"
            : [c1010] "+v"(acc[1][0][1][0]), [c1020] "+v"(acc[1][0][2][0])
            : [vbx0] "v"(vbx[0]), [vbx1] "v"(vbx[1])
            : "memory", "scc");
        SCHED; ARD(s0_[0], 56); ARD(s0_[1], 57); SCHED;
        asm volatile(
            "v_mfma_f32_16x16x32_bf16 %[c1030], a[160:163], a[120:123], %[c1030]\n\t"
            "ds_read_b64_tr_b16 a[140:141], %[vbx1] offset:24576\n\t"
            : [c1030] "+v"(acc[1][0][3][0])
            : [vbx1] "v"(vbx[1])
            : "memory", "scc");
        SCHED; ARD(s0_[2], 58); ARD(s0_[3], 59); SCHED;
        asm volatile(
            "v_mfma_f32_16x16x32_bf16 %[c1030], a[164:167], a[124:127], %[c1030]\n\t"
            "ds_read_b64_tr_b16 a[142:143], %[vbx1] offset:25600\n\t"
            : [c1030] "+v"(acc[1][0][3][0])
            : [vbx1] "v"(vbx[1])
            : "memory", "scc");
        SCHED; s0_[0] = TAGV(s0_[0]); SCHED;
        asm volatile(
            "v_mfma_f32_16x16x32_bf16 %[c1001], a[168:171], a[96:99], %[c1001]\n\t"
            "ds_read_b64_tr_b16 a[144:145], %[vbx2] offset:16384\n\t"
            : [c1001] "+v"(acc[1][0][0][1])
            : [vbx2] "v"(vbx[2])
            : "memory", "scc");
        SCHED; ARD(s1_[0], 60); ARD(s1_[1], 61); SCHED;
        SCHED; PVLD1(0, 0); SCHED;
        asm volatile(
            "v_mfma_f32_16x16x32_bf16 %[c1001], a[172:175], a[100:103], %[c1001]\n\t"
            "ds_read_b64_tr_b16 a[146:147], %[vbx2] offset:17408\n\t"
            : [c1001] "+v"(acc[1][0][0][1])
            : [vbx2] "v"(vbx[2])
            : "memory", "scc");
        SCHED; ARD(s1_[2], 62); ARD(s1_[3], 63); SCHED;
        asm volatile(
            "v_mfma_f32_16x16x32_bf16 %[c1011], a[168:171], a[104:107], %[c1011]\n\t"
            "ds_read_b64_tr_b16 a[148:149], %[vbx2] offset:24576\n\t"
            : [c1011] "+v"(acc[1][0][1][1])
            : [vbx2] "v"(vbx[2])
            : "memory", "scc");
        SCHED; so_.x = cvt_pk_bf16(s0_[0], s0_[1]); SCHED;
        SCHED; PVLD1(0, 1); SCHED;
        asm volatile(
            "v_mfma_f32_16x16x32_bf16 %[c1011], a[172:175], a[108:111], %[c1011]\n\t"
            "ds_read_b64_tr_b16 a[150:151], %[vbx2] offset:25600\n\t"
            : [c1011] "+v"(acc[1][0][1][1])
            : [vbx2] "v"(vbx[2])
            : "memory", "scc");
        SCHED; so_.y = cvt_pk_bf16(s0_[2], s0_[3]); SCHED;
        asm volatile(
            "v_mfma_f32_16x16x32_bf16 %[c1021], a[168:171], a[112:115], %[c1021]\n\t"
            "ds_read_b64_tr_b16 a[152:153], %[vbx3] offset:16384\n\t"
            : [c1021] "+v"(acc[1][0][2][1])
            : [vbx3] "v"(vbx[3])
            : "memory", "scc");
        SCHED; so_.z = cvt_pk_bf16(s1_[0], s1_[1]); SCHED;
        SCHED; PVLD1(0, 2); SCHED;
        asm volatile(
            "v_mfma_f32_16x16x32_bf16 %[c1021], a[172:175], a[116:119], %[c1021]\n\t"
            "ds_read_b64_tr_b16 a[154:155], %[vbx3] offset:17408\n\t"
            : [c1021] "+v"(acc[1][0][2][1])
            : [vbx3] "v"(vbx[3])
            : "memory", "scc");
        SCHED; so_.w = cvt_pk_bf16(s1_[2], s1_[3]); SCHED;
        asm volatile(
            "v_mfma_f32_16x16x32_bf16 %[c1031], a[168:171], a[120:123], %[c1031]\n\t"
            "ds_read_b64_tr_b16 a[156:157], %[vbx3] offset:24576\n\t"
            : [c1031] "+v"(acc[1][0][3][1])
            : [vbx3] "v"(vbx[3])
            : "memory", "scc");
        SCHED; SENDST(15); SCHED;
        SCHED; PVLD1(0, 3); SCHED;
        asm volatile(
            "v_mfma_f32_16x16x32_bf16 %[c1031], a[172:175], a[124:127], %[c1031]\n\t"
            "ds_read_b64_tr_b16 a[158:159], %[vbx3] offset:25600\n\t"
            "v_mfma_f32_16x16x32_bf16 %[c1002], a[176:179], a[96:99], %[c1002]\n\t"
            "ds_read_b128 a[224:227], %[va0] offset:0\n\t"
            : [c1031] "+v"(acc[1][0][3][1]), [c1002] "+v"(acc[1][0][0][2])
            : [vbx3] "v"(vbx[3]), [va0] "v"(va[0])
            : "memory", "scc");
        SCHED; PVLD1(0, 4); SCHED;
        asm volatile(
            "v_mfma_f32_16x16x32_bf16 %[c1002], a[180:183], a[100:103], %[c1002]\n\t"
            "ds_read_b128 a[228:231], %[va1] offset:0\n\t"
            "v_mfma_f32_16x16x32_bf16 %[c1012], a[176:179], a[104:107], %[c1012]\n\t"
            "ds_read_b128 a[232:235], %[va0] offset:2048\n\t"
            : [c1002] "+v"(acc[1][0][0][2]), [c1012] "+v"(acc[1][0][1][2])
            : [va1] "v"(va[1]), [va0] "v"(va[0])
            : "memory", "scc");
        SCHED; PVLD1(0, 5); SCHED;
        asm volatile(
            "v_mfma_f32_16x16x32_bf16 %[c1012], a[180:183], a[108:111], %[c1012]\n\t"
            "ds_read_b128 a[236:239], %[va1] offset:2048\n\t"
            "v_mfma_f32_16x16x32_bf16 %[c1022], a[176:179], a[112:115], %[c1022]\n\t"
            "ds_read_b128 a[240:243], %[va0] offset:4096\n\t"
            : [c1012] "+v"(acc[1][0][1][2]), [c1022] "+v"(acc[1][0][2][2])
            : [va1] "v"(va[1]), [va0] "v"(va[0])
            : "memory", "scc");
        SCHED; PVLD1(0, 6); SCHED;
        asm volatile(
            "v_mfma_f32_16x16x32_bf16 %[c1022], a[180:183], a[116:119], %[c1022]\n\t"
            "ds_read_b128 a[244:247], %[va1] offset:4096\n\t"
            "v_mfma_f32_16x16x32_bf16 %[c1032], a[176:179], a[120:123], %[c1032]\n\t"
            "ds_read_b128 a[248:251], %[va0] offset:6144\n\t"
            : [c1022] "+v"(acc[1][0][2][2]), [c1032] "+v"(acc[1][0][3][2])
            : [va1] "v"(va[1]), [va0] "v"(va[0])
            : "memory", "scc");
        SCHED; PVLD1(0, 7); SCHED;
        asm volatile(
            "v_mfma_f32_16x16x32_bf16 %[c1032], a[180:183], a[124:127], %[c1032]\n\t"
            "ds_read_b128 a[252:255], %[va1] offset:6144\n\t"
            "v_mfma_f32_16x16x32_bf16 %[c1003], a[184:187], a[96:99], %[c1003]\n\t"
            "v_mfma_f32_16x16x32_bf16 %[c1003], a[188:191], a[100:103], %[c1003]\n\t"
            "v_mfma_f32_16x16x32_bf16 %[c1013], a[184:187], a[104:107], %[c1013]\n\t"
            "v_mfma_f32_16x16x32_bf16 %[c1013], a[188:191], a[108:111], %[c1013]\n\t"
            "s_waitcnt lgkmcnt(0)\n\t"
            "v_mfma_f32_16x16x32_bf16 %[c1023], a[184:187], a[112:115], %[c1023]\n\t"
            "v_mfma_f32_16x16x32_bf16 %[c1023], a[188:191], a[116:119], %[c1023]\n\t"
            "v_mfma_f32_16x16x32_bf16 %[c1033], a[184:187], a[120:123], %[c1033]\n\t"
            "v_mfma_f32_16x16x32_bf16 %[c1033], a[188:191], a[124:127], %[c1033]\n\t"
            : [c1032] "+v"(acc[1][0][3][2]), [c1003] "+v"(acc[1][0][0][3]), [c1013] "+v"(acc[1][0][1][3]), [c1023] "+v"(acc[1][0][2][3]), [c1033] "+v"(acc[1][0][3][3])
            : [va1] "v"(va[1])
            : "memory", "scc");
        SCHED; SW(8); SCHED;
        asm volatile(
            "v_mfma_f32_16x16x32_bf16 %[c1000], a[128:131], a[224:227], %[c1000]\n\t"
            "ds_read_b64_tr_b16 a[160:161], %[vb0] offset:49152\n\t"
            "v_mfma_f32_16x16x32_bf16 %[c1000], a[132:135], a[228:231], %[c1000]\n\t"
            "ds_read_b64_tr_b16 a[162:163], %[vb0] offset:50176\n\t"
            "v_mfma_f32_16x16x32_bf16 %[c1010], a[128:131], a[232:235], %[c1010]\n\t"
            "ds_read_b64_tr_b16 a[164:165], %[vb0] offset:57344\n\t"
            "v_mfma_f32_16x16x32_bf16 %[c1010], a[132:135], a[236:239], %[c1010]\n\t"
            "ds_read_b64_tr_b16 a[166:167], %[vb0] offset:58368\n\t"
            "v_mfma_f32_16x16x32_bf16 %[c1020], a[128:131], a[240:243], %[c1020]\n\t"
            "ds_read_b64_tr_b16 a[168:169], %[vb1] offset:49152\n\t"
            "v_mfma_f32_16x16x32_bf16 %[c1020], a[132:135], a[244:247], %[c1020]\n\t"
            "ds_read_b64_tr_b16 a[170:171], %[vb1] offset:50176\n\t"
            "v_mfma_f32_16x16x32_bf16 %[c1030], a[128:131], a[248:251], %[c1030]\n\t"
            "ds_read_b64_tr_b16 a[172:173], %[vb1] offset:57344\n\t"
            "v_mfma_f32_16x16x32_bf16 %[c1030], a[132:135], a[252:255], %[c1030]\n\t"
            "ds_read_b64_tr_b16 a[174:175], %[vb1] offset:58368\n\t"
            "v_mfma_f32_16x16x32_bf16 %[c1001], a[136:139], a[224:227], %[c1001]\n\t"
            "ds_read_b64_tr_b16 a[176:177], %[vb2] offset:49152\n\t"
            "v_mfma_f32_16x16x32_bf16 %[c1001], a[140:143], a[228:231], %[c1001]\n\t"
            "ds_read_b64_tr_b16 a[178:179], %[vb2] offset:50176\n\t"
            "v_mfma_f32_16x16x32_bf16 %[c1011], a[136:139], a[232:235], %[c1011]\n\t"
            "ds_read_b64_tr_b16 a[180:181], %[vb2] offset:57344\n\t"
            "v_mfma_f32_16x16x32_bf16 %[c1011], a[140:143], a[236:239], %[c1011]\n\t"
            "ds_read_b64_tr_b16 a[182:183], %[vb2] offset:58368\n\t"
            "v_mfma_f32_16x16x32_bf16 %[c1021], a[136:139], a[240:243], %[c1021]\n\t"
            "ds_read_b64_tr_b16 a[184:185], %[vb3] offset:49152\n\t"
            "v_mfma_f32_16x16x32_bf16 %[c1021], a[140:143], a[244:247], %[c1021]\n\t"
            "ds_read_b64_tr_b16 a[186:187], %[vb3] offset:50176\n\t"
            "v_mfma_f32_16x16x32_bf16 %[c1031], a[136:139], a[248:251], %[c1031]\n\t"
            "ds_read_b64_tr_b16 a[188:189], %[vb3] offset:57344\n\t"
            "v_mfma_f32_16x16x32_bf16 %[c1031], a[140:143], a[252:255], %[c1031]\n\t"
            "ds_read_b64_tr_b16 a[190:191], %[vb3] offset:58368\n\t"
            "v_mfma_f32_16x16x32_bf16 %[c1002], a[144:147], a[224:227], %[c1002]\n\t"
            "v_mfma_f32_16x16x32_bf16 %[c1002], a[148:151], a[228:231], %[c1002]\n\t"
            : [c1000] "+v"(acc[1][0][0][0]), [c1010] "+v"(acc[1][0][1][0]), [c1020] "+v"(acc[1][0][2][0]), [c1030] "+v"(acc[1][0][3][0]), [c1001] "+v"(acc[1][0][0][1]), [c1011] "+v"(acc[1][0][1][1]), [c1021] "+v"(acc[1][0][2][1]), [c1031] "+v"(acc[1][0][3][1]), [c1002] "+v"(acc[1][0][0][2])
            : [vb0] "v"(vb[0]), [vb1] "v"(vb[1]), [vb2] "v"(vb[2]), [vb3] "v"(vb[3])
            : "memory", "scc");
        SCHED; SETUP0; SCHED;
        asm volatile(
            "v_mfma_f32_16x16x32_bf16 %[c1012], a[144:147], a[232:235], %[c1012]\n\t"
            "v_mfma_f32_16x16x32_bf16 %[c1012], a[148:151], a[236:239], %[c1012]\n\t"
            "v_mfma_f32_16x16x32_bf16 %[c1022], a[144:147], a[240:243], %[c1022]\n\t"
            : [c1012] "+v"(acc[1][0][1][2]), [c1022] "+v"(acc[1][0][2][2])
            :
            : "memory", "scc");
        SCHED; SETUP1; SCHED;
        asm volatile(
            "v_mfma_f32_16x16x32_bf16 %[c1022], a[148:151], a[244:247], %[c1022]\n\t"
            "v_mfma_f32_16x16x32_bf16 %[c1032], a[144:147], a[248:251], %[c1032]\n\t"
            "v_mfma_f32_16x16x32_bf16 %[c1032], a[148:151], a[252:255], %[c1032]\n\t"
            : [c1022] "+v"(acc[1][0][2][2]), [c1032] "+v"(acc[1][0][3][2])
            :
            : "memory", "scc");
        SCHED; SETUP2; SCHED;
        asm volatile(
            "v_mfma_f32_16x16x32_bf16 %[c1003], a[152:155], a[224:227], %[c1003]\n\t"
            "v_mfma_f32_16x16x32_bf16 %[c1003], a[156:159], a[228:231], %[c1003]\n\t"
            "v_mfma_f32_16x16x32_bf16 %[c1013], a[152:155], a[232:235], %[c1013]\n\t"
            "v_mfma_f32_16x16x32_bf16 %[c1013], a[156:159], a[236:239], %[c1013]\n\t"
            "s_waitcnt vmcnt(15) lgkmcnt(0)\n\t"
            "s_barrier\n\t"
            "v_mfma_f32_16x16x32_bf16 %[c1023], a[152:155], a[240:243], %[c1023]\n\t"
            "v_mfma_f32_16x16x32_bf16 %[c1023], a[156:159], a[244:247], %[c1023]\n\t"
            "v_mfma_f32_16x16x32_bf16 %[c1033], a[152:155], a[248:251], %[c1033]\n\t"
            "v_mfma_f32_16x16x32_bf16 %[c1033], a[156:159], a[252:255], %[c1033]\n\t"
            : [c1003] "+v"(acc[1][0][0][3]), [c1013] "+v"(acc[1][0][1][3]), [c1023] "+v"(acc[1][0][2][3]), [c1033] "+v"(acc[1][0][3][3])
            :
            : "memory", "scc");
        SCHED; SW(9); ST(34); SCHED;
        asm volatile(
            "s_waitcnt vmcnt(0)\n\t"
            "v_cmp_eq_u64 vcc, %[fpat2], %[pf]\n\t"
            "s_nop 0\n\t"
            "s_or_b64 %[rdy], %[rdy], vcc\n\t"
            : [pf] "+v"(pf), [rdy] "+s"(rdy)
            : [fpat2] "s"(fpat2)
            : "memory", "scc", "vcc");
        SCHED; badm = (rdy & 4ull) ? 0x7fc07fc0u : 0u; SCHED;
        SCHED; SW(10); SCHED;
        SCHED; ADDP(0); SCHED;
        SCHED; SW(11); SCHED;
        SCHED; E1(0, 0); SCHED;
        asm volatile(
            "v_mfma_f32_16x16x32_bf16 %[c1100], a[160:163], a[192:195], %[c1100]\n\t"
            "ds_read_b64_tr_b16 a[128:129], %[vb0] offset:16384\n\t"
            : [c1100] "+v"(acc[1][1][0][0])
            : [vb0] "v"(vb[0])
            : "memory", "scc");
        SCHED; E2(0, 0); SCHED;
        asm volatile(
            "v_mfma_f32_16x16x32_bf16 %[c1100], a[164:167], a[196:199], %[c1100]\n\t"
            "ds_read_b64_tr_b16 a[130:131], %[vb0] offset:17408\n\t"
            : [c1100] "+v"(acc[1][1][0][0])
            : [vb0] "v"(vb[0])
            : "memory", "scc");
        SCHED; E2(0, 1); SCHED;
        asm volatile(
            "v_mfma_f32_16x16x32_bf16 %[c1110], a[160:163], a[200:203], %[c1110]\n\t"
            "ds_read_b64_tr_b16 a[132:133], %[vb0] offset:24576\n\t"
            : [c1110] "+v"(acc[1][1][1][0])
            : [vb0] "v"(vb[0])
            : "memory", "scc");
        SCHED; E2(0, 2); SCHED;
        asm volatile(
            "v_mfma_f32_16x16x32_bf16 %[c1110], a[164:167], a[204:207], %[c1110]\n\t"
            "ds_read_b64_tr_b16 a[134:135], %[vb0] offset:25600\n\t"
            : [c1110] "+v"(acc[1][1][1][0])
            : [vb0] "v"(vb[0])
            : "memory", "scc");
        SCHED; E2(0, 3); SCHED;
        asm volatile(
            "v_mfma_f32_16x16x32_bf16 %[c1120], a[160:163], a[208:211], %[c1120]\n\t"
            "ds_read_b64_tr_b16 a[136:137], %[vb1] offset:16384\n\t"
            : [c1120] "+v"(acc[1][1][2][0])
            : [vb1] "v"(vb[1])
            : "memory", "scc");
        SCHED; E2(0, 4); SCHED;
        asm volatile(
            "v_mfma_f32_16x16x32_bf16 %[c1120], a[164:167], a[212:215], %[c1120]\n\t"
            "ds_read_b64_tr_b16 a[138:139], %[vb1] offset:17408\n\t"
            : [c1120] "+v"(acc[1][1][2][0])
            : [vb1] "v"(vb[1])
            : "memory", "scc");
        SCHED; E2(0, 5); SCHED;
        asm volatile(
            "v_mfma_f32_16x16x32_bf16 %[c1130], a[160:163], a[216:219], %[c1130]\n\t"
            "ds_read_b64_tr_b16 a[140:141], %[vb1] offset:24576\n\t"
            : [c1130] "+v"(acc[1][1][3][0])
            : [vb1] "v"(vb[1])
            : "memory", "scc");
        SCHED; E2(0, 6); SCHED;
        asm volatile(
            "v_mfma_f32_16x16x32_bf16 %[c1130], a[164:167], a[220:223], %[c1130]\n\t"
            "ds_read_b64_tr_b16 a[142:143], %[vb1] offset:25600\n\t"
            : [c1130] "+v"(acc[1][1][3][0])
            : [vb1] "v"(vb[1])
            : "memory", "scc");
        SCHED; E2(0, 7); SCHED;
        SCHED; E3(0); E1(0, 1); SCHED;
        asm volatile(
            "v_mfma_f32_16x16x32_bf16 %[c1101], a[168:171], a[192:195], %[c1101]\n\t"
            "ds_read_b64_tr_b16 a[144:145], %[vb2] offset:16384\n\t"
            : [c1101] "+v"(acc[1][1][0][1])
            : [vb2] "v"(vb[2])
            : "memory", "scc");
        SCHED; E4(0, 0); SCHED;
        asm volatile(
            "v_mfma_f32_16x16x32_bf16 %[c1101], a[172:175], a[196:199], %[c1101]\n\t"
            "ds_read_b64_tr_b16 a[146:147], %[vb2] offset:17408\n\t"
            : [c1101] "+v"(acc[1][1][0][1])
            : [vb2] "v"(vb[2])
            : "memory", "scc");
        SCHED; E2(1, 0); SCHED;
        asm volatile(
            "v_mfma_f32_16x16x32_bf16 %[c1111], a[168:171], a[200:203], %[c1111]\n\t"
            "ds_read_b64_tr_b16 a[148:149], %[vb2] offset:24576\n\t"
            : [c1111] "+v"(acc[1][1][1][1])
            : [vb2] "v"(vb[2])
            : "memory", "scc");
        SCHED; E4(0, 1); SCHED;
        asm volatile(
            "v_mfma_f32_16x16x32_bf16 %[c1111], a[172:175], a[204:207], %[c1111]\n\t"
            "ds_read_b64_tr_b16 a[150:151], %[vb2] offset:25600\n\t"
            : [c1111] "+v"(acc[1][1][1][1])
            : [vb2] "v"(vb[2])
            : "memory", "scc");
        SCHED; E2(1, 1); SCHED;
        asm volatile(
            "v_mfma_f32_16x16x32_bf16 %[c1121], a[168:171], a[208:211], %[c1121]\n\t"
            "ds_read_b64_tr_b16 a[152:153], %[vb3] offset:16384\n\t"
            : [c1121] "+v"(acc[1][1][2][1])
            : [vb3] "v"(vb[3])
            : "memory", "scc");
        SCHED; E4(0, 2); SCHED;
        asm volatile(
            "v_mfma_f32_16x16x32_bf16 %[c1121], a[172:175], a[212:215], %[c1121]\n\t"
            "ds_read_b64_tr_b16 a[154:155], %[vb3] offset:17408\n\t"
            : [c1121] "+v"(acc[1][1][2][1])
            : [vb3] "v"(vb[3])
            : "memory", "scc");
        SCHED; E2(1, 2); SCHED;
        asm volatile(
            "v_mfma_f32_16x16x32_bf16 %[c1131], a[168:171], a[216:219], %[c1131]\n\t"
            "ds_read_b64_tr_b16 a[156:157], %[vb3] offset:24576\n\t"
            : [c1131] "+v"(acc[1][1][3][1])
            : [vb3] "v"(vb[3])
            : "memory", "scc");
        SCHED; E4(0, 3); SCHED;
        asm volatile(
            "v_mfma_f32_16x16x32_bf16 %[c1131], a[172:175], a[220:223], %[c1131]\n\t"
            "ds_read_b64_tr_b16 a[158:159], %[vb3] offset:25600\n\t"
            : [c1131] "+v"(acc[1][1][3][1])
            : [vb3] "v"(vb[3])
            : "memory", "scc");
        SCHED; E2(1, 3); SCHED;
        asm volatile(
            "v_mfma_f32_16x16x32_bf16 %[c1102], a[176:179], a[192:195], %[c1102]\n\t"
            : [c1102] "+v"(acc[1][1][0][2])
            :
            : "memory", "scc");
        SCHED; E4(0, 4); SCHED;
        asm volatile(
            "v_mfma_f32_16x16x32_bf16 %[c1102], a[180:183], a[196:199], %[c1102]\n\t"
            : [c1102] "+v"(acc[1][1][0][2])
            :
            : "memory", "scc");
        SCHED; E2(1, 4); SCHED;
        asm volatile(
            "v_mfma_f32_16x16x32_bf16 %[c1112], a[176:179], a[200:203], %[c1112]\n\t"
            : [c1112] "+v"(acc[1][1][1][2])
            :
            : "memory", "scc");
        SCHED; E4(0, 5); SCHED;
        asm volatile(
            "v_mfma_f32_16x16x32_bf16 %[c1112], a[180:183], a[204:207], %[c1112]\n\t"
            : [c1112] "+v"(acc[1][1][1][2])
            :
            : "memory", "scc");
        SCHED; E2(1, 5); SCHED;
        asm volatile(
            "v_mfma_f32_16x16x32_bf16 %[c1122], a[176:179], a[208:211], %[c1122]\n\t"
            : [c1122] "+v"(acc[1][1][2][2])
            :
            : "memory", "scc");
        SCHED; E4(0, 6); SCHED;
        asm volatile(
            "v_mfma_f32_16x16x32_bf16 %[c1122], a[180:183], a[212:215], %[c1122]\n\t"
            : [c1122] "+v"(acc[1][1][2][2])
            :
            : "memory", "scc");
        SCHED; E2(1, 6); SCHED;
        asm volatile(
            "v_mfma_f32_16x16x32_bf16 %[c1132], a[176:179], a[216:219], %[c1132]\n\t"
            : [c1132] "+v"(acc[1][1][3][2])
            :
            : "memory", "scc");
        SCHED; E4(0, 7); SCHED;
        asm volatile(
            "v_mfma_f32_16x16x32_bf16 %[c1132], a[180:183], a[220:223], %[c1132]\n\t"
            : [c1132] "+v"(acc[1][1][3][2])
            :
            : "memory", "scc");
        SCHED; E2(1, 7); SCHED;
        SCHED; E5(0, 0); E3(1); E1(0, 2); SCHED;
        asm volatile(
            "v_mfma_f32_16x16x32_bf16 %[c1103], a[184:187], a[192:195], %[c1103]\n\t"
            : [c1103] "+v"(acc[1][1][0][3])
            :
            : "memory", "scc");
        SCHED; E4(1, 0); SCHED;
        asm volatile(
            "v_mfma_f32_16x16x32_bf16 %[c1103], a[188:191], a[196:199], %[c1103]\n\t"
            : [c1103] "+v"(acc[1][1][0][3])
            :
            : "memory", "scc");
        SCHED; E2(2, 0); SCHED;
        asm volatile(
            "v_mfma_f32_16x16x32_bf16 %[c1113], a[184:187], a[200:203], %[c1113]\n\t"
            : [c1113] "+v"(acc[1][1][1][3])
            :
            : "memory", "scc");
        SCHED; E4(1, 1); SCHED;
        asm volatile(
            "v_mfma_f32_16x16x32_bf16 %[c1113], a[188:191], a[204:207], %[c1113]\n\t"
            : [c1113] "+v"(acc[1][1][1][3])
            :
            : "memory", "scc");
        SCHED; E2(2, 1); SCHED;
        asm volatile(
            "v_mfma_f32_16x16x32_bf16 %[c1123], a[184:187], a[208:211], %[c1123]\n\t"
            : [c1123] "+v"(acc[1][1][2][3])
            :
            : "memory", "scc");
        SCHED; E4(1, 2); SCHED;
        asm volatile(
            "v_mfma_f32_16x16x32_bf16 %[c1123], a[188:191], a[212:215], %[c1123]\n\t"
            : [c1123] "+v"(acc[1][1][2][3])
            :
            : "memory", "scc");
        SCHED; E2(2, 2); SCHED;
        asm volatile(
            "v_mfma_f32_16x16x32_bf16 %[c1133], a[184:187], a[216:219], %[c1133]\n\t"
            "s_waitcnt lgkmcnt(0)\n\t"
            : [c1133] "+v"(acc[1][1][3][3])
            :
            : "memory", "scc");
        SCHED; E4(1, 3); SCHED;
        asm volatile(
            "v_mfma_f32_16x16x32_bf16 %[c1133], a[188:191], a[220:223], %[c1133]\n\t"
            : [c1133] "+v"(acc[1][1][3][3])
            :
            : "memory", "scc");
        SCHED; E2(2, 3); SCHED;
        asm volatile(
            "v_mfma_f32_16x16x32_bf16 %[c1100], a[128:131], a[64:67], %[c1100]\n\t"
            "ds_read_b64_tr_b16 a[160:161], %[vbx0] offset:0\n\t"
            : [c1100] "+v"(acc[1][1][0][0])
            : [vbx0] "v"(vbx[0])
            : "memory", "scc");
        SCHED; E4(1, 4); SCHED;
        asm volatile(
            "v_mfma_f32_16x16x32_bf16 %[c1100], a[132:135], a[68:71], %[c1100]\n\t"
            "ds_read_b64_tr_b16 a[162:163], %[vbx0] offset:1024\n\t"
            : [c1100] "+v"(acc[1][1][0][0])
            : [vbx0] "v"(vbx[0])
            : "memory", "scc");
        SCHED; E2(2, 4); SCHED;
        asm volatile(
            "v_mfma_f32_16x16x32_bf16 %[c1110], a[128:131], a[72:75], %[c1110]\n\t"
            "ds_read_b64_tr_b16 a[164:165], %[vbx0] offset:8192\n\t"
            : [c1110] "+v"(acc[1][1][1][0])
            : [vbx0] "v"(vbx[0])
            : "memory", "scc");
        SCHED; E4(1, 5); SCHED;
        asm volatile(
            "v_mfma_f32_16x16x32_bf16 %[c1110], a[132:135], a[76:79], %[c1110]\n\t"
            "ds_read_b64_tr_b16 a[166:167], %[vbx0] offset:9216\n\t"
            : [c1110] "+v"(acc[1][1][1][0])
            : [vbx0] "v"(vbx[0])
            : "memory", "scc");
        SCHED; E2(2, 5); SCHED;
        asm volatile(
            "v_mfma_f32_16x16x32_bf16 %[c1120], a[128:131], a[80:83], %[c1120]\n\t"
            "ds_read_b64_tr_b16 a[168:169], %[vbx1] offset:0\n\t"
            : [c1120] "+v"(acc[1][1][2][0])
            : [vbx1] "v"(vbx[1])
            : "memory", "scc");
        SCHED; E4(1, 6); SCHED;
        asm volatile(
            "v_mfma_f32_16x16x32_bf16 %[c1120], a[132:135], a[84:87], %[c1120]\n\t"
            "ds_read_b64_tr_b16 a[170:171], %[vbx1] offset:1024\n\t"
            : [c1120] "+v"(acc[1][1][2][0])
            : [vbx1] "v"(vbx[1])
            : "memory", "scc");
        SCHED; E2(2, 6); SCHED;
        asm volatile(
            "v_mfma_f32_16x16x32_bf16 %[c1130], a[128:131], a[88:91], %[c1130]\n\t"
            "ds_read_b64_tr_b16 a[172:173], %[vbx1] offset:8192\n\t"
            : [c1130] "+v"(acc[1][1][3][0])
            : [vbx1] "v"(vbx[1])
            : "memory", "scc");
        SCHED; E4(1, 7); SCHED;
        asm volatile(
            "v_mfma_f32_16x16x32_bf16 %[c1130], a[132:135], a[92:95], %[c1130]\n\t"
            "ds_read_b64_tr_b16 a[174:175], %[vbx1] offset:9216\n\t"
            : [c1130] "+v"(acc[1][1][3][0])
            : [vbx1] "v"(vbx[1])
            : "memory", "scc");
        SCHED; E2(2, 7); SCHED;
        SCHED; E5(0, 1); E3(2); E1(0, 3); SCHED;
        asm volatile(
            "v_mfma_f32_16x16x32_bf16 %[c1101], a[136:139], a[64:67], %[c1101]\n\t"
            "ds_read_b64_tr_b16 a[176:177], %[vbx2] offset:0\n\t"
            : [c1101] "+v"(acc[1][1][0][1])
            : [vbx2] "v"(vbx[2])
            : "memory", "scc");
        SCHED; E4(2, 0); SCHED;
        asm volatile(
            "v_mfma_f32_16x16x32_bf16 %[c1101], a[140:143], a[68:71], %[c1101]\n\t"
            "ds_read_b64_tr_b16 a[178:179], %[vbx2] offset:1024\n\t"
            : [c1101] "+v"(acc[1][1][0][1])
            : [vbx2] "v"(vbx[2])
            : "memory", "scc");
        SCHED; E2(3, 0); SCHED;
        asm volatile(
            "v_mfma_f32_16x16x32_bf16 %[c1111], a[136:139], a[72:75], %[c1111]\n\t"
            "ds_read_b64_tr_b16 a[180:181], %[vbx2] offset:8192\n\t"
            : [c1111] "+v"(acc[1][1][1][1])
            : [vbx2] "v"(vbx[2])
            : "memory", "scc");
        SCHED; E4(2, 1); SCHED;
        asm volatile(
            "v_mfma_f32_16x16x32_bf16 %[c1111], a[140:143], a[76:79], %[c1111]\n\t"
            "ds_read_b64_tr_b16 a[182:183], %[vbx2] offset:9216\n\t"
            : [c1111] "+v"(acc[1][1][1][1])
            : [vbx2] "v"(vbx[2])
            : "memory", "scc");
        SCHED; E2(3, 1); SCHED;
        asm volatile(
            "v_mfma_f32_16x16x32_bf16 %[c1121], a[136:139], a[80:83], %[c1121]\n\t"
            "ds_read_b64_tr_b16 a[184:185], %[vbx3] offset:0\n\t"
            : [c1121] "+v"(acc[1][1][2][1])
            : [vbx3] "v"(vbx[3])
            : "memory", "scc");
        SCHED; E4(2, 2); SCHED;
        asm volatile(
            "v_mfma_f32_16x16x32_bf16 %[c1121], a[140:143], a[84:87], %[c1121]\n\t"
            "ds_read_b64_tr_b16 a[186:187], %[vbx3] offset:1024\n\t"
            : [c1121] "+v"(acc[1][1][2][1])
            : [vbx3] "v"(vbx[3])
            : "memory", "scc");
        SCHED; E2(3, 2); SCHED;
        asm volatile(
            "v_mfma_f32_16x16x32_bf16 %[c1131], a[136:139], a[88:91], %[c1131]\n\t"
            "ds_read_b64_tr_b16 a[188:189], %[vbx3] offset:8192\n\t"
            : [c1131] "+v"(acc[1][1][3][1])
            : [vbx3] "v"(vbx[3])
            : "memory", "scc");
        SCHED; E4(2, 3); SCHED;
        asm volatile(
            "v_mfma_f32_16x16x32_bf16 %[c1131], a[140:143], a[92:95], %[c1131]\n\t"
            "ds_read_b64_tr_b16 a[190:191], %[vbx3] offset:9216\n\t"
            : [c1131] "+v"(acc[1][1][3][1])
            : [vbx3] "v"(vbx[3])
            : "memory", "scc");
        SCHED; E2(3, 3); SCHED;
        asm volatile(
            "v_mfma_f32_16x16x32_bf16 %[c1102], a[144:147], a[64:67], %[c1102]\n\t"
            : [c1102] "+v"(acc[1][1][0][2])
            :
            : "memory", "scc");
        SCHED; E4(2, 4); SCHED;
        asm volatile(
            "v_mfma_f32_16x16x32_bf16 %[c1102], a[148:151], a[68:71], %[c1102]\n\t"
            : [c1102] "+v"(acc[1][1][0][2])
            :
            : "memory", "scc");
        SCHED; E2(3, 4); SCHED;
        asm volatile(
            "v_mfma_f32_16x16x32_bf16 %[c1112], a[144:147], a[72:75], %[c1112]\n\t"
            : [c1112] "+v"(acc[1][1][1][2])
            :
            : "memory", "scc");
        SCHED; E4(2, 5); SCHED;
        asm volatile(
            "v_mfma_f32_16x16x32_bf16 %[c1112], a[148:151], a[76:79], %[c1112]\n\t"
            : [c1112] "+v"(acc[1][1][1][2])
            :
            : "memory", "scc");
        SCHED; E2(3, 5); SCHED;
        asm volatile(
            "v_mfma_f32_16x16x32_bf16 %[c1122], a[144:147], a[80:83], %[c1122]\n\t"
            : [c1122] "+v"(acc[1][1][2][2])
            :
            : "memory", "scc");
        SCHED; E4(2, 6); SCHED;
        asm volatile(
            "v_mfma_f32_16x16x32_bf16 %[c1122], a[148:151], a[84:87], %[c1122]\n\t"
            : [c1122] "+v"(acc[1][1][2][2])
            :
            : "memory", "scc");
        SCHED; E2(3, 6); SCHED;
        asm volatile(
            "v_mfma_f32_16x16x32_bf16 %[c1132], a[144:147], a[88:91], %[c1132]\n\t"
            : [c1132] "+v"(acc[1][1][3][2])
            :
            : "memory", "scc");
        SCHED; E4(2, 7); SCHED;
        asm volatile(
            "v_mfma_f32_16x16x32_bf16 %[c1132], a[148:151], a[92:95], %[c1132]\n\t"
            : [c1132] "+v"(acc[1][1][3][2])
            :
            : "memory", "scc");
        SCHED; E2(3, 7); SCHED;
        SCHED; E5(0, 2); E3(3); E1(0, 4); SCHED;
        asm volatile(
            "v_mfma_f32_16x16x32_bf16 %[c1103], a[152:155], a[64:67], %[c1103]\n\t"
            : [c1103] "+v"(acc[1][1][0][3])
            :
            : "memory", "scc");
        SCHED; E4(3, 0); SCHED;
        asm volatile(
            "v_mfma_f32_16x16x32_bf16 %[c1103], a[156:159], a[68:71], %[c1103]\n\t"
            : [c1103] "+v"(acc[1][1][0][3])
            :
            : "memory", "scc");
        SCHED; E2(4, 0); SCHED;
        asm volatile(
            "v_mfma_f32_16x16x32_bf16 %[c1113], a[152:155], a[72:75], %[c1113]\n\t"
            : [c1113] "+v"(acc[1][1][1][3])
            :
            : "memory", "scc");
        SCHED; E4(3, 1); SCHED;
        asm volatile(
            "v_mfma_f32_16x16x32_bf16 %[c1113], a[156:159], a[76:79], %[c1113]\n\t"
            : [c1113] "+v"(acc[1][1][1][3])
            :
            : "memory", "scc");
        SCHED; E2(4, 1); SCHED;
        asm volatile(
            "v_mfma_f32_16x16x32_bf16 %[c1123], a[152:155], a[80:83], %[c1123]\n\t"
            : [c1123] "+v"(acc[1][1][2][3])
            :
            : "memory", "scc");
        SCHED; E4(3, 2); SCHED;
        asm volatile(
            "v_mfma_f32_16x16x32_bf16 %[c1123], a[156:159], a[84:87], %[c1123]\n\t"
            : [c1123] "+v"(acc[1][1][2][3])
            :
            : "memory", "scc");
        SCHED; E2(4, 2); SCHED;
        asm volatile(
            "v_mfma_f32_16x16x32_bf16 %[c1133], a[152:155], a[88:91], %[c1133]\n\t"
            "s_waitcnt lgkmcnt(0)\n\t"
            : [c1133] "+v"(acc[1][1][3][3])
            :
            : "memory", "scc");
        SCHED; E4(3, 3); SCHED;
        asm volatile(
            "v_mfma_f32_16x16x32_bf16 %[c1133], a[156:159], a[92:95], %[c1133]\n\t"
            : [c1133] "+v"(acc[1][1][3][3])
            :
            : "memory", "scc");
        SCHED; E2(4, 3); SCHED;
        asm volatile(
            "v_mfma_f32_16x16x32_bf16 %[c1100], a[160:163], a[96:99], %[c1100]\n\t"
            "ds_read_b64_tr_b16 a[128:129], %[vbl0] offset:32768\n\t"
            : [c1100] "+v"(acc[1][1][0][0])
            : [vbl0] "v"(vbl[0])
            : "memory", "scc");
        SCHED; E4(3, 4); SCHED;
        SCHED; PVLD1(1, 0); SCHED;
        asm volatile(
            "v_mfma_f32_16x16x32_bf16 %[c1100], a[164:167], a[100:103], %[c1100]\n\t"
            "ds_read_b64_tr_b16 a[130:131], %[vbl0] offset:33792\n\t"
            : [c1100] "+v"(acc[1][1][0][0])
            : [vbl0] "v"(vbl[0])
            : "memory", "scc");
        SCHED; E2(4, 4); SCHED;
        SCHED; PVLD1(1, 1); SCHED;
        asm volatile(
            "v_mfma_f32_16x16x32_bf16 %[c1110], a[160:163], a[104:107], %[c1110]\n\t"
            "ds_read_b64_tr_b16 a[132:133], %[vbl0] offset:40960\n\t"
            : [c1110] "+v"(acc[1][1][1][0])
            : [vbl0] "v"(vbl[0])
            : "memory", "scc");
        SCHED; E4(3, 5); SCHED;
        SCHED; PVLD1(1, 2); SCHED;
        asm volatile(
            "v_mfma_f32_16x16x32_bf16 %[c1110], a[164:167], a[108:111], %[c1110]\n\t"
            "ds_read_b64_tr_b16 a[134:135], %[vbl0] offset:41984\n\t"
            : [c1110] "+v"(acc[1][1][1][0])
            : [vbl0] "v"(vbl[0])
            : "memory", "scc");
        SCHED; E2(4, 5); SCHED;
        SCHED; PVLD1(1, 3); SCHED;
        asm volatile(
            "v_mfma_f32_16x16x32_bf16 %[c1120], a[160:163], a[112:115], %[c1120]\n\t"
            "ds_read_b64_tr_b16 a[136:137], %[vbl1] offset:32768\n\t"
            : [c1120] "+v"(acc[1][1][2][0])
            : [vbl1] "v"(vbl[1])
            : "memory", "scc");
        SCHED; E4(3, 6); SCHED;
        SCHED; PVLD1(1, 4); SCHED;
        asm volatile(
            "v_mfma_f32_16x16x32_bf16 %[c1120], a[164:167], a[116:119], %[c1120]\n\t"
            "ds_read_b64_tr_b16 a[138:139], %[vbl1] offset:33792\n\t"
            : [c1120] "+v"(acc[1][1][2][0])
            : [vbl1] "v"(vbl[1])
            : "memory", "scc");
        SCHED; E2(4, 6); SCHED;
        SCHED; PVLD1(1, 5); SCHED;
        asm volatile(
            "v_mfma_f32_16x16x32_bf16 %[c1130], a[160:163], a[120:123], %[c1130]\n\t"
            "ds_read_b64_tr_b16 a[140:141], %[vbl1] offset:40960\n\t"
            : [c1130] "+v"(acc[1][1][3][0])
            : [vbl1] "v"(vbl[1])
            : "memory", "scc");
        SCHED; E4(3, 7); SCHED;
        SCHED; PVLD1(1, 6); SCHED;
        asm volatile(
            "v_mfma_f32_16x16x32_bf16 %[c1130], a[164:167], a[124:127], %[c1130]\n\t"
            "ds_read_b64_tr_b16 a[142:143], %[vbl1] offset:41984\n\t"
            : [c1130] "+v"(acc[1][1][3][0])
            : [vbl1] "v"(vbl[1])
            : "memory", "scc");
        SCHED; E2(4, 7); SCHED;
        SCHED; PVLD1(1, 7); SCHED;
        SCHED; SW(12); SCHED;
        SCHED; E5(0, 3); E3(4); E1(0, 5); SCHED;
        asm volatile(
            "v_mfma_f32_16x16x32_bf16 %[c1101], a[168:171], a[96:99], %[c1101]\n\t"
            "ds_read_b64_tr_b16 a[144:145], %[vbl2] offset:32768\n\t"
            : [c1101] "+v"(acc[1][1][0][1])
            : [vbl2] "v"(vbl[2])
            : "memory", "scc");
        SCHED; E4(4, 0); SCHED;
        asm volatile(
            "v_mfma_f32_16x16x32_bf16 %[c1101], a[172:175], a[100:103], %[c1101]\n\t"
            "ds_read_b64_tr_b16 a[146:147], %[vbl2] offset:33792\n\t"
            : [c1101] "+v"(acc[1][1][0][1])
            : [vbl2] "v"(vbl[2])
            : "memory", "scc");
        SCHED; E2(5, 0); SCHED;
        asm volatile(
            "v_mfma_f32_16x16x32_bf16 %[c1111], a[168:171], a[104:107], %[c1111]\n\t"
            "ds_read_b64_tr_b16 a[148:149], %[vbl2] offset:40960\n\t"
            : [c1111] "+v"(acc[1][1][1][1])
            : [vbl2] "v"(vbl[2])
            : "memory", "scc");
        SCHED; E4(4, 1); SCHED;
        asm volatile(
            "v_mfma_f32_16x16x32_bf16 %[c1111], a[172:175], a[108:111], %[c1111]\n\t"
            "ds_read_b64_tr_b16 a[150:151], %[vbl2] offset:41984\n\t"
            : [c1111] "+v"(acc[1][1][1][1])
            : [vbl2] "v"(vbl[2])
            : "memory", "scc");
        SCHED; E2(5, 1); SCHED;
        asm volatile(
            "v_mfma_f32_16x16x32_bf16 %[c1121], a[168:171], a[112:115], %[c1121]\n\t"
            "ds_read_b64_tr_b16 a[152:153], %[vbl3] offset:32768\n\t"
            : [c1121] "+v"(acc[1][1][2][1])
            : [vbl3] "v"(vbl[3])
            : "memory", "scc");
        SCHED; E4(4, 2); SCHED;
        asm volatile(
            "v_mfma_f32_16x16x32_bf16 %[c1121], a[172:175], a[116:119], %[c1121]\n\t"
            "ds_read_b64_tr_b16 a[154:155], %[vbl3] offset:33792\n\t"
            : [c1121] "+v"(acc[1][1][2][1])
            : [vbl3] "v"(vbl[3])
            : "memory", "scc");
        SCHED; E2(5, 2); SCHED;
        asm volatile(
            "v_mfma_f32_16x16x32_bf16 %[c1131], a[168:171], a[120:123], %[c1131]\n\t"
            "ds_read_b64_tr_b16 a[156:157], %[vbl3] offset:40960\n\t"
            : [c1131] "+v"(acc[1][1][3][1])
            : [vbl3] "v"(vbl[3])
            : "memory", "scc");
        SCHED; E4(4, 3); SCHED;
        asm volatile(
            "v_mfma_f32_16x16x32_bf16 %[c1131], a[172:175], a[124:127], %[c1131]\n\t"
            "ds_read_b64_tr_b16 a[158:159], %[vbl3] offset:41984\n\t"
            : [c1131] "+v"(acc[1][1][3][1])
            : [vbl3] "v"(vbl[3])
            : "memory", "scc");
        SCHED; E2(5, 3); SCHED;
        asm volatile(
            "v_mfma_f32_16x16x32_bf16 %[c1102], a[176:179], a[96:99], %[c1102]\n\t"
            : [c1102] "+v"(acc[1][1][0][2])
            :
            : "memory", "scc");
        SCHED; E4(4, 4); SCHED;
        asm volatile(
            "v_mfma_f32_16x16x32_bf16 %[c1102], a[180:183], a[100:103], %[c1102]\n\t"
            : [c1102] "+v"(acc[1][1][0][2])
            :
            : "memory", "scc");
        SCHED; E2(5, 4); SCHED;
        asm volatile(
            "v_mfma_f32_16x16x32_bf16 %[c1112], a[176:179], a[104:107], %[c1112]\n\t"
            : [c1112] "+v"(acc[1][1][1][2])
            :
            : "memory", "scc");
        SCHED; E4(4, 5); SCHED;
        asm volatile(
            "v_mfma_f32_16x16x32_bf16 %[c1112], a[180:183], a[108:111], %[c1112]\n\t"
            : [c1112] "+v"(acc[1][1][1][2])
            :
            : "memory", "scc");
        SCHED; E2(5, 5); SCHED;
        asm volatile(
            "v_mfma_f32_16x16x32_bf16 %[c1122], a[176:179], a[112:115], %[c1122]\n\t"
            : [c1122] "+v"(acc[1][1][2][2])
            :
            : "memory", "scc");
        SCHED; E4(4, 6); SCHED;
        asm volatile(
            "v_mfma_f32_16x16x32_bf16 %[c1122], a[180:183], a[116:119], %[c1122]\n\t"
            : [c1122] "+v"(acc[1][1][2][2])
            :
            : "memory", "scc");
        SCHED; E2(5, 6); SCHED;
        asm volatile(
            "v_mfma_f32_16x16x32_bf16 %[c1132], a[176:179], a[120:123], %[c1132]\n\t"
            : [c1132] "+v"(acc[1][1][3][2])
            :
            : "memory", "scc");
        SCHED; E4(4, 7); SCHED;
        asm volatile(
            "v_mfma_f32_16x16x32_bf16 %[c1132], a[180:183], a[124:127], %[c1132]\n\t"
            : [c1132] "+v"(acc[1][1][3][2])
            :
            : "memory", "scc");
        SCHED; E2(5, 7); SCHED;
        SCHED; E5(0, 4); E3(5); E1(0, 6); SCHED;
        asm volatile(
            "v_mfma_f32_16x16x32_bf16 %[c1103], a[184:187], a[96:99], %[c1103]\n\t"
            : [c1103] "+v"(acc[1][1][0][3])
            :
            : "memory", "scc");
        SCHED; E4(5, 0); SCHED;
        asm volatile(
            "v_mfma_f32_16x16x32_bf16 %[c1103], a[188:191], a[100:103], %[c1103]\n\t"
            : [c1103] "+v"(acc[1][1][0][3])
            :
            : "memory", "scc");
        SCHED; E2(6, 0); SCHED;
        asm volatile(
            "v_mfma_f32_16x16x32_bf16 %[c1113], a[184:187], a[104:107], %[c1113]\n\t"
            : [c1113] "+v"(acc[1][1][1][3])
            :
            : "memory", "scc");
        SCHED; E4(5, 1); SCHED;
        asm volatile(
            "v_mfma_f32_16x16x32_bf16 %[c1113], a[188:191], a[108:111], %[c1113]\n\t"
            : [c1113] "+v"(acc[1][1][1][3])
            :
            : "memory", "scc");
        SCHED; E2(6, 1); SCHED;
        asm volatile(
            "v_mfma_f32_16x16x32_bf16 %[c1123], a[184:187], a[112:115], %[c1123]\n\t"
            : [c1123] "+v"(acc[1][1][2][3])
            :
            : "memory", "scc");
        SCHED; E4(5, 2); SCHED;
        asm volatile(
            "v_mfma_f32_16x16x32_bf16 %[c1123], a[188:191], a[116:119], %[c1123]\n\t"
            : [c1123] "+v"(acc[1][1][2][3])
            :
            : "memory", "scc");
        SCHED; E2(6, 2); SCHED;
        asm volatile(
            "v_mfma_f32_16x16x32_bf16 %[c1133], a[184:187], a[120:123], %[c1133]\n\t"
            "s_waitcnt lgkmcnt(0)\n\t"
            : [c1133] "+v"(acc[1][1][3][3])
            :
            : "memory", "scc");
        SCHED; E4(5, 3); SCHED;
        asm volatile(
            "v_mfma_f32_16x16x32_bf16 %[c1133], a[188:191], a[124:127], %[c1133]\n\t"
            : [c1133] "+v"(acc[1][1][3][3])
            :
            : "memory", "scc");
        SCHED; E2(6, 3); SCHED;
        asm volatile(
            "v_mfma_f32_16x16x32_bf16 %[c1100], a[128:131], a[224:227], %[c1100]\n\t"
            : [c1100] "+v"(acc[1][1][0][0])
            :
            : "memory", "scc");
        SCHED; E4(5, 4); SCHED;
        asm volatile(
            "v_mfma_f32_16x16x32_bf16 %[c1100], a[132:135], a[228:231], %[c1100]\n\t"
            : [c1100] "+v"(acc[1][1][0][0])
            :
            : "memory", "scc");
        SCHED; E2(6, 4); SCHED;
        asm volatile(
            "v_mfma_f32_16x16x32_bf16 %[c1110], a[128:131], a[232:235], %[c1110]\n\t"
            : [c1110] "+v"(acc[1][1][1][0])
            :
            : "memory", "scc");
        SCHED; E4(5, 5); SCHED;
        asm volatile(
            "v_mfma_f32_16x16x32_bf16 %[c1110], a[132:135], a[236:239], %[c1110]\n\t"
            : [c1110] "+v"(acc[1][1][1][0])
            :
            : "memory", "scc");
        SCHED; E2(6, 5); SCHED;
        asm volatile(
            "v_mfma_f32_16x16x32_bf16 %[c1120], a[128:131], a[240:243], %[c1120]\n\t"
            : [c1120] "+v"(acc[1][1][2][0])
            :
            : "memory", "scc");
        SCHED; E4(5, 6); SCHED;
        asm volatile(
            "v_mfma_f32_16x16x32_bf16 %[c1120], a[132:135], a[244:247], %[c1120]\n\t"
            : [c1120] "+v"(acc[1][1][2][0])
            :
            : "memory", "scc");
        SCHED; E2(6, 6); SCHED;
        asm volatile(
            "v_mfma_f32_16x16x32_bf16 %[c1130], a[128:131], a[248:251], %[c1130]\n\t"
            : [c1130] "+v"(acc[1][1][3][0])
            :
            : "memory", "scc");
        SCHED; E4(5, 7); SCHED;
        asm volatile(
            "v_mfma_f32_16x16x32_bf16 %[c1130], a[132:135], a[252:255], %[c1130]\n\t"
            : [c1130] "+v"(acc[1][1][3][0])
            :
            : "memory", "scc");
        SCHED; E2(6, 7); SCHED;
        SCHED; E5(0, 5); E3(6); E1(0, 7); SCHED;
        asm volatile(
            "v_mfma_f32_16x16x32_bf16 %[c1101], a[136:139], a[224:227], %[c1101]\n\t"
            : [c1101] "+v"(acc[1][1][0][1])
            :
            : "memory", "scc");
        SCHED; E4(6, 0); SCHED;
        asm volatile(
            "v_mfma_f32_16x16x32_bf16 %[c1101], a[140:143], a[228:231], %[c1101]\n\t"
            : [c1101] "+v"(acc[1][1][0][1])
            :
            : "memory", "scc");
        SCHED; E2(7, 0); SCHED;
        asm volatile(
            "v_mfma_f32_16x16x32_bf16 %[c1111], a[136:139], a[232:235], %[c1111]\n\t"
            : [c1111] "+v"(acc[1][1][1][1])
            :
            : "memory", "scc");
        SCHED; E4(6, 1); SCHED;
        asm volatile(
            "v_mfma_f32_16x16x32_bf16 %[c1111], a[140:143], a[236:239], %[c1111]\n\t"
            : [c1111] "+v"(acc[1][1][1][1])
            :
            : "memory", "scc");
        SCHED; E2(7, 1); SCHED;
        asm volatile(
            "v_mfma_f32_16x16x32_bf16 %[c1121], a[136:139], a[240:243], %[c1121]\n\t"
            : [c1121] "+v"(acc[1][1][2][1])
            :
            : "memory", "scc");
        SCHED; E4(6, 2); SCHED;
        asm volatile(
            "v_mfma_f32_16x16x32_bf16 %[c1121], a[140:143], a[244:247], %[c1121]\n\t"
            : [c1121] "+v"(acc[1][1][2][1])
            :
            : "memory", "scc");
        SCHED; E2(7, 2); SCHED;
        asm volatile(
            "v_mfma_f32_16x16x32_bf16 %[c1131], a[136:139], a[248:251], %[c1131]\n\t"
            : [c1131] "+v"(acc[1][1][3][1])
            :
            : "memory", "scc");
        SCHED; E4(6, 3); SCHED;
        asm volatile(
            "v_mfma_f32_16x16x32_bf16 %[c1131], a[140:143], a[252:255], %[c1131]\n\t"
            : [c1131] "+v"(acc[1][1][3][1])
            :
            : "memory", "scc");
        SCHED; E2(7, 3); SCHED;
        asm volatile(
            "v_mfma_f32_16x16x32_bf16 %[c1102], a[144:147], a[224:227], %[c1102]\n\t"
            : [c1102] "+v"(acc[1][1][0][2])
            :
            : "memory", "scc");
        SCHED; E4(6, 4); SCHED;
        asm volatile(
            "v_mfma_f32_16x16x32_bf16 %[c1102], a[148:151], a[228:231], %[c1102]\n\t"
            : [c1102] "+v"(acc[1][1][0][2])
            :
            : "memory", "scc");
        SCHED; E2(7, 4); SCHED;
        asm volatile(
            "v_mfma_f32_16x16x32_bf16 %[c1112], a[144:147], a[232:235], %[c1112]\n\t"
            : [c1112] "+v"(acc[1][1][1][2])
            :
            : "memory", "scc");
        SCHED; E4(6, 5); SCHED;
        asm volatile(
            "v_mfma_f32_16x16x32_bf16 %[c1112], a[148:151], a[236:239], %[c1112]\n\t"
            : [c1112] "+v"(acc[1][1][1][2])
            :
            : "memory", "scc");
        SCHED; E2(7, 5); SCHED;
        asm volatile(
            "v_mfma_f32_16x16x32_bf16 %[c1122], a[144:147], a[240:243], %[c1122]\n\t"
            : [c1122] "+v"(acc[1][1][2][2])
            :
            : "memory", "scc");
        SCHED; E4(6, 6); SCHED;
        asm volatile(
            "v_mfma_f32_16x16x32_bf16 %[c1122], a[148:151], a[244:247], %[c1122]\n\t"
            : [c1122] "+v"(acc[1][1][2][2])
            :
            : "memory", "scc");
        SCHED; E2(7, 6); SCHED;
        asm volatile(
            "v_mfma_f32_16x16x32_bf16 %[c1132], a[144:147], a[248:251], %[c1132]\n\t"
            : [c1132] "+v"(acc[1][1][3][2])
            :
            : "memory", "scc");
        SCHED; E4(6, 7); SCHED;
        asm volatile(
            "v_mfma_f32_16x16x32_bf16 %[c1132], a[148:151], a[252:255], %[c1132]\n\t"
            : [c1132] "+v"(acc[1][1][3][2])
            :
            : "memory", "scc");
        SCHED; E2(7, 7); SCHED;
        SCHED; E5(0, 6); E3(7); SCHED;
        asm volatile(
            "v_mfma_f32_16x16x32_bf16 %[c1103], a[152:155], a[224:227], %[c1103]\n\t"
            : [c1103] "+v"(acc[1][1][0][3])
            :
            : "memory", "scc");
        SCHED; E4(7, 0); SCHED;
        asm volatile(
            "v_mfma_f32_16x16x32_bf16 %[c1103], a[156:159], a[228:231], %[c1103]\n\t"
            : [c1103] "+v"(acc[1][1][0][3])
            :
            : "memory", "scc");
        SCHED; E4(7, 1); SCHED;
        asm volatile(
            "v_mfma_f32_16x16x32_bf16 %[c1113], a[152:155], a[232:235], %[c1113]\n\t"
            : [c1113] "+v"(acc[1][1][1][3])
            :
            : "memory", "scc");
        SCHED; E4(7, 2); SCHED;
        asm volatile(
            "v_mfma_f32_16x16x32_bf16 %[c1113], a[156:159], a[236:239], %[c1113]\n\t"
            : [c1113] "+v"(acc[1][1][1][3])
            :
            : "memory", "scc");
        SCHED; E4(7, 3); SCHED;
        asm volatile(
            "v_mfma_f32_16x16x32_bf16 %[c1123], a[152:155], a[240:243], %[c1123]\n\t"
            : [c1123] "+v"(acc[1][1][2][3])
            :
            : "memory", "scc");
        SCHED; E4(7, 4); SCHED;
        asm volatile(
            "v_mfma_f32_16x16x32_bf16 %[c1123], a[156:159], a[244:247], %[c1123]\n\t"
            : [c1123] "+v"(acc[1][1][2][3])
            :
            : "memory", "scc");
        SCHED; E4(7, 5); SCHED;
        asm volatile(
            "v_mfma_f32_16x16x32_bf16 %[c1133], a[152:155], a[248:251], %[c1133]\n\t"
            : [c1133] "+v"(acc[1][1][3][3])
            :
            : "memory", "scc");
        SCHED; E4(7, 6); SCHED;
        asm volatile(
            "v_mfma_f32_16x16x32_bf16 %[c1133], a[156:159], a[252:255], %[c1133]\n\t"
            : [c1133] "+v"(acc[1][1][3][3])
            :
            : "memory", "scc");
        SCHED; E4(7, 7); SCHED;
        SCHED; SW(13); SR(61); SCHED;
        SCHED; E5(0, 7); SCHED;
        asm volatile(
            "s_waitcnt vmcnt(5)\n\t"
            :
            :
            : "memory", "scc");
        SCHED; SW(14); SCHED;
        SCHED; ADDP(1); SCHED;
        SCHED; SW(15); SCHED;
        E1(1, 0); E2(0, 0); E2(0, 1); E2(0, 2); E2(0, 3); E2(0, 4); E2(0, 5); E2(0, 6); E2(0, 7); E3(0); E4(0, 0); E4(0, 1); E4(0, 2); E4(0, 3); E4(0, 4); E4(0, 5); E4(0, 6); E4(0, 7); E5(1, 0);
        E1(1, 1); E2(1, 0); E2(1, 1); E2(1, 2); E2(1, 3); E2(1, 4); E2(1, 5); E2(1, 6); E2(1, 7); E3(1); E4(1, 0); E4(1, 1); E4(1, 2); E4(1, 3); E4(1, 4); E4(1, 5); E4(1, 6); E4(1, 7); E5(1, 1);
        E1(1, 2); E2(2, 0); E2(2, 1); E2(2, 2); E2(2, 3); E2(2, 4); E2(2, 5); E2(2, 6); E2(2, 7); E3(2); E4(2, 0); E4(2, 1); E4(2, 2); E4(2, 3); E4(2, 4); E4(2, 5); E4(2, 6); E4(2, 7); E5(1, 2);
        E1(1, 3); E2(3, 0); E2(3, 1); E2(3, 2); E2(3, 3); E2(3, 4); E2(3, 5); E2(3, 6); E2(3, 7); E3(3); E4(3, 0); E4(3, 1); E4(3, 2); E4(3, 3); E4(3, 4); E4(3, 5); E4(3, 6); E4(3, 7); E5(1, 3);
        E1(1, 4); E2(4, 0); E2(4, 1); E2(4, 2); E2(4, 3); E2(4, 4); E2(4, 5); E2(4, 6); E2(4, 7); E3(4); E4(4, 0); E4(4, 1); E4(4, 2); E4(4, 3); E4(4, 4); E4(4, 5); E4(4, 6); E4(4, 7); E5(1, 4);
        E1(1, 5); E2(5, 0); E2(5, 1); E2(5, 2); E2(5, 3); E2(5, 4); E2(5, 5); E2(5, 6); E2(5, 7); E3(5); E4(5, 0); E4(5, 1); E4(5, 2); E4(5, 3); E4(5, 4); E4(5, 5); E4(5, 6); E4(5, 7); E5(1, 5);
        E1(1, 6); E2(6, 0); E2(6, 1); E2(6, 2); E2(6, 3); E2(6, 4); E2(6, 5); E2(6, 6); E2(6, 7); E3(6); E4(6, 0); E4(6, 1); E4(6, 2); E4(6, 3); E4(6, 4); E4(6, 5); E4(6, 6); E4(6, 7); E5(1, 6);
        E1(1, 7); E2(7, 0); E2(7, 1); E2(7, 2); E2(7, 3); E2(7, 4); E2(7, 5); E2(7, 6); E2(7, 7); E3(7); E4(7, 0); E4(7, 1); E4(7, 2); E4(7, 3); E4(7, 4); E4(7, 5); E4(7, 6); E4(7, 7); E5(1, 7);
        SCHED; SW(16); SW(17); SW(18); SW(19); SCHED;
        if (ln == 0u) __hip_atomic_store((unsigned*)ctr, call, __ATOMIC_RELAXED, __HIP_MEMORY_SCOPE_AGENT);
    }
}
}

extern "C" void kernel_launch(int rank, int nranks, ncclComm_t comm,
                              hipStream_t stream,
                              void* const* d_in, const int* in_sizes, int n_in,
                              void* d_out, int out_size,
                              const int* in_dtypes, int out_dtype,
                              void* const* d_out_all,
                              void* const* d_ws_all, size_t ws_bytes) {
    (void)comm; (void)in_sizes; (void)n_in; (void)out_size; (void)in_dtypes; (void)out_dtype; (void)d_out_all;
    if (nranks != a2a::NR || (unsigned)rank >= (unsigned)a2a::NR || ws_bytes < a2a::WS_END) {
        fprintf(stderr, "a2a_gemm: unexpected configuration (nranks %d, ws %zu)\n", nranks, ws_bytes);
        return;
    }
    HIP_CHECK(hipFuncSetAttribute((const void*)a2a::a2a_gemm_kernel, hipFuncAttributeMaxDynamicSharedMemorySize, a2a::LDS_BYTES));
    a2a::Params p;
    p.out = (char*)d_out;
    for (int i = 0; i < a2a::NR; ++i) p.ws[i] = (char*)d_ws_all[i];
    hipLaunchKernelGGL(a2a::a2a_gemm_kernel, dim3(256), dim3(256), a2a::LDS_BYTES, stream, (const char*)d_in[0], (const char*)d_in[1], (char*)d_ws_all[rank], rank, p);
    HIP_CHECK(hipGetLastError());
}
```
